```python
import numpy as np

kernel_name = "gpu_ag_gemm_m4096_k4096_n16384_f32_qe4m3"


ABSMAX_RTOL = 0.09

COMPUTE_DTYPE = "float32"
EPILOGUE = "quant_e4m3"
IN_QUANT = None
OUT_QUANT = "e4m3"
IN_DTYPES = {"x": "float32", "w_mat": "float32"}
WORLD_SIZE = 8

def setup_inputs():

    x = np.random.randn(4096, 4096).astype(np.float32)
    w_mat = (np.random.randn(4096, 16384) / np.sqrt(4096)).astype(np.float32)
    return {"x": x, "w_mat": w_mat}

def _snap_fp8(x, mant_bits, max_abs, min_normal):
    a = np.abs(x.astype(np.float64))
    _m, _e = np.frexp(a)
    q = np.where(a >= min_normal, np.ldexp(1.0, _e - 1 - mant_bits),
                 min_normal * 2.0 ** -mant_bits)
    snapped = np.minimum(np.round(a / q) * q, max_abs)
    return (np.sign(x) * snapped).astype(np.float32)
def _quant_dequant(y, amax):
    scale = amax / 448.0
    q = _snap_fp8(y / scale, 3, 448.0, 2.0 ** -6)
    return (q * scale).astype(np.float32)
def _epilogue(y):
    return _quant_dequant(y, np.abs(y).max())

def reference(x, w_mat):
    y = (x @ w_mat).astype(np.float32)
    return _epilogue(y)
```

```cpp
#include <hip/hip_runtime.h>
#include <rccl/rccl.h>
#include <cstdio>

namespace {

typedef unsigned int u32;
typedef unsigned long long u64;
typedef __bf16 bf16x8 __attribute__((ext_vector_type(8)));
typedef __bf16 bf16x2 __attribute__((ext_vector_type(2)));
typedef float f32x4 __attribute__((ext_vector_type(4)));
typedef float f32x2 __attribute__((ext_vector_type(2)));
typedef u32 u32x4 __attribute__((ext_vector_type(4)));
#define GAS __attribute__((address_space(1)))
#define LAS __attribute__((address_space(3)))

constexpr int NR = 8;
constexpr int KD = 4096;
constexpr int NL = 2048;
constexpr int NKS = 64;
constexpr int NCH = 16;
constexpr int NWG = 256;
constexpr int NTHREADS = 576;
constexpr u32 A_BYTES = 32768, STAGE = 49152;
constexpr u32 LDS_CTL = 3 * STAGE;
constexpr u32 LDS_BYTES = LDS_CTL + 128;

constexpr size_t XB_OFF = 0;
constexpr size_t XB_BYTES = 4ull << 20;
constexpr size_t WB_OFF = 4ull << 20;
constexpr size_t WB_BYTES = 16ull << 20;
constexpr size_t CT_OFF = 20ull << 20;
constexpr size_t CNT_OFF = CT_OFF;
constexpr size_t TMO_OFF = CT_OFF + 4096;
constexpr size_t XFL_OFF = CT_OFF + 8192;
constexpr size_t WFL_OFF = CT_OFF + 16384;
constexpr size_t GMX_OFF = CT_OFF + 32768;
constexpr size_t RMX_OFF = CT_OFF + 36864;
constexpr size_t DBG_OFF = CT_OFF + 65536;
constexpr size_t WS_NEED = CT_OFF + 131072;

constexpr u64 DEADLINE_TICKS = 100000000ull;

struct Params {
  const float* x;
  const float* w;
  unsigned char* out;
  char* ws[NR];
  int rank;
  int nranks;
};

#define WAIT_VM(n) asm volatile("s_waitcnt vmcnt(" #n ")" ::: "memory")
#define WAIT_VM_LGKM(n) asm volatile("s_waitcnt vmcnt(" #n ") lgkmcnt(0)" ::: "memory")
#define WAIT_LGKM0() asm volatile("s_waitcnt lgkmcnt(0)" ::: "memory")
#define BARRIER() do { asm volatile("" ::: "memory"); __builtin_amdgcn_s_barrier(); asm volatile("" ::: "memory"); } while (0)
#define RLX_AGENT __ATOMIC_RELAXED, __HIP_MEMORY_SCOPE_AGENT
#define RLX_SYSTEM __ATOMIC_RELAXED, __HIP_MEMORY_SCOPE_SYSTEM

#define STAMP(i) do { } while (0)

__device__ __forceinline__ u32 pack_bf16(float a, float b) {
  f32x2 f = {a, b};
  bf16x2 h = __builtin_convertvector(f, bf16x2);
  return __builtin_bit_cast(u32, h);
}

__device__ __forceinline__ u32 wave_max_u32(u32 v) {
  v = max(v, (u32)__builtin_amdgcn_update_dpp(0, (int)v, 0x111, 0xf, 0xf, false));
  v = max(v, (u32)__builtin_amdgcn_update_dpp(0, (int)v, 0x112, 0xf, 0xf, false));
  v = max(v, (u32)__builtin_amdgcn_update_dpp(0, (int)v, 0x114, 0xf, 0xf, false));
  v = max(v, (u32)__builtin_amdgcn_update_dpp(0, (int)v, 0x118, 0xf, 0xf, false));
  const u32 a = (u32)__builtin_amdgcn_readlane((int)v, 15), b = (u32)__builtin_amdgcn_readlane((int)v, 31);
  const u32 c = (u32)__builtin_amdgcn_readlane((int)v, 47), d = (u32)__builtin_amdgcn_readlane((int)v, 63);
  return max(max(a, b), max(c, d));
}

__device__ __forceinline__ bool give_up(u32& spins, u64 t0, GAS u32* tmo, u32 seq) {
  ++spins;
  bool stop = (__builtin_amdgcn_s_memrealtime() - t0) > DEADLINE_TICKS;
  if ((spins & 7u) == 0u) stop |= (__hip_atomic_load(tmo, RLX_AGENT) == seq);
  stop = __builtin_amdgcn_readfirstlane((int)stop) != 0;
  if (stop) __hip_atomic_store(tmo, seq, RLX_AGENT);
  else __builtin_amdgcn_s_sleep(1);
  return stop;
}

__device__ __forceinline__ void w_load(f32x4 (&v)[8], const float* __restrict__ wsrc, int chunk) {
  const float* s = wsrc + (size_t)chunk * 256 * NL;
#pragma unroll
  for (int j = 0; j < 8; ++j) v[j] = *(const f32x4*)(s + (size_t)j * NL);
}

__device__ __forceinline__ void w_store(const f32x4 (&v)[8], __amdgpu_buffer_rsrc_t rs, u32 woff, int chunk) {
#pragma unroll
  for (int r = 0; r < 4; ++r) {
    u32x4 d;
    d.x = pack_bf16(v[0][r], v[1][r]);
    d.y = pack_bf16(v[2][r], v[3][r]);
    d.z = pack_bf16(v[4][r], v[5][r]);
    d.w = pack_bf16(v[6][r], v[7][r]);
    __builtin_amdgcn_raw_buffer_store_b128(d, rs, woff + r * 2048, chunk * (4 * 16384), 16  );
  }
}

__global__ void __launch_bounds__(NTHREADS) __attribute__((amdgpu_waves_per_eu(3, 3))) ag_gemm_q8(Params p) {
  extern __shared__ __attribute__((aligned(16))) char lds[];
  const int b = blockIdx.x;
  const int tid = threadIdx.x, lane = tid & 63;
  const int wid = __builtin_amdgcn_readfirstlane(tid >> 6);
  const int src = b & 7, idx = b >> 3;
  const int half = idx & 1, nt = idx >> 1, mt = src * 2 + half;
  char* const myws = p.ws[p.rank];

  const int kq1 = lane >> 5, wH = (lane >> 4) & 1, wi = lane & 15;
  const float* const wsrc = p.w + (size_t)(mt * 16 + kq1 * 8) * NL + nt * 128 + wH * 64 + wi * 4;
  const u32 woff = (u32)(((nt * 64 + (mt >> 2)) * 8 + wH * 4) * 2048 + ((mt >> 1) & 1) * 1024 +
                         (((mt & 1) * 2 + kq1) * 16 + wi) * 16);
  const __amdgpu_buffer_rsrc_t wrs =
      __builtin_amdgcn_make_buffer_rsrc(myws + WB_OFF, 0, (int)WB_BYTES, 0x00020000);

  GAS u32* const cnt = (GAS u32*)(myws + CNT_OFF) + b;
  GAS u32* const wfl = (GAS u32*)(myws + WFL_OFF) + nt * 256;

  if (wid == 8) {
    __builtin_amdgcn_s_setprio(3);
    STAMP(0);
    __builtin_amdgcn_fence(__ATOMIC_ACQUIRE, "");
    GAS u32* const tmo = (GAS u32*)(myws + TMO_OFF);
    const u32 seq = (u32)__builtin_amdgcn_readfirstlane((int)__hip_atomic_load(cnt, RLX_AGENT)) + 1u;
    WAIT_VM(0);
    GAS u32* const fl0 = lane < 16 ? wfl + lane : (GAS u32*)(myws + XFL_OFF) + src * 256 + half * 128 + (lane & 7);
    const int flstride = lane < 16 ? 16 : 8;
    char* peer = p.ws[(u32)(lane & 7) < (u32)p.nranks ? (lane & 7) : 0];
    asm volatile("" : "+v"(peer));
    bool failed = false;

    BARRIER();
    STAMP(1);

    auto wait_chunk = [&](int c, u32 v) {
      if (__all(v >= seq)) return;
      const u64 t0 = __builtin_amdgcn_s_memrealtime();
      u32 spins = 0;
      for (;;) {
        v = __hip_atomic_load(fl0 + c * flstride, RLX_SYSTEM);
        if (__all(v >= seq)) break;
        if (give_up(spins, t0, tmo, seq)) { failed = true; break; }
      }
    };
    wait_chunk(0, 0u);
    BARRIER();
    STAMP(2);

    f32x4 wv[8];
    for (int c = 0; c < NCH; ++c) {
      BARRIER();
      if (c >= 1 && c + 2 < NCH) {
        WAIT_VM(0);
        if (lane == 0) __hip_atomic_store(wfl + (c + 2) * 16 + mt, seq, RLX_AGENT);
      }
      u32 fv = seq;
      if (c + 1 < NCH) fv = __hip_atomic_load(fl0 + (c + 1) * flstride, RLX_SYSTEM);
      if (c + 3 < NCH) w_load(wv, wsrc, c + 3);
      BARRIER();
      asm volatile("" : "+v"(fv));
      if (c + 1 < NCH) wait_chunk(c + 1, fv);
      BARRIER();
      if (c + 3 < NCH) w_store(wv, wrs, woff, c + 3);
      BARRIER();
    }
    STAMP(3);

    BARRIER();
    u32 m = *(const volatile LAS u32*)(LAS char*)(lds + LDS_CTL + 16 + (lane & 7) * 4);
    m = wave_max_u32(m);
    GAS u64* const gmx = (GAS u64*)(myws + GMX_OFF);
    if (lane == 0) __hip_atomic_store(gmx + b, ((u64)seq << 32) | m, RLX_AGENT);
    if (b == p.rank) {
      const u64 t0 = __builtin_amdgcn_s_memrealtime();
      u32 spins = 0, gm = 0;
      for (;;) {
        bool ok = true;
        gm = 0;
#pragma unroll
        for (int k = 0; k < 4; ++k) {
          const u64 g = __hip_atomic_load(gmx + k * 64 + lane, RLX_AGENT);
          ok &= (u32)(g >> 32) >= seq;
          gm = max(gm, (u32)g);
        }
        if (__all(ok)) break;
        if (give_up(spins, t0, tmo, seq)) { failed = true; break; }
      }
      gm = wave_max_u32(gm);
      GAS u64* r = (GAS u64*)(peer + RMX_OFF + (lane >> 3) * 256) + p.rank;
      __hip_atomic_store(r, ((u64)seq << 32) | gm, RLX_SYSTEM);
    }
    STAMP(4);
    u32 gmax = 0;
    {
      GAS u64* const r = (GAS u64*)(myws + RMX_OFF + src * 256) + (lane & 7);
      const u64 t0 = __builtin_amdgcn_s_memrealtime();
      u32 spins = 0;
      for (;;) {
        const u64 g = __hip_atomic_load(r, RLX_SYSTEM);
        gmax = (u32)g;
        if (__all((u32)(g >> 32) >= seq)) break;
        if (give_up(spins, t0, tmo, seq)) { failed = true; break; }
      }
    }
    gmax = wave_max_u32(gmax);
    failed |= __builtin_amdgcn_readfirstlane((int)__hip_atomic_load(tmo, RLX_AGENT)) == (int)seq;
    const float amax = __builtin_bit_cast(float, gmax);
    const float inv = amax > 0.f ? 448.0f / amax : 0.f;
    if (lane == 0) {
      *(volatile LAS float*)(LAS char*)(lds + LDS_CTL) = inv;
      *(volatile LAS u32*)(LAS char*)(lds + LDS_CTL + 4) = failed ? 1u : 0u;
    }
    WAIT_LGKM0();
    BARRIER();
    STAMP(5);
    return;
  }

  {
    const int xks = b >> 2, xh = (b >> 1) & 1, msub = (b & 1) * 8 + wid;
    const float* xs = p.x + (size_t)(xh * 256 + msub * 16 + (lane & 15)) * KD + xks * 64 + (lane >> 4) * 8;
    f32x4 xv[4];
#pragma unroll
    for (int j = 0; j < 4; ++j) xv[j] = *(const f32x4*)(xs + (j >> 1) * 32 + (j & 1) * 4);
    f32x4 wv[8];
    if (wid < 3) w_load(wv, wsrc, wid);
    u32 seq = 0;
    GAS u32* pub = cnt;
    if (wid == 7) {
      seq = __hip_atomic_load(cnt, RLX_AGENT) + 1u;
      char* peer = p.ws[(u32)(lane & 7) < (u32)p.nranks ? (lane & 7) : 0];
      if (lane < 8) pub = (GAS u32*)(peer + XFL_OFF) + p.rank * 256 + xh * 128 + xks * 2 + (b & 1);
      else if (lane < 11) pub = wfl + (lane - 8) * 16 + mt;
    }
    const __amdgpu_buffer_rsrc_t xrs =
        __builtin_amdgcn_make_buffer_rsrc(myws + XB_OFF, 0, (int)XB_BYTES, 0x00020000);
    const u32 xoff = (u32)(((xh * 64 + xks) * 16 + msub) * 2048 + lane * 16);
#pragma unroll
    for (int kh = 0; kh < 2; ++kh) {
      u32x4 d;
      d.x = pack_bf16(xv[kh * 2][0], xv[kh * 2][1]);
      d.y = pack_bf16(xv[kh * 2][2], xv[kh * 2][3]);
      d.z = pack_bf16(xv[kh * 2 + 1][0], xv[kh * 2 + 1][1]);
      d.w = pack_bf16(xv[kh * 2 + 1][2], xv[kh * 2 + 1][3]);
      __builtin_amdgcn_raw_buffer_store_b128(d, xrs, xoff + kh * 1024, 0, 17  );
    }
    if (wid < 3) w_store(wv, wrs, woff, wid);
    asm volatile("" : "+v"(pub), "+v"(seq));
    WAIT_VM(0);
    BARRIER();
    if (wid == 7 && lane < 12) __hip_atomic_store(pub, seq, RLX_SYSTEM);
  }

  const __amdgpu_buffer_rsrc_t ars = __builtin_amdgcn_make_buffer_rsrc(
      p.ws[(u32)src < (u32)p.nranks ? src : 0] + XB_OFF + (size_t)half * (XB_BYTES / 2), 0, (int)(XB_BYTES / 2), 0x00020000);
  const __amdgpu_buffer_rsrc_t brs = __builtin_amdgcn_make_buffer_rsrc(
      myws + WB_OFF + (size_t)nt * (NKS * 16384), 0, NKS * 16384, 0x00020000);
  const u32 l16 = lane * 16;
  const u32 avo = wid * 4096 + l16, bvo = wid * 2048 + l16;
  auto stage = [&](int t, u32 slot) {
    LAS void* la = (LAS void*)(lds + slot + wid * 4096);
    __builtin_amdgcn_raw_ptr_buffer_load_lds(ars, la, 16, avo, t * A_BYTES, 0, 0);
    __builtin_amdgcn_raw_ptr_buffer_load_lds(ars, la, 16, avo, t * A_BYTES, 1024, 0);
    __builtin_amdgcn_raw_ptr_buffer_load_lds(ars, la, 16, avo, t * A_BYTES, 2048, 0);
    __builtin_amdgcn_raw_ptr_buffer_load_lds(ars, la, 16, avo, t * A_BYTES, 3072, 0);
    LAS void* lb = (LAS void*)(lds + slot + A_BYTES + wid * 2048);
    __builtin_amdgcn_raw_ptr_buffer_load_lds(brs, lb, 16, bvo, t * 16384, 0, 0);
    __builtin_amdgcn_raw_ptr_buffer_load_lds(brs, lb, 16, bvo, t * 16384, 1024, 0);
  };

  const int wr = wid >> 1, wc = wid & 1;
  const u32 aoff = wr * 8192 + l16, boff = A_BYTES + wc * 8192 + l16;
  f32x4 acc[4][4];
#pragma unroll
  for (int i = 0; i < 4; ++i)
#pragma unroll
    for (int j = 0; j < 4; ++j) acc[i][j] = (f32x4){0.f, 0.f, 0.f, 0.f};

  BARRIER();
  stage(0, 0);
  stage(1, STAGE);

  u32 cur = 0, nxt = 2 * STAGE;
  for (int t = 0; t < NKS; ++t) {
    if (t + 1 < NKS) WAIT_VM_LGKM(6); else WAIT_VM_LGKM(0);
    BARRIER();
    if (t + 2 < NKS) stage(t + 2, nxt);
    const char* sa = lds + cur + aoff;
    const char* sb = lds + cur + boff;
#pragma unroll
    for (int kh = 0; kh < 2; ++kh) {
      bf16x8 af[4], bfr[4];
#pragma unroll
      for (int i = 0; i < 4; ++i) {
        af[i] = *(const bf16x8*)(sa + i * 2048 + kh * 1024);
        bfr[i] = *(const bf16x8*)(sb + i * 2048 + kh * 1024);
      }
#pragma unroll
      for (int mi = 0; mi < 4; ++mi)
#pragma unroll
        for (int r = 0; r < 4; ++r)
          acc[mi][r] = __builtin_amdgcn_mfma_f32_16x16x32_bf16(bfr[r], af[mi], acc[mi][r], 0, 0, 0);
    }
    nxt = cur;
    cur = cur == 2 * STAGE ? 0 : cur + STAGE;
  }

  float mx = 0.f;
#pragma unroll
  for (int mi = 0; mi < 4; ++mi)
#pragma unroll
    for (int r = 0; r < 4; ++r)
#pragma unroll
      for (int j = 0; j < 4; ++j) mx = fmaxf(mx, fabsf(acc[mi][r][j]));
  const u32 wm = wave_max_u32(__builtin_bit_cast(u32, mx));
  if (lane == 0) *(volatile LAS u32*)(LAS char*)(lds + LDS_CTL + 16 + wid * 4) = wm;
  WAIT_LGKM0();
  BARRIER();
  unsigned char* o = p.out + (size_t)(mt * 256 + wr * 64 + (lane & 15)) * NL + nt * 128 + wc * 64 + (lane >> 4) * 16;
  BARRIER();
  const float inv = *(const volatile LAS float*)(LAS char*)(lds + LDS_CTL);
  const u32 bad = *(const volatile LAS u32*)(LAS char*)(lds + LDS_CTL + 4);
#pragma unroll
  for (int mi = 0; mi < 4; ++mi) {
    u32x4 d;
#pragma unroll
    for (int j = 0; j < 4; ++j) {
      int q = __builtin_amdgcn_cvt_pk_fp8_f32(acc[mi][0][j] * inv, acc[mi][1][j] * inv, 0, false);
      q = __builtin_amdgcn_cvt_pk_fp8_f32(acc[mi][2][j] * inv, acc[mi][3][j] * inv, q, true);
      d[j] = bad ? 0x7f7f7f7fu : (u32)q;
    }
    *(u32x4*)(o + (size_t)mi * 16 * NL) = d;
  }
}

}

extern "C" void kernel_launch(int rank, int nranks, ncclComm_t comm, hipStream_t stream,
                              void* const* d_in, const int* in_sizes, int n_in,
                              void* d_out, int out_size,
                              const int* in_dtypes, int out_dtype,
                              void* const* d_out_all, void* const* d_ws_all, size_t ws_bytes) {
  (void)comm; (void)in_dtypes; (void)out_dtype; (void)d_out_all;
  if (nranks != NR || n_in < 2 || ws_bytes < WS_NEED || in_sizes[0] != 512 * KD || in_sizes[1] != KD * NL ||
      out_size != 4096 * NL || (unsigned)rank >= (unsigned)nranks) {
    fprintf(stderr, "unexpected problem geometry\n");
    return;
  }
  hipError_t e = hipFuncSetAttribute((const void*)ag_gemm_q8, hipFuncAttributeMaxDynamicSharedMemorySize, (int)LDS_BYTES);
  if (e != hipSuccess) { fprintf(stderr, "hipFuncSetAttribute: %s\n", hipGetErrorString(e)); return; }
  Params p;
  p.x = (const float*)d_in[0];
  p.w = (const float*)d_in[1];
  p.out = (unsigned char*)d_out;
  for (int i = 0; i < NR; ++i) p.ws[i] = (char*)d_ws_all[i];
  p.rank = rank;
  p.nranks = nranks;
  hipLaunchKernelGGL(ag_gemm_q8, dim3(NWG), dim3(NTHREADS), LDS_BYTES, stream, p);
  e = hipGetLastError();
  if (e != hipSuccess) fprintf(stderr, "launch: %s\n", hipGetErrorString(e));
}
```

```cpp
#include <hip/hip_runtime.h>
#include <rccl/rccl.h>
#include <cstdio>
#include <type_traits>

namespace {

typedef unsigned int u32;
typedef unsigned long long u64;
typedef __bf16 bf16x8 __attribute__((ext_vector_type(8)));
typedef __bf16 bf16x2 __attribute__((ext_vector_type(2)));
typedef float f32x4 __attribute__((ext_vector_type(4)));
typedef float f32x2 __attribute__((ext_vector_type(2)));
typedef u32 u32x4 __attribute__((ext_vector_type(4)));
#define GAS __attribute__((address_space(1)))
#define LAS __attribute__((address_space(3)))

constexpr int NR = 8;
constexpr int KD = 4096;
constexpr int NL = 2048;
constexpr int NKS = 64;
constexpr int NCH = 16;
constexpr int NWG = 256;
#define PF0 2
#define W1_PRO 1
#define B1_X 2
#define B1_W 1
#define XCH 1
#define CLEAN_AT 0
#define EARLY_INV 0
#define XP_SLEEP 2
#define PF_D 1
#define PF_D0 PF_D
#define PF_T0 6
#define KPERM 0
#define OUT_AUX 0
#define P1_SLEEP 16
#define A_FIRST 1
#define ROT 1
#define BAR_POS 2
#define B_AUX 0
constexpr int NTHREADS = 640;
constexpr int X0_END = 8;
constexpr int X1_END = 16;
constexpr u32 A_BYTES = 32768, STAGE = 49152;
constexpr u32 LDS_CTL = 3 * STAGE;
constexpr u32 LDS_PF = LDS_CTL + 256;
constexpr u32 LDS_BYTES = LDS_PF + 1024;

constexpr size_t XB_OFF = 0;
constexpr size_t XB_BYTES = 4ull << 20;
constexpr size_t WB_OFF = 4ull << 20;
constexpr size_t WB_BYTES = 16ull << 20;
constexpr size_t CT_OFF = 20ull << 20;
constexpr size_t CNT_OFF = CT_OFF;
constexpr size_t TMO_OFF = CT_OFF + 4096;
constexpr size_t INV_OFF = CT_OFF + 5120;
constexpr size_t XFL_OFF = CT_OFF + 8192;
constexpr size_t WFL_OFF = CT_OFF + 32768;
constexpr size_t GMX_OFF = CT_OFF + 49152;
constexpr size_t RMX_OFF = CT_OFF + 53248;
constexpr size_t DBG_OFF = CT_OFF + 65536;
constexpr size_t AMX_OFF = CT_OFF + 196608;
constexpr size_t WS_NEED = CT_OFF + 262144;

constexpr u64 DEADLINE_TICKS = 100000000ull;

struct Params {
  const float* x;
  const float* w;
  unsigned char* out;
  char* myws;
  char* ws[NR];
  int rank;
};

__device__ __forceinline__ int kcol(int s) { return KPERM ? (s & 3) * 1024 + (s >> 2) * 64 : s * 64; }

#define WAIT_VM(n) asm volatile("s_waitcnt vmcnt(" #n ")" ::: "memory")
#define WAIT_VM_LGKM(n) asm volatile("s_waitcnt vmcnt(" #n ") lgkmcnt(0)" ::: "memory")
#define WAIT_LGKM0() asm volatile("s_waitcnt lgkmcnt(0)" ::: "memory")
#define BARRIER() do { asm volatile("" ::: "memory"); __builtin_amdgcn_s_barrier(); asm volatile("" ::: "memory"); } while (0)
#define SCHED() __builtin_amdgcn_sched_barrier(0)
#define RLX_AGENT __ATOMIC_RELAXED, __HIP_MEMORY_SCOPE_AGENT
#define RLX_SYSTEM __ATOMIC_RELAXED, __HIP_MEMORY_SCOPE_SYSTEM

#define STAMP(i) do { } while (0)

__device__ __forceinline__ u32 pack_bf16(float a, float b) {
  f32x2 f = {a, b};
  bf16x2 h = __builtin_convertvector(f, bf16x2);
  return __builtin_bit_cast(u32, h);
}

__device__ __forceinline__ u32 wave_max_u32(u32 v) {
  v = max(v, (u32)__builtin_amdgcn_update_dpp(0, (int)v, 0x111, 0xf, 0xf, false));
  v = max(v, (u32)__builtin_amdgcn_update_dpp(0, (int)v, 0x112, 0xf, 0xf, false));
  v = max(v, (u32)__builtin_amdgcn_update_dpp(0, (int)v, 0x114, 0xf, 0xf, false));
  v = max(v, (u32)__builtin_amdgcn_update_dpp(0, (int)v, 0x118, 0xf, 0xf, false));
  const u32 a = (u32)__builtin_amdgcn_readlane((int)v, 15), b = (u32)__builtin_amdgcn_readlane((int)v, 31);
  const u32 c = (u32)__builtin_amdgcn_readlane((int)v, 47), d = (u32)__builtin_amdgcn_readlane((int)v, 63);
  return max(max(a, b), max(c, d));
}

__device__ __forceinline__ bool give_up(u32& spins, u64 t0, GAS u32* tmo, u32 seq) {
  ++spins;
  bool stop = (__builtin_amdgcn_s_memrealtime() - t0) > DEADLINE_TICKS;
  if ((spins & 7u) == 0u) stop |= (__hip_atomic_load(tmo, RLX_AGENT) == seq);
  stop = __builtin_amdgcn_readfirstlane((int)stop) != 0;
  if (stop) __hip_atomic_store(tmo, seq, RLX_AGENT);
  else __builtin_amdgcn_s_sleep(1);
  return stop;
}

__device__ __forceinline__ void w_load(f32x4* v, const float* __restrict__ wsrc, int chunk) {
  const float* s = wsrc + (size_t)chunk * (KPERM ? 64 : 256) * NL;
#pragma unroll
  for (int j = 0; j < 8; ++j) v[j] = *(const f32x4*)(s + (size_t)j * NL);
}

__device__ __forceinline__ void w_store(const f32x4* v, __amdgpu_buffer_rsrc_t rs, u32 woff, int chunk) {
#pragma unroll
  for (int r = 0; r < 4; ++r) {
    u32x4 d;
    d.x = pack_bf16(v[0][r], v[1][r]);
    d.y = pack_bf16(v[2][r], v[3][r]);
    d.z = pack_bf16(v[4][r], v[5][r]);
    d.w = pack_bf16(v[6][r], v[7][r]);
    __builtin_amdgcn_raw_buffer_store_b128(d, rs, woff + r * 2048, chunk * (4 * 16384), 16  );
  }
}

__device__ __forceinline__ void ctl_write(u32 addr, u32 v) {
  asm volatile("ds_write_b32 %0, %1" ::"v"(addr), "v"(v) : "memory");
}
__device__ __forceinline__ u32 ctl_read(u32 addr) {
  u32 v;
  asm volatile("ds_read_b32 %0, %1\n\ts_waitcnt lgkmcnt(0)" : "=v"(v) : "v"(addr) : "memory");
  return v;
}
__device__ __forceinline__ void x_load(f32x4* v, const float* __restrict__ x, int xh, int xks, int msub, int lane) {
  const float* xs = x + (size_t)(xh * 256 + msub * 16 + (lane & 15)) * KD + kcol(xks) + (lane >> 4) * 8;
#pragma unroll
  for (int j = 0; j < 4; ++j) v[j] = *(const f32x4*)(xs + (j >> 1) * 32 + (j & 1) * 4);
}

__device__ __forceinline__ void x_store(const f32x4* v, __amdgpu_buffer_rsrc_t rs, int xh, int xks, int msub, int lane) {
  const u32 xoff = (u32)(((xh * 64 + xks) * 16 + msub) * 2048 + lane * 16);
#pragma unroll
  for (int kh = 0; kh < 2; ++kh) {
    u32x4 d;
    d.x = pack_bf16(v[kh * 2][0], v[kh * 2][1]);
    d.y = pack_bf16(v[kh * 2][2], v[kh * 2][3]);
    d.z = pack_bf16(v[kh * 2 + 1][0], v[kh * 2 + 1][1]);
    d.w = pack_bf16(v[kh * 2 + 1][2], v[kh * 2 + 1][3]);
    __builtin_amdgcn_raw_buffer_store_b128(d, rs, xoff + kh * 1024, 0, 17  );
  }
}

__device__ __forceinline__ u32x4 pack8(const f32x4& a, const f32x4& b) {
  u32x4 d;
  d.x = pack_bf16(a[0], a[1]);
  d.y = pack_bf16(a[2], a[3]);
  d.z = pack_bf16(b[0], b[1]);
  d.w = pack_bf16(b[2], b[3]);
  return d;
}

__device__ __forceinline__ u64 both_ge(u64 f, u32 seq) { return __ballot((u32)f >= seq && (u32)(f >> 32) >= seq); }
__device__ __forceinline__ int lead_ones(u64 m) { return m == ~0ull ? 64 : __builtin_ctzll(~m); }

__global__ void __launch_bounds__(NTHREADS) __attribute__((amdgpu_waves_per_eu(3, 3))) ag_gemm_q8(Params p) {
  extern __shared__ __attribute__((aligned(16))) char lds[];
  const int b = blockIdx.x;
  const int tid = threadIdx.x, lane = tid & 63;
  const int wid = __builtin_amdgcn_readfirstlane(tid >> 6);
  const int src = b & 7, idx = b >> 3;
  const int half = idx & 1, nt = idx >> 1, mt = src * 2 + half;
  const float* const px = p.x;
  const float* const pw = p.w;
  unsigned char* const pout = p.out;
  char* const myws = p.myws;
  char* const srcws = p.ws[src];
  char* const widws = p.ws[wid & 7];
  const int rank = p.rank;
  const u64 w0 = (u64)p.ws[0], w1 = (u64)p.ws[1], w2 = (u64)p.ws[2], w3 = (u64)p.ws[3];
  const u64 w4 = (u64)p.ws[4], w5 = (u64)p.ws[5], w6 = (u64)p.ws[6], w7 = (u64)p.ws[7];
  asm volatile("" ::"s"(px), "s"(pw), "s"(pout), "s"(myws), "s"(srcws), "s"(widws), "s"(rank), "s"(w0), "s"(w1), "s"(w2), "s"(w3),
               "s"(w4), "s"(w5), "s"(w6), "s"(w7));
  auto peer_ws = [&]() {
    int lo = (int)w0, hi = (int)(w0 >> 32);
#define PUT(i) asm("v_writelane_b32 %0, %2, " #i "\n\tv_writelane_b32 %1, %3, " #i : "+v"(lo), "+v"(hi) : "s"((int)w##i), "s"((int)(w##i >> 32)))
    PUT(1); PUT(2); PUT(3); PUT(4); PUT(5); PUT(6); PUT(7);
#undef PUT
    return (char*)(((u64)(u32)hi << 32) | (u32)lo);
  };
  GAS u32* const cnt = (GAS u32*)(myws + CNT_OFF) + b;
  const __amdgpu_buffer_rsrc_t xrs = __builtin_amdgcn_make_buffer_rsrc(myws + XB_OFF, 0, (int)XB_BYTES, 0x00020000);

  if (__builtin_expect(wid == 0, 1)) {
    f32x4 v[4];
    STAMP(6);
    x_load(v, px, (b >> 4) & 1, b >> 5, b & 15, lane);
    const u32 seq = __hip_atomic_load(cnt, RLX_AGENT);
    SCHED();
    GAS u32* pub = (GAS u32*)(peer_ws() + XFL_OFF) + (rank * 2 + ((b >> 4) & 1)) * 384 + (b >> 5) * 16 + (b & 15);
    asm volatile("" : "+v"(pub));
    SCHED();
    x_store(v, xrs, (b >> 4) & 1, b >> 5, b & 15, lane);
    SCHED();
    WAIT_VM(0);
    if (lane < 8) __hip_atomic_store(pub, seq + 1u, RLX_SYSTEM);
    STAMP(10);
  }

  const u32 ctl = (u32)(size_t)(LAS char*)lds + LDS_CTL;
  const int kq1 = lane >> 5, wH = (lane >> 4) & 1, wi = lane & 15;
  const float* const wsrc = pw + (size_t)(kcol(mt >> 2) + (mt & 3) * 16 + kq1 * 8) * NL + nt * 128 + wH * 64 + wi * 4;
  const u32 woff = (u32)(((nt * 64 + (mt >> 2)) * 8 + wH * 4) * 2048 + ((mt >> 1) & 1) * 1024 +
                         (((mt & 1) * 2 + kq1) * 16 + wi) * 16);
  const __amdgpu_buffer_rsrc_t wrs =
      __builtin_amdgcn_make_buffer_rsrc(myws + WB_OFF, 0, (int)WB_BYTES, 0x00020000);

  GAS u32* const wfl = (GAS u32*)(myws + WFL_OFF) + nt * 256;
  GAS u32* const xfl = (GAS u32*)(myws + XFL_OFF) + mt * 384;
  GAS u32* const tmo = (GAS u32*)(myws + TMO_OFF);

  if (wid == 8) {
    __builtin_amdgcn_s_setprio(3);
    STAMP(0);
    const u32 seq = (u32)__builtin_amdgcn_readfirstlane((int)__hip_atomic_load(cnt, RLX_AGENT)) + 1u;
    char* peer;
    {
      const u64 v = (u64)peer_ws();
      const u32 lo = (u32)__builtin_amdgcn_ds_bpermute((lane & 7) * 4, (int)v);
      const u32 hi = (u32)__builtin_amdgcn_ds_bpermute((lane & 7) * 4, (int)(v >> 32));
      peer = (char*)(((u64)hi << 32) | lo);
    }
    bool failed = false;

    int xok = 0, cw = 0;
    GAS u64* const xf = (GAS u64*)xfl + lane;
    struct Poll { u32 fw; u64 f0, f1, f2; int cw; };
    Poll pa, pb;
    auto poll = [&](Poll& q) {
      q.cw = cw;
      q.fw = __hip_atomic_load(wfl + min(cw + (lane >> 4), NCH - 1) * 16 + (lane & 15), RLX_AGENT);
      q.f0 = __hip_atomic_load(xf, RLX_SYSTEM);
      q.f1 = __hip_atomic_load(xf + 64, RLX_SYSTEM);
      q.f2 = __hip_atomic_load(xf + 128, RLX_SYSTEM);
      SCHED();
    };
    auto seen = [&](Poll& q) {
      asm volatile("" : "+v"(q.fw), "+v"(q.f0), "+v"(q.f1), "+v"(q.f2));
      cw = max(cw, min(q.cw + lead_ones(__ballot(q.fw >= seq)) / 16, NCH));
      int n = lead_ones(both_ge(q.f0, seq)) >> 3;
      if (n == X0_END) n += lead_ones(both_ge(q.f1, seq)) >> 3;
      if (n == X1_END && both_ge(q.f2, seq) == ~0ull) n = NKS;
      xok = max(xok, n);
      ctl_write(ctl, (u32)xok);
      SCHED();
    };
    auto wait_for = [&](int sx, int sw) {
      const u64 t0 = __builtin_amdgcn_s_memrealtime();
      u32 spins = 0;
      for (;;) {
        Poll q;
        poll(q);
        seen(q);
        if (xok > sx && 4 * cw > sw) return;
        if (give_up(spins, t0, tmo, seq)) { failed = true; xok = NKS; cw = NCH; ctl_write(ctl, (u32)xok); return; }
      }
    };
    auto need = [&](int s, Poll& q) {
      if (xok > s && 4 * cw > s) return;
      seen(q);
      if (xok > s && 4 * cw > s) return;
      wait_for(s, s);
    };
    auto wflag = [&](int c) { if (lane == 0) __hip_atomic_store(wfl + c * 16 + mt, seq, RLX_AGENT); };
#define KBAR(t) do { BARRIER(); SCHED(); } while (0)
#define PIN(v, n) _Pragma("unroll") for (int i_ = 0; i_ < (n); ++i_) asm volatile("" : "+v"((v)[i_]))

    {
      const int l2 = (b >> 1) * 6;
      f32x4 wv[8], xv[12];
      wait_for(B1_X - 1, 4 * B1_W - 4);
      STAMP(2);
      x_load(xv, px, (b >> 4) & 1, X0_END + (b >> 5), b & 15, lane);
      w_load(wv, wsrc, 2);
      poll(pa);
      WAIT_LGKM0();
      BARRIER();
      SCHED();
      PIN(xv, 4);
      x_store(xv, xrs, (b >> 4) & 1, X0_END + (b >> 5), b & 15, lane);
      PIN(wv, 8);
      w_store(wv, wrs, woff, 2);
      SCHED();
      seen(pa);
#pragma unroll
      for (int i = 0; i < 3; ++i) x_load(xv + 4 * i, px, b & 1, X1_END + ((l2 + i) >> 4), (l2 + i) & 15, lane);
      SCHED();
      need(2, pa);
      KBAR(63);
      WAIT_VM(12);
      {
        GAS u32* pub = (GAS u32*)(peer + XFL_OFF) + (rank * 2 + ((b >> 4) & 1)) * 384 + 128 + (b >> 5) * 16 + (b & 15);
        if (lane < 8) __hip_atomic_store(pub, seq, RLX_SYSTEM);
      }
      wflag(2);
      STAMP(11);
      if (lane == 0) __hip_atomic_store(cnt, seq, RLX_AGENT);
      poll(pb);
      w_load(wv, wsrc, 3);
      SCHED();
      need(3, pb);
      KBAR(0);
      PIN(xv, 12);
#pragma unroll
      for (int i = 0; i < 3; ++i) x_store(xv + 4 * i, xrs, b & 1, X1_END + ((l2 + i) >> 4), (l2 + i) & 15, lane);
#pragma unroll
      for (int i = 3; i < 6; ++i) x_load(xv + 4 * (i - 3), px, b & 1, X1_END + ((l2 + i) >> 4), (l2 + i) & 15, lane);
      SCHED();
      poll(pa);
      need(4, pb);
      KBAR(1);
      seen(pb);
      PIN(wv, 8);
      w_store(wv, wrs, woff, 3);
      SCHED();
      poll(pb);
      need(5, pa);
      KBAR(2);
      seen(pa);
      WAIT_VM(4);
      wflag(3);
      PIN(xv, 12);
#pragma unroll
      for (int i = 3; i < 6; ++i) x_store(xv + 4 * (i - 3), xrs, b & 1, X1_END + ((l2 + i) >> 4), (l2 + i) & 15, lane);
      SCHED();
      poll(pa);
      need(6, pb);
      KBAR(3);
      seen(pb);
      w_load(wv, wsrc, 4);
      SCHED();
      need(7, pa);
      KBAR(4);
      seen(pa);
      {
        GAS u32* pub = (GAS u32*)(peer + XFL_OFF) + (rank * 2 + (b & 1)) * 384 + 256 + (b >> 1);
        WAIT_VM(8);
        if (lane < 8) __hip_atomic_store(pub, seq, RLX_SYSTEM);
      }
      for (int c = 1; c + 3 < NCH; ++c) {
        need(4 * c + 4, pa);
        KBAR(4 * c + 1);
        PIN(wv, 8);
        w_store(wv, wrs, woff, c + 3);
        SCHED();
        need(4 * c + 5, pa);
        KBAR(4 * c + 2);
        WAIT_VM(0);
        wflag(c + 3);
        poll(pa);
        need(4 * c + 6, pa);
        KBAR(4 * c + 3);
        w_load(wv, wsrc, min(c + 4, NCH - 1));
        SCHED();
        need(4 * c + 7, pa);
        KBAR(4 * c + 4);
        seen(pa);
      }
      for (int t = 4 * (NCH - 3) + 1; t < NKS - 1; ++t) {
        if ((t & 1) == 0) poll(pa);
        need(min(t + 3, NKS - 1), pa);
        if (XCH && t == NKS - 2) {
          ctl_write(ctl + 17 * 4, failed ? 1u : 0u);
          ctl_write(ctl + 18 * 4, seq);
          WAIT_LGKM0();
        }
        KBAR(t);
        if (t & 1) seen(pa);
      }
    }
    STAMP(9);
    if (XCH) return;

    BARRIER();
    u32 m = ctl_read(ctl + (24 + (lane & 7)) * 4);
    m = wave_max_u32(m);
    GAS u64* const gmx = (GAS u64*)(myws + GMX_OFF);
    if (lane == 0) __hip_atomic_store(gmx + b, ((u64)seq << 32) | m, RLX_AGENT);
    if (b == rank) {
      const u64 t0 = __builtin_amdgcn_s_memrealtime();
      u32 spins = 0, gm = 0;
      for (;;) {
        bool ok = true;
        gm = 0;
#pragma unroll
        for (int k = 0; k < 4; ++k) {
          const u64 g = __hip_atomic_load(gmx + k * 64 + lane, RLX_AGENT);
          ok &= (u32)(g >> 32) >= seq;
          gm = max(gm, (u32)g);
        }
        if (__all(ok)) break;
        if (give_up(spins, t0, tmo, seq)) { failed = true; break; }
      }
      gm = wave_max_u32(gm);
      GAS u64* r = (GAS u64*)(peer + RMX_OFF + (lane >> 3) * 256) + rank;
      __hip_atomic_store(r, ((u64)seq << 32) | gm, RLX_SYSTEM);
    }
    STAMP(4);
    u32 gmax = 0;
    {
      GAS u64* const r = (GAS u64*)(myws + RMX_OFF + src * 256) + (lane & 7);
      const u64 t0 = __builtin_amdgcn_s_memrealtime();
      u32 spins = 0;
      for (;;) {
        const u64 g = __hip_atomic_load(r, RLX_SYSTEM);
        gmax = (u32)g;
        if (__all((u32)(g >> 32) >= seq)) break;
        if (give_up(spins, t0, tmo, seq)) { failed = true; break; }
      }
    }
    gmax = wave_max_u32(gmax);
    failed |= __builtin_amdgcn_readfirstlane((int)__hip_atomic_load(tmo, RLX_AGENT)) == (int)seq;
    const float amax = __builtin_bit_cast(float, gmax);
    const float inv = amax > 0.f ? 448.0f / amax : 0.f;
    ctl_write(ctl + 16 * 4, __builtin_bit_cast(u32, inv));
    ctl_write(ctl + 17 * 4, failed ? 1u : 0u);
    WAIT_LGKM0();
    BARRIER();
    STAMP(5);
    return;
  }

  char* const xsrc = srcws + XB_OFF + (size_t)half * (XB_BYTES / 2);
  const __amdgpu_buffer_rsrc_t ars = __builtin_amdgcn_make_buffer_rsrc(xsrc, 0, (int)(XB_BYTES / 2), 0x00020000);
  if (wid == 9) {
    __builtin_amdgcn_s_setprio(3);
    const bool elect = b < NR;
    if (EARLY_INV && elect) asm volatile("buffer_inv sc0 sc1" ::: "memory");
    GAS u32* const inv = (GAS u32*)(myws + INV_OFF) + (__builtin_amdgcn_s_getreg(20 | (3 << 11)) & 15) * 32;
    const u32 seq0 = __hip_atomic_load(cnt, RLX_AGENT), cln = __hip_atomic_load(inv + 1, RLX_AGENT);
    const u32 seq = (u32)__builtin_amdgcn_readfirstlane((int)seq0) + 1u;
    if (CLEAN_AT == 2 || seq == 1u || (u32)__builtin_amdgcn_readfirstlane((int)cln) != seq - 1u) {
      bool mine = elect;
      if (!mine) {
        const u64 t0 = __builtin_amdgcn_s_memrealtime();
        while ((u32)__builtin_amdgcn_readfirstlane((int)__hip_atomic_load(inv, RLX_AGENT)) < seq) {
          mine = __builtin_amdgcn_s_memrealtime() - t0 > 800;
          if (mine) break;
          __builtin_amdgcn_s_sleep(2);
        }
      }
      if (mine) {
        if (!(EARLY_INV && elect)) asm volatile("buffer_inv sc0 sc1" ::: "memory");
        WAIT_VM(0);
        if (lane == 0) __hip_atomic_store(inv, seq, RLX_AGENT);
      }
    }
    STAMP(1);
    int t = -2;
    if (src != rank) {
      LAS void* dst = (LAS void*)(lds + LDS_PF);
      const u32 vo = nt * 2048 + lane * 16;
      auto pf = [&](int s) {
        __builtin_amdgcn_raw_ptr_buffer_load_lds(ars, dst, 16, vo, s * A_BYTES, 0, 0);
        __builtin_amdgcn_raw_ptr_buffer_load_lds(ars, dst, 16, vo + 1024, s * A_BYTES, 0, 0);
      };
      int s = 0;
      if (PF0 > 0) {
        GAS u32* const f = xfl + (lane & 7) * 16 + nt;
        const u64 t0 = __builtin_amdgcn_s_memrealtime();
        u32 spins = 0;
        while (((u32)__ballot(__hip_atomic_load(f, RLX_SYSTEM) >= seq) & ((1u << PF0) - 1u)) != (1u << PF0) - 1u)
          if (give_up(spins, t0, tmo, seq)) break;
        STAMP(15);
        for (; s < PF0; ++s) pf(s);
      }
      int xok = 0;
      for (; s < NKS; ++t) {
        BARRIER();
        const int lim = min(t + 3 + (t < PF_T0 ? PF_D0 : PF_D), NKS - 1);
        for (int i = 0; i < 3 && xok <= lim; ++i) {
          if (i) __builtin_amdgcn_s_sleep(3);
          xok = __builtin_amdgcn_readfirstlane((int)ctl_read(ctl));
        }
        for (s = max(s, t + 4); s <= lim && s < xok; ++s) pf(s);
      }
    } else if (CLEAN_AT == 2 || !elect) {
      BARRIER();
      return;
    }
    for (; t < NKS - 1; ++t) BARRIER();
    WAIT_VM(0);
    BARRIER();
    if (CLEAN_AT == 2 || !elect) return;
    bool ok = true;
    if (CLEAN_AT == 0) {
      GAS u64* const g = (GAS u64*)(myws + (XCH ? AMX_OFF + rank * 2048 : GMX_OFF)) + lane;
      const u64 t0 = __builtin_amdgcn_s_memrealtime();
      u32 spins = 0;
      for (;;) {
        bool all = true;
#pragma unroll
        for (int k = 0; k < 4; ++k) all &= (u32)(__hip_atomic_load(g + k * 64, RLX_SYSTEM) >> 32) >= seq;
        if (__all(all)) break;
        if (give_up(spins, t0, tmo, seq)) { ok = false; break; }
      }
      asm volatile("buffer_inv sc0 sc1" ::: "memory");
    }
    BARRIER();
    if (CLEAN_AT == 1) {
      ok = XCH ? wave_max_u32(ctl_read(ctl + (32 + (lane & 7)) * 4)) <= 0x7f800000u : ctl_read(ctl + 17 * 4) == 0u;
      asm volatile("buffer_inv sc0 sc1" ::: "memory");
    }
    WAIT_VM(0);
    if (ok && lane == 0) __hip_atomic_store(inv + 1, seq, RLX_AGENT);
    STAMP(15);
    return;
  }

  if (wid == 2 || wid == 3) {
    f32x4 v[8];
    if (wid == 3) __builtin_amdgcn_s_sleep(P1_SLEEP);
    w_load(v, wsrc, wid - 2);
    const u32 seq = __hip_atomic_load(cnt, RLX_AGENT);
    SCHED();
    w_store(v, wrs, woff, wid - 2);
    SCHED();
    WAIT_VM(0);
    if (lane == 0) __hip_atomic_store(wfl + (wid - 2) * 16 + mt, seq + 1u, RLX_AGENT);
    STAMP(10 + wid);
  }

  const __amdgpu_buffer_rsrc_t brs = __builtin_amdgcn_make_buffer_rsrc(
      myws + WB_OFF + (size_t)nt * (NKS * 16384), 0, NKS * 16384, 0x00020000);
  const u32 l16 = lane * 16;
  const u32 avo = wid * 4096 + l16, bvo = wid * 2048 + l16;
  auto piece = [&](int t, u32 slot, int j) {
    const int jb = A_FIRST ? j - 4 : j, ja = A_FIRST ? j : j - 2;
    if (jb >= 0 && jb < 2) {
      LAS void* lb = (LAS void*)(lds + slot + A_BYTES + wid * 2048);
      if (jb == 0) __builtin_amdgcn_raw_ptr_buffer_load_lds(brs, lb, 16, bvo, t * 16384, 0, B_AUX);
      else __builtin_amdgcn_raw_ptr_buffer_load_lds(brs, lb, 16, bvo, t * 16384, 1024, B_AUX);
    } else {
      const u32 ro = ROT ? ((ja + nt) & 3) * 1024 : ja * 1024;
      LAS void* la = (LAS void*)(lds + slot + wid * 4096 + ro);
      __builtin_amdgcn_raw_ptr_buffer_load_lds(ars, la, 16, avo, t * A_BYTES + ro, 0, 0);
    }
    SCHED();
  };

  const int wr = wid >> 1, wc = wid & 1;
  const u32 aoff = wr * 8192 + l16, boff = A_BYTES + wc * 8192 + l16;
  f32x4 acc[4][4];
#pragma unroll
  for (int i = 0; i < 4; ++i)
#pragma unroll
    for (int j = 0; j < 4; ++j) acc[i][j] = (f32x4){0.f, 0.f, 0.f, 0.f};
  bf16x8 af[2][4], bfr[2][4];
  auto frags = [&](u32 slot, int kh) {
#pragma unroll
    for (int i = 0; i < 4; ++i) {
      bfr[kh][i] = *(const bf16x8*)(lds + slot + boff + i * 2048 + kh * 1024);
      af[kh][i] = *(const bf16x8*)(lds + slot + aoff + i * 2048 + kh * 1024);
    }
    SCHED();
  };
  auto mfma4 = [&](int kh, int mi) {
#pragma unroll
    for (int r = 0; r < 4; ++r)
      acc[mi][r] = __builtin_amdgcn_mfma_f32_16x16x32_bf16(bfr[kh][r], af[kh][mi], acc[mi][r], 0, 0, 0);
    SCHED();
  };

  BARRIER();
#pragma unroll
  for (int j = 0; j < 6; ++j) piece(0, 0, j);
#pragma unroll
  for (int j = 0; j < 6; ++j) piece(1, STAGE, j);
  WAIT_VM(6);
  BARRIER();
  frags(0, 0);
#pragma unroll
  for (int j = 0; j < 4 - BAR_POS; ++j) piece(2, 2 * STAGE, j);

  u32 cur = 0, nx1 = STAGE, nx2 = 2 * STAGE;
  auto step = [&](int t, auto more) {
    constexpr int MORE = decltype(more)::value;
    frags(cur, 1);
#pragma unroll
    for (int g = 0; g < 4; ++g) {
      mfma4(0, g);
      if (MORE >= 2 && 4 - BAR_POS + g < 6) piece(t + 2, nx2, 4 - BAR_POS + g);
    }
    constexpr int BP = MORE == 1 ? 4 : BAR_POS;
#pragma unroll
    for (int g = 0; g < BP; ++g) mfma4(1, g);
    if (MORE >= 1) {
      if (MORE >= 2) WAIT_VM_LGKM(6);
      else WAIT_VM_LGKM(0);
      BARRIER();
      SCHED();
      frags(nx1, 0);
    }
#pragma unroll
    for (int g = BP; g < 4; ++g) {
      mfma4(1, g);
      if (MORE >= 3) piece(t + 3, cur, g - BP);
    }
    const u32 o = cur;
    cur = nx1; nx1 = nx2; nx2 = o;
  };
  for (int t = 0; t < NKS - 3; ++t) step(t, std::integral_constant<int, 3>());
  step(NKS - 3, std::integral_constant<int, 2>());
  step(NKS - 2, std::integral_constant<int, 1>());
  step(NKS - 1, std::integral_constant<int, 0>());

  float mx = 0.f;
#pragma unroll
  for (int mi = 0; mi < 4; ++mi)
#pragma unroll
    for (int r = 0; r < 4; ++r)
#pragma unroll
      for (int j = 0; j < 4; ++j) mx = fmaxf(mx, fabsf(acc[mi][r][j]));
  const u32 wm = wave_max_u32(__builtin_bit_cast(u32, mx));
  ctl_write(ctl + (24 + wid) * 4, wm);
  const u32 fs = ctl_read(ctl + (17 + (lane & 1)) * 4);
  BARRIER();
  const __amdgpu_buffer_rsrc_t ors = __builtin_amdgcn_make_buffer_rsrc(pout, 0, 4096 * NL, 0x00020000);
  const u32 oo = (u32)((mt * 256 + wr * 64 + (lane & 15)) * NL + nt * 128 + wc * 64 + (lane >> 4) * 16);
  float inv;
  u32 bad;
  if (XCH) {
    const u32 seq = (u32)__builtin_amdgcn_readlane((int)fs, 1);
    u32 m = wave_max_u32(ctl_read(ctl + (24 + (lane & 7)) * 4));
    if (__builtin_amdgcn_readlane((int)fs, 0)) m = 0x7fffffffu;
    if (lane == 0) __hip_atomic_store((GAS u64*)(widws + AMX_OFF) + rank * 256 + b, ((u64)seq << 32) | m, RLX_SYSTEM);
    const __amdgpu_buffer_rsrc_t mrs = __builtin_amdgcn_make_buffer_rsrc(myws + AMX_OFF, 0, NR * 2048, 0x00020000);
    const u64 t0 = __builtin_amdgcn_s_memrealtime();
    u32 spins = 0;
    for (;;) {
      const u32x4 g0 = __builtin_amdgcn_raw_buffer_load_b128(mrs, l16, wid * 2048, 17);
      const u32x4 g1 = __builtin_amdgcn_raw_buffer_load_b128(mrs, l16 + 1024, wid * 2048, 17);
      m = max(max(g0.x, g0.z), max(g1.x, g1.z));
      if (__all(min(min(g0.y, g0.w), min(g1.y, g1.w)) >= seq)) break;
      if (give_up(spins, t0, tmo, seq)) { m = 0x7fffffffu; break; }
      if (XP_SLEEP) __builtin_amdgcn_s_sleep(XP_SLEEP);
    }
    ctl_write(ctl + (32 + wid) * 4, wave_max_u32(m));
    WAIT_LGKM0();
    BARRIER();
    const u32 gmax = wave_max_u32(ctl_read(ctl + (32 + (lane & 7)) * 4));
    bad = gmax > 0x7f800000u;
    const float amax = __builtin_bit_cast(float, gmax);
    inv = amax > 0.f ? 448.0f / amax : 0.f;
  } else {
    BARRIER();
    inv = __builtin_bit_cast(float, ctl_read(ctl + 16 * 4));
    bad = ctl_read(ctl + 17 * 4);
  }
#pragma unroll
  for (int mi = 0; mi < 4; ++mi) {
    u32x4 d;
#pragma unroll
    for (int j = 0; j < 4; ++j) {
      int q = __builtin_amdgcn_cvt_pk_fp8_f32(acc[mi][0][j] * inv, acc[mi][1][j] * inv, 0, false);
      q = __builtin_amdgcn_cvt_pk_fp8_f32(acc[mi][2][j] * inv, acc[mi][3][j] * inv, q, true);
      d[j] = bad ? 0x7f7f7f7fu : (u32)q;
    }
    __builtin_amdgcn_raw_buffer_store_b128(d, ors, oo + mi * 16 * NL, 0, OUT_AUX);
  }
}

}

extern "C" void kernel_launch(int rank, int nranks, ncclComm_t comm, hipStream_t stream,
                              void* const* d_in, const int* in_sizes, int n_in,
                              void* d_out, int out_size,
                              const int* in_dtypes, int out_dtype,
                              void* const* d_out_all, void* const* d_ws_all, size_t ws_bytes) {
  (void)comm; (void)in_dtypes; (void)out_dtype; (void)d_out_all;
  if (nranks != NR || n_in < 2 || ws_bytes < WS_NEED || in_sizes[0] != 512 * KD || in_sizes[1] != KD * NL ||
      out_size != 4096 * NL || (unsigned)rank >= (unsigned)nranks) {
    fprintf(stderr, "unexpected problem geometry\n");
    return;
  }
  hipError_t e = hipFuncSetAttribute((const void*)ag_gemm_q8, hipFuncAttributeMaxDynamicSharedMemorySize, (int)LDS_BYTES);
  if (e != hipSuccess) { fprintf(stderr, "hipFuncSetAttribute: %s\n", hipGetErrorString(e)); return; }
  Params p;
  p.x = (const float*)d_in[0];
  p.w = (const float*)d_in[1];
  p.out = (unsigned char*)d_out;
  p.myws = (char*)d_ws_all[rank];
  for (int i = 0; i < NR; ++i) p.ws[i] = (char*)d_ws_all[i];
  p.rank = rank;
  hipLaunchKernelGGL(ag_gemm_q8, dim3(NWG), dim3(NTHREADS), LDS_BYTES, stream, p);
  e = hipGetLastError();
  if (e != hipSuccess) fprintf(stderr, "launch: %s\n", hipGetErrorString(e));
}
```

```cpp
#include <hip/hip_runtime.h>
#include <rccl/rccl.h>
#include <cstdio>
#include <type_traits>

namespace {

typedef unsigned int u32;
typedef unsigned long long u64;
typedef __bf16 bf16x8 __attribute__((ext_vector_type(8)));
typedef __bf16 bf16x2 __attribute__((ext_vector_type(2)));
typedef float f32x4 __attribute__((ext_vector_type(4)));
typedef float f32x2 __attribute__((ext_vector_type(2)));
typedef u32 u32x4 __attribute__((ext_vector_type(4)));
#define GAS __attribute__((address_space(1)))
#define LAS __attribute__((address_space(3)))

constexpr int NR = 8;
constexpr int KD = 4096;
constexpr int NL = 2048;
constexpr int NKS = 64;
constexpr int NCH = 16;
constexpr int NWG = 256;
#define PF0 2
#define W1_PRO 1
#define B1_X 2
#define B1_W 1
#define XCH 1
#define CLEAN_AT 0
#define EARLY_INV 0
#define XP_SLEEP 2
#define PF_D 1
#define PF_D0 PF_D
#define PF_T0 6
#define KPERM 0
#define OUT_AUX 0
#define P1_SLEEP 16
#define A_FIRST 1
#define ROT 1
#define BAR_POS 2
#define B_AUX 0
constexpr int NTHREADS = 640;
constexpr int X0_END = 8;
constexpr int X1_END = 16;
constexpr u32 A_BYTES = 32768, STAGE = 49152;
constexpr u32 LDS_CTL = 3 * STAGE;
constexpr u32 LDS_PF = LDS_CTL + 256;
constexpr u32 LDS_BYTES = LDS_PF + 1024;

constexpr size_t XB_OFF = 0;
constexpr size_t XB_BYTES = 4ull << 20;
constexpr size_t WB_OFF = 4ull << 20;
constexpr size_t WB_BYTES = 16ull << 20;
constexpr size_t CT_OFF = 20ull << 20;
constexpr size_t CNT_OFF = CT_OFF;
constexpr size_t TMO_OFF = CT_OFF + 4096;
constexpr size_t INV_OFF = CT_OFF + 5120;
constexpr size_t XFL_OFF = CT_OFF + 8192;
constexpr size_t WFL_OFF = CT_OFF + 32768;
constexpr size_t GMX_OFF = CT_OFF + 49152;
constexpr size_t RMX_OFF = CT_OFF + 53248;
constexpr size_t DBG_OFF = CT_OFF + 65536;
constexpr size_t AMX_OFF = CT_OFF + 196608;
constexpr size_t WS_NEED = CT_OFF + 262144;

constexpr u64 DEADLINE_TICKS = 100000000ull;

struct Params {
  const float* x;
  const float* w;
  unsigned char* out;
  char* myws;
  char* ws[NR];
  int rank;
};

__device__ __forceinline__ int kcol(int s) { return KPERM ? (s & 3) * 1024 + (s >> 2) * 64 : s * 64; }

#define WAIT_VM(n) asm volatile("s_waitcnt vmcnt(" #n ")" ::: "memory")
#define WAIT_VM_LGKM(n) asm volatile("s_waitcnt vmcnt(" #n ") lgkmcnt(0)" ::: "memory")
#define WAIT_LGKM0() asm volatile("s_waitcnt lgkmcnt(0)" ::: "memory")
#define BARRIER() do { asm volatile("" ::: "memory"); __builtin_amdgcn_s_barrier(); asm volatile("" ::: "memory"); } while (0)
#define SCHED() __builtin_amdgcn_sched_barrier(0)
#define RLX_AGENT __ATOMIC_RELAXED, __HIP_MEMORY_SCOPE_AGENT
#define RLX_SYSTEM __ATOMIC_RELAXED, __HIP_MEMORY_SCOPE_SYSTEM

#define STAMP(i) do { } while (0)

__device__ __forceinline__ u32 pack_bf16(float a, float b) {
  f32x2 f = {a, b};
  bf16x2 h = __builtin_convertvector(f, bf16x2);
  return __builtin_bit_cast(u32, h);
}

__device__ __forceinline__ u32 wave_max_u32(u32 v) {
  v = max(v, (u32)__builtin_amdgcn_update_dpp(0, (int)v, 0x111, 0xf, 0xf, false));
  v = max(v, (u32)__builtin_amdgcn_update_dpp(0, (int)v, 0x112, 0xf, 0xf, false));
  v = max(v, (u32)__builtin_amdgcn_update_dpp(0, (int)v, 0x114, 0xf, 0xf, false));
  v = max(v, (u32)__builtin_amdgcn_update_dpp(0, (int)v, 0x118, 0xf, 0xf, false));
  const u32 a = (u32)__builtin_amdgcn_readlane((int)v, 15), b = (u32)__builtin_amdgcn_readlane((int)v, 31);
  const u32 c = (u32)__builtin_amdgcn_readlane((int)v, 47), d = (u32)__builtin_amdgcn_readlane((int)v, 63);
  return max(max(a, b), max(c, d));
}

__device__ __forceinline__ bool give_up(u32& spins, u64 t0, GAS u32* tmo, u32 seq) {
  ++spins;
  bool stop = (__builtin_amdgcn_s_memrealtime() - t0) > DEADLINE_TICKS;
  if ((spins & 7u) == 0u) stop |= (__hip_atomic_load(tmo, RLX_AGENT) == seq);
  stop = __builtin_amdgcn_readfirstlane((int)stop) != 0;
  if (stop) __hip_atomic_store(tmo, seq, RLX_AGENT);
  else __builtin_amdgcn_s_sleep(1);
  return stop;
}

__device__ __forceinline__ void w_load(f32x4* v, const float* __restrict__ wsrc, int chunk) {
  const float* s = wsrc + (size_t)chunk * (KPERM ? 64 : 256) * NL;
#pragma unroll
  for (int j = 0; j < 8; ++j) v[j] = *(const f32x4*)(s + (size_t)j * NL);
}

__device__ __forceinline__ void w_store(const f32x4* v, __amdgpu_buffer_rsrc_t rs, u32 woff, int chunk) {
#pragma unroll
  for (int r = 0; r < 4; ++r) {
    u32x4 d;
    d.x = pack_bf16(v[0][r], v[1][r]);
    d.y = pack_bf16(v[2][r], v[3][r]);
    d.z = pack_bf16(v[4][r], v[5][r]);
    d.w = pack_bf16(v[6][r], v[7][r]);
    __builtin_amdgcn_raw_buffer_store_b128(d, rs, woff + r * 2048, chunk * (4 * 16384), 16  );
  }
}

__device__ __forceinline__ void ctl_write(u32 addr, u32 v) {
  asm volatile("ds_write_b32 %0, %1" ::"v"(addr), "v"(v) : "memory");
}
__device__ __forceinline__ u32 ctl_read(u32 addr) {
  u32 v;
  asm volatile("ds_read_b32 %0, %1\n\ts_waitcnt lgkmcnt(0)" : "=v"(v) : "v"(addr) : "memory");
  return v;
}
__device__ __forceinline__ void x_load(f32x4* v, const float* __restrict__ x, int xh, int xks, int msub, int lane) {
  const float* xs = x + (size_t)(xh * 256 + msub * 16 + (lane & 15)) * KD + kcol(xks) + (lane >> 4) * 8;
#pragma unroll
  for (int j = 0; j < 4; ++j) v[j] = *(const f32x4*)(xs + (j >> 1) * 32 + (j & 1) * 4);
}

__device__ __forceinline__ void x_store(const f32x4* v, __amdgpu_buffer_rsrc_t rs, int xh, int xks, int msub, int lane) {
  const u32 xoff = (u32)(((xh * 64 + xks) * 16 + msub) * 2048 + lane * 16);
#pragma unroll
  for (int kh = 0; kh < 2; ++kh) {
    u32x4 d;
    d.x = pack_bf16(v[kh * 2][0], v[kh * 2][1]);
    d.y = pack_bf16(v[kh * 2][2], v[kh * 2][3]);
    d.z = pack_bf16(v[kh * 2 + 1][0], v[kh * 2 + 1][1]);
    d.w = pack_bf16(v[kh * 2 + 1][2], v[kh * 2 + 1][3]);
    __builtin_amdgcn_raw_buffer_store_b128(d, rs, xoff + kh * 1024, 0, 17  );
  }
}

__device__ __forceinline__ u32x4 pack8(const f32x4& a, const f32x4& b) {
  u32x4 d;
  d.x = pack_bf16(a[0], a[1]);
  d.y = pack_bf16(a[2], a[3]);
  d.z = pack_bf16(b[0], b[1]);
  d.w = pack_bf16(b[2], b[3]);
  return d;
}

__device__ __forceinline__ u64 both_ge(u64 f, u32 seq) { return __ballot((u32)f >= seq && (u32)(f >> 32) >= seq); }
__device__ __forceinline__ int lead_ones(u64 m) { return m == ~0ull ? 64 : __builtin_ctzll(~m); }

__global__ void __launch_bounds__(NTHREADS) __attribute__((amdgpu_waves_per_eu(3, 3))) ag_gemm_q8(Params p) {
  extern __shared__ __attribute__((aligned(16))) char lds[];
  const int b = blockIdx.x;
  const int tid = threadIdx.x, lane = tid & 63;
  const int wid = __builtin_amdgcn_readfirstlane(tid >> 6);
  const int src = b & 7, idx = b >> 3;
  const int half = idx & 1, nt = idx >> 1, mt = src * 2 + half;
  const float* const px = p.x;
  const float* const pw = p.w;
  unsigned char* const pout = p.out;
  char* const myws = p.myws;
  char* const srcws = p.ws[src];
  char* const widws = p.ws[wid & 7];
  const int rank = p.rank;
  const u64 w0 = (u64)p.ws[0], w1 = (u64)p.ws[1], w2 = (u64)p.ws[2], w3 = (u64)p.ws[3];
  const u64 w4 = (u64)p.ws[4], w5 = (u64)p.ws[5], w6 = (u64)p.ws[6], w7 = (u64)p.ws[7];
  asm volatile("" ::"s"(px), "s"(pw), "s"(pout), "s"(myws), "s"(srcws), "s"(widws), "s"(rank), "s"(w0), "s"(w1), "s"(w2), "s"(w3),
               "s"(w4), "s"(w5), "s"(w6), "s"(w7));
  auto peer_ws = [&]() {
    int lo = (int)w0, hi = (int)(w0 >> 32);
#define PUT(i) asm("v_writelane_b32 %0, %2, " #i "\n\tv_writelane_b32 %1, %3, " #i : "+v"(lo), "+v"(hi) : "s"((int)w##i), "s"((int)(w##i >> 32)))
    PUT(1); PUT(2); PUT(3); PUT(4); PUT(5); PUT(6); PUT(7);
#undef PUT
    return (char*)(((u64)(u32)hi << 32) | (u32)lo);
  };
  GAS u32* const cnt = (GAS u32*)(myws + CNT_OFF) + b;
  const __amdgpu_buffer_rsrc_t xrs = __builtin_amdgcn_make_buffer_rsrc(myws + XB_OFF, 0, (int)XB_BYTES, 0x00020000);

  u32 diag = 0;
  if (__builtin_expect(wid == 0, 1)) {
    f32x4 v[4];
    STAMP(6);
    const u64 ta_ = __builtin_amdgcn_s_memrealtime();
    x_load(v, px, (b >> 4) & 1, b >> 5, b & 15, lane);
    const u32 seq = __hip_atomic_load(cnt, RLX_AGENT);
    SCHED();
    WAIT_VM(0);
    diag = (u32)(__builtin_amdgcn_s_memrealtime() - ta_);
    SCHED();
    GAS u32* pub = (GAS u32*)(peer_ws() + XFL_OFF) + (rank * 2 + ((b >> 4) & 1)) * 384 + (b >> 5) * 16 + (b & 15);
    asm volatile("" : "+v"(pub));
    SCHED();
    x_store(v, xrs, (b >> 4) & 1, b >> 5, b & 15, lane);
    SCHED();
    WAIT_VM(0);
    if (lane < 8) __hip_atomic_store(pub, seq + 1u, RLX_SYSTEM);
    STAMP(10);
  }

  const u32 ctl = (u32)(size_t)(LAS char*)lds + LDS_CTL;
  const int kq1 = lane >> 5, wH = (lane >> 4) & 1, wi = lane & 15;
  const float* const wsrc = pw + (size_t)(kcol(mt >> 2) + (mt & 3) * 16 + kq1 * 8) * NL + nt * 128 + wH * 64 + wi * 4;
  const u32 woff = (u32)(((nt * 64 + (mt >> 2)) * 8 + wH * 4) * 2048 + ((mt >> 1) & 1) * 1024 +
                         (((mt & 1) * 2 + kq1) * 16 + wi) * 16);
  const __amdgpu_buffer_rsrc_t wrs =
      __builtin_amdgcn_make_buffer_rsrc(myws + WB_OFF, 0, (int)WB_BYTES, 0x00020000);

  GAS u32* const wfl = (GAS u32*)(myws + WFL_OFF) + nt * 256;
  GAS u32* const xfl = (GAS u32*)(myws + XFL_OFF) + mt * 384;
  GAS u32* const tmo = (GAS u32*)(myws + TMO_OFF);

  if (wid == 8) {
    __builtin_amdgcn_s_setprio(3);
    STAMP(0);
    const u32 seq = (u32)__builtin_amdgcn_readfirstlane((int)__hip_atomic_load(cnt, RLX_AGENT)) + 1u;
    char* peer;
    {
      const u64 v = (u64)peer_ws();
      const u32 lo = (u32)__builtin_amdgcn_ds_bpermute((lane & 7) * 4, (int)v);
      const u32 hi = (u32)__builtin_amdgcn_ds_bpermute((lane & 7) * 4, (int)(v >> 32));
      peer = (char*)(((u64)hi << 32) | lo);
    }
    bool failed = false;

    int xok = 0, cw = 0;
    GAS u64* const xf = (GAS u64*)xfl + lane;
    struct Poll { u32 fw; u64 f0, f1, f2; int cw; };
    Poll pa, pb;
    auto poll = [&](Poll& q) {
      q.cw = cw;
      q.fw = __hip_atomic_load(wfl + min(cw + (lane >> 4), NCH - 1) * 16 + (lane & 15), RLX_AGENT);
      q.f0 = __hip_atomic_load(xf, RLX_SYSTEM);
      q.f1 = __hip_atomic_load(xf + 64, RLX_SYSTEM);
      q.f2 = __hip_atomic_load(xf + 128, RLX_SYSTEM);
      SCHED();
    };
    auto seen = [&](Poll& q) {
      asm volatile("" : "+v"(q.fw), "+v"(q.f0), "+v"(q.f1), "+v"(q.f2));
      cw = max(cw, min(q.cw + lead_ones(__ballot(q.fw >= seq)) / 16, NCH));
      int n = lead_ones(both_ge(q.f0, seq)) >> 3;
      if (n == X0_END) n += lead_ones(both_ge(q.f1, seq)) >> 3;
      if (n == X1_END && both_ge(q.f2, seq) == ~0ull) n = NKS;
      xok = max(xok, n);
      ctl_write(ctl, (u32)xok);
      SCHED();
    };
    auto wait_for = [&](int sx, int sw) {
      const u64 t0 = __builtin_amdgcn_s_memrealtime();
      u32 spins = 0;
      for (;;) {
        Poll q;
        poll(q);
        seen(q);
        if (xok > sx && 4 * cw > sw) return;
        if (give_up(spins, t0, tmo, seq)) { failed = true; xok = NKS; cw = NCH; ctl_write(ctl, (u32)xok); return; }
      }
    };
    auto need = [&](int s, Poll& q) {
      if (xok > s && 4 * cw > s) return;
      seen(q);
      if (xok > s && 4 * cw > s) return;
      wait_for(s, s);
    };
    auto wflag = [&](int c) { if (lane == 0) __hip_atomic_store(wfl + c * 16 + mt, seq, RLX_AGENT); };
#define KBAR(t) do { BARRIER(); SCHED(); } while (0)
#define PIN(v, n) _Pragma("unroll") for (int i_ = 0; i_ < (n); ++i_) asm volatile("" : "+v"((v)[i_]))

    {
      const int l2 = (b >> 1) * 6;
      f32x4 wv[8], xv[12];
      wait_for(B1_X - 1, 4 * B1_W - 4);
      STAMP(2);
      x_load(xv, px, (b >> 4) & 1, X0_END + (b >> 5), b & 15, lane);
      w_load(wv, wsrc, 2);
      poll(pa);
      WAIT_LGKM0();
      BARRIER();
      SCHED();
      PIN(xv, 4);
      x_store(xv, xrs, (b >> 4) & 1, X0_END + (b >> 5), b & 15, lane);
      PIN(wv, 8);
      w_store(wv, wrs, woff, 2);
      SCHED();
      seen(pa);
#pragma unroll
      for (int i = 0; i < 3; ++i) x_load(xv + 4 * i, px, b & 1, X1_END + ((l2 + i) >> 4), (l2 + i) & 15, lane);
      SCHED();
      need(2, pa);
      KBAR(63);
      WAIT_VM(12);
      {
        GAS u32* pub = (GAS u32*)(peer + XFL_OFF) + (rank * 2 + ((b >> 4) & 1)) * 384 + 128 + (b >> 5) * 16 + (b & 15);
        if (lane < 8) __hip_atomic_store(pub, seq, RLX_SYSTEM);
      }
      wflag(2);
      STAMP(11);
      if (lane == 0) __hip_atomic_store(cnt, seq, RLX_AGENT);
      poll(pb);
      w_load(wv, wsrc, 3);
      SCHED();
      need(3, pb);
      KBAR(0);
      PIN(xv, 12);
#pragma unroll
      for (int i = 0; i < 3; ++i) x_store(xv + 4 * i, xrs, b & 1, X1_END + ((l2 + i) >> 4), (l2 + i) & 15, lane);
#pragma unroll
      for (int i = 3; i < 6; ++i) x_load(xv + 4 * (i - 3), px, b & 1, X1_END + ((l2 + i) >> 4), (l2 + i) & 15, lane);
      SCHED();
      poll(pa);
      need(4, pb);
      KBAR(1);
      seen(pb);
      PIN(wv, 8);
      w_store(wv, wrs, woff, 3);
      SCHED();
      poll(pb);
      need(5, pa);
      KBAR(2);
      seen(pa);
      WAIT_VM(4);
      wflag(3);
      PIN(xv, 12);
#pragma unroll
      for (int i = 3; i < 6; ++i) x_store(xv + 4 * (i - 3), xrs, b & 1, X1_END + ((l2 + i) >> 4), (l2 + i) & 15, lane);
      SCHED();
      poll(pa);
      need(6, pb);
      KBAR(3);
      seen(pb);
      w_load(wv, wsrc, 4);
      SCHED();
      need(7, pa);
      KBAR(4);
      seen(pa);
      {
        GAS u32* pub = (GAS u32*)(peer + XFL_OFF) + (rank * 2 + (b & 1)) * 384 + 256 + (b >> 1);
        WAIT_VM(8);
        if (lane < 8) __hip_atomic_store(pub, seq, RLX_SYSTEM);
      }
      for (int c = 1; c + 3 < NCH; ++c) {
        need(4 * c + 4, pa);
        KBAR(4 * c + 1);
        PIN(wv, 8);
        w_store(wv, wrs, woff, c + 3);
        SCHED();
        need(4 * c + 5, pa);
        KBAR(4 * c + 2);
        WAIT_VM(0);
        wflag(c + 3);
        poll(pa);
        need(4 * c + 6, pa);
        KBAR(4 * c + 3);
        w_load(wv, wsrc, min(c + 4, NCH - 1));
        SCHED();
        need(4 * c + 7, pa);
        KBAR(4 * c + 4);
        seen(pa);
      }
      for (int t = 4 * (NCH - 3) + 1; t < NKS - 1; ++t) {
        if ((t & 1) == 0) poll(pa);
        need(min(t + 3, NKS - 1), pa);
        if (XCH && t == NKS - 2) {
          ctl_write(ctl + 17 * 4, failed ? 1u : 0u);
          ctl_write(ctl + 18 * 4, seq);
          WAIT_LGKM0();
        }
        KBAR(t);
        if (t & 1) seen(pa);
      }
    }
    STAMP(9);
    if (XCH) return;

    BARRIER();
    u32 m = ctl_read(ctl + (24 + (lane & 7)) * 4);
    m = wave_max_u32(m);
    GAS u64* const gmx = (GAS u64*)(myws + GMX_OFF);
    if (lane == 0) __hip_atomic_store(gmx + b, ((u64)seq << 32) | m, RLX_AGENT);
    if (b == rank) {
      const u64 t0 = __builtin_amdgcn_s_memrealtime();
      u32 spins = 0, gm = 0;
      for (;;) {
        bool ok = true;
        gm = 0;
#pragma unroll
        for (int k = 0; k < 4; ++k) {
          const u64 g = __hip_atomic_load(gmx + k * 64 + lane, RLX_AGENT);
          ok &= (u32)(g >> 32) >= seq;
          gm = max(gm, (u32)g);
        }
        if (__all(ok)) break;
        if (give_up(spins, t0, tmo, seq)) { failed = true; break; }
      }
      gm = wave_max_u32(gm);
      GAS u64* r = (GAS u64*)(peer + RMX_OFF + (lane >> 3) * 256) + rank;
      __hip_atomic_store(r, ((u64)seq << 32) | gm, RLX_SYSTEM);
    }
    STAMP(4);
    u32 gmax = 0;
    {
      GAS u64* const r = (GAS u64*)(myws + RMX_OFF + src * 256) + (lane & 7);
      const u64 t0 = __builtin_amdgcn_s_memrealtime();
      u32 spins = 0;
      for (;;) {
        const u64 g = __hip_atomic_load(r, RLX_SYSTEM);
        gmax = (u32)g;
        if (__all((u32)(g >> 32) >= seq)) break;
        if (give_up(spins, t0, tmo, seq)) { failed = true; break; }
      }
    }
    gmax = wave_max_u32(gmax);
    failed |= __builtin_amdgcn_readfirstlane((int)__hip_atomic_load(tmo, RLX_AGENT)) == (int)seq;
    const float amax = __builtin_bit_cast(float, gmax);
    const float inv = amax > 0.f ? 448.0f / amax : 0.f;
    ctl_write(ctl + 16 * 4, __builtin_bit_cast(u32, inv));
    ctl_write(ctl + 17 * 4, failed ? 1u : 0u);
    WAIT_LGKM0();
    BARRIER();
    STAMP(5);
    return;
  }

  char* const xsrc = srcws + XB_OFF + (size_t)half * (XB_BYTES / 2);
  const __amdgpu_buffer_rsrc_t ars = __builtin_amdgcn_make_buffer_rsrc(xsrc, 0, (int)(XB_BYTES / 2), 0x00020000);
  if (wid == 9) {
    __builtin_amdgcn_s_setprio(3);
    const bool elect = b < NR;
    if (EARLY_INV && elect) asm volatile("buffer_inv sc0 sc1" ::: "memory");
    GAS u32* const inv = (GAS u32*)(myws + INV_OFF) + (__builtin_amdgcn_s_getreg(20 | (3 << 11)) & 15) * 32;
    const u32 seq0 = __hip_atomic_load(cnt, RLX_AGENT), cln = __hip_atomic_load(inv + 1, RLX_AGENT);
    const u32 seq = (u32)__builtin_amdgcn_readfirstlane((int)seq0) + 1u;
    if (CLEAN_AT == 2 || seq == 1u || (u32)__builtin_amdgcn_readfirstlane((int)cln) != seq - 1u) {
      bool mine = elect;
      if (!mine) {
        const u64 t0 = __builtin_amdgcn_s_memrealtime();
        while ((u32)__builtin_amdgcn_readfirstlane((int)__hip_atomic_load(inv, RLX_AGENT)) < seq) {
          mine = __builtin_amdgcn_s_memrealtime() - t0 > 800;
          if (mine) break;
          __builtin_amdgcn_s_sleep(2);
        }
      }
      if (mine) {
        if (!(EARLY_INV && elect)) asm volatile("buffer_inv sc0 sc1" ::: "memory");
        WAIT_VM(0);
        if (lane == 0) __hip_atomic_store(inv, seq, RLX_AGENT);
      }
    }
    STAMP(1);
    int t = -2;
    if (src != rank) {
      LAS void* dst = (LAS void*)(lds + LDS_PF);
      const u32 vo = nt * 2048 + lane * 16;
      auto pf = [&](int s) {
        __builtin_amdgcn_raw_ptr_buffer_load_lds(ars, dst, 16, vo, s * A_BYTES, 0, 0);
        __builtin_amdgcn_raw_ptr_buffer_load_lds(ars, dst, 16, vo + 1024, s * A_BYTES, 0, 0);
      };
      int s = 0;
      if (PF0 > 0) {
        GAS u32* const f = xfl + (lane & 7) * 16 + nt;
        const u64 t0 = __builtin_amdgcn_s_memrealtime();
        u32 spins = 0;
        while (((u32)__ballot(__hip_atomic_load(f, RLX_SYSTEM) >= seq) & ((1u << PF0) - 1u)) != (1u << PF0) - 1u)
          if (give_up(spins, t0, tmo, seq)) break;
        STAMP(15);
        for (; s < PF0; ++s) pf(s);
      }
      int xok = 0;
      for (; s < NKS; ++t) {
        BARRIER();
        const int lim = min(t + 3 + (t < PF_T0 ? PF_D0 : PF_D), NKS - 1);
        for (int i = 0; i < 3 && xok <= lim; ++i) {
          if (i) __builtin_amdgcn_s_sleep(3);
          xok = __builtin_amdgcn_readfirstlane((int)ctl_read(ctl));
        }
        for (s = max(s, t + 4); s <= lim && s < xok; ++s) pf(s);
      }
    } else if (CLEAN_AT == 2 || !elect) {
      BARRIER();
      return;
    }
    for (; t < NKS - 1; ++t) BARRIER();
    WAIT_VM(0);
    BARRIER();
    if (CLEAN_AT == 2 || !elect) return;
    bool ok = true;
    if (CLEAN_AT == 0) {
      GAS u64* const g = (GAS u64*)(myws + (XCH ? AMX_OFF + rank * 2048 : GMX_OFF)) + lane;
      const u64 t0 = __builtin_amdgcn_s_memrealtime();
      u32 spins = 0;
      for (;;) {
        bool all = true;
#pragma unroll
        for (int k = 0; k < 4; ++k) all &= (u32)(__hip_atomic_load(g + k * 64, RLX_SYSTEM) >> 32) >= seq;
        if (__all(all)) break;
        if (give_up(spins, t0, tmo, seq)) { ok = false; break; }
      }
      asm volatile("buffer_inv sc0 sc1" ::: "memory");
    }
    BARRIER();
    if (CLEAN_AT == 1) {
      ok = XCH ? wave_max_u32(ctl_read(ctl + (32 + (lane & 7)) * 4)) <= 0x7f800000u : ctl_read(ctl + 17 * 4) == 0u;
      asm volatile("buffer_inv sc0 sc1" ::: "memory");
    }
    WAIT_VM(0);
    if (ok && lane == 0) __hip_atomic_store(inv + 1, seq, RLX_AGENT);
    STAMP(15);
    return;
  }

  if (wid == 2 || wid == 3) {
    f32x4 v[8];
    if (wid == 3) __builtin_amdgcn_s_sleep(P1_SLEEP);
    w_load(v, wsrc, wid - 2);
    const u32 seq = __hip_atomic_load(cnt, RLX_AGENT);
    SCHED();
    w_store(v, wrs, woff, wid - 2);
    SCHED();
    WAIT_VM(0);
    if (lane == 0) __hip_atomic_store(wfl + (wid - 2) * 16 + mt, seq + 1u, RLX_AGENT);
    STAMP(10 + wid);
  }

  const __amdgpu_buffer_rsrc_t brs = __builtin_amdgcn_make_buffer_rsrc(
      myws + WB_OFF + (size_t)nt * (NKS * 16384), 0, NKS * 16384, 0x00020000);
  const u32 l16 = lane * 16;
  const u32 avo = wid * 4096 + l16, bvo = wid * 2048 + l16;
  auto piece = [&](int t, u32 slot, int j) {
    const int jb = A_FIRST ? j - 4 : j, ja = A_FIRST ? j : j - 2;
    if (jb >= 0 && jb < 2) {
      LAS void* lb = (LAS void*)(lds + slot + A_BYTES + wid * 2048);
      if (jb == 0) __builtin_amdgcn_raw_ptr_buffer_load_lds(brs, lb, 16, bvo, t * 16384, 0, B_AUX);
      else __builtin_amdgcn_raw_ptr_buffer_load_lds(brs, lb, 16, bvo, t * 16384, 1024, B_AUX);
    } else {
      const u32 ro = ROT ? ((ja + nt) & 3) * 1024 : ja * 1024;
      LAS void* la = (LAS void*)(lds + slot + wid * 4096 + ro);
      __builtin_amdgcn_raw_ptr_buffer_load_lds(ars, la, 16, avo, t * A_BYTES + ro, 0, 0);
    }
    SCHED();
  };

  const int wr = wid >> 1, wc = wid & 1;
  const u32 aoff = wr * 8192 + l16, boff = A_BYTES + wc * 8192 + l16;
  f32x4 acc[4][4];
#pragma unroll
  for (int i = 0; i < 4; ++i)
#pragma unroll
    for (int j = 0; j < 4; ++j) acc[i][j] = (f32x4){0.f, 0.f, 0.f, 0.f};
  bf16x8 af[2][4], bfr[2][4];
  auto frags = [&](u32 slot, int kh) {
#pragma unroll
    for (int i = 0; i < 4; ++i) {
      bfr[kh][i] = *(const bf16x8*)(lds + slot + boff + i * 2048 + kh * 1024);
      af[kh][i] = *(const bf16x8*)(lds + slot + aoff + i * 2048 + kh * 1024);
    }
    SCHED();
  };
  auto mfma4 = [&](int kh, int mi) {
#pragma unroll
    for (int r = 0; r < 4; ++r)
      acc[mi][r] = __builtin_amdgcn_mfma_f32_16x16x32_bf16(bfr[kh][r], af[kh][mi], acc[mi][r], 0, 0, 0);
    SCHED();
  };

  BARRIER();
#pragma unroll
  for (int j = 0; j < 6; ++j) piece(0, 0, j);
#pragma unroll
  for (int j = 0; j < 6; ++j) piece(1, STAGE, j);
  WAIT_VM(6);
  BARRIER();
  frags(0, 0);
#pragma unroll
  for (int j = 0; j < 4 - BAR_POS; ++j) piece(2, 2 * STAGE, j);

  u32 cur = 0, nx1 = STAGE, nx2 = 2 * STAGE;
  auto step = [&](int t, auto more) {
    constexpr int MORE = decltype(more)::value;
    frags(cur, 1);
#pragma unroll
    for (int g = 0; g < 4; ++g) {
      mfma4(0, g);
      if (MORE >= 2 && 4 - BAR_POS + g < 6) piece(t + 2, nx2, 4 - BAR_POS + g);
    }
    constexpr int BP = MORE == 1 ? 4 : BAR_POS;
#pragma unroll
    for (int g = 0; g < BP; ++g) mfma4(1, g);
    if (MORE >= 1) {
      if (MORE >= 2) WAIT_VM_LGKM(6);
      else WAIT_VM_LGKM(0);
      BARRIER();
      SCHED();
      frags(nx1, 0);
    }
#pragma unroll
    for (int g = BP; g < 4; ++g) {
      mfma4(1, g);
      if (MORE >= 3) piece(t + 3, cur, g - BP);
    }
    const u32 o = cur;
    cur = nx1; nx1 = nx2; nx2 = o;
  };
  for (int t = 0; t < NKS - 3; ++t) step(t, std::integral_constant<int, 3>());
  step(NKS - 3, std::integral_constant<int, 2>());
  step(NKS - 2, std::integral_constant<int, 1>());
  step(NKS - 1, std::integral_constant<int, 0>());

  float mx = 0.f;
#pragma unroll
  for (int mi = 0; mi < 4; ++mi)
#pragma unroll
    for (int r = 0; r < 4; ++r)
#pragma unroll
      for (int j = 0; j < 4; ++j) mx = fmaxf(mx, fabsf(acc[mi][r][j]));
  const u32 wm = wave_max_u32(__builtin_bit_cast(u32, mx));
  ctl_write(ctl + (24 + wid) * 4, wm);
  const u32 fs = ctl_read(ctl + (17 + (lane & 1)) * 4);
  if (wid == 0 && b == 0) {
    u32 d = min(diag * 20u, 9000u);
    if (__builtin_amdgcn_readlane((int)fs, 1) == 1) d += 10000u;
    const u64 t0 = __builtin_amdgcn_s_memrealtime();
    while (__builtin_amdgcn_s_memrealtime() - t0 < d) __builtin_amdgcn_s_sleep(1);
  }
  BARRIER();
  const __amdgpu_buffer_rsrc_t ors = __builtin_amdgcn_make_buffer_rsrc(pout, 0, 4096 * NL, 0x00020000);
  const u32 oo = (u32)((mt * 256 + wr * 64 + (lane & 15)) * NL + nt * 128 + wc * 64 + (lane >> 4) * 16);
  float inv;
  u32 bad;
  if (XCH) {
    const u32 seq = (u32)__builtin_amdgcn_readlane((int)fs, 1);
    u32 m = wave_max_u32(ctl_read(ctl + (24 + (lane & 7)) * 4));
    if (__builtin_amdgcn_readlane((int)fs, 0)) m = 0x7fffffffu;
    if (lane == 0) __hip_atomic_store((GAS u64*)(widws + AMX_OFF) + rank * 256 + b, ((u64)seq << 32) | m, RLX_SYSTEM);
    const __amdgpu_buffer_rsrc_t mrs = __builtin_amdgcn_make_buffer_rsrc(myws + AMX_OFF, 0, NR * 2048, 0x00020000);
    const u64 t0 = __builtin_amdgcn_s_memrealtime();
    u32 spins = 0;
    for (;;) {
      const u32x4 g0 = __builtin_amdgcn_raw_buffer_load_b128(mrs, l16, wid * 2048, 17);
      const u32x4 g1 = __builtin_amdgcn_raw_buffer_load_b128(mrs, l16 + 1024, wid * 2048, 17);
      m = max(max(g0.x, g0.z), max(g1.x, g1.z));
      if (__all(min(min(g0.y, g0.w), min(g1.y, g1.w)) >= seq)) break;
      if (give_up(spins, t0, tmo, seq)) { m = 0x7fffffffu; break; }
      if (XP_SLEEP) __builtin_amdgcn_s_sleep(XP_SLEEP);
    }
    ctl_write(ctl + (32 + wid) * 4, wave_max_u32(m));
    WAIT_LGKM0();
    BARRIER();
    const u32 gmax = wave_max_u32(ctl_read(ctl + (32 + (lane & 7)) * 4));
    bad = gmax > 0x7f800000u;
    const float amax = __builtin_bit_cast(float, gmax);
    inv = amax > 0.f ? 448.0f / amax : 0.f;
  } else {
    BARRIER();
    inv = __builtin_bit_cast(float, ctl_read(ctl + 16 * 4));
    bad = ctl_read(ctl + 17 * 4);
  }
#pragma unroll
  for (int mi = 0; mi < 4; ++mi) {
    u32x4 d;
#pragma unroll
    for (int j = 0; j < 4; ++j) {
      int q = __builtin_amdgcn_cvt_pk_fp8_f32(acc[mi][0][j] * inv, acc[mi][1][j] * inv, 0, false);
      q = __builtin_amdgcn_cvt_pk_fp8_f32(acc[mi][2][j] * inv, acc[mi][3][j] * inv, q, true);
      d[j] = bad ? 0x7f7f7f7fu : (u32)q;
    }
    __builtin_amdgcn_raw_buffer_store_b128(d, ors, oo + mi * 16 * NL, 0, OUT_AUX);
  }
}

}

extern "C" void kernel_launch(int rank, int nranks, ncclComm_t comm, hipStream_t stream,
                              void* const* d_in, const int* in_sizes, int n_in,
                              void* d_out, int out_size,
                              const int* in_dtypes, int out_dtype,
                              void* const* d_out_all, void* const* d_ws_all, size_t ws_bytes) {
  (void)comm; (void)in_dtypes; (void)out_dtype; (void)d_out_all;
  if (nranks != NR || n_in < 2 || ws_bytes < WS_NEED || in_sizes[0] != 512 * KD || in_sizes[1] != KD * NL ||
      out_size != 4096 * NL || (unsigned)rank >= (unsigned)nranks) {
    fprintf(stderr, "unexpected problem geometry\n");
    return;
  }
  hipError_t e = hipFuncSetAttribute((const void*)ag_gemm_q8, hipFuncAttributeMaxDynamicSharedMemorySize, (int)LDS_BYTES);
  if (e != hipSuccess) { fprintf(stderr, "hipFuncSetAttribute: %s\n", hipGetErrorString(e)); return; }
  Params p;
  p.x = (const float*)d_in[0];
  p.w = (const float*)d_in[1];
  p.out = (unsigned char*)d_out;
  p.myws = (char*)d_ws_all[rank];
  for (int i = 0; i < NR; ++i) p.ws[i] = (char*)d_ws_all[i];
  p.rank = rank;
  hipLaunchKernelGGL(ag_gemm_q8, dim3(NWG), dim3(NTHREADS), LDS_BYTES, stream, p);
  e = hipGetLastError();
  if (e != hipSuccess) fprintf(stderr, "launch: %s\n", hipGetErrorString(e));
}
```

```cpp
#include <hip/hip_runtime.h>
#include <rccl/rccl.h>
#include <cstdio>
#include <type_traits>

namespace {

typedef unsigned int u32;
typedef unsigned long long u64;
typedef __bf16 bf16x8 __attribute__((ext_vector_type(8)));
typedef __bf16 bf16x2 __attribute__((ext_vector_type(2)));
typedef float f32x4 __attribute__((ext_vector_type(4)));
typedef float f32x2 __attribute__((ext_vector_type(2)));
typedef u32 u32x4 __attribute__((ext_vector_type(4)));
#define GAS __attribute__((address_space(1)))
#define LAS __attribute__((address_space(3)))

constexpr int NR = 8;
constexpr int KD = 4096;
constexpr int NL = 2048;
constexpr int NKS = 64;
constexpr int NCH = 16;
constexpr int NWG = 256;
#define PUSH 0
#define WIN 2
#define DMAX 1
#define FP_K 64
#define X1_SLEEP 8
#define X2_SLEEP 0
#define W2_SLEEP 0
#define DW 2
#define DW_SLEEP 45
#define LATE_FLAG 1
#define LATE_CV 1
#define LATE_DW 1
#define PRE_POLL 1
#define PF0 2
#define PF_SLEEP 20
#define SKIP_X2 0
#define W1_PRE 0
#define PEEL 0
#define LEAN 1
#define FP_N 4
#define PERM 1
#define B1_X 2
#define B1_W (DW >= 2 ? 0 : 1)
#define XP_SLEEP 2
#define KPERM 0
#define OUT_AUX 16
#define P1_SLEEP 16
#define A_FIRST 1
#define ROT 1
#define BAR_POS 2
#define B_AUX 0
#define WLAY 1
constexpr u32 W_KS = WLAY ? 16 * 16384 : 16384, W_NT = WLAY ? 16384 : 64 * 16384;
constexpr int NTHREADS = 640;
constexpr int X0_END = 8;
constexpr int X1_END = 16;
constexpr u32 A_BYTES = 32768, STAGE = 49152;
constexpr u32 LDS_CTL = 3 * STAGE;
constexpr u32 LDS_FLG = LDS_CTL + 256;
constexpr u32 LDS_PF = LDS_FLG + 256;
constexpr u32 LDS_BYTES = LDS_PF + 1024;

constexpr size_t XB_OFF = 0;
constexpr size_t XB_BYTES = 4ull << 20;
constexpr size_t WB_OFF = 4ull << 20;
constexpr size_t WB_BYTES = 16ull << 20;
constexpr size_t CT_OFF = 20ull << 20;
constexpr size_t CNT_OFF = CT_OFF;
constexpr size_t TMO_OFF = CT_OFF + 4096;
constexpr size_t INV_OFF = CT_OFF + 5120;
constexpr size_t XFL_OFF = CT_OFF + 8192;
constexpr size_t WFL_OFF = CT_OFF + 32768;
constexpr size_t GMX_OFF = CT_OFF + 49152;
constexpr size_t RMX_OFF = CT_OFF + 53248;
constexpr size_t DBG_OFF = CT_OFF + 65536;
constexpr size_t AMX_OFF = CT_OFF + 196608;
constexpr size_t LX_OFF = CT_OFF + 262144;
constexpr size_t LX_BYTES = 1ull << 20;
constexpr size_t WS_NEED = LX_OFF + LX_BYTES;

constexpr u64 DEADLINE_TICKS = 100000000ull;

struct Params {
  const float* x;
  const float* w;
  unsigned char* out;
  char* myws;
  char* ws[NR];
  char* sws[NR];
  u32 perm;
  int rank;
};

__device__ __forceinline__ int kcol(int s) { return KPERM ? (s & 3) * 1024 + (s >> 2) * 64 : s * 64; }

#define WAIT_VM(n) asm volatile("s_waitcnt vmcnt(" #n ")" ::: "memory")
#define WAIT_VM_LGKM(n) asm volatile("s_waitcnt vmcnt(" #n ") lgkmcnt(0)" ::: "memory")
#define WAIT_LGKM0() asm volatile("s_waitcnt lgkmcnt(0)" ::: "memory")
#define BARRIER() do { asm volatile("" ::: "memory"); __builtin_amdgcn_s_barrier(); asm volatile("" ::: "memory"); } while (0)
#define SCHED() __builtin_amdgcn_sched_barrier(0)
#define RLX_AGENT __ATOMIC_RELAXED, __HIP_MEMORY_SCOPE_AGENT
#define RLX_SYSTEM __ATOMIC_RELAXED, __HIP_MEMORY_SCOPE_SYSTEM

#define STAMP(i) do { } while (0)

__device__ __forceinline__ u32 pack_bf16(float a, float b) {
  f32x2 f = {a, b};
  bf16x2 h = __builtin_convertvector(f, bf16x2);
  return __builtin_bit_cast(u32, h);
}

__device__ __forceinline__ u32 wave_max_u32(u32 v) {
  v = max(v, (u32)__builtin_amdgcn_update_dpp(0, (int)v, 0x111, 0xf, 0xf, false));
  v = max(v, (u32)__builtin_amdgcn_update_dpp(0, (int)v, 0x112, 0xf, 0xf, false));
  v = max(v, (u32)__builtin_amdgcn_update_dpp(0, (int)v, 0x114, 0xf, 0xf, false));
  v = max(v, (u32)__builtin_amdgcn_update_dpp(0, (int)v, 0x118, 0xf, 0xf, false));
  const u32 a = (u32)__builtin_amdgcn_readlane((int)v, 15), b = (u32)__builtin_amdgcn_readlane((int)v, 31);
  const u32 c = (u32)__builtin_amdgcn_readlane((int)v, 47), d = (u32)__builtin_amdgcn_readlane((int)v, 63);
  return max(max(a, b), max(c, d));
}

__device__ __forceinline__ bool give_up(u32& spins, u64 t0, GAS u32* tmo, u32 seq) {
  ++spins;
  bool stop = (__builtin_amdgcn_s_memrealtime() - t0) > DEADLINE_TICKS;
  if ((spins & 7u) == 0u) stop |= (__hip_atomic_load(tmo, RLX_AGENT) == seq);
  stop = __builtin_amdgcn_readfirstlane((int)stop) != 0;
  if (stop) __hip_atomic_store(tmo, seq, RLX_AGENT);
  else __builtin_amdgcn_s_sleep(1);
  return stop;
}

__device__ __forceinline__ void w_load(f32x4* v, const float* __restrict__ wsrc, int chunk) {
  const float* s = wsrc + (size_t)chunk * (KPERM ? 64 : 256) * NL;
#pragma unroll
  for (int j = 0; j < 8; ++j) v[j] = *(const f32x4*)(s + (size_t)j * NL);
}

__device__ __forceinline__ void w_store(const f32x4* v, __amdgpu_buffer_rsrc_t rs, u32 woff, int chunk) {
#pragma unroll
  for (int r = 0; r < 4; ++r) {
    u32x4 d;
    d.x = pack_bf16(v[0][r], v[1][r]);
    d.y = pack_bf16(v[2][r], v[3][r]);
    d.z = pack_bf16(v[4][r], v[5][r]);
    d.w = pack_bf16(v[6][r], v[7][r]);
    __builtin_amdgcn_raw_buffer_store_b128(d, rs, woff + r * 2048, chunk * (4 * W_KS), 16  );
  }
}

__device__ __forceinline__ void ctl_write(u32 addr, u32 v) {
  asm volatile("ds_write_b32 %0, %1" ::"v"(addr), "v"(v) : "memory");
}
__device__ __forceinline__ u32 ctl_read(u32 addr) {
  u32 v;
  asm volatile("ds_read_b32 %0, %1\n\ts_waitcnt lgkmcnt(0)" : "=v"(v) : "v"(addr) : "memory");
  return v;
}
__device__ __forceinline__ void x_load(f32x4* v, const float* __restrict__ x, int xh, int xks, int msub, int lane) {
  const float* xs = x + (size_t)(xh * 256 + msub * 16 + (lane & 15)) * KD + kcol(xks) + (lane >> 4) * 8;
#pragma unroll
  for (int j = 0; j < 4; ++j) v[j] = *(const f32x4*)(xs + (j >> 1) * 32 + (j & 1) * 4);
}

__device__ __forceinline__ void x_store(const f32x4* v, __amdgpu_buffer_rsrc_t rs, int xh, int xks, int msub, int lane) {
  const u32 xoff = (u32)(((xh * 64 + xks) * 16 + msub) * 2048 + lane * 16);
#pragma unroll
  for (int kh = 0; kh < 2; ++kh) {
    u32x4 d;
    d.x = pack_bf16(v[kh * 2][0], v[kh * 2][1]);
    d.y = pack_bf16(v[kh * 2][2], v[kh * 2][3]);
    d.z = pack_bf16(v[kh * 2 + 1][0], v[kh * 2 + 1][1]);
    d.w = pack_bf16(v[kh * 2 + 1][2], v[kh * 2 + 1][3]);
    __builtin_amdgcn_raw_buffer_store_b128(d, rs, xoff + kh * 1024, 0, 17  );
  }
}

__device__ __forceinline__ u32x4 pack8(const f32x4& a, const f32x4& b) {
  u32x4 d;
  d.x = pack_bf16(a[0], a[1]);
  d.y = pack_bf16(a[2], a[3]);
  d.z = pack_bf16(b[0], b[1]);
  d.w = pack_bf16(b[2], b[3]);
  return d;
}

__device__ __forceinline__ u64 both_ge(u64 f, u32 seq) { return __ballot((u32)f >= seq && (u32)(f >> 32) >= seq); }
__device__ __forceinline__ int lead_ones(u64 m) { return m == ~0ull ? 64 : __builtin_ctzll(~m); }

__global__ void __launch_bounds__(NTHREADS) __attribute__((amdgpu_waves_per_eu(3, 3))) ag_gemm_q8(Params p) {
  extern __shared__ __attribute__((aligned(16))) char lds[];
  const int b = blockIdx.x;
  const int tid = threadIdx.x, lane = tid & 63;
  const int wid = __builtin_amdgcn_readfirstlane(tid >> 6);
  const int idx = b >> 3;
  const int half = idx & 1, nt = idx >> 1, shr = (b & 7) * 2 + half;
  const float* const px = p.x;
  const float* const pw = p.w;
  unsigned char* const pout = p.out;
  char* const myws = p.myws;
  char* const srcws = p.sws[b & 7];
  char* const widws = p.ws[wid & 7];
  const int rank = p.rank;
  const u32 perm = p.perm;
  const u64 w0 = (u64)p.ws[0], w1 = (u64)p.ws[1], w2 = (u64)p.ws[2], w3 = (u64)p.ws[3];
  const u64 w4 = (u64)p.ws[4], w5 = (u64)p.ws[5], w6 = (u64)p.ws[6], w7 = (u64)p.ws[7];
  asm volatile("" ::"s"(px), "s"(pw), "s"(pout), "s"(myws), "s"(srcws), "s"(widws), "s"(rank), "s"(perm), "s"(w0), "s"(w1), "s"(w2), "s"(w3),
               "s"(w4), "s"(w5), "s"(w6), "s"(w7));
  const int src = (perm >> (4 * (b & 7))) & 7, mt = src * 2 + half;
  auto peer_ws = [&]() {
    int lo = (int)w0, hi = (int)(w0 >> 32);
#define PUT(i) asm("v_writelane_b32 %0, %2, " #i "\n\tv_writelane_b32 %1, %3, " #i : "+v"(lo), "+v"(hi) : "s"((int)w##i), "s"((int)(w##i >> 32)))
    PUT(1); PUT(2); PUT(3); PUT(4); PUT(5); PUT(6); PUT(7);
#undef PUT
    return (char*)(((u64)(u32)hi << 32) | (u32)lo);
  };
  GAS u32* const cnt = (GAS u32*)(myws + CNT_OFF) + b;
  const __amdgpu_buffer_rsrc_t xrs = __builtin_amdgcn_make_buffer_rsrc(myws + XB_OFF, 0, (int)XB_BYTES, 0x00020000);

  if (__builtin_expect(wid < 2, 1)) {
    f32x4 v[4];
    const int xh = (b >> 4) & 1, xks = wid * X0_END + (b >> 5), msub = b & 15;
    if (wid) __builtin_amdgcn_s_sleep(X1_SLEEP);
    x_load(v, px, xh, xks, msub, lane);
    const u32 seq = __hip_atomic_load(cnt, RLX_AGENT);
    SCHED();
    GAS u32* pub = (GAS u32*)(peer_ws() + XFL_OFF) + (rank * 2 + xh) * 384 + xks * 16 + msub;
    asm volatile("" : "+v"(pub));
    SCHED();
    if (PUSH > 0 && xks < PUSH) {
      const u32x4 d0 = pack8(v[0], v[1]), d1 = pack8(v[2], v[3]);
      const u32 lo = (u32)((((rank * 2 + xh) * 2 + xks) * 16 + msub) * 2048 + lane * 16);
      const u64 wsp[NR] = {w0, w1, w2, w3, w4, w5, w6, w7};
#pragma unroll
      for (int i = 0; i < NR; ++i) {
        const __amdgpu_buffer_rsrc_t rs = __builtin_amdgcn_make_buffer_rsrc((char*)wsp[i] + LX_OFF, 0, (int)LX_BYTES, 0x00020000);
        __builtin_amdgcn_raw_buffer_store_b128(d0, rs, lo, 0, 17);
        __builtin_amdgcn_raw_buffer_store_b128(d1, rs, lo + 1024, 0, 17);
      }
    } else {
      x_store(v, xrs, xh, xks, msub, lane);
    }
    SCHED();
    WAIT_VM(0);
    if (lane < 8) __hip_atomic_store(pub, seq + 1u, RLX_SYSTEM);
  }

  const u32 ctl = (u32)(size_t)(LAS char*)lds + LDS_CTL;
  const int kq1 = lane >> 5, wH = (lane >> 4) & 1, wi = lane & 15;
  const float* const wsrc = pw + (size_t)(kcol(shr >> 2) + (shr & 3) * 16 + kq1 * 8) * NL + nt * 128 + wH * 64 + wi * 4;
  const u32 woff = (u32)(nt * W_NT + (shr >> 2) * W_KS + wH * 8192 + ((shr >> 1) & 1) * 1024 +
                         (((shr & 1) * 2 + kq1) * 16 + wi) * 16);
  const __amdgpu_buffer_rsrc_t wrs =
      __builtin_amdgcn_make_buffer_rsrc(myws + WB_OFF, 0, (int)WB_BYTES, 0x00020000);

  GAS u32* const wfl = (GAS u32*)(myws + WFL_OFF) + nt * 256;
  GAS u32* const xfl = (GAS u32*)(myws + XFL_OFF) + mt * 384;
  GAS u32* const tmo = (GAS u32*)(myws + TMO_OFF);

  if (wid == 8) {
    __builtin_amdgcn_s_setprio(3);
    STAMP(0);
    const u32 seq = (u32)__builtin_amdgcn_readfirstlane((int)__hip_atomic_load(cnt, RLX_AGENT)) + 1u;
    bool failed = false;

    int xok = 0, cw = 0;
    GAS u64* const xf = (GAS u64*)xfl + lane;
    struct Poll { u32 fw; u64 f0, f1, f2; int cw; };
    Poll pa;
    auto poll = [&](Poll& q) {
      q.cw = cw;
      q.fw = __hip_atomic_load(wfl + min(cw + (lane >> 4), NCH - 1) * 16 + (lane & 15), RLX_AGENT);
      q.f0 = __hip_atomic_load(xf, RLX_SYSTEM);
      q.f1 = __hip_atomic_load(xf + 64, RLX_SYSTEM);
      q.f2 = __hip_atomic_load(xf + 128, RLX_SYSTEM);
      SCHED();
    };
    auto seen = [&](Poll& q) {
      asm volatile("" : "+v"(q.fw), "+v"(q.f0), "+v"(q.f1), "+v"(q.f2));
      cw = max(cw, min(q.cw + lead_ones(__ballot(q.fw >= seq)) / 16, NCH));
      int n = lead_ones(both_ge(q.f0, seq)) >> 3;
      if (n == X0_END) n += lead_ones(both_ge(q.f1, seq)) >> 3;
      if (n == X1_END && both_ge(q.f2, seq) == ~0ull) n = NKS;
      xok = max(xok, n);
      SCHED();
    };
    auto wait_for = [&](int sx, int sw) {
      const u64 t0 = __builtin_amdgcn_s_memrealtime();
      u32 spins = 0;
      for (;;) {
        Poll q;
        poll(q);
        seen(q);
        if (xok > sx && 4 * cw > sw) return;
        if (give_up(spins, t0, tmo, seq)) { failed = true; xok = NKS; cw = NCH; return; }
      }
    };
    auto need = [&](int s, Poll& q) {
      if (xok > s && 4 * cw > s) return;
      seen(q);
      if (xok > s && 4 * cw > s) return;
      wait_for(s, s);
    };
    auto wflag = [&](int c) { if (lane == 0) __hip_atomic_store(wfl + c * 16 + shr, seq, RLX_AGENT); };
#define KBAR(t) do { BARRIER(); SCHED(); } while (0)
#define PIN(v, n) _Pragma("unroll") for (int i_ = 0; i_ < (n); ++i_) asm volatile("" : "+v"((v)[i_]))

    {
      f32x4 wv[8];
      if (LEAN && B1_W == 0) {
        const u64 t0 = __builtin_amdgcn_s_memrealtime();
        u32 spins = 0;
        for (;;) {
          u64 f = ~0ull;
          if (lane < 8 * B1_X) f = __hip_atomic_load(xf, RLX_SYSTEM);
          if (both_ge(f, seq) == ~0ull) break;
          if (give_up(spins, t0, tmo, seq)) { failed = true; xok = NKS; cw = NCH; break; }
        }
        xok = max(xok, B1_X);
      } else {
        wait_for(B1_X - 1, 4 * B1_W - 4);
      }
      STAMP(2);
      poll(pa);
      BARRIER();
      SCHED();
      need(2, pa);
      KBAR(-1);
      if (lane == 0) __hip_atomic_store(cnt, seq, RLX_AGENT);
      seen(pa);
      poll(pa);
      if (!PEEL) w_load(wv, wsrc, 3);
      SCHED();
      need(3, pa);
      KBAR(0);
      if (PEEL) {
        need(4, pa);
        w_load(wv, wsrc, 3);
        SCHED();
        KBAR(1);
        need(5, pa);
        KBAR(2);
        PIN(wv, 8);
        w_store(wv, wrs, woff, 3);
        poll(pa);
        need(6, pa);
        KBAR(3);
        w_load(wv, wsrc, 4);
        SCHED();
        need(7, pa);
        KBAR(4);
        WAIT_VM(12);
        wflag(3);
        seen(pa);
      }
      for (int c = PEEL; c + 3 < NCH; ++c) {
        need(4 * c + 4, pa);
        KBAR(4 * c + 1);
        PIN(wv, 8);
        w_store(wv, wrs, woff, c + 3);
        SCHED();
        need(4 * c + 5, pa);
        KBAR(4 * c + 2);
        if (!LATE_FLAG) {
          WAIT_VM(0);
          wflag(c + 3);
        }
        poll(pa);
        need(4 * c + 6, pa);
        KBAR(4 * c + 3);
        if (LATE_FLAG) {
          WAIT_VM(4);
          wflag(c + 3);
        }
        w_load(wv, wsrc, min(c + 4, NCH - 1));
        SCHED();
        need(4 * c + 7, pa);
        KBAR(4 * c + 4);
        seen(pa);
      }
      for (int t = 4 * (NCH - 3) + 1; t < NKS - 1; ++t) {
        if ((t & 1) == 0) poll(pa);
        need(min(t + 3, NKS - 1), pa);
        if (t == NKS - 2) {
          ctl_write(ctl + 17 * 4, failed ? 1u : 0u);
          ctl_write(ctl + 18 * 4, seq);
          WAIT_LGKM0();
        }
        KBAR(t);
        if (t & 1) seen(pa);
      }
    }
    return;
  }

  char* const xsrc = srcws + XB_OFF + (size_t)half * (XB_BYTES / 2);
  const __amdgpu_buffer_rsrc_t ars = __builtin_amdgcn_make_buffer_rsrc(xsrc, 0, (int)(XB_BYTES / 2), 0x00020000);
  if (wid == 9) {
    __builtin_amdgcn_s_setprio(3);
    const u32 seq0 = __hip_atomic_load(cnt, RLX_AGENT);
    const u32 arr = ctl + (LDS_FLG - LDS_CTL) + lane * 4;
    ctl_write(arr, 0u);
    GAS u32* pub = (GAS u32*)(peer_ws() + XFL_OFF) + (rank * 2 + (b & 1)) * 384 + 256 + (b >> 1);
    const __amdgpu_buffer_rsrc_t frs = __builtin_amdgcn_make_buffer_rsrc((void*)xfl, 0, 384 * 4, 0x00020000);
    const u32 fo = (lane < X1_END ? lane * 16 + nt : 256 + ((lane - X1_END) * 16 + nt) / 6) * 4;
    u32 fom = lane < FP_N ? fo : 0x7fffffffu;
    u32 hist = 0;
    auto fpoll = [&]() {
      __builtin_amdgcn_raw_ptr_buffer_load_lds(frs, (LAS void*)(lds + LDS_FLG), 4, fom, 0, 0, 17);
      hist <<= 1;
    };
    WAIT_LGKM0();
    const u32 seq = (u32)__builtin_amdgcn_readfirstlane((int)seq0) + 1u;
    auto late_flags = [&]() {
      if (lane < 8) __hip_atomic_store(pub, seq, RLX_SYSTEM);
      if (lane == 0) __hip_atomic_store(wfl + 2 * 16 + shr, seq, RLX_AGENT);
      hist <<= 2;
    };
    STAMP(1);
    if (src == rank) {
      BARRIER();
      BARRIER();
      BARRIER();
      late_flags();
      return;
    }
    GAS u32* const tick = (GAS u32*)(myws + INV_OFF) + (__builtin_amdgcn_s_getreg(20 | (3 << 11)) & 15) * 32;
    int k = 1;
    {
      LAS void* dst = (LAS void*)(lds + LDS_PF);
      const u32 vo = nt * 2048 + lane * 16;
      u64 todo = ~0ull << PUSH;
      auto pull = [&](int s) {
        __builtin_amdgcn_raw_ptr_buffer_load_lds(ars, dst, 16, vo, s * A_BYTES, 0, 0);
        __builtin_amdgcn_raw_ptr_buffer_load_lds(ars, dst, 16, vo + 1024, s * A_BYTES, 0, 0);
        hist = hist * 4u + 3u;
        todo &= ~(1ull << s);
      };
      if (PF0 > 0) {
        const u64 t0 = __builtin_amdgcn_s_memrealtime();
        const u64 want = ((1ull << PF0) - 1ull) << PUSH;
        if (PF_SLEEP) __builtin_amdgcn_s_sleep(PF_SLEEP);
        for (;;) {
          fpoll();
          WAIT_VM(0);
          if ((__ballot(ctl_read(arr) >= seq) & want) == want) {
#pragma unroll
            for (int s = PUSH; s < PUSH + PF0; ++s) pull(s);
            break;
          }
          if (__builtin_amdgcn_s_memrealtime() - t0 > 1000) break;
        }
      } else if (PRE_POLL) {
        fpoll();
      }
      for (; todo; ++k) {
        if (k > 1) todo &= ~0ull << min(k, NKS - 1) << (k >= NKS);
        const u64 near = ~0ull >> (NKS - 1 - min(k - 1 + DMAX, NKS - 1));
        if (k == (PF0 > 0 ? 1 : 2)) fom = fo;
        const u64 pubd = __ballot(ctl_read(arr) >= seq);
        for (u64 cand = todo & pubd & near; cand; cand &= cand - 1) {
          const u32 ib = __builtin_amdgcn_s_getreg(7 | (31 << 11));
          const u32 vm = (ib & 15u) | ((ib >> 18) & 48u);
          if (__builtin_popcount(hist & ((1u << min(vm, 31u)) - 1u)) > WIN) break;
          pull(__builtin_ctzll(cand));
        }
        if ((todo & ~pubd) && (k < FP_K || (k & 1))) fpoll();
        if (k == 4) late_flags();
        BARRIER();
      }
    }
    u32 old = 0;
    if (lane == 0) old = __hip_atomic_exchange(tick, seq, RLX_SYSTEM);
    SCHED();
    for (; k < NKS + 2; ++k) BARRIER();
    WAIT_VM(0);
    BARRIER();
    if ((u32)__builtin_amdgcn_readfirstlane((int)old) == seq) return;
    {
      GAS u64* const g = (GAS u64*)(myws + AMX_OFF + rank * 2048) + lane;
      const u64 t0 = __builtin_amdgcn_s_memrealtime();
      u32 spins = 0;
      for (;;) {
        bool all = true;
#pragma unroll
        for (int k = 0; k < 4; ++k) all &= (u32)(__hip_atomic_load(g + k * 64, RLX_SYSTEM) >> 32) >= seq;
        if (__all(all)) break;
        if (give_up(spins, t0, tmo, seq)) break;
      }
      WAIT_VM(0);
      asm volatile("buffer_inv sc0 sc1" ::: "memory");
    }
    BARRIER();
    WAIT_VM(0);
    STAMP(15);
    return;
  }

  f32x4 dv[DW > 0 ? DW : 1][8];
  if (DW > 0 && wid >= 4) {
    if (DW_SLEEP) __builtin_amdgcn_s_sleep(DW_SLEEP);
#pragma unroll
    for (int d = 0; d < DW; ++d)
      w_load(dv[d], pw + (size_t)(kcol(d) + (wid - 4) * 16 + kq1 * 8) * NL + nt * 128 + wH * 64 + wi * 4, 0);
    SCHED();
  }
  auto dw_store = [&]() {
    char* const dst = lds + A_BYTES + wH * 8192 + ((wid >> 1) & 1) * 1024 + (((wid & 1) * 2 + kq1) * 16 + wi) * 16;
#pragma unroll
    for (int d = 0; d < DW; ++d)
#pragma unroll
      for (int r = 0; r < 4; ++r) {
        u32x4 q;
        q.x = pack_bf16(dv[d][0][r], dv[d][1][r]);
        q.y = pack_bf16(dv[d][2][r], dv[d][3][r]);
        q.z = pack_bf16(dv[d][4][r], dv[d][5][r]);
        q.w = pack_bf16(dv[d][6][r], dv[d][7][r]);
        *(u32x4*)(dst + d * STAGE + r * 2048) = q;
      }
    SCHED();
  };
  if (DW > 0 && wid >= 4 && !(LATE_DW && LATE_CV)) dw_store();
  f32x4 cv[8];
  const int xl = (b >> 1) * 6 + (wid - 4) * 2;
  u32 seqw = 0;
  if (wid == 2 || wid == 3) {
    const bool skip = wid == 2 && (shr >> 2) < DW;
    if (wid == 3) __builtin_amdgcn_s_sleep(P1_SLEEP);
    if (!skip) w_load(cv, wsrc, wid - 2);
    seqw = __hip_atomic_load(cnt, RLX_AGENT) + 1u;
    SCHED();
    if (!skip) w_store(cv, wrs, woff, wid - 2);
    SCHED();
    if (wid == 2 || !LATE_CV || W1_PRE) {
      WAIT_VM(0);
      if (lane == 0) __hip_atomic_store(wfl + (wid - 2) * 16 + shr, seqw, RLX_AGENT);
      STAMP(10 + wid);
    }
  } else if (wid == 7) {
    if (W2_SLEEP) __builtin_amdgcn_s_sleep(W2_SLEEP);
    w_load(cv, wsrc, 2);
    SCHED();
  } else if (wid >= 4 && !SKIP_X2) {
    if (X2_SLEEP) __builtin_amdgcn_s_sleep(X2_SLEEP);
    x_load(cv, px, b & 1, X1_END + (xl >> 4), xl & 15, lane);
    x_load(cv + 4, px, b & 1, X1_END + (xl >> 4), (xl & 15) + 1, lane);
    SCHED();
  }
  auto cv_store = [&]() {
    if (wid == 7) {
      w_store(cv, wrs, woff, 2);
    } else if (wid >= 4 && !SKIP_X2) {
      x_store(cv, xrs, b & 1, X1_END + (xl >> 4), xl & 15, lane);
      x_store(cv + 4, xrs, b & 1, X1_END + (xl >> 4), (xl & 15) + 1, lane);
    }
    SCHED();
  };
  if (!LATE_CV) cv_store();

  const __amdgpu_buffer_rsrc_t brs = __builtin_amdgcn_make_buffer_rsrc(
      myws + WB_OFF + (size_t)nt * W_NT, 0, (int)(WB_BYTES - nt * W_NT), 0x00020000);
  const u32 l16 = lane * 16;
  const u32 avo = wid * 4096 + l16, bvo = wid * 2048 + l16;
  const __amdgpu_buffer_rsrc_t lrs = __builtin_amdgcn_make_buffer_rsrc(myws + LX_OFF + mt * 2 * A_BYTES, 0, 2 * A_BYTES, 0x00020000);
  auto piece = [&](int t, u32 slot, int j, bool pushed = false) {
    const int jb = A_FIRST ? j - 4 : j, ja = A_FIRST ? j : j - 2;
    if (jb >= 0 && jb < 2) {
      LAS void* lb = (LAS void*)(lds + slot + A_BYTES + wid * 2048);
      if (jb == 0) __builtin_amdgcn_raw_ptr_buffer_load_lds(brs, lb, 16, bvo, t * W_KS, 0, B_AUX);
      else __builtin_amdgcn_raw_ptr_buffer_load_lds(brs, lb, 16, bvo, t * W_KS, 1024, B_AUX);
    } else {
      const u32 ro = ROT ? ((ja + nt) & 3) * 1024 : ja * 1024;
      LAS void* la = (LAS void*)(lds + slot + wid * 4096 + ro);
      __builtin_amdgcn_raw_ptr_buffer_load_lds(pushed ? lrs : ars, la, 16, avo, t * A_BYTES + ro, 0, 0);
    }
    SCHED();
  };

  const int wr = wid >> 1, wc = wid & 1;
  const u32 aoff = wr * 8192 + l16, boff = A_BYTES + wc * 8192 + l16;
  f32x4 acc[4][4];
#pragma unroll
  for (int i = 0; i < 4; ++i)
#pragma unroll
    for (int j = 0; j < 4; ++j) acc[i][j] = (f32x4){0.f, 0.f, 0.f, 0.f};
  bf16x8 af[2][4], bfr[2][4];
  auto frags = [&](u32 slot, int kh) {
#pragma unroll
    for (int i = 0; i < 4; ++i) {
      bfr[kh][i] = *(const bf16x8*)(lds + slot + boff + i * 2048 + kh * 1024);
      af[kh][i] = *(const bf16x8*)(lds + slot + aoff + i * 2048 + kh * 1024);
    }
    SCHED();
  };
  auto mfma4 = [&](int kh, int mi) {
#pragma unroll
    for (int r = 0; r < 4; ++r)
      acc[mi][r] = __builtin_amdgcn_mfma_f32_16x16x32_bf16(bfr[kh][r], af[kh][mi], acc[mi][r], 0, 0, 0);
    SCHED();
  };

  BARRIER();
  static_assert(A_FIRST && DW <= 2, "piece order");
#pragma unroll
  for (int j = 0; j < (DW > 0 ? 4 : 6); ++j) piece(0, 0, j, PUSH > 0);
#pragma unroll
  for (int j = 0; j < (DW > 1 ? 4 : 6); ++j) piece(1, STAGE, j, PUSH > 1);
  static_assert(!LATE_CV || DW == 2, "wait counts");
  if (LATE_CV && wid >= 4) {
    if (DW > 0 && LATE_DW) dw_store();
    cv_store();
    if (SKIP_X2 && wid < 7) WAIT_VM_LGKM(4);
    else WAIT_VM_LGKM(8);
  } else if (LATE_CV && !W1_PRE && wid == 3) {
    WAIT_VM(8);
    if (lane == 0) __hip_atomic_store(wfl + 16 + shr, seqw, RLX_AGENT);
    STAMP(13);
    WAIT_VM(5);
  } else if (DW > 1) {
    WAIT_VM_LGKM(4);
  } else {
    WAIT_VM_LGKM(6);
  }
  BARRIER();
  frags(0, 0);
#pragma unroll
  for (int j = 0; j < 4 - BAR_POS; ++j) piece(2, 2 * STAGE, j);

  u32 cur = 0, nx1 = STAGE, nx2 = 2 * STAGE;
  auto step = [&](int t, auto more) {
    constexpr int MORE = decltype(more)::value;
    frags(cur, 1);
#pragma unroll
    for (int g = 0; g < 4; ++g) {
      mfma4(0, g);
      if (MORE >= 2 && 4 - BAR_POS + g < 6) piece(t + 2, nx2, 4 - BAR_POS + g);
    }
    constexpr int BP = MORE == 1 ? 4 : BAR_POS;
#pragma unroll
    for (int g = 0; g < BP; ++g) mfma4(1, g);
    if (MORE >= 1) {
      if (MORE >= 2) WAIT_VM_LGKM(6);
      else WAIT_VM_LGKM(0);
      BARRIER();
      SCHED();
      frags(nx1, 0);
    }
#pragma unroll
    for (int g = BP; g < 4; ++g) {
      mfma4(1, g);
      if (MORE >= 3) piece(t + 3, cur, g - BP);
    }
    const u32 o = cur;
    cur = nx1; nx1 = nx2; nx2 = o;
  };
  for (int t = 0; t < NKS - 3; ++t) step(t, std::integral_constant<int, 3>());
  step(NKS - 3, std::integral_constant<int, 2>());
  step(NKS - 2, std::integral_constant<int, 1>());
  step(NKS - 1, std::integral_constant<int, 0>());

  float mx = 0.f;
#pragma unroll
  for (int mi = 0; mi < 4; ++mi)
#pragma unroll
    for (int r = 0; r < 4; ++r)
#pragma unroll
      for (int j = 0; j < 4; ++j) mx = fmaxf(mx, fabsf(acc[mi][r][j]));
  const u32 wm = wave_max_u32(__builtin_bit_cast(u32, mx));
  ctl_write(ctl + (24 + wid) * 4, wm);
  const u32 fs = ctl_read(ctl + (17 + (lane & 1)) * 4);
  BARRIER();
  const __amdgpu_buffer_rsrc_t ors = __builtin_amdgcn_make_buffer_rsrc(pout, 0, 4096 * NL, 0x00020000);
  const u32 oo = (u32)((mt * 256 + wr * 64 + (lane & 15)) * NL + nt * 128 + wc * 64 + (lane >> 4) * 16);
  const u32 seq = (u32)__builtin_amdgcn_readlane((int)fs, 1);
  {
    u32 m = wave_max_u32(ctl_read(ctl + (24 + (lane & 7)) * 4));
    if (__builtin_amdgcn_readlane((int)fs, 0)) m = 0x7fffffffu;
    u32 tm = 0;
    if (wid == 0) tm = __hip_atomic_load(tmo, RLX_AGENT);
    if (lane == 0) __hip_atomic_store((GAS u64*)(widws + AMX_OFF) + rank * 256 + b, ((u64)seq << 32) | m, RLX_SYSTEM);
    const __amdgpu_buffer_rsrc_t mrs = __builtin_amdgcn_make_buffer_rsrc(myws + AMX_OFF, 0, NR * 2048, 0x00020000);
    const u64 t0 = __builtin_amdgcn_s_memrealtime();
    u32 spins = 0;
    for (;;) {
      const u32x4 g0 = __builtin_amdgcn_raw_buffer_load_b128(mrs, l16, wid * 2048, 17);
      const u32x4 g1 = __builtin_amdgcn_raw_buffer_load_b128(mrs, l16 + 1024, wid * 2048, 17);
      m = max(max(g0.x, g0.z), max(g1.x, g1.z));
      if (__all(min(min(g0.y, g0.w), min(g1.y, g1.w)) >= seq)) break;
      if (give_up(spins, t0, tmo, seq)) { m = 0x7fffffffu; break; }
      if (XP_SLEEP) __builtin_amdgcn_s_sleep(XP_SLEEP);
    }
    if (tm == seq) m = 0x7fffffffu;
    ctl_write(ctl + (32 + wid) * 4, wave_max_u32(m));
    WAIT_LGKM0();
  }
  BARRIER();
  const u32 gmax = wave_max_u32(ctl_read(ctl + (32 + (lane & 7)) * 4));
  const bool bad = gmax > 0x7f800000u;
  const float amax = __builtin_bit_cast(float, gmax);
  const float inv = amax > 0.f ? 448.0f / amax : 0.f;
#pragma unroll
  for (int mi = 0; mi < 4; ++mi) {
    u32x4 d;
#pragma unroll
    for (int j = 0; j < 4; ++j) {
      int q = __builtin_amdgcn_cvt_pk_fp8_f32(acc[mi][0][j] * inv, acc[mi][1][j] * inv, 0, false);
      q = __builtin_amdgcn_cvt_pk_fp8_f32(acc[mi][2][j] * inv, acc[mi][3][j] * inv, q, true);
      d[j] = bad ? 0x7f7f7f7fu : (u32)q;
    }
    __builtin_amdgcn_raw_buffer_store_b128(d, ors, oo + mi * 16 * NL, 0, OUT_AUX);
  }
  if (bad) asm volatile("buffer_inv sc0 sc1" ::: "memory");
}

}

extern "C" void kernel_launch(int rank, int nranks, ncclComm_t comm, hipStream_t stream,
                              void* const* d_in, const int* in_sizes, int n_in,
                              void* d_out, int out_size,
                              const int* in_dtypes, int out_dtype,
                              void* const* d_out_all, void* const* d_ws_all, size_t ws_bytes) {
  (void)comm; (void)in_dtypes; (void)out_dtype; (void)d_out_all;
  if (nranks != NR || n_in < 2 || ws_bytes < WS_NEED || in_sizes[0] != 512 * KD || in_sizes[1] != KD * NL ||
      out_size != 4096 * NL || (unsigned)rank >= (unsigned)nranks) {
    fprintf(stderr, "unexpected problem geometry\n");
    return;
  }
  hipError_t e = hipFuncSetAttribute((const void*)ag_gemm_q8, hipFuncAttributeMaxDynamicSharedMemorySize, (int)LDS_BYTES);
  if (e != hipSuccess) { fprintf(stderr, "hipFuncSetAttribute: %s\n", hipGetErrorString(e)); return; }
  Params p;
  p.x = (const float*)d_in[0];
  p.w = (const float*)d_in[1];
  p.out = (unsigned char*)d_out;
  p.myws = (char*)d_ws_all[rank];
  for (int i = 0; i < NR; ++i) p.ws[i] = (char*)d_ws_all[i];
  p.rank = rank;
  p.perm = 0;
  for (int i = 0, lo = 0, hi = 4; i < NR; ++i) {
    const bool same = ((0x4d >> i) & 1) == ((0x4d >> rank) & 1);
    const int xc = PERM ? (same ? lo++ : hi++) : i;
    p.sws[xc] = p.ws[i];
    p.perm |= (u32)i << (4 * xc);
  }
  hipLaunchKernelGGL(ag_gemm_q8, dim3(NWG), dim3(NTHREADS), LDS_BYTES, stream, p);
  e = hipGetLastError();
  if (e != hipSuccess) fprintf(stderr, "launch: %s\n", hipGetErrorString(e));
}
```

```cpp
#include <hip/hip_runtime.h>
#include <rccl/rccl.h>
#include <cstdio>
#include <type_traits>

namespace {

typedef unsigned int u32;
typedef unsigned long long u64;
typedef __bf16 bf16x8 __attribute__((ext_vector_type(8)));
typedef __bf16 bf16x2 __attribute__((ext_vector_type(2)));
typedef float f32x4 __attribute__((ext_vector_type(4)));
typedef float f32x2 __attribute__((ext_vector_type(2)));
typedef u32 u32x4 __attribute__((ext_vector_type(4)));
typedef u32 u32x16 __attribute__((ext_vector_type(16)));
#define GAS __attribute__((address_space(1)))
#define LAS __attribute__((address_space(3)))

constexpr int NR = 8;
constexpr int KD = 4096;
constexpr int NL = 2048;
constexpr int NKS = 64;
constexpr int NCH = 16;
constexpr int NWG = 256;
#define WIN 2
#define FREE_K 2
#define X1_SLEEP 8
#define P1_SLEEP 16
#define DW_SLEEP 45
#define H_SLEEP 30
#define PF_SLEEP 30
#define POLL_GAP 2
#define P2_SLEEP 8
#define PF0 2
#define XP_SLEEP 2
#define PERM 1
constexpr int BAR_POS = 2;
constexpr u32 W_KS = 16 * 16384, W_NT = 16384;
constexpr int NTHREADS = 640;
constexpr int X0_END = 8;
constexpr int X1_END = 16;
constexpr u32 A_BYTES = 32768, STAGE = 49152;
constexpr u32 LDS_CTL = 3 * STAGE;
constexpr u32 LDS_PF = LDS_CTL + 256;
constexpr u32 LDS_BYTES = LDS_PF + 1024;

constexpr size_t XB_OFF = 0;
constexpr size_t XB_BYTES = 4ull << 20;
constexpr size_t WB_OFF = 4ull << 20;
constexpr size_t WB_BYTES = 16ull << 20;
constexpr size_t CT_OFF = 20ull << 20;
constexpr size_t CNT_OFF = CT_OFF;
constexpr size_t TMO_OFF = CT_OFF + 4096;
constexpr size_t INV_OFF = CT_OFF + 5120;
constexpr size_t XFL_OFF = CT_OFF + 8192;
constexpr size_t WFL_OFF = CT_OFF + 32768;
constexpr size_t DBG_OFF = CT_OFF + 65536;
constexpr size_t AMX_OFF = CT_OFF + 196608;
constexpr size_t WS_NEED = CT_OFF + 262144;

constexpr u64 DEADLINE_TICKS = 100000000ull;

struct Params {
  const float* x;
  const float* w;
  unsigned char* out;
  char* myws;
  char* ws[NR];
  char* sws[NR];
  u32 perm;
  int rank;
};

#define WAIT_VM(n) asm volatile("s_waitcnt vmcnt(" #n ")" ::: "memory")
#define WAIT_VM_LGKM(n) asm volatile("s_waitcnt vmcnt(" #n ") lgkmcnt(0)" ::: "memory")
#define WAIT_LGKM0() asm volatile("s_waitcnt lgkmcnt(0)" ::: "memory")
#define BARRIER() do { asm volatile("" ::: "memory"); __builtin_amdgcn_s_barrier(); asm volatile("" ::: "memory"); } while (0)
#define SCHED() __builtin_amdgcn_sched_barrier(0)
#define RLX_AGENT __ATOMIC_RELAXED, __HIP_MEMORY_SCOPE_AGENT
#define RLX_SYSTEM __ATOMIC_RELAXED, __HIP_MEMORY_SCOPE_SYSTEM

#define ARRV(t) do { } while (0)
#define ARRV_OUT() do { } while (0)
#define STAMP(i) do { } while (0)

__device__ __forceinline__ u32 pack_bf16(float a, float b) {
  f32x2 f = {a, b};
  bf16x2 h = __builtin_convertvector(f, bf16x2);
  return __builtin_bit_cast(u32, h);
}

__device__ __forceinline__ u32 wave_max_u32(u32 v) {
  v = max(v, (u32)__builtin_amdgcn_update_dpp(0, (int)v, 0x111, 0xf, 0xf, false));
  v = max(v, (u32)__builtin_amdgcn_update_dpp(0, (int)v, 0x112, 0xf, 0xf, false));
  v = max(v, (u32)__builtin_amdgcn_update_dpp(0, (int)v, 0x114, 0xf, 0xf, false));
  v = max(v, (u32)__builtin_amdgcn_update_dpp(0, (int)v, 0x118, 0xf, 0xf, false));
  const u32 a = (u32)__builtin_amdgcn_readlane((int)v, 15), b = (u32)__builtin_amdgcn_readlane((int)v, 31);
  const u32 c = (u32)__builtin_amdgcn_readlane((int)v, 47), d = (u32)__builtin_amdgcn_readlane((int)v, 63);
  return max(max(a, b), max(c, d));
}

__device__ __forceinline__ u32 sword(const GAS u32* p) {
  u32 v;
  asm volatile("s_load_dword %0, %1, 0x0 glc\n\ts_waitcnt lgkmcnt(0)" : "=&s"(v) : "s"(p) : "memory");
  return v;
}
__device__ __forceinline__ void sflags3(u32x16& v, u32x16& w, u32x16& z, const GAS u32* p, u32 off, const GAS u32* q, u32 off2) {
  asm volatile("s_load_dwordx16 %0, %3, %4 glc\n\ts_load_dwordx16 %1, %3, %4 offset:0x40 glc\n\ts_load_dwordx16 %2, %5, %6 glc\n\ts_waitcnt lgkmcnt(0)"
               : "=&s"(v), "=&s"(w), "=&s"(z)
               : "s"(p), "s"(__builtin_amdgcn_readfirstlane((int)off)), "s"(q), "s"(__builtin_amdgcn_readfirstlane((int)off2)) : "memory");
}
__device__ __forceinline__ u32 min16(const u32x16& v) {
  u32 m, n;
  asm("s_min_u32 %0, %2, %3\n\ts_min_u32 %1, %4, %5\n\ts_min_u32 %0, %0, %6\n\ts_min_u32 %1, %1, %7\n\t"
      "s_min_u32 %0, %0, %8\n\ts_min_u32 %1, %1, %9\n\ts_min_u32 %0, %0, %10\n\ts_min_u32 %1, %1, %11\n\t"
      "s_min_u32 %0, %0, %12\n\ts_min_u32 %1, %1, %13\n\ts_min_u32 %0, %0, %14\n\ts_min_u32 %1, %1, %15\n\t"
      "s_min_u32 %0, %0, %16\n\ts_min_u32 %1, %1, %17\n\ts_min_u32 %0, %0, %1"
      : "=&s"(m), "=&s"(n)
      : "s"(v[0]), "s"(v[1]), "s"(v[2]), "s"(v[3]), "s"(v[4]), "s"(v[5]), "s"(v[6]), "s"(v[7]), "s"(v[8]), "s"(v[9]), "s"(v[10]),
        "s"(v[11]), "s"(v[12]), "s"(v[13]), "s"(v[14]), "s"(v[15])
      : "scc");
  return m;
}

__device__ __forceinline__ bool give_up(u32 t0, GAS u32* tmo, u32 seq) {
  const u32 dt = (u32)__builtin_amdgcn_s_memrealtime() - t0;
  if (__builtin_expect(dt < 100000u, 1)) return false;
  __builtin_amdgcn_s_sleep(64);
  const bool stop = dt > (u32)DEADLINE_TICKS || (u32)__builtin_amdgcn_readfirstlane((int)__hip_atomic_load(tmo, RLX_AGENT)) == seq;
  if (stop) __hip_atomic_store(tmo, seq, RLX_AGENT);
  return stop;
}

__device__ __forceinline__ void w_load(f32x4* v, __amdgpu_buffer_rsrc_t rs, u32 off, int row) {
#pragma unroll
  for (int j = 0; j < 8; ++j)
    v[j] = __builtin_bit_cast(f32x4, __builtin_amdgcn_raw_buffer_load_b128(rs, off, (row + j) * (NL * 4), 0));
}

__device__ __forceinline__ u32x4 w_pack(const f32x4* v, int r) {
  u32x4 d;
  d.x = pack_bf16(v[0][r], v[1][r]);
  d.y = pack_bf16(v[2][r], v[3][r]);
  d.z = pack_bf16(v[4][r], v[5][r]);
  d.w = pack_bf16(v[6][r], v[7][r]);
  return d;
}

__device__ __forceinline__ void w_store(const f32x4* v, __amdgpu_buffer_rsrc_t rs, u32 woff, int chunk) {
#pragma unroll
  for (int r = 0; r < 4; ++r) __builtin_amdgcn_raw_buffer_store_b128(w_pack(v, r), rs, woff + r * 2048, chunk * (4 * W_KS), 16  );
}

__device__ __forceinline__ void ctl_write(u32 addr, u32 v) {
  asm volatile("ds_write_b32 %0, %1" ::"v"(addr), "v"(v) : "memory");
}
__device__ __forceinline__ u32 ctl_read(u32 addr) {
  u32 v;
  asm volatile("ds_read_b32 %0, %1\n\ts_waitcnt lgkmcnt(0)" : "=v"(v) : "v"(addr) : "memory");
  return v;
}
__device__ __forceinline__ void x_load(f32x4* v, const float* __restrict__ x, int xh, int xks, int msub, int lane) {
  const float* xs = x + (size_t)(xh * 256 + msub * 16 + (lane & 15)) * KD + xks * 64 + (lane >> 4) * 8;
#pragma unroll
  for (int j = 0; j < 4; ++j) v[j] = *(const f32x4*)(xs + (j >> 1) * 32 + (j & 1) * 4);
}

__device__ __forceinline__ void x_store(const f32x4* v, __amdgpu_buffer_rsrc_t rs, int xh, int xks, int msub, int lane) {
  const u32 xoff = (u32)(((xh * 64 + xks) * 16 + msub) * 2048 + lane * 16);
#pragma unroll
  for (int kh = 0; kh < 2; ++kh) {
    u32x4 d;
    d.x = pack_bf16(v[kh * 2][0], v[kh * 2][1]);
    d.y = pack_bf16(v[kh * 2][2], v[kh * 2][3]);
    d.z = pack_bf16(v[kh * 2 + 1][0], v[kh * 2 + 1][1]);
    d.w = pack_bf16(v[kh * 2 + 1][2], v[kh * 2 + 1][3]);
    __builtin_amdgcn_raw_buffer_store_b128(d, rs, xoff + kh * 1024, 0, 17  );
  }
}

__device__ __forceinline__ u64 both_ge(u64 f, u32 seq) { return __ballot((u32)f >= seq && (u32)(f >> 32) >= seq); }
__device__ __forceinline__ int lead_ones(u64 m) { return m == ~0ull ? 64 : __builtin_ctzll(~m); }

__global__ void __launch_bounds__(NTHREADS) __attribute__((amdgpu_waves_per_eu(3, 3))) ag_gemm_q8(Params p) {
  extern __shared__ __attribute__((aligned(16))) char lds[];
  const int b = blockIdx.x;
  const int tid = threadIdx.x, lane = tid & 63;
  const int wid = __builtin_amdgcn_readfirstlane(tid >> 6);
  const int idx = b >> 3;
  const int half = idx & 1, nt = idx >> 1, shr = (b & 7) * 2 + half;
  const float* const px = p.x;
  const float* const pw = p.w;
  unsigned char* const pout = p.out;
  char* const myws = p.myws;
  char* const srcws = p.sws[b & 7];
  char* const widws = p.ws[wid & 7];
  const int rank = p.rank;
  const u32 perm = p.perm;
  const u64 w0 = (u64)p.ws[0], w1 = (u64)p.ws[1], w2 = (u64)p.ws[2], w3 = (u64)p.ws[3];
  const u64 w4 = (u64)p.ws[4], w5 = (u64)p.ws[5], w6 = (u64)p.ws[6], w7 = (u64)p.ws[7];
  asm volatile("" ::"s"(px), "s"(pw), "s"(pout), "s"(myws), "s"(srcws), "s"(widws), "s"(rank), "s"(perm), "s"(w0), "s"(w1), "s"(w2), "s"(w3),
               "s"(w4), "s"(w5), "s"(w6), "s"(w7));
  const int src = (perm >> (4 * (b & 7))) & 7, mt = src * 2 + half;
  auto peer_ws = [&]() {
    int lo = (int)w0, hi = (int)(w0 >> 32);
#define PUT(i) asm("v_writelane_b32 %0, %2, " #i "\n\tv_writelane_b32 %1, %3, " #i : "+v"(lo), "+v"(hi) : "s"((int)w##i), "s"((int)(w##i >> 32)))
    PUT(1); PUT(2); PUT(3); PUT(4); PUT(5); PUT(6); PUT(7);
#undef PUT
    return (char*)(((u64)(u32)hi << 32) | (u32)lo);
  };
  GAS u32* const cnt = (GAS u32*)(myws + CNT_OFF) + b;
  const __amdgpu_buffer_rsrc_t xrs = __builtin_amdgcn_make_buffer_rsrc(myws + XB_OFF, 0, (int)XB_BYTES, 0x00020000);

  if (__builtin_expect(wid < 2, 1)) {
    f32x4 v[4];
    const int xh = (b >> 4) & 1, xks = wid * X0_END + (b >> 5), msub = b & 15;
    if (wid) __builtin_amdgcn_s_sleep(X1_SLEEP);
    x_load(v, px, xh, xks, msub, lane);
    const u32 seq = __hip_atomic_load(cnt, RLX_AGENT);
    SCHED();
    GAS u32* pub = (GAS u32*)(peer_ws() + XFL_OFF) + (rank * 2 + xh) * 384 + xks * 16 + msub;
    asm volatile("" : "+v"(pub));
    SCHED();
    x_store(v, xrs, xh, xks, msub, lane);
    SCHED();
    WAIT_VM(0);
    if (lane < 8) __hip_atomic_store(pub, seq + 1u, RLX_SYSTEM);
  }

  const u32 ctl = (u32)(size_t)(LAS char*)lds + LDS_CTL;
  const int kq1 = lane >> 5, wH = (lane >> 4) & 1, wi = lane & 15;
  const __amdgpu_buffer_rsrc_t wfr = __builtin_amdgcn_make_buffer_rsrc((void*)pw, 0, KD * NL * 4, 0x00020000);
  const u32 wfo = (u32)(kq1 * 8 * NL + nt * 128 + wH * 64 + wi * 4) * 4u;
  const u32 woff = (u32)(nt * W_NT + (shr >> 2) * W_KS + wH * 8192 + ((shr >> 1) & 1) * 1024 +
                         (((shr & 1) * 2 + kq1) * 16 + wi) * 16);
  const __amdgpu_buffer_rsrc_t wrs =
      __builtin_amdgcn_make_buffer_rsrc(myws + WB_OFF, 0, (int)WB_BYTES, 0x00020000);

  GAS u32* const wfl = (GAS u32*)(myws + WFL_OFF) + nt * 256;
  GAS u32* const xfl = (GAS u32*)(myws + XFL_OFF) + mt * 384;
  GAS u32* const tmo = (GAS u32*)(myws + TMO_OFF);

  if (wid == 8) {
    __builtin_amdgcn_s_setprio(3);
    STAMP(0);
    const u32 seq = (u32)__builtin_amdgcn_readfirstlane((int)__hip_atomic_load(cnt, RLX_AGENT)) + 1u;
    bool failed = false;
    int xn = 0, cw = 0;
#define KBAR(t) do { BARRIER(); SCHED(); } while (0)
    f32x4 wv[8];
#pragma unroll
    for (int j = 0; j < 8; ++j) wv[j] = (f32x4){0.f, 0.f, 0.f, 0.f};
    if (H_SLEEP) __builtin_amdgcn_s_sleep(H_SLEEP);
#pragma nounroll
    for (int t = -2; t < NKS - 1; ++t) {
      const int c = (t >> 2) + 3;
      if ((t & 3) == 0) {
        __builtin_amdgcn_s_waitcnt(0x0f70);
        asm volatile("" ::: "memory");
        if (lane == 0 && c <= NCH) __hip_atomic_store(t ? wfl + (c - 1) * 16 + shr : cnt, seq, RLX_AGENT);
        if (c < NCH) w_load(wv, wfr, wfo, c * 256 + shr * 16);
      } else if ((t & 3) == 2 && t > 0 && c < NCH) {
        w_store(wv, wrs, woff, c);
      }
      SCHED();
      const int s = min(t + 3, NKS - 1), gx = s < X1_END ? s + 1 : 24, gw = t < -1 ? 0 : (s >> 2) + 1;
      if (xn < 24 || cw <= min(gw, NCH - 1)) {
        u32 t0 = 0, spins = 0;
        for (;;) {
          u32x16 f0, f1, fw;
          const int x0 = min(xn, 22), on = xn + cw;
          sflags3(f0, f1, fw, xfl, (u32)x0 * 64u, wfl, (u32)min(cw, NCH - 1) * 64u);
          if (min16(f0) >= seq) xn = max(xn, min16(f1) >= seq ? x0 + 2 : x0 + 1);
          if (min16(fw) >= seq) cw = min(cw + 1, NCH);
          if (xn >= gx && cw >= gw) break;
          if (xn + cw != on) continue;
          if (spins == 0) t0 = (u32)__builtin_amdgcn_s_memrealtime();
          if ((spins & 63u) == 63u) {
            GAS u64* const xf = (GAS u64*)xfl + lane;
            const u32 vw = __hip_atomic_load(wfl + min(cw, NCH - 1) * 16 + (lane & 15), RLX_AGENT);
            const u64 v0 = __hip_atomic_load(xf, RLX_SYSTEM), v1 = __hip_atomic_load(xf + 64, RLX_SYSTEM), v2 = __hip_atomic_load(xf + 128, RLX_SYSTEM);
            if (__all(vw >= seq)) cw = min(cw + 1, NCH);
            int n = lead_ones(both_ge(v0, seq)) >> 3;
            if (n == 8) n += lead_ones(both_ge(v1, seq)) >> 3;
            if (n == 16) n += lead_ones(both_ge(v2, seq)) >> 3;
            xn = max(xn, n);
          }
          ++spins;
          if (give_up(t0, tmo, seq)) { failed = true; xn = 24; cw = NCH; break; }
          if (POLL_GAP) __builtin_amdgcn_s_sleep(POLL_GAP);
        }
      }
      if (t == NKS - 2) {
        ctl_write(ctl + 17 * 4, failed ? 1u : 0u);
        ctl_write(ctl + 18 * 4, seq);
        WAIT_LGKM0();
      }
      ARRV(t == -2 ? -1 : t);
      KBAR(t == -2 ? -1 : t);
    }
    ARRV_OUT();
    return;
  }

  char* const xsrc = srcws + XB_OFF + (size_t)half * (XB_BYTES / 2);
  const __amdgpu_buffer_rsrc_t ars = __builtin_amdgcn_make_buffer_rsrc(xsrc, 0, (int)(XB_BYTES / 2), 0x00020000);
  if (wid == 9) {
    __builtin_amdgcn_s_setprio(3);
    const u32 seq0 = __hip_atomic_load(cnt, RLX_AGENT);
    GAS u32* pub = (GAS u32*)(peer_ws() + XFL_OFF) + (rank * 2 + (b & 1)) * 384 + 256 + (b >> 1);
    u32 hist = 0;
    const u32 seq = (u32)__builtin_amdgcn_readfirstlane((int)seq0) + 1u;
    auto late_flags = [&]() {
      if (lane < 8) __hip_atomic_store(pub, seq, RLX_SYSTEM);
      if (lane == 0) __hip_atomic_store(wfl + 2 * 16 + shr, seq, RLX_AGENT);
      hist <<= 2;
    };
    STAMP(1);
    if (src == rank) {
      BARRIER();
      BARRIER();
      BARRIER();
      late_flags();
      return;
    }
    GAS u32* const tick = (GAS u32*)(myws + INV_OFF) + (__builtin_amdgcn_s_getreg(20 | (3 << 11)) & 15) * 32;
    {
      LAS void* dst = (LAS void*)(lds + LDS_PF);
      const u32 vo = nt * 2048 + lane * 16;
      auto pull = [&](int s) {
        __builtin_amdgcn_raw_ptr_buffer_load_lds(ars, dst, 16, vo, s * A_BYTES, 0, 0);
        __builtin_amdgcn_raw_ptr_buffer_load_lds(ars, dst, 16, vo + 1024, s * A_BYTES, 0, 0);
        hist = hist * 4u + 3u;
      };
      auto out = [&](int s) { return sword(xfl + (s < X1_END ? s * 16 + nt : 256 + ((s - X1_END) * 16 + nt) / 6)) >= seq; };
      if (PF_SLEEP) __builtin_amdgcn_s_sleep(PF_SLEEP);
      {
        const u32 t0 = (u32)__builtin_amdgcn_s_memrealtime();
        for (int s = 0; s < PF0;) {
          if (out(s)) pull(s++);
          else if ((u32)__builtin_amdgcn_s_memrealtime() - t0 > 1000u) break;
          else if (POLL_GAP) __builtin_amdgcn_s_sleep(POLL_GAP);
        }
      }
      bool ok = false;
#pragma nounroll
      for (int k = 1; k < NKS; ++k) {
        if (k >= PF0) {
          if (k == 2 && P2_SLEEP) __builtin_amdgcn_s_sleep(P2_SLEEP);
          if (!ok) ok = out(k);
          const u32 ib = __builtin_amdgcn_s_getreg(7 | (31 << 11));
          const u32 vm = (ib & 15u) | ((ib >> 18) & 48u);
          if (ok && (k <= FREE_K || __builtin_popcount(hist & ((1u << min(vm, 31u)) - 1u)) <= WIN)) pull(k);
        }
        if (k == 4) late_flags();
        ok = out(min(k + 1, NKS - 1));
        ARRV(max(k - 3, -1));
        BARRIER();
      }
      ARRV_OUT();
    }
    u32 old = 0;
    if (lane == 0) old = __hip_atomic_exchange(tick, seq, RLX_SYSTEM);
    SCHED();
    BARRIER();
    BARRIER();
    WAIT_VM(0);
    BARRIER();
    if ((u32)__builtin_amdgcn_readfirstlane((int)old) == seq) return;
    {
      GAS u64* const g = (GAS u64*)(myws + AMX_OFF + rank * 2048) + lane;
      const u32 t0 = (u32)__builtin_amdgcn_s_memrealtime();
      for (;;) {
        bool all = true;
#pragma unroll
        for (int k = 0; k < 4; ++k) all &= (u32)(__hip_atomic_load(g + k * 64, RLX_SYSTEM) >> 32) >= seq;
        if (__all(all)) break;
        if (give_up(t0, tmo, seq)) break;
        __builtin_amdgcn_s_sleep(1);
      }
      WAIT_VM(0);
      asm volatile("buffer_inv sc0 sc1" ::: "memory");
    }
    BARRIER();
    WAIT_VM(0);
    STAMP(15);
    return;
  }

  f32x4 dv[2][8];
  if (wid >= 4) {
    if (DW_SLEEP) __builtin_amdgcn_s_sleep(DW_SLEEP);
#pragma unroll
    for (int d = 0; d < 2; ++d) w_load(dv[d], wfr, wfo, d * 64 + (wid - 4) * 16);
    SCHED();
  }
  f32x4 cv[8];
  const int xl = (b >> 1) * 6 + (wid - 4) * 2;
  u32 seqw = 0;
  if (wid == 2 || wid == 3) {
    const bool skip = wid == 2 && shr < 8;
    if (wid == 3) __builtin_amdgcn_s_sleep(P1_SLEEP);
    if (!skip) w_load(cv, wfr, wfo, (wid - 2) * 256 + shr * 16);
    seqw = __hip_atomic_load(cnt, RLX_AGENT) + 1u;
    SCHED();
    if (!skip) w_store(cv, wrs, woff, wid - 2);
    SCHED();
    if (wid == 2) {
      WAIT_VM(0);
      if (lane == 0) __hip_atomic_store(wfl + shr, seqw, RLX_AGENT);
      STAMP(12);
    }
  } else if (wid == 7) {
    w_load(cv, wfr, wfo, 2 * 256 + shr * 16);
    SCHED();
  } else if (wid >= 4) {
    x_load(cv, px, b & 1, X1_END + (xl >> 4), xl & 15, lane);
    x_load(cv + 4, px, b & 1, X1_END + (xl >> 4), (xl & 15) + 1, lane);
    SCHED();
  }

  const __amdgpu_buffer_rsrc_t brs = __builtin_amdgcn_make_buffer_rsrc(
      myws + WB_OFF + (size_t)nt * W_NT, 0, (int)(WB_BYTES - nt * W_NT), 0x00020000);
  const u32 l16 = lane * 16;
  const u32 avo = wid * 4096 + l16, bvo = wid * 2048 + l16;
  auto piece = [&](int t, u32 slot, int j) {
    if (j >= 4) {
      LAS void* lb = (LAS void*)(lds + slot + A_BYTES + wid * 2048);
      if (j == 4) __builtin_amdgcn_raw_ptr_buffer_load_lds(brs, lb, 16, bvo, t * W_KS, 0, 0);
      else __builtin_amdgcn_raw_ptr_buffer_load_lds(brs, lb, 16, bvo, t * W_KS, 1024, 0);
    } else {
      const u32 ro = ((j + nt) & 3) * 1024;
      LAS void* la = (LAS void*)(lds + slot + wid * 4096 + ro);
      __builtin_amdgcn_raw_ptr_buffer_load_lds(ars, la, 16, avo, t * A_BYTES + ro, 0, 0);
    }
    SCHED();
  };

  const int wr = wid >> 1, wc = wid & 1;
  const u32 aoff = wr * 8192 + l16, boff = A_BYTES + wc * 8192 + l16;
  f32x4 acc[4][4];
#pragma unroll
  for (int i = 0; i < 4; ++i)
#pragma unroll
    for (int j = 0; j < 4; ++j) acc[i][j] = (f32x4){0.f, 0.f, 0.f, 0.f};
  bf16x8 af[2][4], bfr[2][4];
  auto frags = [&](u32 slot, int kh) {
#pragma unroll
    for (int i = 0; i < 4; ++i) {
      bfr[kh][i] = *(const bf16x8*)(lds + slot + boff + i * 2048 + kh * 1024);
      af[kh][i] = *(const bf16x8*)(lds + slot + aoff + i * 2048 + kh * 1024);
    }
    SCHED();
  };
  auto mfma4 = [&](int kh, int mi) {
#pragma unroll
    for (int r = 0; r < 4; ++r)
      acc[mi][r] = __builtin_amdgcn_mfma_f32_16x16x32_bf16(bfr[kh][r], af[kh][mi], acc[mi][r], 0, 0, 0);
    SCHED();
  };

  BARRIER();
#pragma unroll
  for (int j = 0; j < 4; ++j) piece(0, 0, j);
#pragma unroll
  for (int j = 0; j < 4; ++j) piece(1, STAGE, j);
  if (wid >= 4) {
    char* const dst = lds + A_BYTES + wH * 8192 + ((wid >> 1) & 1) * 1024 + (((wid & 1) * 2 + kq1) * 16 + wi) * 16;
#pragma unroll
    for (int d = 0; d < 2; ++d)
#pragma unroll
      for (int r = 0; r < 4; ++r) *(u32x4*)(dst + d * STAGE + r * 2048) = w_pack(dv[d], r);
    SCHED();
    if (wid == 7) {
      w_store(cv, wrs, woff, 2);
    } else {
      x_store(cv, xrs, b & 1, X1_END + (xl >> 4), xl & 15, lane);
      x_store(cv + 4, xrs, b & 1, X1_END + (xl >> 4), (xl & 15) + 1, lane);
    }
    SCHED();
    WAIT_VM_LGKM(8);
  } else if (wid == 3) {
    WAIT_VM(8);
    if (lane == 0) __hip_atomic_store(wfl + 16 + shr, seqw, RLX_AGENT);
    STAMP(13);
    WAIT_VM(5);
  } else {
    WAIT_VM_LGKM(4);
  }
  ARRV(-1);
  BARRIER();
  frags(0, 0);
#pragma unroll
  for (int j = 0; j < 4 - BAR_POS; ++j) piece(2, 2 * STAGE, j);

  u32 cur = 0, nx1 = STAGE, nx2 = 2 * STAGE;
  auto step = [&](int t, auto more) {
    constexpr int MORE = decltype(more)::value;
    frags(cur, 1);
#pragma unroll
    for (int g = 0; g < 4; ++g) {
      mfma4(0, g);
      if (MORE >= 2 && 4 - BAR_POS + g < 6) piece(t + 2, nx2, 4 - BAR_POS + g);
    }
    constexpr int BP = MORE == 1 ? 4 : BAR_POS;
#pragma unroll
    for (int g = 0; g < BP; ++g) mfma4(1, g);
    if (MORE >= 1) {
      if (MORE >= 2) WAIT_VM_LGKM(6);
      else WAIT_VM_LGKM(0);
      ARRV(t);
      BARRIER();
      SCHED();
      frags(nx1, 0);
    }
#pragma unroll
    for (int g = BP; g < 4; ++g) {
      mfma4(1, g);
      if (MORE >= 3) piece(t + 3, cur, g - BP);
    }
    const u32 o = cur;
    cur = nx1; nx1 = nx2; nx2 = o;
  };
  for (int t = 0; t < NKS - 3; ++t) step(t, std::integral_constant<int, 3>());
  step(NKS - 3, std::integral_constant<int, 2>());
  step(NKS - 2, std::integral_constant<int, 1>());
  step(NKS - 1, std::integral_constant<int, 0>());
  ARRV_OUT();

  float mx = 0.f;
#pragma unroll
  for (int mi = 0; mi < 4; ++mi)
#pragma unroll
    for (int r = 0; r < 4; ++r)
#pragma unroll
      for (int j = 0; j < 4; ++j) mx = fmaxf(mx, fabsf(acc[mi][r][j]));
  const u32 wm = wave_max_u32(__builtin_bit_cast(u32, mx));
  ctl_write(ctl + (24 + wid) * 4, wm);
  const u32 fs = ctl_read(ctl + (17 + (lane & 1)) * 4);
  BARRIER();
  const __amdgpu_buffer_rsrc_t ors = __builtin_amdgcn_make_buffer_rsrc(pout, 0, 4096 * NL, 0x00020000);
  const u32 oo = (u32)((mt * 256 + wr * 64 + (lane & 15)) * NL + nt * 128 + wc * 64 + (lane >> 4) * 16);
  const u32 seq = (u32)__builtin_amdgcn_readlane((int)fs, 1);
  {
    u32 m = wave_max_u32(ctl_read(ctl + (24 + (lane & 7)) * 4));
    if (__builtin_amdgcn_readlane((int)fs, 0)) m = 0x7fffffffu;
    u32 tm = 0;
    if (wid == 0) tm = __hip_atomic_load(tmo, RLX_AGENT);
    if (lane == 0) __hip_atomic_store((GAS u64*)(widws + AMX_OFF) + rank * 256 + b, ((u64)seq << 32) | m, RLX_SYSTEM);
    const __amdgpu_buffer_rsrc_t mrs = __builtin_amdgcn_make_buffer_rsrc(myws + AMX_OFF, 0, NR * 2048, 0x00020000);
    const u32 t0 = (u32)__builtin_amdgcn_s_memrealtime();
    for (;;) {
      const u32x4 g0 = __builtin_amdgcn_raw_buffer_load_b128(mrs, l16, wid * 2048, 17);
      const u32x4 g1 = __builtin_amdgcn_raw_buffer_load_b128(mrs, l16 + 1024, wid * 2048, 17);
      m = max(max(g0.x, g0.z), max(g1.x, g1.z));
      if (__all(min(min(g0.y, g0.w), min(g1.y, g1.w)) >= seq)) break;
      if (give_up(t0, tmo, seq)) { m = 0x7fffffffu; break; }
      __builtin_amdgcn_s_sleep(XP_SLEEP + 1);
    }
    if (tm == seq) m = 0x7fffffffu;
    ctl_write(ctl + (32 + wid) * 4, wave_max_u32(m));
    WAIT_LGKM0();
  }
  BARRIER();
  const u32 gmax = wave_max_u32(ctl_read(ctl + (32 + (lane & 7)) * 4));
  const bool bad = gmax > 0x7f800000u;
  const float amax = __builtin_bit_cast(float, gmax);
  const float inv = amax > 0.f ? 448.0f / amax : 0.f;
#pragma unroll
  for (int mi = 0; mi < 4; ++mi) {
    u32x4 d;
#pragma unroll
    for (int j = 0; j < 4; ++j) {
      int q = __builtin_amdgcn_cvt_pk_fp8_f32(acc[mi][0][j] * inv, acc[mi][1][j] * inv, 0, false);
      q = __builtin_amdgcn_cvt_pk_fp8_f32(acc[mi][2][j] * inv, acc[mi][3][j] * inv, q, true);
      d[j] = bad ? 0x7f7f7f7fu : (u32)q;
    }
    __builtin_amdgcn_raw_buffer_store_b128(d, ors, oo + mi * 16 * NL, 0, 16  );
  }
  if (bad) asm volatile("buffer_inv sc0 sc1" ::: "memory");
}

}

extern "C" void kernel_launch(int rank, int nranks, ncclComm_t comm, hipStream_t stream,
                              void* const* d_in, const int* in_sizes, int n_in,
                              void* d_out, int out_size,
                              const int* in_dtypes, int out_dtype,
                              void* const* d_out_all, void* const* d_ws_all, size_t ws_bytes) {
  (void)comm; (void)in_dtypes; (void)out_dtype; (void)d_out_all;
  if (nranks != NR || n_in < 2 || ws_bytes < WS_NEED || in_sizes[0] != 512 * KD || in_sizes[1] != KD * NL ||
      out_size != 4096 * NL || (unsigned)rank >= (unsigned)nranks) {
    fprintf(stderr, "unexpected problem geometry\n");
    return;
  }
  hipError_t e = hipFuncSetAttribute((const void*)ag_gemm_q8, hipFuncAttributeMaxDynamicSharedMemorySize, (int)LDS_BYTES);
  if (e != hipSuccess) { fprintf(stderr, "hipFuncSetAttribute: %s\n", hipGetErrorString(e)); return; }
  Params p;
  p.x = (const float*)d_in[0];
  p.w = (const float*)d_in[1];
  p.out = (unsigned char*)d_out;
  p.myws = (char*)d_ws_all[rank];
  for (int i = 0; i < NR; ++i) p.ws[i] = (char*)d_ws_all[i];
  p.rank = rank;
  p.perm = 0;
  for (int i = 0, lo = 0, hi = 4; i < NR; ++i) {
    const bool same = ((0x4d >> i) & 1) == ((0x4d >> rank) & 1);
    const int xc = PERM ? (same ? lo++ : hi++) : i;
    p.sws[xc] = p.ws[i];
    p.perm |= (u32)i << (4 * xc);
  }
  hipLaunchKernelGGL(ag_gemm_q8, dim3(NWG), dim3(NTHREADS), LDS_BYTES, stream, p);
  e = hipGetLastError();
  if (e != hipSuccess) fprintf(stderr, "launch: %s\n", hipGetErrorString(e));
}
```

```cpp
#include <hip/hip_runtime.h>
#include <rccl/rccl.h>
#include <cstdio>
#include <type_traits>

namespace {

typedef unsigned int u32;
typedef unsigned long long u64;
typedef __bf16 bf16x8 __attribute__((ext_vector_type(8)));
typedef __bf16 bf16x2 __attribute__((ext_vector_type(2)));
typedef float f32x4 __attribute__((ext_vector_type(4)));
typedef float f32x2 __attribute__((ext_vector_type(2)));
typedef u32 u32x2 __attribute__((ext_vector_type(2)));
typedef u32 u32x4 __attribute__((ext_vector_type(4)));
typedef u32 u32x16 __attribute__((ext_vector_type(16)));
#define GAS __attribute__((address_space(1)))
#define LAS __attribute__((address_space(3)))

constexpr int NR = 8;
constexpr int KD = 4096;
constexpr int NL = 2048;
constexpr int NKS = 64;
constexpr int NCH = 16;
constexpr int NWG = 256;
constexpr int NGW = 240;
#define DED 1
#define NPW 2
#define PWIN 3
#define PUSH 2
#define D_SLEEP 40
#define X1_SLEEP 8
#define P1_SLEEP 16
#define DW_SLEEP 45
#define H_SLEEP 0
#define POLL_GAP 2
#define C_SLEEP 12
#define E_AUX 0
#define PU_SLEEP 30
#define XP_SLEEP 2
#define PERM 1
constexpr int BAR_POS = 2;
constexpr u32 W_KS = 16 * 16384, W_NT = 16384;
constexpr int NTHREADS = 576;
constexpr int X0_END = 8;
constexpr int X1_END = 16;
constexpr u32 A_BYTES = 32768, E_POS = 49152, STAGE = 51200;
constexpr u32 LDS_CTL = 3 * STAGE;
constexpr u32 LDS_BYTES = LDS_CTL + 256;
constexpr u32 NOWHERE = 0x80000000u;

constexpr size_t XB_OFF = 0;
constexpr size_t XB_BYTES = 4ull << 20;
constexpr size_t WB_OFF = 4ull << 20;
constexpr size_t WB_BYTES = 16ull << 20;
constexpr size_t CT_OFF = 20ull << 20;
constexpr size_t CNT_OFF = CT_OFF;
constexpr size_t TMO_OFF = CT_OFF + 4096;
constexpr size_t INV_OFF = CT_OFF + 5120;
constexpr size_t XFL_OFF = CT_OFF + 8192;
constexpr size_t WFL_OFF = CT_OFF + 40960;
constexpr size_t DBG_OFF = CT_OFF + 65536;
constexpr size_t AMX_OFF = CT_OFF + 196608;
constexpr size_t WS_NEED = CT_OFF + 262144;
constexpr size_t XP_OFF = 28ull << 20;
constexpr size_t XP_SRC = 262144;
constexpr size_t WS_END = XP_OFF + NR * XP_SRC;

constexpr u64 DEADLINE_TICKS = 100000000ull;

struct Params {
  const float* x;
  const float* w;
  unsigned char* out;
  char* myws;
  char* ws[NR];
  char* sws[NR];
  u32 perm;
  int rank;
};

#define WAIT_VM(n) asm volatile("s_waitcnt vmcnt(%0)" ::"n"(n) : "memory")
#define WAIT_VM_LGKM(n) asm volatile("s_waitcnt vmcnt(" #n ") lgkmcnt(0)" ::: "memory")
#define WAIT_LGKM0() asm volatile("s_waitcnt lgkmcnt(0)" ::: "memory")
#define BARRIER() do { asm volatile("" ::: "memory"); __builtin_amdgcn_s_barrier(); asm volatile("" ::: "memory"); } while (0)
#define SCHED() __builtin_amdgcn_sched_barrier(0)
#define RLX_AGENT __ATOMIC_RELAXED, __HIP_MEMORY_SCOPE_AGENT
#define RLX_SYSTEM __ATOMIC_RELAXED, __HIP_MEMORY_SCOPE_SYSTEM

#define ARRV(t) do { } while (0)
#define ARRV_OUT() do { } while (0)
#define STAMP(i) do { } while (0)

__device__ __forceinline__ u32 pack_bf16(float a, float b) {
  f32x2 f = {a, b};
  bf16x2 h = __builtin_convertvector(f, bf16x2);
  return __builtin_bit_cast(u32, h);
}

__device__ __forceinline__ u32 wave_max_u32(u32 v) {
  v = max(v, (u32)__builtin_amdgcn_update_dpp(0, (int)v, 0x111, 0xf, 0xf, false));
  v = max(v, (u32)__builtin_amdgcn_update_dpp(0, (int)v, 0x112, 0xf, 0xf, false));
  v = max(v, (u32)__builtin_amdgcn_update_dpp(0, (int)v, 0x114, 0xf, 0xf, false));
  v = max(v, (u32)__builtin_amdgcn_update_dpp(0, (int)v, 0x118, 0xf, 0xf, false));
  const u32 a = (u32)__builtin_amdgcn_readlane((int)v, 15), b = (u32)__builtin_amdgcn_readlane((int)v, 31);
  const u32 c = (u32)__builtin_amdgcn_readlane((int)v, 47), d = (u32)__builtin_amdgcn_readlane((int)v, 63);
  return max(max(a, b), max(c, d));
}

__device__ __forceinline__ u32 sword(const GAS u32* p) {
  u32 v;
  asm volatile("s_load_dword %0, %1, 0x0 glc\n\ts_waitcnt lgkmcnt(0)" : "=&s"(v) : "s"(p) : "memory");
  return v;
}
__device__ __forceinline__ void sflags3(u32x16& v, u32x16& w, u32x16& z, const GAS u32* p, u32 off, const GAS u32* q, u32 off2) {
  asm volatile("s_load_dwordx16 %0, %3, %4 glc\n\ts_load_dwordx16 %1, %3, %4 offset:0x40 glc\n\ts_load_dwordx16 %2, %5, %6 glc\n\ts_waitcnt lgkmcnt(0)"
               : "=&s"(v), "=&s"(w), "=&s"(z)
               : "s"(p), "s"(__builtin_amdgcn_readfirstlane((int)off)), "s"(q), "s"(__builtin_amdgcn_readfirstlane((int)off2)) : "memory");
}
__device__ __forceinline__ void sflags2q(u32x16& v, u32x16& w, u32x4& z, const GAS u32* p, u32 off, u32 off2) {
  asm volatile("s_load_dwordx16 %0, %3, %4 glc\n\ts_load_dwordx16 %1, %3, %4 offset:0x40 glc\n\ts_load_dwordx4 %2, %3, %5 glc\n\ts_waitcnt lgkmcnt(0)"
               : "=&s"(v), "=&s"(w), "=&s"(z)
               : "s"(p), "s"(__builtin_amdgcn_readfirstlane((int)off)), "s"(__builtin_amdgcn_readfirstlane((int)off2)) : "memory");
}
__device__ __forceinline__ u32 min16(const u32x16& v) {
  u32 m, n;
  asm("s_min_u32 %0, %2, %3\n\ts_min_u32 %1, %4, %5\n\ts_min_u32 %0, %0, %6\n\ts_min_u32 %1, %1, %7\n\t"
      "s_min_u32 %0, %0, %8\n\ts_min_u32 %1, %1, %9\n\ts_min_u32 %0, %0, %10\n\ts_min_u32 %1, %1, %11\n\t"
      "s_min_u32 %0, %0, %12\n\ts_min_u32 %1, %1, %13\n\ts_min_u32 %0, %0, %14\n\ts_min_u32 %1, %1, %15\n\t"
      "s_min_u32 %0, %0, %16\n\ts_min_u32 %1, %1, %17\n\ts_min_u32 %0, %0, %1"
      : "=&s"(m), "=&s"(n)
      : "s"(v[0]), "s"(v[1]), "s"(v[2]), "s"(v[3]), "s"(v[4]), "s"(v[5]), "s"(v[6]), "s"(v[7]), "s"(v[8]), "s"(v[9]), "s"(v[10]),
        "s"(v[11]), "s"(v[12]), "s"(v[13]), "s"(v[14]), "s"(v[15])
      : "scc");
  return m;
}

__device__ __forceinline__ int x_known(int xn, const u32x16& v, const u32x16& w, u32 seq) {
  const int x0 = min(xn, X0_END - 1);
  if (xn < X0_END && min16(v) >= seq) xn = (x0 < X0_END - 1 && min16(w) >= seq) ? x0 + 2 : x0 + 1;
  if (xn >= X0_END && x0 == X0_END - 1 && w[0] >= seq) xn = w[1] >= seq ? NKS : X1_END;
  return xn;
}

__device__ __forceinline__ bool give_up(u32 t0, GAS u32* tmo, u32 seq) {
  const u32 dt = (u32)__builtin_amdgcn_s_memrealtime() - t0;
  if (__builtin_expect(dt < 100000u, 1)) return false;
  __builtin_amdgcn_s_sleep(64);
  const bool stop = dt > (u32)DEADLINE_TICKS || (u32)__builtin_amdgcn_readfirstlane((int)__hip_atomic_load(tmo, RLX_AGENT)) == seq;
  if (stop) __hip_atomic_store(tmo, seq, RLX_AGENT);
  return stop;
}

__device__ __forceinline__ void w_load(f32x4* v, __amdgpu_buffer_rsrc_t rs, u32 off, int row) {
#pragma unroll
  for (int j = 0; j < 8; ++j)
    v[j] = __builtin_bit_cast(f32x4, __builtin_amdgcn_raw_buffer_load_b128(rs, off, (row + j) * (NL * 4), 0));
}

__device__ __forceinline__ u32x4 w_pack(const f32x4* v, int r) {
  u32x4 d;
  d.x = pack_bf16(v[0][r], v[1][r]);
  d.y = pack_bf16(v[2][r], v[3][r]);
  d.z = pack_bf16(v[4][r], v[5][r]);
  d.w = pack_bf16(v[6][r], v[7][r]);
  return d;
}

__device__ __forceinline__ void w_store(const f32x4* v, __amdgpu_buffer_rsrc_t rs, u32 woff, int chunk) {
#pragma unroll
  for (int r = 0; r < 4; ++r) __builtin_amdgcn_raw_buffer_store_b128(w_pack(v, r), rs, woff + r * 2048, chunk * (4 * W_KS), 16  );
}

__device__ __forceinline__ f32x4 e_load(__amdgpu_buffer_rsrc_t rs, u32 off, int row) {
  f32x4 v;
#pragma unroll
  for (int j = 0; j < 4; ++j) v[j] = __builtin_bit_cast(float, __builtin_amdgcn_raw_buffer_load_b32(rs, off, (row + j) * (NL * 4), 0));
  return v;
}
__device__ __forceinline__ u32x2 e_pack(const f32x4& v) { return (u32x2){pack_bf16(v[0], v[1]), pack_bf16(v[2], v[3])}; }
__device__ __forceinline__ void e_store(const f32x4& v, __amdgpu_buffer_rsrc_t rs, u32 off, int chunk) {
  __builtin_amdgcn_raw_buffer_store_b64(e_pack(v), rs, off, chunk * (4 * W_KS), 16  );
}
__device__ __forceinline__ void ticket(u32& old, GAS u32* p, u32 v) {
  asm volatile("global_atomic_swap %0, %1, %2, off sc0 sc1" : "+v"(old) : "v"(p), "v"(v) : "memory");
}

__device__ __forceinline__ void ctl_write(u32 addr, u32 v) {
  asm volatile("ds_write_b32 %0, %1" ::"v"(addr), "v"(v) : "memory");
}
__device__ __forceinline__ u32 ctl_read(u32 addr) {
  u32 v;
  asm volatile("ds_read_b32 %0, %1\n\ts_waitcnt lgkmcnt(0)" : "=v"(v) : "v"(addr) : "memory");
  return v;
}
__device__ __forceinline__ void x_load(f32x4* v, const float* __restrict__ x, int xh, int xks, int msub, int lane) {
  const float* xs = x + (size_t)(xh * 256 + msub * 16 + (lane & 15)) * KD + xks * 64 + (lane >> 4) * 8;
#pragma unroll
  for (int j = 0; j < 4; ++j) v[j] = *(const f32x4*)(xs + (j >> 1) * 32 + (j & 1) * 4);
}

__device__ __forceinline__ u32x4 x_pack(const f32x4* v, int kh) {
  u32x4 d;
  d.x = pack_bf16(v[kh * 2][0], v[kh * 2][1]);
  d.y = pack_bf16(v[kh * 2][2], v[kh * 2][3]);
  d.z = pack_bf16(v[kh * 2 + 1][0], v[kh * 2 + 1][1]);
  d.w = pack_bf16(v[kh * 2 + 1][2], v[kh * 2 + 1][3]);
  return d;
}

__device__ __forceinline__ void x_store(const f32x4* v, __amdgpu_buffer_rsrc_t rs, int xh, int xks, int msub, int lane) {
  const u32 xoff = (u32)(((xh * 64 + xks) * 16 + msub) * 2048 + lane * 16);
#pragma unroll
  for (int kh = 0; kh < 2; ++kh) __builtin_amdgcn_raw_buffer_store_b128(x_pack(v, kh), rs, xoff + kh * 1024, 0, 17  );
}

__device__ __forceinline__ u64 both_ge(u64 f, u32 seq) { return __ballot((u32)f >= seq && (u32)(f >> 32) >= seq); }
__device__ __forceinline__ int lead_ones(u64 m) { return m == ~0ull ? 64 : __builtin_ctzll(~m); }

__global__ void __launch_bounds__(NTHREADS) __attribute__((amdgpu_waves_per_eu(3, 3))) ag_gemm_q8(Params p) {
  extern __shared__ __attribute__((aligned(16))) char lds[];
  const int b = blockIdx.x;
  const int tid = threadIdx.x, lane = tid & 63;
  const int wid = __builtin_amdgcn_readfirstlane(tid >> 6);
  const int idx = b >> 3;
  const int half = idx & 1, nt = idx >> 1, shr = (b & 7) * 2 + half;
  const float* const px = p.x;
  const float* const pw = p.w;
  unsigned char* const pout = p.out;
  char* const myws = p.myws;
  char* const srcws = p.sws[b & 7];
  char* const widws = p.ws[wid & 7];
  char* const dst0 = p.ws[PUSH == 2 ? (b >> 6) * 2 : b >> 5];
  char* const dst1 = p.ws[(b >> 6) * 2 + 1];
  const int rank = p.rank;
  const u32 perm = p.perm;
  const u64 w0 = (u64)p.ws[0], w1 = (u64)p.ws[1], w2 = (u64)p.ws[2], w3 = (u64)p.ws[3];
  const u64 w4 = (u64)p.ws[4], w5 = (u64)p.ws[5], w6 = (u64)p.ws[6], w7 = (u64)p.ws[7];
  asm volatile("" ::"s"(px), "s"(pw), "s"(pout), "s"(myws), "s"(srcws), "s"(widws), "s"(dst0), "s"(dst1), "s"(rank), "s"(perm), "s"(w0), "s"(w1), "s"(w2), "s"(w3),
               "s"(w4), "s"(w5), "s"(w6), "s"(w7));
  const int src = (perm >> (4 * (b & 7))) & 7, mt = src * 2 + half;
  auto peer_ws = [&]() {
    int lo = (int)w0, hi = (int)(w0 >> 32);
#define PUT(i) asm("v_writelane_b32 %0, %2, " #i "\n\tv_writelane_b32 %1, %3, " #i : "+v"(lo), "+v"(hi) : "s"((int)w##i), "s"((int)(w##i >> 32)))
    PUT(1); PUT(2); PUT(3); PUT(4); PUT(5); PUT(6); PUT(7);
#undef PUT
    return (char*)(((u64)(u32)hi << 32) | (u32)lo);
  };
  GAS u32* const cnt = (GAS u32*)(myws + CNT_OFF) + b;
  GAS u32* const ownfl = (GAS u32*)(myws + XFL_OFF) + rank * 1024;
  const __amdgpu_buffer_rsrc_t xrs = __builtin_amdgcn_make_buffer_rsrc(myws + XB_OFF, 0, (int)XB_BYTES, 0x00020000);

  if (wid < 2 && (wid || (b >> 5) >= PUSH)) {
    f32x4 v[4];
    const int xh = (b >> 4) & 1, xks = wid * X0_END + (b >> 5), msub = b & 15;
    if (wid) __builtin_amdgcn_s_sleep(X1_SLEEP);
    x_load(v, px, xh, xks, msub, lane);
    const u32 seq = __hip_atomic_load(cnt, RLX_AGENT);
    SCHED();
    GAS u32* pub = wid ? ownfl + xh * 512 + 128 + xks * 16 + msub : (GAS u32*)(peer_ws() + XFL_OFF) + (rank * 2 + xh) * 512 + xks * 16 + msub;
    asm volatile("" : "+v"(pub));
    SCHED();
    x_store(v, xrs, xh, xks, msub, lane);
    SCHED();
    WAIT_VM(0);
    if (lane < (wid ? 1 : 8)) __hip_atomic_store(pub, seq + 1u, RLX_SYSTEM);
  }
  u32 seq8 = 0;
  if (wid == 8) {
    f32x4 v[4];
    const int blk = b & (32 * PUSH - 1), xh = (blk >> 4) & 1, xks = blk >> 5, msub = blk & 15;
    STAMP(0);
    if (PUSH) x_load(v, px, xh, xks, msub, lane);
    seq8 = __hip_atomic_load(cnt, RLX_AGENT) + 1u;
    SCHED();
    if (PUSH) {
      const u32 off = (u32)(XP_OFF + rank * XP_SRC) + (u32)(xh * (XP_SRC / 2) + xks * A_BYTES + msub * 2048) + lane * 16;
      const u32 flo = lane == 0 ? (u32)XFL_OFF + (u32)((rank * 2 + xh) * 512 + xks * 16 + msub) * 4u : NOWHERE;
      const __amdgpu_buffer_rsrc_t r0 = __builtin_amdgcn_make_buffer_rsrc(dst0, 0, (int)WS_END, 0x00020000);
      const __amdgpu_buffer_rsrc_t r1 = __builtin_amdgcn_make_buffer_rsrc(dst1, 0, (int)WS_END, 0x00020000);
      const u32x4 d0 = x_pack(v, 0), d1 = x_pack(v, 1);
      if (PU_SLEEP && (xks & 1)) __builtin_amdgcn_s_sleep(PU_SLEEP);
      __builtin_amdgcn_raw_buffer_store_b128(d0, r0, off, 0, 17);
      __builtin_amdgcn_raw_buffer_store_b128(d1, r0, off + 1024, 0, 17);
      if (PUSH > 1) {
        __builtin_amdgcn_raw_buffer_store_b128(d0, r1, off, 0, 17);
        __builtin_amdgcn_raw_buffer_store_b128(d1, r1, off + 1024, 0, 17);
        WAIT_VM(2);
      } else {
        WAIT_VM(0);
      }
      __builtin_amdgcn_raw_buffer_store_b32(seq8, r0, flo, 0, 17);
      if (PUSH > 1) {
        WAIT_VM(1);
        __builtin_amdgcn_raw_buffer_store_b32(seq8, r1, flo, 0, 17);
      }
      STAMP(1);
    }
  }

  GAS u32* const xfl = (GAS u32*)(myws + XFL_OFF) + mt * 512;
  GAS u32* const tmo = (GAS u32*)(myws + TMO_OFF);
  GAS u32* const tick = (GAS u32*)(myws + INV_OFF) + (__builtin_amdgcn_s_getreg(20 | (3 << 11)) & 15) * 32;
  auto clean = [&](u32 seq) {
    GAS u64* const g = (GAS u64*)(myws + AMX_OFF + rank * 2048) + lane;
    const u32 t0 = (u32)__builtin_amdgcn_s_memrealtime();
    for (;;) {
      bool all = true;
#pragma unroll
      for (int k = 0; k < 4; ++k) all &= (u32)(__hip_atomic_load(g + k * 64, RLX_SYSTEM) >> 32) >= seq;
      if (__all(all)) break;
      if (give_up(t0, tmo, seq)) break;
      __builtin_amdgcn_s_sleep(C_SLEEP);
    }
    WAIT_VM(0);
    asm volatile("buffer_inv sc0 sc1" ::: "memory");
    WAIT_VM(0);
  };
  char* const xsrc = srcws + XB_OFF + (size_t)half * (XB_BYTES / 2);
  const __amdgpu_buffer_rsrc_t ars = __builtin_amdgcn_make_buffer_rsrc(xsrc, 0, (int)(XB_BYTES / 2), 0x00020000);

  if (nt == 15) {
    if (wid < 2) return;
    if (wid >= 2 + NPW) {
      if (wid < 5 || wid == 8) return;
      f32x4 cv[8];
      const int xl = (b >> 1) * 6 + (wid - 5) * 2;
      x_load(cv, px, b & 1, X1_END + (xl >> 4), xl & 15, lane);
      x_load(cv + 4, px, b & 1, X1_END + (xl >> 4), (xl & 15) + 1, lane);
      SCHED();
      x_store(cv, xrs, b & 1, X1_END + (xl >> 4), xl & 15, lane);
      x_store(cv + 4, xrs, b & 1, X1_END + (xl >> 4), (xl & 15) + 1, lane);
      WAIT_VM(0);
      return;
    }
    __builtin_amdgcn_s_setprio(3);
    const u32 seq = (u32)__builtin_amdgcn_readfirstlane((int)__hip_atomic_load(cnt, RLX_AGENT)) + 1u;
    const int pv = wid - 2;
    u32 old = seq;
    bool failed = false, passed = false;
    auto late_flag = [&]() {
      if (pv == 0 && lane == 0) {
        __hip_atomic_store(ownfl + (b & 1) * 512 + 384 + (b >> 1), seq, RLX_SYSTEM);
        __hip_atomic_store(cnt, seq, RLX_AGENT);
        ticket(old, tick, seq);
      }
    };
    if (src != rank) {
      LAS void* dst = (LAS void*)(lds + pv * 256);
      const u32 vo = lane * 128;
      if (D_SLEEP) __builtin_amdgcn_s_sleep(D_SLEEP);
      const u32 t0 = (u32)__builtin_amdgcn_s_memrealtime();
      int xn = PUSH;
#pragma nounroll
      for (int u = PUSH * 4 + pv; u < NKS * 4; u += NPW) {
        const int s = u >> 2;
        if (xn < NKS && (xn <= s + 2 || (u & 3) < NPW)) {
          for (;;) {
            u32x16 f0, f1;
            u32x4 fq;
            sflags2q(f0, f1, fq, xfl, (u32)min(xn, X0_END - 1) * 64u, (u32)min(u, X0_END * 4 - 1) * 16u);
            xn = x_known(xn, f0, f1, seq);
            if (xn > s || (s < X0_END && min(min(fq.x, fq.y), min(fq.z, fq.w)) >= seq)) break;
            if (give_up(t0, tmo, seq)) { failed = true; break; }
            if (POLL_GAP) __builtin_amdgcn_s_sleep(POLL_GAP);
          }
          if (failed) break;
        }
        WAIT_VM(PWIN - 1);
        __builtin_amdgcn_raw_ptr_buffer_load_lds(ars, dst, 4, vo, u * 8192, 0, 0);
        if (u - pv == 16) {
          BARRIER();
          late_flag();
          passed = true;
        }
      }
      if (!passed) {
        BARRIER();
        late_flag();
      }
      WAIT_VM(0);
      if (failed) asm volatile("buffer_inv sc0 sc1" ::: "memory");
    } else {
      BARRIER();
      late_flag();
      if (pv == 0) {
        GAS u32* const pub = (GAS u32*)(p.ws[lane & 7] + XFL_OFF) + (rank * 2 + half) * 512 + 128;
        const u32 t0 = (u32)__builtin_amdgcn_s_memrealtime();
        for (int ph = 0; ph < 2;) {
          if (both_ge(__hip_atomic_load((GAS u64*)(ownfl + half * 512 + 256 + ph * 128) + lane, RLX_SYSTEM), seq) == ~0ull) {
            if (lane < 8) __hip_atomic_store(pub + ph, seq, RLX_SYSTEM);
            ++ph;
            continue;
          }
          if (give_up(t0, tmo, seq)) break;
          __builtin_amdgcn_s_sleep(4);
        }
      }
    }
    BARRIER();
    if (pv != 0) return;
    if (lane == 0) __hip_atomic_store((GAS u64*)(myws + AMX_OFF) + rank * 256 + b, (u64)seq << 32, RLX_SYSTEM);
    WAIT_VM(0);
    asm volatile("" : "+v"(old));
    if ((u32)__builtin_amdgcn_readfirstlane((int)old) != seq) clean(seq);
    STAMP(15);
    return;
  }

  const u32 ctl = (u32)(size_t)(LAS char*)lds + LDS_CTL;
  const int kq1 = lane >> 5, wH = (lane >> 4) & 1, wi = lane & 15;
  const __amdgpu_buffer_rsrc_t wfr = __builtin_amdgcn_make_buffer_rsrc((void*)pw, 0, KD * NL * 4, 0x00020000);
  const u32 wfo = (u32)(kq1 * 8 * NL + nt * 128 + wH * 64 + wi * 4) * 4u;
  const u32 woff = (u32)(nt * W_NT + (shr >> 2) * W_KS + wH * 8192 + ((shr >> 1) & 1) * 1024 +
                         (((shr & 1) * 2 + kq1) * 16 + wi) * 16);
  const bool ext = nt < 8;
  const u32 efo = ext ? (u32)((wH * 8 + kq1 * 4) * NL + 15 * 128 + nt * 16 + wi) * 4u : NOWHERE;
  const u32 eoff = ext ? (u32)(15 * W_NT + nt * 2048 + (shr >> 2) * W_KS + ((shr >> 1) & 1) * 1024 +
                               (((shr & 1) * 2 + wH) * 16 + wi) * 16 + kq1 * 8) : NOWHERE;
  const __amdgpu_buffer_rsrc_t wrs =
      __builtin_amdgcn_make_buffer_rsrc(myws + WB_OFF, 0, (int)WB_BYTES, 0x00020000);
  GAS u32* const wfl = (GAS u32*)(myws + WFL_OFF) + nt * 256;

  if (wid == 8) {
    __builtin_amdgcn_s_setprio(3);
    const u32 seq = (u32)__builtin_amdgcn_readfirstlane((int)seq8);
    bool failed = false, cleaner = false;
    int xn = 0, cw = 0;
#define KBAR(t) do { BARRIER(); SCHED(); } while (0)
    f32x4 wv[9];
#pragma unroll
    for (int j = 0; j < 9; ++j) wv[j] = (f32x4){0.f, 0.f, 0.f, 0.f};
    if (H_SLEEP) __builtin_amdgcn_s_sleep(H_SLEEP);
#pragma nounroll
    for (int t = -2; t < NKS - 1; ++t) {
      const int c = (t >> 2) + 3;
      if ((t & 3) == 0) {
        __builtin_amdgcn_s_waitcnt(0x0f70);
        asm volatile("" ::: "memory");
        if (lane == 0 && c <= NCH) __hip_atomic_store(t ? wfl + (c - 1) * 16 + shr : cnt, seq, RLX_AGENT);
        if (c < NCH) {
          w_load(wv, wfr, wfo, c * 256 + shr * 16);
          wv[8] = e_load(wfr, efo, c * 256 + shr * 16);
        }
      } else if ((t & 3) == 2 && t > 0 && c < NCH) {
        w_store(wv, wrs, woff, c);
        e_store(wv[8], wrs, eoff, c);
      }
      if (t == 1 && lane < 2)
        __hip_atomic_store(lane ? ownfl + (b & 1) * 512 + 384 + (b >> 1) : wfl + 2 * 16 + shr, seq, RLX_SYSTEM);
      if (t == 25 + idx && sword(tick) != seq) {
        u32 old = 0;
        if (lane == 0) old = __hip_atomic_exchange(tick, seq, RLX_SYSTEM);
        cleaner = (u32)__builtin_amdgcn_readfirstlane((int)old) != seq;
      }
      SCHED();
      const int s = min(t + 3, NKS - 1), gw = t < -1 ? 0 : (s >> 2) + 1;
      if (xn < NKS || cw <= min(gw, NCH - 1)) {
        u32 t0 = 0, spins = 0;
        for (;;) {
          u32x16 f0, f1, fw;
          const int on = xn + cw;
          sflags3(f0, f1, fw, xfl, (u32)min(xn, X0_END - 1) * 64u, wfl, (u32)min(cw, NCH - 1) * 64u);
          xn = x_known(xn, f0, f1, seq);
          if (min16(fw) >= seq) cw = min(cw + 1, NCH);
          if (xn > s && cw >= gw) break;
          if (xn + cw != on) continue;
          if (spins == 0) t0 = (u32)__builtin_amdgcn_s_memrealtime();
          if ((spins & 63u) == 63u) {
            GAS u64* const xf = (GAS u64*)xfl;
            const u32 vw = __hip_atomic_load(wfl + min(cw, NCH - 1) * 16 + (lane & 15), RLX_AGENT);
            const u64 v0 = __hip_atomic_load(xf + lane, RLX_SYSTEM), v1 = __hip_atomic_load(xf + 64, RLX_SYSTEM);
            if (__all(vw >= seq)) cw = min(cw + 1, NCH);
            int n = lead_ones(both_ge(v0, seq)) >> 3;
            if (n == X0_END && __all((u32)v1 >= seq)) n = __all((u32)(v1 >> 32) >= seq) ? NKS : X1_END;
            xn = max(xn, n);
          }
          ++spins;
          if (give_up(t0, tmo, seq)) { failed = true; xn = NKS; cw = NCH; break; }
          if (POLL_GAP) __builtin_amdgcn_s_sleep(POLL_GAP);
        }
      }
      if (t == NKS - 2) {
        ctl_write(ctl + 17 * 4, failed ? 1u : 0u);
        ctl_write(ctl + 18 * 4, seq);
        WAIT_LGKM0();
      }
      ARRV(t == -2 ? -1 : t);
      KBAR(t == -2 ? -1 : t);
    }
    ARRV_OUT();
    if (cleaner) {
      BARRIER();
      BARRIER();
      clean(seq);
    }
    return;
  }

  f32x4 dv[2][9];
  if (wid >= 4) {
    if (DW_SLEEP) __builtin_amdgcn_s_sleep(DW_SLEEP);
#pragma unroll
    for (int d = 0; d < 2; ++d) {
      w_load(dv[d], wfr, wfo, d * 64 + (wid - 4) * 16);
      dv[d][8] = e_load(wfr, efo, d * 64 + (wid - 4) * 16);
    }
    SCHED();
  }
  f32x4 cv[9];
  const int xl = (b >> 1) * 6 + (wid - 4) * 2;
  u32 seqw = 0;
  if (wid == 2 || wid == 3) {
    const bool skip = wid == 2 && shr < 8;
    if (wid == 3) __builtin_amdgcn_s_sleep(P1_SLEEP);
    if (!skip) {
      w_load(cv, wfr, wfo, (wid - 2) * 256 + shr * 16);
      cv[8] = e_load(wfr, efo, (wid - 2) * 256 + shr * 16);
    }
    seqw = __hip_atomic_load(cnt, RLX_AGENT) + 1u;
    SCHED();
    if (!skip) {
      w_store(cv, wrs, woff, wid - 2);
      e_store(cv[8], wrs, eoff, wid - 2);
    }
    SCHED();
    if (wid == 2) {
      WAIT_VM(0);
      if (lane == 0) __hip_atomic_store(wfl + shr, seqw, RLX_AGENT);
      STAMP(12);
    }
  } else if (wid == 7) {
    w_load(cv, wfr, wfo, 2 * 256 + shr * 16);
    cv[8] = e_load(wfr, efo, 2 * 256 + shr * 16);
    SCHED();
  } else if (wid >= 4) {
    x_load(cv, px, b & 1, X1_END + (xl >> 4), xl & 15, lane);
    x_load(cv + 4, px, b & 1, X1_END + (xl >> 4), (xl & 15) + 1, lane);
    SCHED();
  }

  const __amdgpu_buffer_rsrc_t brs = __builtin_amdgcn_make_buffer_rsrc(
      myws + WB_OFF + (size_t)nt * W_NT, 0, (int)(WB_BYTES - nt * W_NT), 0x00020000);
  const __amdgpu_buffer_rsrc_t prs = __builtin_amdgcn_make_buffer_rsrc(myws + XP_OFF + (size_t)mt * (XP_SRC / 2), 0, (int)(XP_SRC / 2), 0x00020000);
  const u32 l16 = lane * 16;
  const u32 avo = wid * 4096 + l16, bvo = wid * 2048 + l16;
  const u32 evo = ext ? (u32)((15 - nt) * W_NT + nt * 2048 + wid * 256 + lane * 4) : NOWHERE;
  auto piece = [&](int t, u32 slot, int j, bool pushed = false) {
    if (j == 6) {
      __builtin_amdgcn_raw_ptr_buffer_load_lds(brs, (LAS void*)(lds + slot + E_POS + wid * 256), 4, evo, t * W_KS, 0, 0);
    } else if (j >= 4) {
      LAS void* lb = (LAS void*)(lds + slot + A_BYTES + wid * 2048);
      if (j == 4) __builtin_amdgcn_raw_ptr_buffer_load_lds(brs, lb, 16, bvo, t * W_KS, 0, 0);
      else __builtin_amdgcn_raw_ptr_buffer_load_lds(brs, lb, 16, bvo, t * W_KS, 1024, 0);
    } else {
      const u32 ro = ((j + nt) & 3) * 1024;
      LAS void* la = (LAS void*)(lds + slot + wid * 4096 + ro);
      __builtin_amdgcn_raw_ptr_buffer_load_lds(pushed ? prs : ars, la, 16, avo, t * A_BYTES + ro, 0, 0);
    }
    SCHED();
  };

  const int wr = wid >> 1, wc = wid & 1;
  const u32 aoff0 = wr * 8192 + wc * 4096 + l16, aoff1 = wr * 8192 + (1 - wc) * 4096 + l16, boff = A_BYTES + wc * 8192 + l16;
  f32x4 acc[4][4], ace[2];
#pragma unroll
  for (int i = 0; i < 4; ++i)
#pragma unroll
    for (int j = 0; j < 4; ++j) acc[i][j] = (f32x4){0.f, 0.f, 0.f, 0.f};
  ace[0] = ace[1] = (f32x4){0.f, 0.f, 0.f, 0.f};
  bf16x8 af[2][4], bfr[2][4], ef[2];
  auto frags = [&](u32 slot, int kh) {
#pragma unroll
    for (int i = 0; i < 4; ++i) {
      bfr[kh][i] = *(const bf16x8*)(lds + slot + boff + i * 2048 + kh * 1024);
      af[kh][i] = *(const bf16x8*)(lds + slot + (i < 2 ? aoff0 : aoff1) + (i & 1) * 2048 + kh * 1024);
    }
    ef[kh] = *(const bf16x8*)(lds + slot + E_POS + l16 + kh * 1024);
    SCHED();
  };
  auto mfma4 = [&](int kh, int mi) {
#pragma unroll
    for (int r = 0; r < 4; ++r)
      acc[mi][r] = __builtin_amdgcn_mfma_f32_16x16x32_bf16(bfr[kh][r], af[kh][mi], acc[mi][r], 0, 0, 0);
    if (mi >= 2) ace[mi - 2] = __builtin_amdgcn_mfma_f32_16x16x32_bf16(ef[kh], af[kh][mi - 2], ace[mi - 2], 0, 0, 0);
    SCHED();
  };

  BARRIER();
#pragma unroll
  for (int j = 0; j < 4; ++j) piece(0, 0, j, PUSH > 0);
#pragma unroll
  for (int j = 0; j < 4; ++j) piece(1, STAGE, j, PUSH > 1);
  if (wid >= 4) {
    char* const dst = lds + A_BYTES + wH * 8192 + ((wid >> 1) & 1) * 1024 + (((wid & 1) * 2 + kq1) * 16 + wi) * 16;
    char* const dse = lds + E_POS + ((wid >> 1) & 1) * 1024 + (((wid & 1) * 2 + wH) * 16 + wi) * 16 + kq1 * 8;
#pragma unroll
    for (int d = 0; d < 2; ++d) {
#pragma unroll
      for (int r = 0; r < 4; ++r) *(u32x4*)(dst + d * STAGE + r * 2048) = w_pack(dv[d], r);
      *(u32x2*)(dse + d * STAGE) = e_pack(dv[d][8]);
    }
    SCHED();
    if (wid == 7) {
      w_store(cv, wrs, woff, 2);
      e_store(cv[8], wrs, eoff, 2);
      SCHED();
      WAIT_VM_LGKM(9);
    } else {
      x_store(cv, xrs, b & 1, X1_END + (xl >> 4), xl & 15, lane);
      x_store(cv + 4, xrs, b & 1, X1_END + (xl >> 4), (xl & 15) + 1, lane);
      SCHED();
      WAIT_VM_LGKM(8);
    }
  } else if (wid == 3) {
    WAIT_VM(8);
    if (lane == 0) __hip_atomic_store(wfl + 16 + shr, seqw, RLX_AGENT);
    STAMP(13);
    WAIT_VM(5);
  } else {
    WAIT_VM_LGKM(4);
  }
  ARRV(-1);
  BARRIER();
  frags(0, 0);
#pragma unroll
  for (int j = 0; j < 4 - BAR_POS; ++j) piece(2, 2 * STAGE, j, PUSH > 2);

  u32 cur = 0, nx1 = STAGE, nx2 = 2 * STAGE;
  auto step = [&](int t, auto more) {
    constexpr int MORE = decltype(more)::value;
    frags(cur, 1);
    if (MORE >= 2) piece(t + 2, nx2, 6);
#pragma unroll
    for (int g = 0; g < 4; ++g) {
      mfma4(0, g);
      if (MORE >= 2 && 4 - BAR_POS + g < 6) piece(t + 2, nx2, 4 - BAR_POS + g);
    }
    constexpr int BP = MORE == 1 ? 4 : BAR_POS;
#pragma unroll
    for (int g = 0; g < BP; ++g) mfma4(1, g);
    if (MORE >= 1) {
      if (MORE >= 2) WAIT_VM_LGKM(7);
      else WAIT_VM_LGKM(0);
      ARRV(t);
      BARRIER();
      SCHED();
      frags(nx1, 0);
    }
#pragma unroll
    for (int g = BP; g < 4; ++g) {
      mfma4(1, g);
      if (MORE >= 3) piece(t + 3, cur, g - BP);
    }
    const u32 o = cur;
    cur = nx1; nx1 = nx2; nx2 = o;
  };
  static_assert(PUSH <= 2, "the K loop reads the peers");
  for (int t = 0; t < NKS - 3; ++t) step(t, std::integral_constant<int, 3>());
  step(NKS - 3, std::integral_constant<int, 2>());
  step(NKS - 2, std::integral_constant<int, 1>());
  step(NKS - 1, std::integral_constant<int, 0>());
  ARRV_OUT();

  float mx = 0.f;
#pragma unroll
  for (int mi = 0; mi < 4; ++mi)
#pragma unroll
    for (int r = 0; r < 4; ++r)
#pragma unroll
      for (int j = 0; j < 4; ++j) mx = fmaxf(mx, fabsf(acc[mi][r][j]));
  float me = 0.f;
#pragma unroll
  for (int i = 0; i < 8; ++i) me = fmaxf(me, fabsf(ace[i >> 2][i & 3]));
  if (ext) mx = fmaxf(mx, me);
  const u32 wm = wave_max_u32(__builtin_bit_cast(u32, mx));
  ctl_write(ctl + (24 + wid) * 4, wm);
  const u32 fs = ctl_read(ctl + (17 + (lane & 1)) * 4);
  BARRIER();
  const __amdgpu_buffer_rsrc_t ors = __builtin_amdgcn_make_buffer_rsrc(pout, 0, 4096 * NL, 0x00020000);
  const u32 orow = (u32)(mt * 256 + wr * 64 + (lane & 15)) * NL;
  const u32 oo0 = orow + wc * 32 * NL + nt * 128 + wc * 64 + (lane >> 4) * 16, oo1 = oo0 + (1 - 2 * wc) * 32 * NL;
  const u32 oe = ext ? orow + wc * 32 * NL + 15 * 128 + nt * 16 + (lane >> 4) * 4 : NOWHERE;
  const u32 seq = (u32)__builtin_amdgcn_readlane((int)fs, 1);
  {
    u32 m = wave_max_u32(ctl_read(ctl + (24 + (lane & 7)) * 4));
    if (__builtin_amdgcn_readlane((int)fs, 0)) m = 0x7fffffffu;
    u32 tm = 0;
    if (wid == 0) tm = __hip_atomic_load(tmo, RLX_AGENT);
    if (lane == 0) __hip_atomic_store((GAS u64*)(widws + AMX_OFF) + rank * 256 + b, ((u64)seq << 32) | m, RLX_SYSTEM);
    const __amdgpu_buffer_rsrc_t mrs = __builtin_amdgcn_make_buffer_rsrc(myws + AMX_OFF, 0, NR * 2048, 0x00020000);
    const u32 l2 = min(l16 + 1024u, (u32)NGW * 8u - 16u);
    const u32 t0 = (u32)__builtin_amdgcn_s_memrealtime();
    for (;;) {
      const u32x4 g0 = __builtin_amdgcn_raw_buffer_load_b128(mrs, l16, wid * 2048, 17);
      const u32x4 g1 = __builtin_amdgcn_raw_buffer_load_b128(mrs, l2, wid * 2048, 17);
      m = max(max(g0.x, g0.z), max(g1.x, g1.z));
      if (__all(min(min(g0.y, g0.w), min(g1.y, g1.w)) >= seq)) break;
      if (give_up(t0, tmo, seq)) { m = 0x7fffffffu; break; }
      __builtin_amdgcn_s_sleep(XP_SLEEP + 1);
    }
    if (tm == seq) m = 0x7fffffffu;
    ctl_write(ctl + (32 + wid) * 4, wave_max_u32(m));
    WAIT_LGKM0();
  }
  BARRIER();
  const u32 gmax = wave_max_u32(ctl_read(ctl + (32 + (lane & 7)) * 4));
  const bool bad = gmax > 0x7f800000u;
  const float amax = __builtin_bit_cast(float, gmax);
  const float inv = amax > 0.f ? 448.0f / amax : 0.f;
  auto q4 = [&](float a, float c, float d, float e) {
    int q = __builtin_amdgcn_cvt_pk_fp8_f32(a * inv, c * inv, 0, false);
    q = __builtin_amdgcn_cvt_pk_fp8_f32(d * inv, e * inv, q, true);
    return bad ? 0x7f7f7f7fu : (u32)q;
  };
#pragma unroll
  for (int mi = 0; mi < 4; ++mi) {
    u32x4 d;
#pragma unroll
    for (int j = 0; j < 4; ++j) d[j] = q4(acc[mi][0][j], acc[mi][1][j], acc[mi][2][j], acc[mi][3][j]);
    __builtin_amdgcn_raw_buffer_store_b128(d, ors, (mi < 2 ? oo0 : oo1) + (mi & 1) * 16 * NL, 0, 16  );
  }
#pragma unroll
  for (int i = 0; i < 2; ++i) __builtin_amdgcn_raw_buffer_store_b32(q4(ace[i][0], ace[i][1], ace[i][2], ace[i][3]), ors, oe, i * 16 * NL, E_AUX);
  if (bad) asm volatile("buffer_inv sc0 sc1" ::: "memory");
}

}

extern "C" void kernel_launch(int rank, int nranks, ncclComm_t comm, hipStream_t stream,
                              void* const* d_in, const int* in_sizes, int n_in,
                              void* d_out, int out_size,
                              const int* in_dtypes, int out_dtype,
                              void* const* d_out_all, void* const* d_ws_all, size_t ws_bytes) {
  (void)comm; (void)in_dtypes; (void)out_dtype; (void)d_out_all;
  if (nranks != NR || n_in < 2 || ws_bytes < WS_END || in_sizes[0] != 512 * KD || in_sizes[1] != KD * NL ||
      out_size != 4096 * NL || (unsigned)rank >= (unsigned)nranks) {
    fprintf(stderr, "unexpected problem geometry\n");
    return;
  }
  hipError_t e = hipFuncSetAttribute((const void*)ag_gemm_q8, hipFuncAttributeMaxDynamicSharedMemorySize, (int)LDS_BYTES);
  if (e != hipSuccess) { fprintf(stderr, "hipFuncSetAttribute: %s\n", hipGetErrorString(e)); return; }
  Params p;
  p.x = (const float*)d_in[0];
  p.w = (const float*)d_in[1];
  p.out = (unsigned char*)d_out;
  p.myws = (char*)d_ws_all[rank];
  for (int i = 0; i < NR; ++i) p.ws[i] = (char*)d_ws_all[i];
  p.rank = rank;
  p.perm = 0;
  for (int i = 0, lo = 0, hi = 4; i < NR; ++i) {
    const bool same = ((0x4d >> i) & 1) == ((0x4d >> rank) & 1);
    const int xc = PERM ? (same ? lo++ : hi++) : i;
    p.sws[xc] = p.ws[i];
    p.perm |= (u32)i << (4 * xc);
  }
  hipLaunchKernelGGL(ag_gemm_q8, dim3(NWG), dim3(NTHREADS), LDS_BYTES, stream, p);
  e = hipGetLastError();
  if (e != hipSuccess) fprintf(stderr, "launch: %s\n", hipGetErrorString(e));
}
```

```cpp
#include <hip/hip_runtime.h>
#include <rccl/rccl.h>
#include <cstdio>
#include <type_traits>

namespace {

typedef unsigned int u32;
typedef unsigned long long u64;
typedef __bf16 bf16x8 __attribute__((ext_vector_type(8)));
typedef __bf16 bf16x2 __attribute__((ext_vector_type(2)));
typedef float f32x4 __attribute__((ext_vector_type(4)));
typedef float f32x2 __attribute__((ext_vector_type(2)));
typedef u32 u32x2 __attribute__((ext_vector_type(2)));
typedef u32 u32x4 __attribute__((ext_vector_type(4)));
typedef u32 u32x16 __attribute__((ext_vector_type(16)));
#define GAS __attribute__((address_space(1)))
#define LAS __attribute__((address_space(3)))

constexpr int NR = 8;
constexpr int KD = 4096;
constexpr int NL = 2048;
constexpr int NKS = 64;
constexpr int NCH = 16;
constexpr int NWG = 256;
constexpr int NGW = 240;
#define EPI_OLD
#define VCNT
#define WIN 2
#define NPW 2
#define PWIN 3
#define PQ 6
#define PUSH (PQ / 4)
#define D_SLEEP 40
#define X1_SLEEP 50
#define P1_SLEEP 80
#define W2_SLEEP 50
#define DW_SLEEP 100
#define H_SLEEP 0
#define POLL_GAP 2
#define C_SLEEP 12
#define E_AUX 0
#define PU_SLEEP 36
#define CODE_PF 0
#define CODE_BYTES 17408
#define LS_SLEEP 1
#define PACE 0
#define PACE_U 24
#define PU0_SLEEP 0
#define L_SLEEP 100
#define XP_SLEEP 2
#define PERM 1
constexpr int BAR_POS = 2;
constexpr u32 W_KS = 16 * 16384, W_NT = 16384;
constexpr int NTHREADS = 576;
constexpr int X0_END = 8;
constexpr int X1_END = 16;
constexpr u32 A_BYTES = 32768, E_POS = 49152, STAGE = 51200;
constexpr u32 LDS_CTL = 3 * STAGE;
constexpr u32 LDS_BYTES = LDS_CTL + 512;
constexpr u32 NOWHERE = 0x80000000u;

constexpr size_t XB_OFF = 0;
constexpr size_t XB_BYTES = 4ull << 20;
constexpr size_t WB_OFF = 4ull << 20;
constexpr size_t WB_BYTES = 16ull << 20;
constexpr size_t CT_OFF = 20ull << 20;
constexpr size_t CNT_OFF = CT_OFF;
constexpr size_t TMO_OFF = CT_OFF + 4096;
constexpr size_t INV_OFF = CT_OFF + 5120;
constexpr size_t XFL_OFF = CT_OFF + 8192;
constexpr size_t WFL_OFF = CT_OFF + 40960;
constexpr size_t DBG_OFF = CT_OFF + 65536;
constexpr size_t AMX_OFF = CT_OFF + 196608;
#define REC(b) (b)
constexpr size_t WS_NEED = CT_OFF + 262144;
constexpr size_t XP_OFF = 28ull << 20;
constexpr size_t XP_SRC = 262144;
constexpr size_t WS_END = XP_OFF + NR * XP_SRC;

constexpr u64 DEADLINE_TICKS = 100000000ull;

struct Params {
  const float* x;
  const float* w;
  unsigned char* out;
  char* myws;
  char* ws[NR];
  char* sws[NR];
  u32 perm;
  int rank;
};

#define WAIT_VM(n) asm volatile("s_waitcnt vmcnt(%0)" ::"n"(n) : "memory")
#define WAIT_VM_LGKM(n) asm volatile("s_waitcnt vmcnt(" #n ") lgkmcnt(0)" ::: "memory")
#define WAIT_LGKM0() asm volatile("s_waitcnt lgkmcnt(0)" ::: "memory")
#define BARRIER() do { asm volatile("" ::: "memory"); __builtin_amdgcn_s_barrier(); asm volatile("" ::: "memory"); } while (0)
#define SCHED() __builtin_amdgcn_sched_barrier(0)
#define RLX_AGENT __ATOMIC_RELAXED, __HIP_MEMORY_SCOPE_AGENT
#define RLX_SYSTEM __ATOMIC_RELAXED, __HIP_MEMORY_SCOPE_SYSTEM

#define ARRV(t) do { } while (0)
#define ARRV_OUT() do { } while (0)
#define STAMP(i) do { } while (0)

__device__ __forceinline__ u32 pack_bf16(float a, float b) {
  f32x2 f = {a, b};
  bf16x2 h = __builtin_convertvector(f, bf16x2);
  return __builtin_bit_cast(u32, h);
}

__device__ __forceinline__ u32 wave_max_u32(u32 v) {
  v = max(v, (u32)__builtin_amdgcn_update_dpp(0, (int)v, 0x111, 0xf, 0xf, false));
  v = max(v, (u32)__builtin_amdgcn_update_dpp(0, (int)v, 0x112, 0xf, 0xf, false));
  v = max(v, (u32)__builtin_amdgcn_update_dpp(0, (int)v, 0x114, 0xf, 0xf, false));
  v = max(v, (u32)__builtin_amdgcn_update_dpp(0, (int)v, 0x118, 0xf, 0xf, false));
  const u32 a = (u32)__builtin_amdgcn_readlane((int)v, 15), b = (u32)__builtin_amdgcn_readlane((int)v, 31);
  const u32 c = (u32)__builtin_amdgcn_readlane((int)v, 47), d = (u32)__builtin_amdgcn_readlane((int)v, 63);
  return max(max(a, b), max(c, d));
}

__device__ __forceinline__ u32 sword(const GAS u32* p) {
  u32 v;
  asm volatile("s_load_dword %0, %1, 0x0 glc\n\ts_waitcnt lgkmcnt(0)" : "=&s"(v) : "s"(p) : "memory");
  return v;
}
__device__ __forceinline__ void sflags3(u32x16& v, u32x16& w, u32x16& z, const GAS u32* p, u32 off, const GAS u32* q, u32 off2) {
  asm volatile("s_load_dwordx16 %0, %3, %4 glc\n\ts_load_dwordx16 %1, %3, %4 offset:0x40 glc\n\ts_load_dwordx16 %2, %5, %6 glc\n\ts_waitcnt lgkmcnt(0)"
               : "=&s"(v), "=&s"(w), "=&s"(z)
               : "s"(p), "s"(__builtin_amdgcn_readfirstlane((int)off)), "s"(q), "s"(__builtin_amdgcn_readfirstlane((int)off2)) : "memory");
}
__device__ __forceinline__ void sflags3p(u32x16& v, u32x16& w, u32x16& z, u32& m0, u32& m1, const GAS u32* p, u32 off) {
  u64 m;
  asm volatile("s_load_dwordx16 %0, %4, %5 glc\n\ts_load_dwordx16 %1, %4, %5 offset:0x40 glc\n\ts_load_dwordx16 %2, %4, %5 offset:0x80 glc\n\t"
               "s_load_dwordx2 %3, %4, 0x200 glc\n\ts_waitcnt lgkmcnt(0)"
               : "=&s"(v), "=&s"(w), "=&s"(z), "=&s"(m)
               : "s"(p), "s"(__builtin_amdgcn_readfirstlane((int)off)) : "memory");
  m0 = (u32)m;
  m1 = (u32)(m >> 32);
}
__device__ __forceinline__ int lead4(const u32x16& v, u32 seq, int skip = 0) {
  const bool a = skip > 0 || min(min(v[0], v[1]), min(v[2], v[3])) >= seq, b = skip > 1 || min(min(v[4], v[5]), min(v[6], v[7])) >= seq;
  const bool c = skip > 2 || min(min(v[8], v[9]), min(v[10], v[11])) >= seq, d = min(min(v[12], v[13]), min(v[14], v[15])) >= seq;
  return a ? (b ? (c ? (d ? 4 : 3) : 2) : 1) : 0;
}
__device__ __forceinline__ u32 min16(const u32x16& v) {
  u32 m, n;
  asm("s_min_u32 %0, %2, %3\n\ts_min_u32 %1, %4, %5\n\ts_min_u32 %0, %0, %6\n\ts_min_u32 %1, %1, %7\n\t"
      "s_min_u32 %0, %0, %8\n\ts_min_u32 %1, %1, %9\n\ts_min_u32 %0, %0, %10\n\ts_min_u32 %1, %1, %11\n\t"
      "s_min_u32 %0, %0, %12\n\ts_min_u32 %1, %1, %13\n\ts_min_u32 %0, %0, %14\n\ts_min_u32 %1, %1, %15\n\t"
      "s_min_u32 %0, %0, %16\n\ts_min_u32 %1, %1, %17\n\ts_min_u32 %0, %0, %1"
      : "=&s"(m), "=&s"(n)
      : "s"(v[0]), "s"(v[1]), "s"(v[2]), "s"(v[3]), "s"(v[4]), "s"(v[5]), "s"(v[6]), "s"(v[7]), "s"(v[8]), "s"(v[9]), "s"(v[10]),
        "s"(v[11]), "s"(v[12]), "s"(v[13]), "s"(v[14]), "s"(v[15])
      : "scc");
  return m;
}

__device__ __forceinline__ int x_known(int xn, const u32x16& v, const u32x16& w, u32 seq) {
  const int x0 = min(xn, X0_END - 1);
  if (xn < X0_END && min16(v) >= seq) xn = (x0 < X0_END - 1 && min16(w) >= seq) ? x0 + 2 : x0 + 1;
  if (xn >= X0_END && x0 == X0_END - 1 && w[0] >= seq) xn = w[1] >= seq ? NKS : X1_END;
  return xn;
}

__device__ __forceinline__ bool give_up(u32 t0, GAS u32* tmo, u32 seq) {
  const u32 dt = (u32)__builtin_amdgcn_s_memrealtime() - t0;
  if (__builtin_expect(dt < 100000u, 1)) return false;
  __builtin_amdgcn_s_sleep(64);
  const bool stop = dt > (u32)DEADLINE_TICKS || sword(tmo) == seq;
  if (stop) __hip_atomic_store(tmo, seq, RLX_AGENT);
  return stop;
}

__device__ __forceinline__ void w_load(f32x4* v, __amdgpu_buffer_rsrc_t rs, u32 off, int row) {
#pragma unroll
  for (int j = 0; j < 8; ++j)
    v[j] = __builtin_bit_cast(f32x4, __builtin_amdgcn_raw_buffer_load_b128(rs, off, (row + j) * (NL * 4), 0));
}

__device__ __forceinline__ u32x4 w_pack(const f32x4* v, int r) {
  u32x4 d;
  d.x = pack_bf16(v[0][r], v[1][r]);
  d.y = pack_bf16(v[2][r], v[3][r]);
  d.z = pack_bf16(v[4][r], v[5][r]);
  d.w = pack_bf16(v[6][r], v[7][r]);
  return d;
}

__device__ __forceinline__ void w_store(const f32x4* v, __amdgpu_buffer_rsrc_t rs, u32 woff, int chunk) {
#pragma unroll
  for (int r = 0; r < 4; ++r) __builtin_amdgcn_raw_buffer_store_b128(w_pack(v, r), rs, woff + r * 2048, chunk * (4 * W_KS), 16  );
}

__device__ __forceinline__ f32x4 e_load(__amdgpu_buffer_rsrc_t rs, u32 off, int row) {
  f32x4 v;
#pragma unroll
  for (int j = 0; j < 4; ++j) v[j] = __builtin_bit_cast(float, __builtin_amdgcn_raw_buffer_load_b32(rs, off, (row + j) * (NL * 4), 0));
  return v;
}
__device__ __forceinline__ u32x2 e_pack(const f32x4& v) { return (u32x2){pack_bf16(v[0], v[1]), pack_bf16(v[2], v[3])}; }
__device__ __forceinline__ void e_store(const f32x4& v, __amdgpu_buffer_rsrc_t rs, u32 off, int chunk) {
  __builtin_amdgcn_raw_buffer_store_b64(e_pack(v), rs, off, chunk * (4 * W_KS), 16  );
}
__device__ __forceinline__ void ticket(u32& old, GAS u32* p, u32 v) {
  asm volatile("global_atomic_swap %0, %1, %2, off sc0 sc1" : "+v"(old) : "v"(p), "v"(v) : "memory");
}

__device__ __forceinline__ void ctl_write(u32 addr, u32 v) {
  asm volatile("ds_write_b32 %0, %1" ::"v"(addr), "v"(v) : "memory");
}
__device__ __forceinline__ u32 ctl_read(u32 addr) {
  u32 v;
  asm volatile("ds_read_b32 %0, %1\n\ts_waitcnt lgkmcnt(0)" : "=v"(v) : "v"(addr) : "memory");
  return v;
}
__device__ __forceinline__ void x_load(f32x4* v, const float* __restrict__ x, int xh, int xks, int msub, int lane) {
  const float* xs = x + (size_t)(xh * 256 + msub * 16 + (lane & 15)) * KD + xks * 64 + (lane >> 4) * 8;
#pragma unroll
  for (int j = 0; j < 4; ++j) v[j] = *(const f32x4*)(xs + (j >> 1) * 32 + (j & 1) * 4);
}

__device__ __forceinline__ u32x4 x_pack(const f32x4* v, int kh) {
  u32x4 d;
  d.x = pack_bf16(v[kh * 2][0], v[kh * 2][1]);
  d.y = pack_bf16(v[kh * 2][2], v[kh * 2][3]);
  d.z = pack_bf16(v[kh * 2 + 1][0], v[kh * 2 + 1][1]);
  d.w = pack_bf16(v[kh * 2 + 1][2], v[kh * 2 + 1][3]);
  return d;
}

__device__ __forceinline__ void x_store(const f32x4* v, __amdgpu_buffer_rsrc_t rs, int xh, int xks, int msub, int lane) {
  const u32 xoff = (u32)(((xh * 64 + xks) * 16 + msub) * 2048 + lane * 16);
#pragma unroll
  for (int kh = 0; kh < 2; ++kh) __builtin_amdgcn_raw_buffer_store_b128(x_pack(v, kh), rs, xoff + kh * 1024, 0, 17  );
}

__device__ __forceinline__ u64 both_ge(u64 f, u32 seq) { return __ballot((u32)f >= seq && (u32)(f >> 32) >= seq); }
__device__ __forceinline__ int lead_ones(u64 m) { return m == ~0ull ? 64 : __builtin_ctzll(~m); }

__global__ void __launch_bounds__(NTHREADS) __attribute__((amdgpu_waves_per_eu(3, 3))) ag_gemm_q8(Params p) {
  extern __shared__ __attribute__((aligned(16))) char lds[];
  const int b = blockIdx.x;
  const int tid = threadIdx.x, lane = tid & 63;
  const int wid = __builtin_amdgcn_readfirstlane(tid >> 6);
  const int idx = b >> 3;
  const int half = idx & 1, nt = idx >> 1, shr = (b & 7) * 2 + half;
  const float* const px = p.x;
  const float* const pw = p.w;
  unsigned char* const pout = p.out;
  char* const myws = p.myws;
  char* const srcws = p.sws[b & 7];
  char* const widws = p.ws[wid & 7];
  const int rank = p.rank;
  const u32 perm = p.perm;
  const u64 w0 = (u64)p.ws[0], w1 = (u64)p.ws[1], w2 = (u64)p.ws[2], w3 = (u64)p.ws[3];
  const u64 w4 = (u64)p.ws[4], w5 = (u64)p.ws[5], w6 = (u64)p.ws[6], w7 = (u64)p.ws[7];
  asm volatile("" ::"s"(px), "s"(pw), "s"(pout), "s"(myws), "s"(srcws), "s"(widws), "s"(rank), "s"(perm), "s"(w0), "s"(w1), "s"(w2), "s"(w3),
               "s"(w4), "s"(w5), "s"(w6), "s"(w7));
  const int src = (perm >> (4 * (b & 7))) & 7, mt = src * 2 + half;
  auto peer_ws = [&]() {
    int lo = (int)w0, hi = (int)(w0 >> 32);
#define PUT(i) asm("v_writelane_b32 %0, %2, " #i "\n\tv_writelane_b32 %1, %3, " #i : "+v"(lo), "+v"(hi) : "s"((int)w##i), "s"((int)(w##i >> 32)))
    PUT(1); PUT(2); PUT(3); PUT(4); PUT(5); PUT(6); PUT(7);
#undef PUT
    return (char*)(((u64)(u32)hi << 32) | (u32)lo);
  };
  GAS u32* const cnt = (GAS u32*)(myws + CNT_OFF) + b;
  GAS u32* const ownfl = (GAS u32*)(myws + XFL_OFF) + rank * 1024;
  const __amdgpu_buffer_rsrc_t xrs = __builtin_amdgcn_make_buffer_rsrc(myws + XB_OFF, 0, (int)XB_BYTES, 0x00020000);

  const u32 ctl = (u32)(size_t)(LAS char*)lds + LDS_CTL;
  if (nt == 15) {
    if (wid == 4) {
      ctl_write(ctl, PQ);
      WAIT_LGKM0();
    }
    BARRIER();
  }
  if (wid < 2 && (wid || (b >> 5) * 16 + (b & 15) >= PQ * 4)) {
    f32x4 v[4];
    const int xh = (b >> 4) & 1, xks = wid * X0_END + (b >> 5), msub = b & 15;
    if (wid) __builtin_amdgcn_s_sleep(X1_SLEEP);
    x_load(v, px, xh, xks, msub, lane);
    const u32 seq = __hip_atomic_load(cnt, RLX_AGENT);
    SCHED();
    GAS u32* pub = wid ? ownfl + xh * 512 + 128 + xks * 16 + msub : (GAS u32*)(peer_ws() + XFL_OFF) + (rank * 2 + xh) * 512 + xks * 16 + msub;
    asm volatile("" : "+v"(pub));
    SCHED();
    x_store(v, xrs, xh, xks, msub, lane);
    SCHED();
    WAIT_VM(0);
    if (lane < (wid ? 1 : 8)) __hip_atomic_store(pub, seq + 1u, RLX_SYSTEM);
  }
  u32 seq8 = 0;
  if (wid == 8) {
    f32x4 v[8];
    const int xh = idx >> 4, msub = idx & 15;
    const bool one = msub < PQ * 4, two = msub + 16 < PQ * 4;
    STAMP(0);
    if (one) x_load(v, px, xh, 0, msub, lane);
    if (two) x_load(v + 4, px, xh, 1, msub, lane);
    seq8 = __hip_atomic_load(cnt, RLX_AGENT) + 1u;
    SCHED();
    if (one) {
      const u32 off = (u32)(XP_OFF + rank * XP_SRC) + (u32)(xh * (XP_SRC / 2) + msub * 2048) + lane * 16;
      const u32 flo = lane == 0 ? (u32)XFL_OFF + (u32)((rank * 2 + xh) * 512 + msub) * 4u : NOWHERE;
      const __amdgpu_buffer_rsrc_t r0 = __builtin_amdgcn_make_buffer_rsrc(srcws, 0, (int)WS_END, 0x00020000);
      const u32x4 d0 = x_pack(v, 0), d1 = x_pack(v, 1);
      if (PU0_SLEEP) __builtin_amdgcn_s_sleep(PU0_SLEEP);
      __builtin_amdgcn_raw_buffer_store_b128(d0, r0, off, 0, 17);
      __builtin_amdgcn_raw_buffer_store_b128(d1, r0, off + 1024, 0, 17);
      if (two) {
        const u32x4 d2 = x_pack(v + 4, 0), d3 = x_pack(v + 4, 1);
        SCHED();
        __builtin_amdgcn_s_sleep(PU_SLEEP);
        __builtin_amdgcn_raw_buffer_store_b128(d2, r0, off, A_BYTES, 17);
        __builtin_amdgcn_raw_buffer_store_b128(d3, r0, off + 1024, A_BYTES, 17);
        WAIT_VM(2);
        __builtin_amdgcn_raw_buffer_store_b32(seq8, r0, flo, 0, 17);
        WAIT_VM(1);
        __builtin_amdgcn_raw_buffer_store_b32(seq8, r0, flo, 64, 17);
      } else {
        WAIT_VM(0);
        __builtin_amdgcn_raw_buffer_store_b32(seq8, r0, flo, 0, 17);
      }
      STAMP(1);
    }
  }

  GAS u32* const xfl = (GAS u32*)(myws + XFL_OFF) + mt * 512;
  GAS u32* const tmo = (GAS u32*)(myws + TMO_OFF);
  GAS u32* const tick = (GAS u32*)(myws + INV_OFF) + (__builtin_amdgcn_s_getreg(20 | (3 << 11)) & 15) * 32;
  auto clean = [&](u32 seq) {
    GAS u64* const g = (GAS u64*)(myws + AMX_OFF + rank * 2048) + lane;
    const u32 t0 = (u32)__builtin_amdgcn_s_memrealtime();
    for (;;) {
      bool all = true;
#pragma unroll
      for (int k = 0; k < 4; ++k) all &= (u32)(__hip_atomic_load(g + k * 64, RLX_SYSTEM) >> 32) >= seq;
      if (__all(all)) break;
      if (give_up(t0, tmo, seq)) break;
      __builtin_amdgcn_s_sleep(C_SLEEP);
    }
    WAIT_VM(0);
    asm volatile("buffer_inv sc0 sc1" ::: "memory");
    WAIT_VM(0);
  };
  char* const xsrc = srcws + XB_OFF + (size_t)half * (XB_BYTES / 2);
  const __amdgpu_buffer_rsrc_t ars = __builtin_amdgcn_make_buffer_rsrc(xsrc, 0, (int)(XB_BYTES / 2), 0x00020000);

  if (nt == 15) {
    if (wid < 2 || wid > 5 || (wid < 4 && src == rank)) return;
    const u32 seq = (u32)__builtin_amdgcn_readfirstlane((int)__hip_atomic_load(cnt, RLX_AGENT)) + 1u;
    if (wid == 5) {
      f32x4 cv[24];
      const int xl = (b >> 1) * 6;
      __builtin_amdgcn_s_sleep(L_SLEEP);
#pragma unroll
      for (int j = 0; j < 6; ++j) x_load(cv + 4 * j, px, b & 1, X1_END + ((xl + j) >> 4), (xl + j) & 15, lane);
      SCHED();
#pragma unroll
      for (int j = 0; j < 6; ++j) x_store(cv + 4 * j, xrs, b & 1, X1_END + ((xl + j) >> 4), (xl + j) & 15, lane);
      WAIT_VM(0);
      if (lane == 0) __hip_atomic_store(ownfl + (b & 1) * 512 + 384 + (b >> 1), seq, RLX_SYSTEM);
      return;
    }
    __builtin_amdgcn_s_setprio(3);
    const u32 t0 = (u32)__builtin_amdgcn_s_memrealtime();
    if (wid < 4) {
      const int pv = wid - 2;
      LAS void* dst = (LAS void*)(lds + pv * 256);
      const u32 vo = lane * 128;
      int np = PQ;
#pragma nounroll
      for (int u = PQ + pv; u < NKS * 4; u += NPW) {
        for (u32 spins = 1; u >= np; ++spins) {
          np = __builtin_amdgcn_readfirstlane((int)ctl_read(ctl));
          if (u < np) break;
          __builtin_amdgcn_s_sleep(LS_SLEEP);
          if ((spins & 255u) == 0 && give_up(t0, tmo, seq)) np = NKS * 4 + 1;
        }
        if (np > NKS * 4) break;
        if (PACE && u < PACE_U) __builtin_amdgcn_s_sleep(PACE);
        WAIT_VM(PWIN - 1);
        __builtin_amdgcn_raw_ptr_buffer_load_lds(ars, dst, 4, vo, u * 8192, 0, 0);
      }
      WAIT_VM(0);
      if (np > NKS * 4) asm volatile("buffer_inv sc0 sc1" ::: "memory");
      BARRIER();
      return;
    }
    if (src != rank) {
      if (D_SLEEP) __builtin_amdgcn_s_sleep(D_SLEEP);
      for (int np = PQ; np < NKS * 4;) {
        u32x16 f0, f1, f2;
        u32 m0, m1;
        const int l0 = min(np >> 2, X0_END - 3);
        sflags3p(f0, f1, f2, m0, m1, xfl, (u32)l0 * 64u);
        int n = l0 * 4 + lead4(f0, seq, l0 == PQ / 4 ? PQ & 3 : 0);
        if (n == l0 * 4 + 4) n += lead4(f1, seq);
        if (n == l0 * 4 + 8) n += lead4(f2, seq);
        if (n == X0_END * 4 && m0 >= seq) n = m1 >= seq ? NKS * 4 : X1_END * 4;
        if (n > np) {
          np = n;
          ctl_write(ctl, (u32)np);
          continue;
        }
        if (give_up(t0, tmo, seq)) {
          ctl_write(ctl, NKS * 4 + 1);
          break;
        }
        if (POLL_GAP) __builtin_amdgcn_s_sleep(POLL_GAP);
      }
    } else {
      GAS u32* const pub = (GAS u32*)(p.ws[lane & 7] + XFL_OFF) + (rank * 2 + half) * 512 + 128;
      for (int ph = 0; ph < 2;) {
        if (both_ge(__hip_atomic_load((GAS u64*)(ownfl + half * 512 + 256 + ph * 128) + lane, RLX_SYSTEM), seq) == ~0ull) {
          if (lane < 8) __hip_atomic_store(pub + ph, seq, RLX_SYSTEM);
          ++ph;
          continue;
        }
        if (give_up(t0, tmo, seq)) break;
        __builtin_amdgcn_s_sleep(4);
      }
    }
    u32 old = seq;
    if (lane == 0) ticket(old, tick, seq);
    BARRIER();
    if (lane == 0) {
      __hip_atomic_store((GAS u64*)(myws + AMX_OFF) + rank * 256 + REC(b), (u64)seq << 32, RLX_SYSTEM);
      __hip_atomic_store(cnt, seq, RLX_AGENT);
    }
    WAIT_VM(0);
    asm volatile("" : "+v"(old));
    if ((u32)__builtin_amdgcn_readfirstlane((int)old) != seq) clean(seq);
    STAMP(15);
    return;
  }

  const int kq1 = lane >> 5, wH = (lane >> 4) & 1, wi = lane & 15;
  const __amdgpu_buffer_rsrc_t wfr = __builtin_amdgcn_make_buffer_rsrc((void*)pw, 0, KD * NL * 4, 0x00020000);
  const u32 wfo = (u32)(kq1 * 8 * NL + nt * 128 + wH * 64 + wi * 4) * 4u;
  const u32 woff = (u32)(nt * W_NT + (shr >> 2) * W_KS + wH * 8192 + ((shr >> 1) & 1) * 1024 +
                         (((shr & 1) * 2 + kq1) * 16 + wi) * 16);
  const bool ext = nt < 8;
  const u32 efo = ext ? (u32)((wH * 8 + kq1 * 4) * NL + 15 * 128 + nt * 16 + wi) * 4u : NOWHERE;
  const u32 eoff = ext ? (u32)(15 * W_NT + nt * 2048 + (shr >> 2) * W_KS + ((shr >> 1) & 1) * 1024 +
                               (((shr & 1) * 2 + wH) * 16 + wi) * 16 + kq1 * 8) : NOWHERE;
  const __amdgpu_buffer_rsrc_t wrs =
      __builtin_amdgcn_make_buffer_rsrc(myws + WB_OFF, 0, (int)WB_BYTES, 0x00020000);
  GAS u32* const wfl = (GAS u32*)(myws + WFL_OFF) + nt * 256;

  if (wid == 8) {
    __builtin_amdgcn_s_setprio(3);
    const u32 seq = (u32)__builtin_amdgcn_readfirstlane((int)seq8);
    bool failed = false, cleaner = false;
    int xn = 0, cw = 0;
#define KBAR(t) do { BARRIER(); SCHED(); } while (0)
    f32x4 wv[9];
#pragma unroll
    for (int j = 0; j < 9; ++j) wv[j] = (f32x4){0.f, 0.f, 0.f, 0.f};
    if (H_SLEEP) __builtin_amdgcn_s_sleep(H_SLEEP);
#pragma nounroll
    for (int t = -2; t < NKS - 1; ++t) {
      const int c = (t >> 2) + 3;
      if ((t & 3) == 0) {
        __builtin_amdgcn_s_waitcnt(0x0f70);
        asm volatile("" ::: "memory");
        if (lane == 0 && c <= NCH) __hip_atomic_store(t ? wfl + (c - 1) * 16 + shr : cnt, seq, RLX_AGENT);
        if (c < NCH) {
          w_load(wv, wfr, wfo, c * 256 + shr * 16);
          wv[8] = e_load(wfr, efo, c * 256 + shr * 16);
        }
      } else if ((t & 3) == 2 && t > 0 && c < NCH) {
        w_store(wv, wrs, woff, c);
        e_store(wv[8], wrs, eoff, c);
      }
      if (t == 1 && lane < 2)
        __hip_atomic_store(lane ? ownfl + (b & 1) * 512 + 384 + (b >> 1) : wfl + 2 * 16 + shr, seq, RLX_SYSTEM);
      if (t == 25 + idx && sword(tick) != seq) {
        u32 old = 0;
        if (lane == 0) old = __hip_atomic_exchange(tick, seq, RLX_SYSTEM);
        cleaner = (u32)__builtin_amdgcn_readfirstlane((int)old) != seq;
      }
      SCHED();
      const int s = min(t + 3, NKS - 1), gw = t < -1 ? 0 : (s >> 2) + 1;
      if (xn < NKS || cw <= min(gw, NCH - 1)) {
        u32 t0 = 0, spins = 0;
        for (;;) {
          u32x16 f0, f1, fw;
          const int on = xn + cw;
          sflags3(f0, f1, fw, xfl, (u32)min(xn, X0_END - 1) * 64u, wfl, (u32)min(cw, NCH - 1) * 64u);
          xn = x_known(xn, f0, f1, seq);
          if (min16(fw) >= seq) cw = min(cw + 1, NCH);
          if (xn > s && cw >= gw) break;
          if (xn + cw != on) continue;
          if (spins == 0) t0 = (u32)__builtin_amdgcn_s_memrealtime();
          if ((spins & 63u) == 63u) {
            GAS u64* const xf = (GAS u64*)xfl;
            const u32 vw = __hip_atomic_load(wfl + min(cw, NCH - 1) * 16 + (lane & 15), RLX_AGENT);
            const u64 v0 = __hip_atomic_load(xf + lane, RLX_SYSTEM), v1 = __hip_atomic_load(xf + 64, RLX_SYSTEM);
            if (__all(vw >= seq)) cw = min(cw + 1, NCH);
            int n = lead_ones(both_ge(v0, seq)) >> 3;
            if (n == X0_END && __all((u32)v1 >= seq)) n = __all((u32)(v1 >> 32) >= seq) ? NKS : X1_END;
            xn = max(xn, n);
          }
          ++spins;
          if (give_up(t0, tmo, seq)) { failed = true; xn = NKS; cw = NCH; break; }
          if (POLL_GAP) __builtin_amdgcn_s_sleep(POLL_GAP);
        }
      }
      if (t == NKS - 2) {
        ctl_write(ctl + 17 * 4, failed ? 1u : 0u);
        ctl_write(ctl + 18 * 4, seq);
        WAIT_LGKM0();
      }
      ARRV(t == -2 ? -1 : t);
      KBAR(t == -2 ? -1 : t);
    }
    ARRV_OUT();
    if (cleaner) {
      BARRIER();
      BARRIER();
      clean(seq);
    }
    return;
  }

  f32x4 dv[2][9];
  if (wid >= 4) {
    if (DW_SLEEP) __builtin_amdgcn_s_sleep(DW_SLEEP);
#pragma unroll
    for (int d = 0; d < 2; ++d) {
      w_load(dv[d], wfr, wfo, d * 64 + (wid - 4) * 16);
      dv[d][8] = e_load(wfr, efo, d * 64 + (wid - 4) * 16);
    }
    SCHED();
  }
  f32x4 cv[9];
  const int xl = (b >> 1) * 6 + (wid - 4) * 2;
  u32 seqw = 0;
  if (wid == 2 || wid == 3) {
    const bool skip = wid == 2 && shr < 8;
    if (wid == 3) __builtin_amdgcn_s_sleep(P1_SLEEP);
    else if (W2_SLEEP) __builtin_amdgcn_s_sleep(W2_SLEEP);
    if (!skip) {
      w_load(cv, wfr, wfo, (wid - 2) * 256 + shr * 16);
      cv[8] = e_load(wfr, efo, (wid - 2) * 256 + shr * 16);
    }
    seqw = __hip_atomic_load(cnt, RLX_AGENT) + 1u;
    SCHED();
    if (!skip) {
      w_store(cv, wrs, woff, wid - 2);
      e_store(cv[8], wrs, eoff, wid - 2);
    }
    SCHED();
    if (wid == 2) {
      WAIT_VM(0);
      if (lane == 0) __hip_atomic_store(wfl + shr, seqw, RLX_AGENT);
      STAMP(12);
    }
  } else if (wid == 7) {
    w_load(cv, wfr, wfo, 2 * 256 + shr * 16);
    cv[8] = e_load(wfr, efo, 2 * 256 + shr * 16);
    SCHED();
  } else if (wid >= 4) {
    x_load(cv, px, b & 1, X1_END + (xl >> 4), xl & 15, lane);
    x_load(cv + 4, px, b & 1, X1_END + (xl >> 4), (xl & 15) + 1, lane);
    SCHED();
  }

  const __amdgpu_buffer_rsrc_t brs = __builtin_amdgcn_make_buffer_rsrc(
      myws + WB_OFF + (size_t)nt * W_NT, 0, (int)(WB_BYTES - nt * W_NT), 0x00020000);
  const __amdgpu_buffer_rsrc_t prs = __builtin_amdgcn_make_buffer_rsrc(myws + XP_OFF + (size_t)mt * (XP_SRC / 2), 0, (int)(XP_SRC / 2), 0x00020000);
  const u32 l16 = lane * 16;
  const u32 avo = wid * 4096 + l16, bvo = wid * 2048 + l16;
  const u32 evo = ext ? (u32)((15 - nt) * W_NT + nt * 2048 + wid * 256 + lane * 4) : NOWHERE;
  auto piece = [&](int t, u32 slot, int j, bool pushed = false) {
    if (j == 6) {
      __builtin_amdgcn_raw_ptr_buffer_load_lds(brs, (LAS void*)(lds + slot + E_POS + wid * 256), 4, evo, t * W_KS, 0, 0);
    } else if (j >= 4) {
      LAS void* lb = (LAS void*)(lds + slot + A_BYTES + wid * 2048);
      if (j == 4) __builtin_amdgcn_raw_ptr_buffer_load_lds(brs, lb, 16, bvo, t * W_KS, 0, 0);
      else __builtin_amdgcn_raw_ptr_buffer_load_lds(brs, lb, 16, bvo, t * W_KS, 1024, 0);
    } else {
      const u32 ro = ((j + nt) & 3) * 1024;
      LAS void* la = (LAS void*)(lds + slot + wid * 4096 + ro);
      __builtin_amdgcn_raw_ptr_buffer_load_lds(pushed ? prs : ars, la, 16, avo, t * A_BYTES + ro, 0, 0);
    }
    SCHED();
  };

  const int wr = wid >> 1, wc = wid & 1;
  const u32 aoff0 = wr * 8192 + wc * 4096 + l16, aoff1 = wr * 8192 + (1 - wc) * 4096 + l16, boff = A_BYTES + wc * 8192 + l16;
  f32x4 acc[4][4], ace[2];
#pragma unroll
  for (int i = 0; i < 4; ++i)
#pragma unroll
    for (int j = 0; j < 4; ++j) acc[i][j] = (f32x4){0.f, 0.f, 0.f, 0.f};
  ace[0] = ace[1] = (f32x4){0.f, 0.f, 0.f, 0.f};
  bf16x8 af[2][4], bfr[2][4], ef[2];
  auto frags = [&](u32 slot, int kh) {
#pragma unroll
    for (int i = 0; i < 4; ++i) {
      bfr[kh][i] = *(const bf16x8*)(lds + slot + boff + i * 2048 + kh * 1024);
      af[kh][i] = *(const bf16x8*)(lds + slot + (i < 2 ? aoff0 : aoff1) + (i & 1) * 2048 + kh * 1024);
    }
    ef[kh] = *(const bf16x8*)(lds + slot + E_POS + l16 + kh * 1024);
    SCHED();
  };
  auto mfma4 = [&](int kh, int mi) {
#pragma unroll
    for (int r = 0; r < 4; ++r)
      acc[mi][r] = __builtin_amdgcn_mfma_f32_16x16x32_bf16(bfr[kh][r], af[kh][mi], acc[mi][r], 0, 0, 0);
    if (mi >= 2) ace[mi - 2] = __builtin_amdgcn_mfma_f32_16x16x32_bf16(ef[kh], af[kh][mi - 2], ace[mi - 2], 0, 0, 0);
    SCHED();
  };

  BARRIER();
#pragma unroll
  for (int j = 0; j < 4; ++j) piece(0, 0, j, wid < PQ * 2);
#pragma unroll
  for (int j = 0; j < 4; ++j) piece(1, STAGE, j, wid + 8 < PQ * 2);
  if (wid >= 4) {
    char* const dst = lds + A_BYTES + wH * 8192 + ((wid >> 1) & 1) * 1024 + (((wid & 1) * 2 + kq1) * 16 + wi) * 16;
    char* const dse = lds + E_POS + ((wid >> 1) & 1) * 1024 + (((wid & 1) * 2 + wH) * 16 + wi) * 16 + kq1 * 8;
#pragma unroll
    for (int d = 0; d < 2; ++d) {
#pragma unroll
      for (int r = 0; r < 4; ++r) *(u32x4*)(dst + d * STAGE + r * 2048) = w_pack(dv[d], r);
      *(u32x2*)(dse + d * STAGE) = e_pack(dv[d][8]);
    }
    SCHED();
    if (wid == 7) {
      w_store(cv, wrs, woff, 2);
      e_store(cv[8], wrs, eoff, 2);
      SCHED();
      WAIT_VM_LGKM(9);
    } else {
      x_store(cv, xrs, b & 1, X1_END + (xl >> 4), xl & 15, lane);
      x_store(cv + 4, xrs, b & 1, X1_END + (xl >> 4), (xl & 15) + 1, lane);
      SCHED();
      WAIT_VM_LGKM(8);
    }
  } else if (wid == 3) {
    WAIT_VM(8);
    if (lane == 0) __hip_atomic_store(wfl + 16 + shr, seqw, RLX_AGENT);
    STAMP(13);
    WAIT_VM(5);
  } else {
    WAIT_VM_LGKM(4);
  }
  ARRV(-1);
  BARRIER();
  frags(0, 0);
#pragma unroll
  for (int j = 0; j < 4 - BAR_POS; ++j) piece(2, 2 * STAGE, j);

  u32 cur = 0, nx1 = STAGE, nx2 = 2 * STAGE;
  auto step = [&](int t, auto more) {
    constexpr int MORE = decltype(more)::value;
    frags(cur, 1);
    if (MORE >= 2) piece(t + 2, nx2, 6);
#pragma unroll
    for (int g = 0; g < 4; ++g) {
      mfma4(0, g);
      if (MORE >= 2 && 4 - BAR_POS + g < 6) piece(t + 2, nx2, 4 - BAR_POS + g);
    }
    constexpr int BP = MORE == 1 ? 4 : BAR_POS;
#pragma unroll
    for (int g = 0; g < BP; ++g) mfma4(1, g);
    if (MORE >= 1) {
      if (MORE >= 2) WAIT_VM_LGKM(7);
      else WAIT_VM_LGKM(0);
      ARRV(t);
      BARRIER();
      SCHED();
      frags(nx1, 0);
    }
#pragma unroll
    for (int g = BP; g < 4; ++g) {
      mfma4(1, g);
      if (MORE >= 3) piece(t + 3, cur, g - BP);
    }
    const u32 o = cur;
    cur = nx1; nx1 = nx2; nx2 = o;
  };
  static_assert(PQ <= 8, "the K loop reads the peers");
  for (int t = 0; t < NKS - 3; ++t) step(t, std::integral_constant<int, 3>());
  step(NKS - 3, std::integral_constant<int, 2>());
  step(NKS - 2, std::integral_constant<int, 1>());
  step(NKS - 1, std::integral_constant<int, 0>());
  ARRV_OUT();

  float mx = 0.f;
#pragma unroll
  for (int mi = 0; mi < 4; ++mi)
#pragma unroll
    for (int r = 0; r < 4; ++r)
#pragma unroll
      for (int j = 0; j < 4; ++j) mx = fmaxf(mx, fabsf(acc[mi][r][j]));
  float me = 0.f;
#pragma unroll
  for (int i = 0; i < 8; ++i) me = fmaxf(me, fabsf(ace[i >> 2][i & 3]));
  if (ext) mx = fmaxf(mx, me);
  const u32 wm = wave_max_u32(__builtin_bit_cast(u32, mx));
  ctl_write(ctl + (24 + wid) * 4, wm);
  const u32 fs = ctl_read(ctl + (17 + (lane & 1)) * 4);
  BARRIER();
  const __amdgpu_buffer_rsrc_t ors = __builtin_amdgcn_make_buffer_rsrc(pout, 0, 4096 * NL, 0x00020000);
  const u32 orow = (u32)(mt * 256 + wr * 64 + (lane & 15)) * NL;
  const u32 oo0 = orow + wc * 32 * NL + nt * 128 + wc * 64 + (lane >> 4) * 16, oo1 = oo0 + (1 - 2 * wc) * 32 * NL;
  const u32 oe = ext ? orow + wc * 32 * NL + 15 * 128 + nt * 16 + (lane >> 4) * 4 : NOWHERE;
  const u32 seq = (u32)__builtin_amdgcn_readlane((int)fs, 1);
  {
    u32 m = wave_max_u32(ctl_read(ctl + (24 + (lane & 7)) * 4));
    if (__builtin_amdgcn_readlane((int)fs, 0)) m = 0x7fffffffu;
    u32 tm = 0;
    if (wid == 0) tm = __hip_atomic_load(tmo, RLX_AGENT);
    if (lane == 0) __hip_atomic_store((GAS u64*)(widws + AMX_OFF) + rank * 256 + REC(b), ((u64)seq << 32) | m, RLX_SYSTEM);
    const __amdgpu_buffer_rsrc_t mrs = __builtin_amdgcn_make_buffer_rsrc(myws + AMX_OFF, 0, NR * 2048, 0x00020000);
    const u32 l2 = min(l16 + 1024u, (u32)NGW * 8u - 16u);
    const u32 t0 = (u32)__builtin_amdgcn_s_memrealtime();
    for (;;) {
      const u32x4 g0 = __builtin_amdgcn_raw_buffer_load_b128(mrs, l16, wid * 2048, 17);
      const u32x4 g1 = __builtin_amdgcn_raw_buffer_load_b128(mrs, l2, wid * 2048, 17);
      m = max(max(g0.x, g0.z), max(g1.x, g1.z));
      if (__all(min(min(g0.y, g0.w), min(g1.y, g1.w)) >= seq)) break;
      if (give_up(t0, tmo, seq)) { m = 0x7fffffffu; break; }
      __builtin_amdgcn_s_sleep(XP_SLEEP + 1);
    }
    if (tm == seq) m = 0x7fffffffu;
    ctl_write(ctl + (32 + wid) * 4, wave_max_u32(m));
    WAIT_LGKM0();
  }
  BARRIER();
  const u32 gmax = wave_max_u32(ctl_read(ctl + (32 + (lane & 7)) * 4));
  const bool bad = gmax > 0x7f800000u;
  const float amax = __builtin_bit_cast(float, gmax);
  const float inv = amax > 0.f ? 448.0f / amax : 0.f;
  auto q4 = [&](float a, float c, float d, float e) {
    int q = __builtin_amdgcn_cvt_pk_fp8_f32(a * inv, c * inv, 0, false);
    q = __builtin_amdgcn_cvt_pk_fp8_f32(d * inv, e * inv, q, true);
    return bad ? 0x7f7f7f7fu : (u32)q;
  };
#pragma unroll
  for (int mi = 0; mi < 4; ++mi) {
    u32x4 d;
#pragma unroll
    for (int j = 0; j < 4; ++j) d[j] = q4(acc[mi][0][j], acc[mi][1][j], acc[mi][2][j], acc[mi][3][j]);
    __builtin_amdgcn_raw_buffer_store_b128(d, ors, (mi < 2 ? oo0 : oo1) + (mi & 1) * 16 * NL, 0, 16  );
  }
#pragma unroll
  for (int i = 0; i < 2; ++i) __builtin_amdgcn_raw_buffer_store_b32(q4(ace[i][0], ace[i][1], ace[i][2], ace[i][3]), ors, oe, i * 16 * NL, E_AUX);
  if (bad) asm volatile("buffer_inv sc0 sc1" ::: "memory");
}

}

extern "C" void kernel_launch(int rank, int nranks, ncclComm_t comm, hipStream_t stream,
                              void* const* d_in, const int* in_sizes, int n_in,
                              void* d_out, int out_size,
                              const int* in_dtypes, int out_dtype,
                              void* const* d_out_all, void* const* d_ws_all, size_t ws_bytes) {
  (void)comm; (void)in_dtypes; (void)out_dtype; (void)d_out_all;
  if (nranks != NR || n_in < 2 || ws_bytes < WS_END || in_sizes[0] != 512 * KD || in_sizes[1] != KD * NL ||
      out_size != 4096 * NL || (unsigned)rank >= (unsigned)nranks) {
    fprintf(stderr, "unexpected problem geometry\n");
    return;
  }
  hipError_t e = hipFuncSetAttribute((const void*)ag_gemm_q8, hipFuncAttributeMaxDynamicSharedMemorySize, (int)LDS_BYTES);
  if (e != hipSuccess) { fprintf(stderr, "hipFuncSetAttribute: %s\n", hipGetErrorString(e)); return; }
  Params p;
  p.x = (const float*)d_in[0];
  p.w = (const float*)d_in[1];
  p.out = (unsigned char*)d_out;
  p.myws = (char*)d_ws_all[rank];
  for (int i = 0; i < NR; ++i) p.ws[i] = (char*)d_ws_all[i];
  p.rank = rank;
  p.perm = 0;
  for (int i = 0, lo = 0, hi = 4; i < NR; ++i) {
    const bool same = ((0x4d >> i) & 1) == ((0x4d >> rank) & 1);
    const int xc = PERM ? (same ? lo++ : hi++) : i;
    p.sws[xc] = p.ws[i];
    p.perm |= (u32)i << (4 * xc);
  }
  hipLaunchKernelGGL(ag_gemm_q8, dim3(NWG), dim3(NTHREADS), LDS_BYTES, stream, p);
  e = hipGetLastError();
  if (e != hipSuccess) fprintf(stderr, "launch: %s\n", hipGetErrorString(e));
}
```

```cpp
#include <hip/hip_runtime.h>
#include <rccl/rccl.h>
#include <cstdio>
#include <type_traits>

namespace {

typedef unsigned int u32;
typedef unsigned long long u64;
typedef __bf16 bf16x8 __attribute__((ext_vector_type(8)));
typedef __bf16 bf16x2 __attribute__((ext_vector_type(2)));
typedef float f32x4 __attribute__((ext_vector_type(4)));
typedef float f32x2 __attribute__((ext_vector_type(2)));
typedef u32 u32x2 __attribute__((ext_vector_type(2)));
typedef u32 u32x3 __attribute__((ext_vector_type(3)));
typedef u32 u32x4 __attribute__((ext_vector_type(4)));
typedef u32 u32x16 __attribute__((ext_vector_type(16)));
#define GAS __attribute__((address_space(1)))
#define LAS __attribute__((address_space(3)))

constexpr int NR = 8;
constexpr int KD = 4096;
constexpr int NL = 2048;
constexpr int NKS = 64;
constexpr int NCH = 16;
constexpr int NWG = 256;
constexpr int NGW = 240;
#define SPEC_THR 40
#define PART 1
#define TOP_T 59
#define TOP_MIN 4
#define HP_SLEEP 16
#define AG_SLEEP 6
#define GS_SLEEP 1
#define LOC_AUX 0
#define NPW 2
#define PWIN 3
#define PQ 6
#define PUSH (PQ / 4)
#define D_SLEEP 40
#define X1_SLEEP 50
#define P1_SLEEP 80
#define W2_SLEEP 50
#define DW_SLEEP 100
#define H_SLEEP 0
#define POLL_GAP 2
#define C_SLEEP 12
#define E_AUX 0
#define PU_SLEEP 36
#define CODE_PF 0
#define CODE_BYTES 17408
#define LS_SLEEP 1
#define PACE 0
#define PACE_U 24
#define PU0_SLEEP 0
#define L_SLEEP 100
#define XP_SLEEP 2
#define PERM 1
constexpr int BAR_POS = 2;
constexpr u32 W_KS = 16 * 16384, W_NT = 16384;
constexpr int NTHREADS = 576;
constexpr int X0_END = 8;
constexpr int X1_END = 16;
constexpr u32 A_BYTES = 32768, E_POS = 49152, STAGE = 51200;
constexpr u32 LDS_CTL = 3 * STAGE;
constexpr u32 LDS_BYTES = LDS_CTL + 512;
constexpr u32 NOWHERE = 0x80000000u;

constexpr size_t XB_OFF = 0;
constexpr size_t XB_BYTES = 4ull << 20;
constexpr size_t WB_OFF = 4ull << 20;
constexpr size_t WB_BYTES = 16ull << 20;
constexpr size_t CT_OFF = 20ull << 20;
constexpr size_t CNT_OFF = CT_OFF;
constexpr size_t TMO_OFF = CT_OFF + 4096;
constexpr size_t INV_OFF = CT_OFF + 5120;
constexpr size_t XFL_OFF = CT_OFF + 8192;
constexpr size_t WFL_OFF = CT_OFF + 40960;
constexpr size_t DBG_OFF = CT_OFF + 65536;
constexpr size_t AMX_OFF = CT_OFF + 196608;
constexpr u32 LOC_A = 0, LOC_B = 16384;
constexpr u32 GRP = 32768;
constexpr u32 PDN = 40960;
constexpr u32 FIN = 0x80000000u, BAD = 0x7fffffffu;
constexpr size_t WS_NEED = CT_OFF + 262144;
constexpr size_t XP_OFF = 28ull << 20;
constexpr size_t XP_SRC = 262144;
constexpr size_t WS_END = XP_OFF + NR * XP_SRC;

constexpr u64 DEADLINE_TICKS = 100000000ull;

struct Params {
  const float* x;
  const float* w;
  unsigned char* out;
  char* myws;
  char* ws[NR];
  char* sws[NR];
  u32 perm;
  int rank;
};

#define WAIT_VM(n) asm volatile("s_waitcnt vmcnt(%0)" ::"n"(n) : "memory")
#define WAIT_VM_LGKM(n) asm volatile("s_waitcnt vmcnt(" #n ") lgkmcnt(0)" ::: "memory")
#define WAIT_LGKM0() asm volatile("s_waitcnt lgkmcnt(0)" ::: "memory")
#define BARRIER() do { asm volatile("" ::: "memory"); __builtin_amdgcn_s_barrier(); asm volatile("" ::: "memory"); } while (0)
#define SCHED() __builtin_amdgcn_sched_barrier(0)
#define RLX_AGENT __ATOMIC_RELAXED, __HIP_MEMORY_SCOPE_AGENT
#define RLX_SYSTEM __ATOMIC_RELAXED, __HIP_MEMORY_SCOPE_SYSTEM

#define ARRV(t) do { } while (0)
#define ARRV_OUT() do { } while (0)
#define STAMP(i) do { } while (0)

__device__ __forceinline__ u32 pack_bf16(float a, float b) {
  f32x2 f = {a, b};
  bf16x2 h = __builtin_convertvector(f, bf16x2);
  return __builtin_bit_cast(u32, h);
}

__device__ __forceinline__ u32 wave_max_u32(u32 v) {
  v = max(v, (u32)__builtin_amdgcn_update_dpp(0, (int)v, 0x111, 0xf, 0xf, false));
  v = max(v, (u32)__builtin_amdgcn_update_dpp(0, (int)v, 0x112, 0xf, 0xf, false));
  v = max(v, (u32)__builtin_amdgcn_update_dpp(0, (int)v, 0x114, 0xf, 0xf, false));
  v = max(v, (u32)__builtin_amdgcn_update_dpp(0, (int)v, 0x118, 0xf, 0xf, false));
  const u32 a = (u32)__builtin_amdgcn_readlane((int)v, 15), b = (u32)__builtin_amdgcn_readlane((int)v, 31);
  const u32 c = (u32)__builtin_amdgcn_readlane((int)v, 47), d = (u32)__builtin_amdgcn_readlane((int)v, 63);
  return max(max(a, b), max(c, d));
}

__device__ __forceinline__ u32 wave_max_63(u32 v) {
  v = max(v, (u32)__builtin_amdgcn_update_dpp(0, (int)v, 0x111, 0xf, 0xf, false));
  v = max(v, (u32)__builtin_amdgcn_update_dpp(0, (int)v, 0x112, 0xf, 0xf, false));
  v = max(v, (u32)__builtin_amdgcn_update_dpp(0, (int)v, 0x114, 0xf, 0xf, false));
  v = max(v, (u32)__builtin_amdgcn_update_dpp(0, (int)v, 0x118, 0xf, 0xf, false));
  v = max(v, (u32)__builtin_amdgcn_update_dpp(0, (int)v, 0x142, 0xa, 0xf, false));
  v = max(v, (u32)__builtin_amdgcn_update_dpp(0, (int)v, 0x143, 0xc, 0xf, false));
  return v;
}

__device__ __forceinline__ u32 sword(const GAS u32* p) {
  u32 v;
  asm volatile("s_load_dword %0, %1, 0x0 glc\n\ts_waitcnt lgkmcnt(0)" : "=&s"(v) : "s"(p) : "memory");
  return v;
}
__device__ __forceinline__ void sflags3(u32x16& v, u32x16& w, u32x16& z, const GAS u32* p, u32 off, const GAS u32* q, u32 off2) {
  asm volatile("s_load_dwordx16 %0, %3, %4 glc\n\ts_load_dwordx16 %1, %3, %4 offset:0x40 glc\n\ts_load_dwordx16 %2, %5, %6 glc\n\ts_waitcnt lgkmcnt(0)"
               : "=&s"(v), "=&s"(w), "=&s"(z)
               : "s"(p), "s"(__builtin_amdgcn_readfirstlane((int)off)), "s"(q), "s"(__builtin_amdgcn_readfirstlane((int)off2)) : "memory");
}
__device__ __forceinline__ void sflags3p(u32x16& v, u32x16& w, u32x16& z, u32& m0, u32& m1, const GAS u32* p, u32 off) {
  u64 m;
  asm volatile("s_load_dwordx16 %0, %4, %5 glc\n\ts_load_dwordx16 %1, %4, %5 offset:0x40 glc\n\ts_load_dwordx16 %2, %4, %5 offset:0x80 glc\n\t"
               "s_load_dwordx2 %3, %4, 0x200 glc\n\ts_waitcnt lgkmcnt(0)"
               : "=&s"(v), "=&s"(w), "=&s"(z), "=&s"(m)
               : "s"(p), "s"(__builtin_amdgcn_readfirstlane((int)off)) : "memory");
  m0 = (u32)m;
  m1 = (u32)(m >> 32);
}
__device__ __forceinline__ int lead4(const u32x16& v, u32 seq, int skip = 0) {
  const bool a = skip > 0 || min(min(v[0], v[1]), min(v[2], v[3])) >= seq, b = skip > 1 || min(min(v[4], v[5]), min(v[6], v[7])) >= seq;
  const bool c = skip > 2 || min(min(v[8], v[9]), min(v[10], v[11])) >= seq, d = min(min(v[12], v[13]), min(v[14], v[15])) >= seq;
  return a ? (b ? (c ? (d ? 4 : 3) : 2) : 1) : 0;
}
__device__ __forceinline__ u32 min16(const u32x16& v) {
  u32 m, n;
  asm("s_min_u32 %0, %2, %3\n\ts_min_u32 %1, %4, %5\n\ts_min_u32 %0, %0, %6\n\ts_min_u32 %1, %1, %7\n\t"
      "s_min_u32 %0, %0, %8\n\ts_min_u32 %1, %1, %9\n\ts_min_u32 %0, %0, %10\n\ts_min_u32 %1, %1, %11\n\t"
      "s_min_u32 %0, %0, %12\n\ts_min_u32 %1, %1, %13\n\ts_min_u32 %0, %0, %14\n\ts_min_u32 %1, %1, %15\n\t"
      "s_min_u32 %0, %0, %16\n\ts_min_u32 %1, %1, %17\n\ts_min_u32 %0, %0, %1"
      : "=&s"(m), "=&s"(n)
      : "s"(v[0]), "s"(v[1]), "s"(v[2]), "s"(v[3]), "s"(v[4]), "s"(v[5]), "s"(v[6]), "s"(v[7]), "s"(v[8]), "s"(v[9]), "s"(v[10]),
        "s"(v[11]), "s"(v[12]), "s"(v[13]), "s"(v[14]), "s"(v[15])
      : "scc");
  return m;
}

__device__ __forceinline__ int x_known(int xn, const u32x16& v, const u32x16& w, u32 seq) {
  const int x0 = min(xn, X0_END - 1);
  if (xn < X0_END && min16(v) >= seq) xn = (x0 < X0_END - 1 && min16(w) >= seq) ? x0 + 2 : x0 + 1;
  if (xn >= X0_END && x0 == X0_END - 1 && w[0] >= seq) xn = w[1] >= seq ? NKS : X1_END;
  return xn;
}

__device__ __forceinline__ bool give_up(u32 t0, GAS u32* tmo, u32 seq) {
  const u32 dt = (u32)__builtin_amdgcn_s_memrealtime() - t0;
  if (__builtin_expect(dt < 100000u, 1)) return false;
  __builtin_amdgcn_s_sleep(64);
  const bool stop = dt > (u32)DEADLINE_TICKS || sword(tmo) == seq;
  if (stop) __hip_atomic_store(tmo, seq, RLX_AGENT);
  return stop;
}

__device__ __forceinline__ void w_load(f32x4* v, __amdgpu_buffer_rsrc_t rs, u32 off, int row) {
#pragma unroll
  for (int j = 0; j < 8; ++j)
    v[j] = __builtin_bit_cast(f32x4, __builtin_amdgcn_raw_buffer_load_b128(rs, off, (row + j) * (NL * 4), 0));
}

__device__ __forceinline__ u32x4 w_pack(const f32x4* v, int r) {
  u32x4 d;
  d.x = pack_bf16(v[0][r], v[1][r]);
  d.y = pack_bf16(v[2][r], v[3][r]);
  d.z = pack_bf16(v[4][r], v[5][r]);
  d.w = pack_bf16(v[6][r], v[7][r]);
  return d;
}

__device__ __forceinline__ void w_store(const f32x4* v, __amdgpu_buffer_rsrc_t rs, u32 woff, int chunk) {
#pragma unroll
  for (int r = 0; r < 4; ++r) __builtin_amdgcn_raw_buffer_store_b128(w_pack(v, r), rs, woff + r * 2048, chunk * (4 * W_KS), 16  );
}

__device__ __forceinline__ f32x4 e_load(__amdgpu_buffer_rsrc_t rs, u32 off, int row) {
  f32x4 v;
#pragma unroll
  for (int j = 0; j < 4; ++j) v[j] = __builtin_bit_cast(float, __builtin_amdgcn_raw_buffer_load_b32(rs, off, (row + j) * (NL * 4), 0));
  return v;
}
__device__ __forceinline__ u32x2 e_pack(const f32x4& v) { return (u32x2){pack_bf16(v[0], v[1]), pack_bf16(v[2], v[3])}; }
__device__ __forceinline__ void e_store(const f32x4& v, __amdgpu_buffer_rsrc_t rs, u32 off, int chunk) {
  __builtin_amdgcn_raw_buffer_store_b64(e_pack(v), rs, off, chunk * (4 * W_KS), 16  );
}
__device__ __forceinline__ void ticket(u32& old, GAS u32* p, u32 v) {
  asm volatile("global_atomic_swap %0, %1, %2, off sc0 sc1" : "+v"(old) : "v"(p), "v"(v) : "memory");
}

__device__ __forceinline__ void ctl_write(u32 addr, u32 v) {
  asm volatile("ds_write_b32 %0, %1" ::"v"(addr), "v"(v) : "memory");
}
__device__ __forceinline__ u32 ctl_read(u32 addr) {
  u32 v;
  asm volatile("ds_read_b32 %0, %1\n\ts_waitcnt lgkmcnt(0)" : "=v"(v) : "v"(addr) : "memory");
  return v;
}
__device__ __forceinline__ void x_load(f32x4* v, const float* __restrict__ x, int xh, int xks, int msub, int lane) {
  const float* xs = x + (size_t)(xh * 256 + msub * 16 + (lane & 15)) * KD + xks * 64 + (lane >> 4) * 8;
#pragma unroll
  for (int j = 0; j < 4; ++j) v[j] = *(const f32x4*)(xs + (j >> 1) * 32 + (j & 1) * 4);
}

__device__ __forceinline__ u32x4 x_pack(const f32x4* v, int kh) {
  u32x4 d;
  d.x = pack_bf16(v[kh * 2][0], v[kh * 2][1]);
  d.y = pack_bf16(v[kh * 2][2], v[kh * 2][3]);
  d.z = pack_bf16(v[kh * 2 + 1][0], v[kh * 2 + 1][1]);
  d.w = pack_bf16(v[kh * 2 + 1][2], v[kh * 2 + 1][3]);
  return d;
}

__device__ __forceinline__ void x_store(const f32x4* v, __amdgpu_buffer_rsrc_t rs, int xh, int xks, int msub, int lane) {
  const u32 xoff = (u32)(((xh * 64 + xks) * 16 + msub) * 2048 + lane * 16);
#pragma unroll
  for (int kh = 0; kh < 2; ++kh) __builtin_amdgcn_raw_buffer_store_b128(x_pack(v, kh), rs, xoff + kh * 1024, 0, 17  );
}

__device__ __forceinline__ u64 both_ge(u64 f, u32 seq) { return __ballot((u32)f >= seq && (u32)(f >> 32) >= seq); }
__device__ __forceinline__ int lead_ones(u64 m) { return m == ~0ull ? 64 : __builtin_ctzll(~m); }

__global__ void __launch_bounds__(NTHREADS) __attribute__((amdgpu_waves_per_eu(3, 3))) ag_gemm_q8(Params p) {
  extern __shared__ __attribute__((aligned(16))) char lds[];
  const int b = blockIdx.x;
  const int tid = threadIdx.x, lane = tid & 63;
  const int wid = __builtin_amdgcn_readfirstlane(tid >> 6);
  const int idx = b >> 3;
  const int half = idx & 1, nt = idx >> 1, shr = (b & 7) * 2 + half;
  const float* const px = p.x;
  const float* const pw = p.w;
  unsigned char* const pout = p.out;
  char* const myws = p.myws;
  char* const srcws = p.sws[b & 7];
  const int rank = p.rank;
  const u32 perm = p.perm;
  const u64 w0 = (u64)p.ws[0], w1 = (u64)p.ws[1], w2 = (u64)p.ws[2], w3 = (u64)p.ws[3];
  const u64 w4 = (u64)p.ws[4], w5 = (u64)p.ws[5], w6 = (u64)p.ws[6], w7 = (u64)p.ws[7];
  asm volatile("" ::"s"(px), "s"(pw), "s"(pout), "s"(myws), "s"(srcws), "s"(rank), "s"(perm), "s"(w0), "s"(w1), "s"(w2), "s"(w3),
               "s"(w4), "s"(w5), "s"(w6), "s"(w7));
  const int src = (perm >> (4 * (b & 7))) & 7, mt = src * 2 + half;
  auto peer_ws = [&]() {
    int lo = (int)w0, hi = (int)(w0 >> 32);
#define PUT(i) asm("v_writelane_b32 %0, %2, " #i "\n\tv_writelane_b32 %1, %3, " #i : "+v"(lo), "+v"(hi) : "s"((int)w##i), "s"((int)(w##i >> 32)))
    PUT(1); PUT(2); PUT(3); PUT(4); PUT(5); PUT(6); PUT(7);
#undef PUT
    return (char*)(((u64)(u32)hi << 32) | (u32)lo);
  };
  GAS u32* const cnt = (GAS u32*)(myws + CNT_OFF) + b;
  GAS u32* const ownfl = (GAS u32*)(myws + XFL_OFF) + rank * 1024;
  const __amdgpu_buffer_rsrc_t xrs = __builtin_amdgcn_make_buffer_rsrc(myws + XB_OFF, 0, (int)XB_BYTES, 0x00020000);

  const u32 ctl = (u32)(size_t)(LAS char*)lds + LDS_CTL;
  if (nt == 15) {
    if (wid == 4) {
      ctl_write(ctl, PQ);
      WAIT_LGKM0();
    }
    BARRIER();
  }
  if (wid < 2 && (wid || (b >> 5) * 16 + (b & 15) >= PQ * 4)) {
    f32x4 v[4];
    const int xh = (b >> 4) & 1, xks = wid * X0_END + (b >> 5), msub = b & 15;
    if (wid) __builtin_amdgcn_s_sleep(X1_SLEEP);
    x_load(v, px, xh, xks, msub, lane);
    const u32 seq = __hip_atomic_load(cnt, RLX_AGENT);
    SCHED();
    GAS u32* pub = wid ? ownfl + xh * 512 + 128 + xks * 16 + msub : (GAS u32*)(peer_ws() + XFL_OFF) + (rank * 2 + xh) * 512 + xks * 16 + msub;
    asm volatile("" : "+v"(pub));
    SCHED();
    x_store(v, xrs, xh, xks, msub, lane);
    SCHED();
    WAIT_VM(0);
    if (lane < (wid ? 1 : 8)) __hip_atomic_store(pub, seq + 1u, RLX_SYSTEM);
  }
  u32 seq8 = 0;
  if (wid == 8) {
    f32x4 v[8];
    const int xh = idx >> 4, msub = idx & 15;
    const bool one = msub < PQ * 4, two = msub + 16 < PQ * 4;
    STAMP(0);
    if (one) x_load(v, px, xh, 0, msub, lane);
    if (two) x_load(v + 4, px, xh, 1, msub, lane);
    seq8 = __hip_atomic_load(cnt, RLX_AGENT) + 1u;
    if (PART) {
      char* const pw8 = peer_ws() + AMX_OFF + GRP + (rank * 8 + (b & 7)) * 64 + 8;
      if (lane < 8) asm volatile("ds_write_b64 %0, %1" ::"v"(ctl + 32 * 4 + lane * 8), "v"(pw8) : "memory");
    }
    SCHED();
    if (one) {
      const u32 off = (u32)(XP_OFF + rank * XP_SRC) + (u32)(xh * (XP_SRC / 2) + msub * 2048) + lane * 16;
      const u32 flo = lane == 0 ? (u32)XFL_OFF + (u32)((rank * 2 + xh) * 512 + msub) * 4u : NOWHERE;
      const __amdgpu_buffer_rsrc_t r0 = __builtin_amdgcn_make_buffer_rsrc(srcws, 0, (int)WS_END, 0x00020000);
      const u32x4 d0 = x_pack(v, 0), d1 = x_pack(v, 1);
      if (PU0_SLEEP) __builtin_amdgcn_s_sleep(PU0_SLEEP);
      __builtin_amdgcn_raw_buffer_store_b128(d0, r0, off, 0, 17);
      __builtin_amdgcn_raw_buffer_store_b128(d1, r0, off + 1024, 0, 17);
      if (two) {
        const u32x4 d2 = x_pack(v + 4, 0), d3 = x_pack(v + 4, 1);
        SCHED();
        __builtin_amdgcn_s_sleep(PU_SLEEP);
        __builtin_amdgcn_raw_buffer_store_b128(d2, r0, off, A_BYTES, 17);
        __builtin_amdgcn_raw_buffer_store_b128(d3, r0, off + 1024, A_BYTES, 17);
        WAIT_VM(2);
        __builtin_amdgcn_raw_buffer_store_b32(seq8, r0, flo, 0, 17);
        WAIT_VM(1);
        __builtin_amdgcn_raw_buffer_store_b32(seq8, r0, flo, 64, 17);
      } else {
        WAIT_VM(0);
        __builtin_amdgcn_raw_buffer_store_b32(seq8, r0, flo, 0, 17);
      }
      STAMP(1);
    }
  }

  GAS u32* const xfl = (GAS u32*)(myws + XFL_OFF) + mt * 512;
  GAS u32* const tmo = (GAS u32*)(myws + TMO_OFF);
  GAS u32* const tick = (GAS u32*)(myws + INV_OFF) + (__builtin_amdgcn_s_getreg(20 | (3 << 11)) & 15) * 32;
  const __amdgpu_buffer_rsrc_t mrs = __builtin_amdgcn_make_buffer_rsrc(myws + AMX_OFF, 0, 65536, 0x00020000);
  auto drop = [&]() {
    WAIT_VM(0);
    asm volatile("buffer_inv sc0 sc1" ::: "memory");
    WAIT_VM(0);
  };
  char* const xsrc = srcws + XB_OFF + (size_t)half * (XB_BYTES / 2);
  const __amdgpu_buffer_rsrc_t ars = __builtin_amdgcn_make_buffer_rsrc(xsrc, 0, (int)(XB_BYTES / 2), 0x00020000);

  if (nt == 15) {
    if (wid < 2 || wid > 5 || (wid < 4 && src == rank)) return;
    const u32 seq = (u32)__builtin_amdgcn_readfirstlane((int)__hip_atomic_load(cnt, RLX_AGENT)) + 1u;
    if (wid == 5) {
      f32x4 cv[24];
      const int xl = (b >> 1) * 6;
      __builtin_amdgcn_s_sleep(L_SLEEP);
#pragma unroll
      for (int j = 0; j < 6; ++j) x_load(cv + 4 * j, px, b & 1, X1_END + ((xl + j) >> 4), (xl + j) & 15, lane);
      SCHED();
#pragma unroll
      for (int j = 0; j < 6; ++j) x_store(cv + 4 * j, xrs, b & 1, X1_END + ((xl + j) >> 4), (xl + j) & 15, lane);
      WAIT_VM(0);
      if (lane == 0) __hip_atomic_store(ownfl + (b & 1) * 512 + 384 + (b >> 1), seq, RLX_SYSTEM);
      return;
    }
    __builtin_amdgcn_s_setprio(3);
    const u32 t0 = (u32)__builtin_amdgcn_s_memrealtime();
    if (wid < 4) {
      const int pv = wid - 2;
      LAS void* dst = (LAS void*)(lds + pv * 256);
      const u32 vo = lane * 128;
      int np = PQ;
#pragma nounroll
      for (int u = PQ + pv; u < NKS * 4; u += NPW) {
        for (u32 spins = 1; u >= np; ++spins) {
          np = __builtin_amdgcn_readfirstlane((int)ctl_read(ctl));
          if (u < np) break;
          __builtin_amdgcn_s_sleep(LS_SLEEP);
          if ((spins & 255u) == 0 && give_up(t0, tmo, seq)) np = NKS * 4 + 1;
        }
        if (np > NKS * 4) break;
        if (PACE && u < PACE_U) __builtin_amdgcn_s_sleep(PACE);
        WAIT_VM(PWIN - 1);
        __builtin_amdgcn_raw_ptr_buffer_load_lds(ars, dst, 4, vo, u * 8192, 0, 0);
      }
      WAIT_VM(0);
      if (np > NKS * 4) asm volatile("buffer_inv sc0 sc1" ::: "memory");
      BARRIER();
      return;
    }
    if (src != rank) {
      if (D_SLEEP) __builtin_amdgcn_s_sleep(D_SLEEP);
      for (int np = PQ; np < NKS * 4;) {
        u32x16 f0, f1, f2;
        u32 m0, m1;
        const int l0 = min(np >> 2, X0_END - 3);
        sflags3p(f0, f1, f2, m0, m1, xfl, (u32)l0 * 64u);
        int n = l0 * 4 + lead4(f0, seq, l0 == PQ / 4 ? PQ & 3 : 0);
        if (n == l0 * 4 + 4) n += lead4(f1, seq);
        if (n == l0 * 4 + 8) n += lead4(f2, seq);
        if (n == X0_END * 4 && m0 >= seq) n = m1 >= seq ? NKS * 4 : X1_END * 4;
        if (n > np) {
          np = n;
          ctl_write(ctl, (u32)np);
          continue;
        }
        if (give_up(t0, tmo, seq)) {
          ctl_write(ctl, NKS * 4 + 1);
          break;
        }
        if (POLL_GAP) __builtin_amdgcn_s_sleep(POLL_GAP);
      }
    } else {
      GAS u32* const pub = (GAS u32*)(p.ws[lane & 7] + XFL_OFF) + (rank * 2 + half) * 512 + 128;
      for (int ph = 0; ph < 2;) {
        if (both_ge(__hip_atomic_load((GAS u64*)(ownfl + half * 512 + 256 + ph * 128) + lane, RLX_SYSTEM), seq) == ~0ull) {
          if (lane < 8) __hip_atomic_store(pub + ph, seq, RLX_SYSTEM);
          ++ph;
          continue;
        }
        if (give_up(t0, tmo, seq)) break;
        __builtin_amdgcn_s_sleep(4);
      }
    }
    u32 old = seq;
    if (lane == 0) ticket(old, tick, seq);
    BARRIER();
    if (lane == 0) __hip_atomic_store(cnt, seq, RLX_AGENT);
    const u32 g = b & 7;
    if (half) {
      __builtin_amdgcn_raw_buffer_store_b32(seq, mrs, lane == 0 ? 0u : NOWHERE, PDN + g * 64, 17);
    } else {
      GAS u64* post = (GAS u64*)(p.ws[lane & 7] + AMX_OFF + GRP + (rank * 8 + g) * 64);
      asm volatile("" : "+v"(post));
      bool bad = false;
      while (!bad && __any(__builtin_amdgcn_raw_buffer_load_b32(mrs, 0, PDN + g * 64, 17) < seq)) {
        bad = give_up(t0, tmo, seq);
        __builtin_amdgcn_s_sleep(AG_SLEEP);
      }
      const u32 l1 = lane * 16, l2 = min(l1 + 1024u, (u32)NGW * 8u - 16u);
      u32 m = 0, s0 = 0, s1 = 0, s2 = 0, s3 = 0;
      for (u32 spins = 1; !bad; ++spins) {
        const u32 so = g * 2048 + ((spins & 31u) ? LOC_A : LOC_B);
        const u32x4 g0 = __builtin_amdgcn_raw_buffer_load_b128(mrs, l1, so, 17);
        const u32x4 g1 = __builtin_amdgcn_raw_buffer_load_b128(mrs, l2, so, 17);
        m = max(max(m, g0.y >= seq ? g0.x : 0u), g0.w >= seq ? g0.z : 0u);
        m = max(max(m, g1.y >= seq ? g1.x : 0u), g1.w >= seq ? g1.z : 0u);
        s0 = max(s0, g0.y); s1 = max(s1, g0.w); s2 = max(s2, g1.y); s3 = max(s3, g1.w);
        if (__all(min(min(s0, s1), min(s2, s3)) >= seq)) break;
        if ((spins & 31u) == 0) bad = give_up(t0, tmo, seq);
        if (!__any(max(max(s0, s1), max(s2, s3)) >= seq)) __builtin_amdgcn_s_sleep(AG_SLEEP);
      }
      m = bad ? BAD : wave_max_u32(m);
      if (lane < 8) __hip_atomic_store(post, ((u64)seq << 32) | m, RLX_SYSTEM);
      STAMP(11);
    }
    WAIT_VM(0);
    asm volatile("" : "+v"(old));
    if ((u32)__builtin_amdgcn_readfirstlane((int)old) != seq) {
      while (!__all(__builtin_amdgcn_raw_buffer_load_b32(mrs, (lane & 7) * 64, GRP + rank * 512 + 4, 17) >= seq)) {
        if (give_up(t0, tmo, seq)) break;
        __builtin_amdgcn_s_sleep(C_SLEEP);
      }
      drop();
    }
    STAMP(15);
    return;
  }

  const int kq1 = lane >> 5, wH = (lane >> 4) & 1, wi = lane & 15;
  const __amdgpu_buffer_rsrc_t wfr = __builtin_amdgcn_make_buffer_rsrc((void*)pw, 0, KD * NL * 4, 0x00020000);
  const u32 wfo = (u32)(kq1 * 8 * NL + nt * 128 + wH * 64 + wi * 4) * 4u;
  const u32 woff = (u32)(nt * W_NT + (shr >> 2) * W_KS + wH * 8192 + ((shr >> 1) & 1) * 1024 +
                         (((shr & 1) * 2 + kq1) * 16 + wi) * 16);
  const bool ext = nt < 8;
  const u32 efo = ext ? (u32)((wH * 8 + kq1 * 4) * NL + 15 * 128 + nt * 16 + wi) * 4u : NOWHERE;
  const u32 eoff = ext ? (u32)(15 * W_NT + nt * 2048 + (shr >> 2) * W_KS + ((shr >> 1) & 1) * 1024 +
                               (((shr & 1) * 2 + wH) * 16 + wi) * 16 + kq1 * 8) : NOWHERE;
  const __amdgpu_buffer_rsrc_t wrs =
      __builtin_amdgcn_make_buffer_rsrc(myws + WB_OFF, 0, (int)WB_BYTES, 0x00020000);
  GAS u32* const wfl = (GAS u32*)(myws + WFL_OFF) + nt * 256;

  if (wid == 8) {
    __builtin_amdgcn_s_setprio(3);
    const u32 seq = (u32)__builtin_amdgcn_readfirstlane((int)seq8);
    bool failed = false, cleaner = false;
    int xn = 0, cw = 0;
#define KBAR(t) do { BARRIER(); SCHED(); } while (0)
    f32x4 wv[9];
#pragma unroll
    for (int j = 0; j < 9; ++j) wv[j] = (f32x4){0.f, 0.f, 0.f, 0.f};
    if (H_SLEEP) __builtin_amdgcn_s_sleep(H_SLEEP);
#pragma nounroll
    for (int t = -2; t < NKS - 1; ++t) {
      const int c = (t >> 2) + 3;
      if ((t & 3) == 0) {
        __builtin_amdgcn_s_waitcnt(0x0f70);
        asm volatile("" ::: "memory");
        if (lane == 0 && c <= NCH) __hip_atomic_store(t ? wfl + (c - 1) * 16 + shr : cnt, seq, RLX_AGENT);
        if (c < NCH) {
          w_load(wv, wfr, wfo, c * 256 + shr * 16);
          wv[8] = e_load(wfr, efo, c * 256 + shr * 16);
        }
      } else if ((t & 3) == 2 && t > 0 && c < NCH) {
        w_store(wv, wrs, woff, c);
        e_store(wv[8], wrs, eoff, c);
      }
      if (t == 1 && lane < 2)
        __hip_atomic_store(lane ? ownfl + (b & 1) * 512 + 384 + (b >> 1) : wfl + 2 * 16 + shr, seq, RLX_SYSTEM);
      if (t == 25 + idx && sword(tick) != seq) {
        u32 old = 0;
        if (lane == 0) old = __hip_atomic_exchange(tick, seq, RLX_SYSTEM);
        cleaner = (u32)__builtin_amdgcn_readfirstlane((int)old) != seq;
      }
      SCHED();
      const int s = min(t + 3, NKS - 1), gw = t < -1 ? 0 : (s >> 2) + 1;
      if (xn < NKS || cw <= min(gw, NCH - 1)) {
        u32 t0 = 0, spins = 0;
        for (;;) {
          u32x16 f0, f1, fw;
          const int on = xn + cw;
          sflags3(f0, f1, fw, xfl, (u32)min(xn, X0_END - 1) * 64u, wfl, (u32)min(cw, NCH - 1) * 64u);
          xn = x_known(xn, f0, f1, seq);
          if (min16(fw) >= seq) cw = min(cw + 1, NCH);
          if (xn > s && cw >= gw) break;
          if (xn + cw != on) continue;
          if (spins == 0) t0 = (u32)__builtin_amdgcn_s_memrealtime();
          if ((spins & 63u) == 63u) {
            GAS u64* const xf = (GAS u64*)xfl;
            const u32 vw = __hip_atomic_load(wfl + min(cw, NCH - 1) * 16 + (lane & 15), RLX_AGENT);
            const u64 v0 = __hip_atomic_load(xf + lane, RLX_SYSTEM), v1 = __hip_atomic_load(xf + 64, RLX_SYSTEM);
            if (__all(vw >= seq)) cw = min(cw + 1, NCH);
            int n = lead_ones(both_ge(v0, seq)) >> 3;
            if (n == X0_END && __all((u32)v1 >= seq)) n = __all((u32)(v1 >> 32) >= seq) ? NKS : X1_END;
            xn = max(xn, n);
          }
          ++spins;
          if (give_up(t0, tmo, seq)) { failed = true; xn = NKS; cw = NCH; break; }
          if (POLL_GAP) __builtin_amdgcn_s_sleep(POLL_GAP);
        }
      }
      if (PART && t == TOP_T) {
        const u32x4 a = __builtin_amdgcn_raw_buffer_load_b128(mrs, lane * 64, GRP, 17);
        const u32 c = wave_max_u32(max(a.y >= seq ? a.x : 0u, a.w >= seq ? a.z : 0u));
        ctl_write(ctl + 18 * 4, __builtin_popcountll(__ballot(a.y >= seq)) >= TOP_MIN ? c : ~0u);
      }
      if (t == NKS - 2) {
        ctl_write(ctl + 16 * 4, failed ? BAD : 0u);
        ctl_write(ctl + 17 * 4, seq);
        ctl_write(ctl + 20 * 4, 0u);
        WAIT_LGKM0();
      }
      ARRV(t == -2 ? -1 : t);
      KBAR(t == -2 ? -1 : t);
    }
    ARRV_OUT();
    if (HP_SLEEP) __builtin_amdgcn_s_sleep(HP_SLEEP);
    const u32 t0 = (u32)__builtin_amdgcn_s_memrealtime();
    u32 off = lane * 64, m = __hip_atomic_load(tmo, RLX_AGENT) == seq ? BAD : 0u;
    int n = 0;
    bool go = false;
    for (u32 spins = 1;; ++spins) {
      const u32x4 g = __builtin_amdgcn_raw_buffer_load_b128(mrs, off, GRP, 17);
      const bool ok = g.y >= seq;
      u32 c = ok ? g.x : 0u;
      off = ok ? NOWHERE : off;
      n += __builtin_popcountll(__ballot(ok));
      if (n == 64) {
        const u32 a = wave_max_63(max(m, c)) | FIN;
        if (lane == 63) ctl_write(ctl + 20 * 4, a);
        break;
      }
      c = max(c, g.w >= seq ? g.z : 0u);
      const bool up = __any(c > m);
      m = max(m, c);
      if (n >= SPEC_THR && (up || !go)) {
        go = true;
        const u32 a = wave_max_63(m);
        if (lane == 63) ctl_write(ctl + 20 * 4, a);
      }
      if ((spins & 15u) == 0 && give_up(t0, tmo, seq)) {
        ctl_write(ctl + 20 * 4, FIN | BAD);
        break;
      }
    }
    if (cleaner) drop();
    return;
  }

  f32x4 dv[2][9];
  if (wid >= 4) {
    if (DW_SLEEP) __builtin_amdgcn_s_sleep(DW_SLEEP);
#pragma unroll
    for (int d = 0; d < 2; ++d) {
      w_load(dv[d], wfr, wfo, d * 64 + (wid - 4) * 16);
      dv[d][8] = e_load(wfr, efo, d * 64 + (wid - 4) * 16);
    }
    SCHED();
  }
  f32x4 cv[9];
  const int xl = (b >> 1) * 6 + (wid - 4) * 2;
  u32 seqw = 0;
  if (wid == 2 || wid == 3) {
    const bool skip = wid == 2 && shr < 8;
    if (wid == 3) __builtin_amdgcn_s_sleep(P1_SLEEP);
    else if (W2_SLEEP) __builtin_amdgcn_s_sleep(W2_SLEEP);
    if (!skip) {
      w_load(cv, wfr, wfo, (wid - 2) * 256 + shr * 16);
      cv[8] = e_load(wfr, efo, (wid - 2) * 256 + shr * 16);
    }
    seqw = __hip_atomic_load(cnt, RLX_AGENT) + 1u;
    SCHED();
    if (!skip) {
      w_store(cv, wrs, woff, wid - 2);
      e_store(cv[8], wrs, eoff, wid - 2);
    }
    SCHED();
    if (wid == 2) {
      WAIT_VM(0);
      if (lane == 0) __hip_atomic_store(wfl + shr, seqw, RLX_AGENT);
      STAMP(12);
    }
  } else if (wid == 7) {
    w_load(cv, wfr, wfo, 2 * 256 + shr * 16);
    cv[8] = e_load(wfr, efo, 2 * 256 + shr * 16);
    SCHED();
  } else if (wid >= 4) {
    x_load(cv, px, b & 1, X1_END + (xl >> 4), xl & 15, lane);
    x_load(cv + 4, px, b & 1, X1_END + (xl >> 4), (xl & 15) + 1, lane);
    SCHED();
  }

  const __amdgpu_buffer_rsrc_t brs = __builtin_amdgcn_make_buffer_rsrc(
      myws + WB_OFF + (size_t)nt * W_NT, 0, (int)(WB_BYTES - nt * W_NT), 0x00020000);
  const __amdgpu_buffer_rsrc_t prs = __builtin_amdgcn_make_buffer_rsrc(myws + XP_OFF + (size_t)mt * (XP_SRC / 2), 0, (int)(XP_SRC / 2), 0x00020000);
  const u32 l16 = lane * 16;
  const u32 avo = wid * 4096 + l16, bvo = wid * 2048 + l16;
  const u32 evo = ext ? (u32)((15 - nt) * W_NT + nt * 2048 + wid * 256 + lane * 4) : NOWHERE;
  auto piece = [&](int t, u32 slot, int j, bool pushed = false) {
    if (j == 6) {
      __builtin_amdgcn_raw_ptr_buffer_load_lds(brs, (LAS void*)(lds + slot + E_POS + wid * 256), 4, evo, t * W_KS, 0, 0);
    } else if (j >= 4) {
      LAS void* lb = (LAS void*)(lds + slot + A_BYTES + wid * 2048);
      if (j == 4) __builtin_amdgcn_raw_ptr_buffer_load_lds(brs, lb, 16, bvo, t * W_KS, 0, 0);
      else __builtin_amdgcn_raw_ptr_buffer_load_lds(brs, lb, 16, bvo, t * W_KS, 1024, 0);
    } else {
      const u32 ro = ((j + nt) & 3) * 1024;
      LAS void* la = (LAS void*)(lds + slot + wid * 4096 + ro);
      __builtin_amdgcn_raw_ptr_buffer_load_lds(pushed ? prs : ars, la, 16, avo, t * A_BYTES + ro, 0, 0);
    }
    SCHED();
  };

  const int wr = wid >> 1, wc = wid & 1;
  const u32 aoff0 = wr * 8192 + wc * 4096 + l16, aoff1 = wr * 8192 + (1 - wc) * 4096 + l16, boff = A_BYTES + wc * 8192 + l16;
  f32x4 acc[4][4], ace[2];
#pragma unroll
  for (int i = 0; i < 4; ++i)
#pragma unroll
    for (int j = 0; j < 4; ++j) acc[i][j] = (f32x4){0.f, 0.f, 0.f, 0.f};
  ace[0] = ace[1] = (f32x4){0.f, 0.f, 0.f, 0.f};
  bf16x8 af[2][4], bfr[2][4], ef[2];
  auto frags = [&](u32 slot, int kh) {
#pragma unroll
    for (int i = 0; i < 4; ++i) {
      bfr[kh][i] = *(const bf16x8*)(lds + slot + boff + i * 2048 + kh * 1024);
      af[kh][i] = *(const bf16x8*)(lds + slot + (i < 2 ? aoff0 : aoff1) + (i & 1) * 2048 + kh * 1024);
    }
    ef[kh] = *(const bf16x8*)(lds + slot + E_POS + l16 + kh * 1024);
    SCHED();
  };
  auto mfma4 = [&](int kh, int mi) {
#pragma unroll
    for (int r = 0; r < 4; ++r)
      acc[mi][r] = __builtin_amdgcn_mfma_f32_16x16x32_bf16(bfr[kh][r], af[kh][mi], acc[mi][r], 0, 0, 0);
    if (mi >= 2) ace[mi - 2] = __builtin_amdgcn_mfma_f32_16x16x32_bf16(ef[kh], af[kh][mi - 2], ace[mi - 2], 0, 0, 0);
    SCHED();
  };

  BARRIER();
#pragma unroll
  for (int j = 0; j < 4; ++j) piece(0, 0, j, wid < PQ * 2);
#pragma unroll
  for (int j = 0; j < 4; ++j) piece(1, STAGE, j, wid + 8 < PQ * 2);
  if (wid >= 4) {
    char* const dst = lds + A_BYTES + wH * 8192 + ((wid >> 1) & 1) * 1024 + (((wid & 1) * 2 + kq1) * 16 + wi) * 16;
    char* const dse = lds + E_POS + ((wid >> 1) & 1) * 1024 + (((wid & 1) * 2 + wH) * 16 + wi) * 16 + kq1 * 8;
#pragma unroll
    for (int d = 0; d < 2; ++d) {
#pragma unroll
      for (int r = 0; r < 4; ++r) *(u32x4*)(dst + d * STAGE + r * 2048) = w_pack(dv[d], r);
      *(u32x2*)(dse + d * STAGE) = e_pack(dv[d][8]);
    }
    SCHED();
    if (wid == 7) {
      w_store(cv, wrs, woff, 2);
      e_store(cv[8], wrs, eoff, 2);
      SCHED();
      WAIT_VM_LGKM(9);
    } else {
      x_store(cv, xrs, b & 1, X1_END + (xl >> 4), xl & 15, lane);
      x_store(cv + 4, xrs, b & 1, X1_END + (xl >> 4), (xl & 15) + 1, lane);
      SCHED();
      WAIT_VM_LGKM(8);
    }
  } else if (wid == 3) {
    WAIT_VM(8);
    if (lane == 0) __hip_atomic_store(wfl + 16 + shr, seqw, RLX_AGENT);
    STAMP(13);
    WAIT_VM(5);
  } else {
    WAIT_VM_LGKM(4);
  }
  ARRV(-1);
  BARRIER();
  frags(0, 0);
#pragma unroll
  for (int j = 0; j < 4 - BAR_POS; ++j) piece(2, 2 * STAGE, j);

  u32 cur = 0, nx1 = STAGE, nx2 = 2 * STAGE;
  auto step = [&](int t, auto more) {
    constexpr int MORE = decltype(more)::value;
    frags(cur, 1);
    if (MORE >= 2) piece(t + 2, nx2, 6);
#pragma unroll
    for (int g = 0; g < 4; ++g) {
      mfma4(0, g);
      if (MORE >= 2 && 4 - BAR_POS + g < 6) piece(t + 2, nx2, 4 - BAR_POS + g);
    }
    constexpr int BP = MORE == 1 ? 4 : BAR_POS;
#pragma unroll
    for (int g = 0; g < BP; ++g) mfma4(1, g);
    if (MORE >= 1) {
      if (MORE >= 2) WAIT_VM_LGKM(7);
      else WAIT_VM_LGKM(0);
      ARRV(t);
      BARRIER();
      SCHED();
      frags(nx1, 0);
    }
#pragma unroll
    for (int g = BP; g < 4; ++g) {
      mfma4(1, g);
      if (MORE >= 3) piece(t + 3, cur, g - BP);
    }
    const u32 o = cur;
    cur = nx1; nx1 = nx2; nx2 = o;
  };
  static_assert(PQ <= 8, "the K loop reads the peers");
  for (int t = 0; t < NKS - 3; ++t) step(t, std::integral_constant<int, 3>());
  step(NKS - 3, std::integral_constant<int, 2>());
  step(NKS - 2, std::integral_constant<int, 1>());
  step(NKS - 1, std::integral_constant<int, 0>());
  ARRV_OUT();

  u32x3 f3;
  asm volatile("ds_read_b96 %0, %1" : "=v"(f3) : "v"(ctl + 16 * 4) : "memory");
  SCHED();
  float mx = 0.f;
#pragma unroll
  for (int mi = 0; mi < 4; ++mi)
#pragma unroll
    for (int r = 0; r < 4; ++r)
#pragma unroll
      for (int j = 0; j < 4; ++j) mx = fmaxf(mx, fabsf(acc[mi][r][j]));
  float me = 0.f;
#pragma unroll
  for (int i = 0; i < 8; ++i) me = fmaxf(me, fabsf(ace[i >> 2][i & 3]));
  if (ext) mx = fmaxf(mx, me);
  asm volatile("s_waitcnt lgkmcnt(0)" : "+v"(f3)::"memory");
  const u32x2 fs = {wave_max_63(max(__builtin_bit_cast(u32, mx), f3.x)), f3.y};
  {
    const u32 ro = lane == 63 ? (u32)((b & 7) * 2048 + idx * 64 + wid * 8) : NOWHERE;
    __builtin_amdgcn_raw_buffer_store_b64(fs, mrs, ro, LOC_A, LOC_AUX);
    __builtin_amdgcn_raw_buffer_store_b64(fs, mrs, ro, LOC_B, 17);
    SCHED();
  }
  STAMP(4);
  const u32 wm = (u32)__builtin_amdgcn_readlane((int)fs.x, 63), seq = (u32)__builtin_amdgcn_readfirstlane((int)fs.y);
  if (PART && wm > (u32)__builtin_amdgcn_readfirstlane((int)f3.z)) {
    GAS u64* pw8;
    asm volatile("ds_read_b64 %0, %1\n\ts_waitcnt lgkmcnt(0)" : "=v"(pw8) : "v"(ctl + 32 * 4 + (lane & 7) * 8) : "memory");
    if (lane < 8) __hip_atomic_store(pw8, ((u64)seq << 32) | wm, RLX_SYSTEM);
  }
  const __amdgpu_buffer_rsrc_t ors = __builtin_amdgcn_make_buffer_rsrc(pout, 0, 4096 * NL, 0x00020000);
  const u32 orow = (u32)(mt * 256 + wr * 64 + (lane & 15)) * NL;
  const u32 oo0 = orow + wc * 32 * NL + nt * 128 + wc * 64 + (lane >> 4) * 16, oo1 = oo0 + (1 - 2 * wc) * 32 * NL;
  const u32 oe = ext ? orow + wc * 32 * NL + 15 * 128 + nt * 16 + (lane >> 4) * 4 : NOWHERE;
  const u32 t0 = (u32)__builtin_amdgcn_s_memrealtime();
  u32 done = FIN;
#pragma nounroll
  for (u32 spins = 1;; ++spins) {
    u32 a = (u32)__builtin_amdgcn_readfirstlane((int)ctl_read(ctl + 20 * 4));
    if ((spins & 255u) == 0 && give_up(t0, tmo, seq)) a = FIN | BAD;
    const u32 gmax = a & ~FIN;
    if (gmax != done && a >= wm) {
      done = gmax;
      const float amax = __builtin_bit_cast(float, gmax);
      const float inv = amax > 0.f ? 448.0f / amax : 0.f;
      auto scaled = [&](const f32x4& v) {
        f32x2 lo, hi;
        asm("v_pk_mul_f32 %0, %1, %2 op_sel_hi:[1,0]" : "=v"(lo) : "v"(__builtin_shufflevector(v, v, 0, 1)), "v"((f32x2){inv, inv}));
        asm("v_pk_mul_f32 %0, %1, %2 op_sel_hi:[1,0]" : "=v"(hi) : "v"(__builtin_shufflevector(v, v, 2, 3)), "v"((f32x2){inv, inv}));
        return __builtin_shufflevector(lo, hi, 0, 1, 2, 3);
      };
      auto q4 = [&](float a, float c, float d, float e) {
        const int q = __builtin_amdgcn_cvt_pk_fp8_f32(a, c, __builtin_bit_cast(int, a), false);
        return (u32)__builtin_amdgcn_cvt_pk_fp8_f32(d, e, q, true);
      };
      if (gmax > 0x7f800000u) {
        const u32 z = 0x7f7f7f7fu;
#pragma unroll
        for (int mi = 0; mi < 4; ++mi)
          __builtin_amdgcn_raw_buffer_store_b128((u32x4){z, z, z, z}, ors, (mi < 2 ? oo0 : oo1) + (mi & 1) * 16 * NL, 0, 16);
#pragma unroll
        for (int i = 0; i < 2; ++i) __builtin_amdgcn_raw_buffer_store_b32(z, ors, oe, i * 16 * NL, E_AUX);
      } else {
#pragma unroll
        for (int mi = 0; mi < 4; ++mi) {
          const f32x4 s0 = scaled(acc[mi][0]), s1 = scaled(acc[mi][1]), s2 = scaled(acc[mi][2]), s3 = scaled(acc[mi][3]);
          u32x4 d;
#pragma unroll
          for (int j = 0; j < 4; ++j) d[j] = q4(s0[j], s1[j], s2[j], s3[j]);
          __builtin_amdgcn_raw_buffer_store_b128(d, ors, (mi < 2 ? oo0 : oo1) + (mi & 1) * 16 * NL, 0, 16  );
          SCHED();
        }
#pragma unroll
        for (int i = 0; i < 2; ++i) {
          const f32x4 s = scaled(ace[i]);
          __builtin_amdgcn_raw_buffer_store_b32(q4(s[0], s[1], s[2], s[3]), ors, oe, i * 16 * NL, E_AUX);
        }
      }
    }
    if (a >= FIN) break;
    __builtin_amdgcn_s_sleep(GS_SLEEP);
  }
  if (done > 0x7f800000u) asm volatile("buffer_inv sc0 sc1" ::: "memory");
}

}

extern "C" void kernel_launch(int rank, int nranks, ncclComm_t comm, hipStream_t stream,
                              void* const* d_in, const int* in_sizes, int n_in,
                              void* d_out, int out_size,
                              const int* in_dtypes, int out_dtype,
                              void* const* d_out_all, void* const* d_ws_all, size_t ws_bytes) {
  (void)comm; (void)in_dtypes; (void)out_dtype; (void)d_out_all;
  if (nranks != NR || n_in < 2 || ws_bytes < WS_END || in_sizes[0] != 512 * KD || in_sizes[1] != KD * NL ||
      out_size != 4096 * NL || (unsigned)rank >= (unsigned)nranks) {
    fprintf(stderr, "unexpected problem geometry\n");
    return;
  }
  hipError_t e = hipFuncSetAttribute((const void*)ag_gemm_q8, hipFuncAttributeMaxDynamicSharedMemorySize, (int)LDS_BYTES);
  if (e != hipSuccess) { fprintf(stderr, "hipFuncSetAttribute: %s\n", hipGetErrorString(e)); return; }
  Params p;
  p.x = (const float*)d_in[0];
  p.w = (const float*)d_in[1];
  p.out = (unsigned char*)d_out;
  p.myws = (char*)d_ws_all[rank];
  for (int i = 0; i < NR; ++i) p.ws[i] = (char*)d_ws_all[i];
  p.rank = rank;
  p.perm = 0;
  for (int i = 0, lo = 0, hi = 4; i < NR; ++i) {
    const bool same = ((0x4d >> i) & 1) == ((0x4d >> rank) & 1);
    const int xc = PERM ? (same ? lo++ : hi++) : i;
    p.sws[xc] = p.ws[i];
    p.perm |= (u32)i << (4 * xc);
  }
  hipLaunchKernelGGL(ag_gemm_q8, dim3(NWG), dim3(NTHREADS), LDS_BYTES, stream, p);
  e = hipGetLastError();
  if (e != hipSuccess) fprintf(stderr, "launch: %s\n", hipGetErrorString(e));
}
```

```cpp
#include <hip/hip_runtime.h>
#include <rccl/rccl.h>
#include <cstdio>
#include <type_traits>

namespace {

typedef unsigned int u32;
typedef unsigned long long u64;
typedef __bf16 bf16x8 __attribute__((ext_vector_type(8)));
typedef __bf16 bf16x2 __attribute__((ext_vector_type(2)));
typedef float f32x4 __attribute__((ext_vector_type(4)));
typedef float f32x2 __attribute__((ext_vector_type(2)));
typedef u32 u32x2 __attribute__((ext_vector_type(2)));
typedef u32 u32x3 __attribute__((ext_vector_type(3)));
typedef u32 u32x4 __attribute__((ext_vector_type(4)));
typedef u32 u32x16 __attribute__((ext_vector_type(16)));
#define GAS __attribute__((address_space(1)))
#define LAS __attribute__((address_space(3)))

constexpr int NR = 8;
constexpr int KD = 4096;
constexpr int NL = 2048;
constexpr int NKS = 64;
constexpr int NCH = 16;
constexpr int NWG = 256;
constexpr int NGW = 240;
#define SPEC_THR 9
#define SPEC_REM 0
#define CAND 1
#define ALPHA 0.94f
#define TMO_S 0
#define BEST_S 1
#define WGATE 2
#define WDELAY 0
#define APRE 4
#define BP61 2
#define WPACE 0
#define OUT_AUX 16
#define WBL2 0
#define RD_SLEEP 1
#define HP_SLEEP 0
#define AG_SLEEP 6
#define GS_SLEEP 1
#define AROT 1
#define DROPV 0
#define HNT_AUX 16
#define NPW 2
#define PWIN 3
#define PQ 6
#define PUSH (PQ / 4)
#define D_SLEEP 40
#define X1_SLEEP 50
#define P1_SLEEP 80
#define W2_SLEEP 50
#define DW_SLEEP 100
#define H_SLEEP 0
#define POLL_GAP 2
#define C_SLEEP 12
#define E_AUX 0
#define PU_SLEEP 36
#define LS_SLEEP 1
#define PACE 0
#define PACE_U 24
#define PU0_SLEEP 0
#define L_SLEEP 100
#define XP_SLEEP 2
#define PERM 1
constexpr int BAR_POS = 2;
constexpr u32 W_KS = 16 * 16384, W_NT = 16384;
constexpr int NTHREADS = 576;
constexpr int X0_END = 8;
constexpr int X1_END = 16;
constexpr u32 A_BYTES = 32768, E_POS = 49152, STAGE = 51200;
#define LOOK_AT 57
#define HINT 1
#define HT_SLEEP 2
constexpr int T0 = LOOK_AT;
constexpr int NH = 2 * (NKS - T0) - 1;
constexpr int NNEAR = 7;
constexpr u32 LDS_CTL = 3 * STAGE;
constexpr u32 LDS_BYTES = LDS_CTL + 512 + 8192;
constexpr u32 NOWHERE = 0x80000000u;

constexpr size_t XB_OFF = 0;
constexpr size_t XB_BYTES = 4ull << 20;
constexpr size_t WB_OFF = 4ull << 20;
constexpr size_t WB_BYTES = 16ull << 20;
constexpr size_t CT_OFF = 20ull << 20;
constexpr size_t CNT_OFF = CT_OFF;
constexpr size_t TMO_OFF = CT_OFF + 4096;
constexpr size_t INV_OFF = CT_OFF + 5120;
constexpr size_t XFL_OFF = CT_OFF + 8192;
constexpr size_t WFL_OFF = CT_OFF + 40960;
constexpr size_t DBG_OFF = CT_OFF + 65536;
constexpr size_t AMX_OFF = CT_OFF + 196608;
constexpr u32 LOC = 0;
constexpr u32 GRP = 32768;
constexpr u32 PDN = 40960;
constexpr u32 PRE = 45056;
constexpr u32 HNT = 47104;
constexpr u32 BEST = 49152;
constexpr u32 FIN = 0x80000000u, BAD = 0x7fffffffu;
constexpr size_t WS_NEED = CT_OFF + 262144;
constexpr size_t XP_OFF = 28ull << 20;
constexpr size_t XP_SRC = 262144;
constexpr size_t WS_END = XP_OFF + NR * XP_SRC;

constexpr u64 DEADLINE_TICKS = 100000000ull;

struct Params {
  const float* x;
  const float* w;
  unsigned char* out;
  char* myws;
  char* ws[NR];
  char* sws[NR];
  u32 perm;
  int rank;
};

#define WAIT_VM(n) asm volatile("s_waitcnt vmcnt(%0)" ::"n"(n) : "memory")
#define WAIT_VM_LGKM(n) asm volatile("s_waitcnt vmcnt(" #n ") lgkmcnt(0)" ::: "memory")
#define WAIT_LGKM0() asm volatile("s_waitcnt lgkmcnt(0)" ::: "memory")
#define BARRIER() do { asm volatile("" ::: "memory"); __builtin_amdgcn_s_barrier(); asm volatile("" ::: "memory"); } while (0)
#define SCHED() __builtin_amdgcn_sched_barrier(0)
#define RLX_AGENT __ATOMIC_RELAXED, __HIP_MEMORY_SCOPE_AGENT
#define RLX_SYSTEM __ATOMIC_RELAXED, __HIP_MEMORY_SCOPE_SYSTEM

#define ARRV(t) do { } while (0)
#define ARRV_OUT() do { } while (0)
#define STAMP(i) do { } while (0)

__device__ __forceinline__ u32 pack_bf16(float a, float b) {
  f32x2 f = {a, b};
  bf16x2 h = __builtin_convertvector(f, bf16x2);
  return __builtin_bit_cast(u32, h);
}

__device__ __forceinline__ u32 wave_max_u32(u32 v) {
  v = max(v, (u32)__builtin_amdgcn_update_dpp(0, (int)v, 0x111, 0xf, 0xf, false));
  v = max(v, (u32)__builtin_amdgcn_update_dpp(0, (int)v, 0x112, 0xf, 0xf, false));
  v = max(v, (u32)__builtin_amdgcn_update_dpp(0, (int)v, 0x114, 0xf, 0xf, false));
  v = max(v, (u32)__builtin_amdgcn_update_dpp(0, (int)v, 0x118, 0xf, 0xf, false));
  const u32 a = (u32)__builtin_amdgcn_readlane((int)v, 15), b = (u32)__builtin_amdgcn_readlane((int)v, 31);
  const u32 c = (u32)__builtin_amdgcn_readlane((int)v, 47), d = (u32)__builtin_amdgcn_readlane((int)v, 63);
  return max(max(a, b), max(c, d));
}

__device__ __forceinline__ u32 wave_max_63(u32 v) {
  v = max(v, (u32)__builtin_amdgcn_update_dpp(0, (int)v, 0x111, 0xf, 0xf, false));
  v = max(v, (u32)__builtin_amdgcn_update_dpp(0, (int)v, 0x112, 0xf, 0xf, false));
  v = max(v, (u32)__builtin_amdgcn_update_dpp(0, (int)v, 0x114, 0xf, 0xf, false));
  v = max(v, (u32)__builtin_amdgcn_update_dpp(0, (int)v, 0x118, 0xf, 0xf, false));
  v = max(v, (u32)__builtin_amdgcn_update_dpp(0, (int)v, 0x142, 0xa, 0xf, false));
  v = max(v, (u32)__builtin_amdgcn_update_dpp(0, (int)v, 0x143, 0xc, 0xf, false));
  return v;
}

__device__ __forceinline__ u32 sword(const GAS u32* p) {
  u32 v;
  asm volatile("s_load_dword %0, %1, 0x0 glc\n\ts_waitcnt lgkmcnt(0)" : "=&s"(v) : "s"(p) : "memory");
  return v;
}
__device__ __forceinline__ void sflags3(u32x16& v, u32x16& w, u32x16& z, const GAS u32* p, u32 off, const GAS u32* q, u32 off2) {
  asm volatile("s_load_dwordx16 %0, %3, %4 glc\n\ts_load_dwordx16 %1, %3, %4 offset:0x40 glc\n\ts_load_dwordx16 %2, %5, %6 glc\n\ts_waitcnt lgkmcnt(0)"
               : "=&s"(v), "=&s"(w), "=&s"(z)
               : "s"(p), "s"(__builtin_amdgcn_readfirstlane((int)off)), "s"(q), "s"(__builtin_amdgcn_readfirstlane((int)off2)) : "memory");
}
__device__ __forceinline__ void sflags3p(u32x16& v, u32x16& w, u32x16& z, u32& m0, u32& m1, const GAS u32* p, u32 off) {
  u64 m;
  asm volatile("s_load_dwordx16 %0, %4, %5 glc\n\ts_load_dwordx16 %1, %4, %5 offset:0x40 glc\n\ts_load_dwordx16 %2, %4, %5 offset:0x80 glc\n\t"
               "s_load_dwordx2 %3, %4, 0x200 glc\n\ts_waitcnt lgkmcnt(0)"
               : "=&s"(v), "=&s"(w), "=&s"(z), "=&s"(m)
               : "s"(p), "s"(__builtin_amdgcn_readfirstlane((int)off)) : "memory");
  m0 = (u32)m;
  m1 = (u32)(m >> 32);
}
__device__ __forceinline__ int lead4(const u32x16& v, u32 seq, int skip = 0) {
  const bool a = skip > 0 || min(min(v[0], v[1]), min(v[2], v[3])) >= seq, b = skip > 1 || min(min(v[4], v[5]), min(v[6], v[7])) >= seq;
  const bool c = skip > 2 || min(min(v[8], v[9]), min(v[10], v[11])) >= seq, d = min(min(v[12], v[13]), min(v[14], v[15])) >= seq;
  return a ? (b ? (c ? (d ? 4 : 3) : 2) : 1) : 0;
}
__device__ __forceinline__ u32 min16(const u32x16& v) {
  u32 m, n;
  asm("s_min_u32 %0, %2, %3\n\ts_min_u32 %1, %4, %5\n\ts_min_u32 %0, %0, %6\n\ts_min_u32 %1, %1, %7\n\t"
      "s_min_u32 %0, %0, %8\n\ts_min_u32 %1, %1, %9\n\ts_min_u32 %0, %0, %10\n\ts_min_u32 %1, %1, %11\n\t"
      "s_min_u32 %0, %0, %12\n\ts_min_u32 %1, %1, %13\n\ts_min_u32 %0, %0, %14\n\ts_min_u32 %1, %1, %15\n\t"
      "s_min_u32 %0, %0, %16\n\ts_min_u32 %1, %1, %17\n\ts_min_u32 %0, %0, %1"
      : "=&s"(m), "=&s"(n)
      : "s"(v[0]), "s"(v[1]), "s"(v[2]), "s"(v[3]), "s"(v[4]), "s"(v[5]), "s"(v[6]), "s"(v[7]), "s"(v[8]), "s"(v[9]), "s"(v[10]),
        "s"(v[11]), "s"(v[12]), "s"(v[13]), "s"(v[14]), "s"(v[15])
      : "scc");
  return m;
}

__device__ __forceinline__ int x_known(int xn, const u32x16& v, const u32x16& w, u32 seq) {
  const int x0 = min(xn, X0_END - 1);
  if (xn < X0_END && min16(v) >= seq) xn = (x0 < X0_END - 1 && min16(w) >= seq) ? x0 + 2 : x0 + 1;
  if (xn >= X0_END && x0 == X0_END - 1 && w[0] >= seq) xn = w[1] >= seq ? NKS : X1_END;
  return xn;
}

__device__ __forceinline__ bool give_up(u32 t0, GAS u32* tmo, u32 seq) {
  const u32 dt = (u32)__builtin_amdgcn_s_memrealtime() - t0;
  if (__builtin_expect(dt < 100000u, 1)) return false;
  __builtin_amdgcn_s_sleep(64);
  const bool stop = dt > (u32)DEADLINE_TICKS || sword(tmo) == seq;
  if (stop) __hip_atomic_store(tmo, seq, RLX_AGENT);
  return stop;
}

__device__ __forceinline__ void w_load(f32x4* v, __amdgpu_buffer_rsrc_t rs, u32 off, int row) {
#pragma unroll
  for (int j = 0; j < 8; ++j)
    v[j] = __builtin_bit_cast(f32x4, __builtin_amdgcn_raw_buffer_load_b128(rs, off, (row + j) * (NL * 4), 0));
}

__device__ __forceinline__ u32x4 w_pack(const f32x4* v, int r) {
  u32x4 d;
  d.x = pack_bf16(v[0][r], v[1][r]);
  d.y = pack_bf16(v[2][r], v[3][r]);
  d.z = pack_bf16(v[4][r], v[5][r]);
  d.w = pack_bf16(v[6][r], v[7][r]);
  return d;
}

__device__ __forceinline__ void w_store(const f32x4* v, __amdgpu_buffer_rsrc_t rs, u32 woff, int chunk) {
#pragma unroll
  for (int r = 0; r < 4; ++r) __builtin_amdgcn_raw_buffer_store_b128(w_pack(v, r), rs, woff + r * 2048, chunk * (4 * W_KS), 16  );
}

__device__ __forceinline__ f32x4 e_load(__amdgpu_buffer_rsrc_t rs, u32 off, int row) {
  f32x4 v;
#pragma unroll
  for (int j = 0; j < 4; ++j) v[j] = __builtin_bit_cast(float, __builtin_amdgcn_raw_buffer_load_b32(rs, off, (row + j) * (NL * 4), 0));
  return v;
}
__device__ __forceinline__ u32x2 e_pack(const f32x4& v) { return (u32x2){pack_bf16(v[0], v[1]), pack_bf16(v[2], v[3])}; }
__device__ __forceinline__ void e_store(const f32x4& v, __amdgpu_buffer_rsrc_t rs, u32 off, int chunk) {
  __builtin_amdgcn_raw_buffer_store_b64(e_pack(v), rs, off, chunk * (4 * W_KS), 16  );
}
__device__ __forceinline__ void ticket(u32& old, GAS u32* p, u32 v) {
  asm volatile("global_atomic_swap %0, %1, %2, off sc0 sc1" : "+v"(old) : "v"(p), "v"(v) : "memory");
}

__device__ __forceinline__ void raise(u64 p, u32 v, u32 seq) {
  asm volatile("global_atomic_umax_x2 %0, %1, off sc1" ::"v"(p), "v"((u32x2){v, seq}) : "memory");
}
__device__ __forceinline__ u32 valid(u64 v, u32 seq) { return (u32)(v >> 32) >= seq ? (u32)v : 0u; }

__device__ __forceinline__ void ctl_write(u32 addr, u32 v) {
  asm volatile("ds_write_b32 %0, %1" ::"v"(addr), "v"(v) : "memory");
}
__device__ __forceinline__ u32 ctl_read(u32 addr) {
  u32 v;
  asm volatile("ds_read_b32 %0, %1\n\ts_waitcnt lgkmcnt(0)" : "=v"(v) : "v"(addr) : "memory");
  return v;
}
__device__ __forceinline__ void x_load(f32x4* v, const float* __restrict__ x, int xh, int xks, int msub, int lane) {
  const float* xs = x + (size_t)(xh * 256 + msub * 16 + (lane & 15)) * KD + xks * 64 + (lane >> 4) * 8;
#pragma unroll
  for (int j = 0; j < 4; ++j) v[j] = *(const f32x4*)(xs + (j >> 1) * 32 + (j & 1) * 4);
}

__device__ __forceinline__ u32x4 x_pack(const f32x4* v, int kh) {
  u32x4 d;
  d.x = pack_bf16(v[kh * 2][0], v[kh * 2][1]);
  d.y = pack_bf16(v[kh * 2][2], v[kh * 2][3]);
  d.z = pack_bf16(v[kh * 2 + 1][0], v[kh * 2 + 1][1]);
  d.w = pack_bf16(v[kh * 2 + 1][2], v[kh * 2 + 1][3]);
  return d;
}

__device__ __forceinline__ void x_store(const f32x4* v, __amdgpu_buffer_rsrc_t rs, int xh, int xks, int msub, int lane) {
  const u32 xoff = (u32)(((xh * 64 + xks) * 16 + msub) * 2048 + lane * 16);
#pragma unroll
  for (int kh = 0; kh < 2; ++kh) __builtin_amdgcn_raw_buffer_store_b128(x_pack(v, kh), rs, xoff + kh * 1024, 0, 17  );
}

__device__ __forceinline__ u64 both_ge(u64 f, u32 seq) { return __ballot((u32)f >= seq && (u32)(f >> 32) >= seq); }
__device__ __forceinline__ int lead_ones(u64 m) { return m == ~0ull ? 64 : __builtin_ctzll(~m); }

__global__ void __launch_bounds__(NTHREADS) __attribute__((amdgpu_waves_per_eu(3, 3))) ag_gemm_q8(Params p) {
  extern __shared__ __attribute__((aligned(16))) char lds[];
  const int b = blockIdx.x;
  const int tid = threadIdx.x, lane = tid & 63;
  const int wid = __builtin_amdgcn_readfirstlane(tid >> 6);
  const int idx = b >> 3;
  const int half = idx & 1, nt = idx >> 1, shr = (b & 7) * 2 + half;
  const float* const px = p.x;
  const float* const pw = p.w;
  unsigned char* const pout = p.out;
  char* const myws = p.myws;
  char* const srcws = p.sws[b & 7];
  const int rank = p.rank;
  const u32 perm = p.perm;
  const u64 w0 = (u64)p.ws[0], w1 = (u64)p.ws[1], w2 = (u64)p.ws[2], w3 = (u64)p.ws[3];
  const u64 w4 = (u64)p.ws[4], w5 = (u64)p.ws[5], w6 = (u64)p.ws[6], w7 = (u64)p.ws[7];
  asm volatile("" ::"s"(px), "s"(pw), "s"(pout), "s"(myws), "s"(srcws), "s"(rank), "s"(perm), "s"(w0), "s"(w1), "s"(w2), "s"(w3),
               "s"(w4), "s"(w5), "s"(w6), "s"(w7));
  const int src = (perm >> (4 * (b & 7))) & 7, mt = src * 2 + half;
  auto peer_ws = [&]() {
    int lo = (int)w0, hi = (int)(w0 >> 32);
#define PUT(i) asm("v_writelane_b32 %0, %2, " #i "\n\tv_writelane_b32 %1, %3, " #i : "+v"(lo), "+v"(hi) : "s"((int)w##i), "s"((int)(w##i >> 32)))
    PUT(1); PUT(2); PUT(3); PUT(4); PUT(5); PUT(6); PUT(7);
#undef PUT
    return (char*)(((u64)(u32)hi << 32) | (u32)lo);
  };
  GAS u32* const cnt = (GAS u32*)(myws + CNT_OFF) + b;
  GAS u32* const ownfl = (GAS u32*)(myws + XFL_OFF) + rank * 1024;
  const __amdgpu_buffer_rsrc_t xrs = __builtin_amdgcn_make_buffer_rsrc(myws + XB_OFF, 0, (int)XB_BYTES, 0x00020000);

  const u32 ctl = (u32)(size_t)(LAS char*)lds + LDS_CTL;
  if (nt == 15) {
    if (wid == 4) {
      ctl_write(ctl, PQ);
      ctl_write(ctl + 4, 0u);
      WAIT_LGKM0();
    }
    BARRIER();
  }
  if (wid < 2 && (wid || (b >> 5) * 16 + (b & 15) >= PQ * 4)) {
    f32x4 v[4];
    const int xh = (b >> 4) & 1, xks = wid * X0_END + (b >> 5), msub = b & 15;
    if (wid) __builtin_amdgcn_s_sleep(X1_SLEEP);
    x_load(v, px, xh, xks, msub, lane);
    const u32 seq = __hip_atomic_load(cnt, RLX_AGENT);
    SCHED();
    GAS u32* pub = wid ? ownfl + xh * 512 + 128 + xks * 16 + msub : (GAS u32*)(peer_ws() + XFL_OFF) + (rank * 2 + xh) * 512 + xks * 16 + msub;
    asm volatile("" : "+v"(pub));
    SCHED();
    x_store(v, xrs, xh, xks, msub, lane);
    SCHED();
    WAIT_VM(0);
    if (lane < (wid ? 1 : 8)) __hip_atomic_store(pub, seq + 1u, RLX_SYSTEM);
  }
  u32 seq8 = 0;
  if (wid == 8) {
    f32x4 v[8];
    const int xh = idx >> 4, msub = idx & 15;
    const bool one = msub < PQ * 4, two = msub + 16 < PQ * 4;
    STAMP(0);
    if (one) x_load(v, px, xh, 0, msub, lane);
    if (two) x_load(v + 4, px, xh, 1, msub, lane);
    seq8 = __hip_atomic_load(cnt, RLX_AGENT) + 1u;
    {
      char* const pw8 = peer_ws() + AMX_OFF + BEST;
      if (lane < 8) asm volatile("ds_write_b64 %0, %1" ::"v"(ctl + 32 * 4 + lane * 8), "v"(pw8) : "memory");
    }
    SCHED();
    if (one) {
      const u32 off = (u32)(XP_OFF + rank * XP_SRC) + (u32)(xh * (XP_SRC / 2) + msub * 2048) + lane * 16;
      const u32 flo = lane == 0 ? (u32)XFL_OFF + (u32)((rank * 2 + xh) * 512 + msub) * 4u : NOWHERE;
      const __amdgpu_buffer_rsrc_t r0 = __builtin_amdgcn_make_buffer_rsrc(srcws, 0, (int)WS_END, 0x00020000);
      const u32x4 d0 = x_pack(v, 0), d1 = x_pack(v, 1);
      if (PU0_SLEEP) __builtin_amdgcn_s_sleep(PU0_SLEEP);
      __builtin_amdgcn_raw_buffer_store_b128(d0, r0, off, 0, 17);
      __builtin_amdgcn_raw_buffer_store_b128(d1, r0, off + 1024, 0, 17);
      if (two) {
        const u32x4 d2 = x_pack(v + 4, 0), d3 = x_pack(v + 4, 1);
        SCHED();
        __builtin_amdgcn_s_sleep(PU_SLEEP);
        __builtin_amdgcn_raw_buffer_store_b128(d2, r0, off, A_BYTES, 17);
        __builtin_amdgcn_raw_buffer_store_b128(d3, r0, off + 1024, A_BYTES, 17);
        WAIT_VM(2);
        __builtin_amdgcn_raw_buffer_store_b32(seq8, r0, flo, 0, 17);
        WAIT_VM(1);
        __builtin_amdgcn_raw_buffer_store_b32(seq8, r0, flo, 64, 17);
      } else {
        WAIT_VM(0);
        __builtin_amdgcn_raw_buffer_store_b32(seq8, r0, flo, 0, 17);
      }
      STAMP(1);
    }
    seq8 = (u32)__builtin_amdgcn_readfirstlane((int)seq8);
  }

  GAS u32* const xfl = (GAS u32*)(myws + XFL_OFF) + mt * 512;
  GAS u32* const tmo = (GAS u32*)(myws + TMO_OFF);
  GAS u32* const tick = (GAS u32*)(myws + INV_OFF) + (__builtin_amdgcn_s_getreg(20 | (3 << 11)) & 15) * 32;
  const __amdgpu_buffer_rsrc_t mrs = __builtin_amdgcn_make_buffer_rsrc(myws + AMX_OFF, 0, 65536, 0x00020000);
  auto drop = [&]() {
    WAIT_VM(0);
    asm volatile("buffer_inv sc0 sc1" ::: "memory");
    WAIT_VM(0);
  };
  char* const xsrc = srcws + XB_OFF + (size_t)half * (XB_BYTES / 2);
  const __amdgpu_buffer_rsrc_t ars = __builtin_amdgcn_make_buffer_rsrc(xsrc, 0, (int)(XB_BYTES / 2), 0x00020000);

  if (nt == 15) {
    if (wid < 2 || wid > 5 || (wid < 4 && src == rank)) return;
    const u32 seq = (u32)__builtin_amdgcn_readfirstlane((int)__hip_atomic_load(cnt, RLX_AGENT)) + 1u;
    if (wid == 5) {
      f32x4 cv[24];
      const int xl = (b >> 1) * 6;
      __builtin_amdgcn_s_sleep(L_SLEEP);
#pragma unroll
      for (int j = 0; j < 6; ++j) x_load(cv + 4 * j, px, b & 1, X1_END + ((xl + j) >> 4), (xl + j) & 15, lane);
      SCHED();
#pragma unroll
      for (int j = 0; j < 6; ++j) x_store(cv + 4 * j, xrs, b & 1, X1_END + ((xl + j) >> 4), (xl + j) & 15, lane);
      WAIT_VM(0);
      if (lane == 0) __hip_atomic_store(ownfl + (b & 1) * 512 + 384 + (b >> 1), seq, RLX_SYSTEM);
      return;
    }
    __builtin_amdgcn_s_setprio(3);
    const u32 t0 = (u32)__builtin_amdgcn_s_memrealtime();
    if (wid < 4) {
      const int pv = wid - 2;
      LAS void* dst = (LAS void*)(lds + pv * 256);
      const u32 vo = lane * 128;
      int np = PQ;
#pragma nounroll
      for (int u = PQ + pv; u < NKS * 4; u += NPW) {
        for (u32 spins = 1; u >= np; ++spins) {
          np = __builtin_amdgcn_readfirstlane((int)ctl_read(ctl));
          if (u < np) break;
          __builtin_amdgcn_s_sleep(LS_SLEEP);
          if ((spins & 255u) == 0 && give_up(t0, tmo, seq)) np = NKS * 4 + 1;
        }
        if (np > NKS * 4) break;
        if (PACE && u < PACE_U) __builtin_amdgcn_s_sleep(PACE);
        WAIT_VM(PWIN - 1);
        __builtin_amdgcn_raw_ptr_buffer_load_lds(ars, dst, 4, vo, u * 8192, 0, 0);
        if (HINT && pv == 0) ctl_write(ctl + 4, (u32)u);
      }
      if (HINT && pv == 0) ctl_write(ctl + 4, NKS * 4);
      WAIT_VM(0);
      if (np > NKS * 4) asm volatile("buffer_inv sc0 sc1" ::: "memory");
      BARRIER();
      return;
    }
    if (src != rank) {
      if (D_SLEEP) __builtin_amdgcn_s_sleep(D_SLEEP);
      for (int np = PQ; np < NKS * 4;) {
        u32x16 f0, f1, f2;
        u32 m0, m1;
        const int l0 = min(np >> 2, X0_END - 3);
        sflags3p(f0, f1, f2, m0, m1, xfl, (u32)l0 * 64u);
        int n = l0 * 4 + lead4(f0, seq, l0 == PQ / 4 ? PQ & 3 : 0);
        if (n == l0 * 4 + 4) n += lead4(f1, seq);
        if (n == l0 * 4 + 8) n += lead4(f2, seq);
        if (n == X0_END * 4 && m0 >= seq) n = m1 >= seq ? NKS * 4 : X1_END * 4;
        if (n > np) {
          np = n;
          ctl_write(ctl, (u32)np);
          continue;
        }
        if (give_up(t0, tmo, seq)) {
          ctl_write(ctl, NKS * 4 + 1);
          break;
        }
        if (POLL_GAP) __builtin_amdgcn_s_sleep(POLL_GAP);
      }
    } else {
      GAS u32* const pub = (GAS u32*)(p.ws[lane & 7] + XFL_OFF) + (rank * 2 + half) * 512 + 128;
      for (int ph = 0; ph < 2;) {
        if (both_ge(__hip_atomic_load((GAS u64*)(ownfl + half * 512 + 256 + ph * 128) + lane, RLX_SYSTEM), seq) == ~0ull) {
          if (lane < 8) __hip_atomic_store(pub + ph, seq, RLX_SYSTEM);
          ++ph;
          continue;
        }
        if (give_up(t0, tmo, seq)) break;
        __builtin_amdgcn_s_sleep(4);
      }
    }
    u32 old = seq;
    if (lane == 0) ticket(old, tick, seq);
    if (HINT && src != rank) {
      const GAS u32* const hint = (const GAS u32*)(myws + AMX_OFF + HNT) + ((b & 7) * 2 + half) * 16;
      const u32 far = (u32)(NKS - 4 + (lane >> 4)) * A_BYTES + (lane & 15) * 128;
      u32 seen = 0;
      for (u32 spins = 1;; ++spins) {
        const int u = __builtin_amdgcn_readfirstlane((int)ctl_read(ctl + 4));
        if (u >= NKS * 4 - 4 || ((spins & 63u) == 0 && give_up(t0, tmo, seq))) break;
        if (u < (T0 + 1) * 4) {
          __builtin_amdgcn_s_sleep(16);
          continue;
        }
        u32x16 f;
        asm volatile("s_load_dwordx16 %0, %1, 0x0 glc\n\ts_waitcnt lgkmcnt(0)" : "=&s"(f) : "s"(hint) : "memory");
        u32 want = 0;
#pragma unroll
        for (int i = 0; i < 16; ++i) want |= f[i] == seq ? 1u << i : 0u;
        want &= ~seen;
        seen |= want;
        for (; want; want &= want - 1)
          __builtin_amdgcn_raw_ptr_buffer_load_lds(ars, (LAS void*)(lds + 512), 4, far, __builtin_ctz(want) * 2048, 0, 0);
        if (HT_SLEEP) __builtin_amdgcn_s_sleep(HT_SLEEP);
      }
    }
    BARRIER();
    if (lane == 0) __hip_atomic_store(cnt, seq, RLX_AGENT);
    const u32 g = b & 7;
    if (half) {
      __builtin_amdgcn_raw_buffer_store_b32(seq, mrs, lane == 0 ? 0u : NOWHERE, PDN + g * 64, 17);
    } else {
      GAS u64* post = (GAS u64*)(p.ws[lane & 7] + AMX_OFF + GRP + (rank * 8 + g) * 64);
      asm volatile("" : "+v"(post));
      bool bad = false;
      while (!bad && __any(__builtin_amdgcn_raw_buffer_load_b32(mrs, 0, PDN + g * 64, 17) < seq)) {
        bad = give_up(t0, tmo, seq);
        __builtin_amdgcn_s_sleep(AG_SLEEP);
      }
      const u32 l1 = lane * 16, l2 = min(l1 + 1024u, (u32)NGW * 8u - 16u);
      u32 m = 0, s0 = 0, s1 = 0, s2 = 0, s3 = 0;
      for (u32 spins = 1; !bad; ++spins) {
        const u32 so = LOC + g * 2048;
        const u32x4 g0 = __builtin_amdgcn_raw_buffer_load_b128(mrs, l1, so, 17);
        const u32x4 g1 = __builtin_amdgcn_raw_buffer_load_b128(mrs, l2, so, 17);
        m = max(max(m, g0.y >= seq ? g0.x : 0u), g0.w >= seq ? g0.z : 0u);
        m = max(max(m, g1.y >= seq ? g1.x : 0u), g1.w >= seq ? g1.z : 0u);
        s0 = max(s0, g0.y); s1 = max(s1, g0.w); s2 = max(s2, g1.y); s3 = max(s3, g1.w);
        if (__all(min(min(s0, s1), min(s2, s3)) >= seq)) break;
        if ((spins & 31u) == 0) bad = give_up(t0, tmo, seq);
        if (!__any(max(max(s0, s1), max(s2, s3)) >= seq)) __builtin_amdgcn_s_sleep(AG_SLEEP);
      }
      m = bad ? BAD : wave_max_u32(m);
      if (lane < 8) __hip_atomic_store(post, ((u64)seq << 32) | m, RLX_SYSTEM);
      if (src == rank) raise((u64)post + (BEST - GRP + (lane >> 3) * 128 - (rank * 8 + g) * 64), m, seq);
      STAMP(11);
      if (WBL2) asm volatile("buffer_wbl2 sc1" ::: "memory");
    }
    WAIT_VM(0);
    asm volatile("" : "+v"(old));
    if ((u32)__builtin_amdgcn_readfirstlane((int)old) != seq) {
      while (!__all(__builtin_amdgcn_raw_buffer_load_b32(mrs, (DROPV == 2 ? lane : lane & 7) * 64, GRP + (DROPV == 2 ? 0 : rank * 512) + 4, 17) >= seq)) {
        if (give_up(t0, tmo, seq)) break;
        __builtin_amdgcn_s_sleep(C_SLEEP);
      }
      if (DROPV != 1) drop();
    }
    STAMP(15);
    return;
  }

  const int kq1 = lane >> 5, wH = (lane >> 4) & 1, wi = lane & 15;
  const __amdgpu_buffer_rsrc_t wfr = __builtin_amdgcn_make_buffer_rsrc((void*)pw, 0, KD * NL * 4, 0x00020000);
  const u32 wfo = (u32)(kq1 * 8 * NL + nt * 128 + wH * 64 + wi * 4) * 4u;
  const u32 woff = (u32)(nt * W_NT + (shr >> 2) * W_KS + wH * 8192 + ((shr >> 1) & 1) * 1024 +
                         (((shr & 1) * 2 + kq1) * 16 + wi) * 16);
  const bool ext = nt < 8;
  const u32 efo = ext ? (u32)((wH * 8 + kq1 * 4) * NL + 15 * 128 + nt * 16 + wi) * 4u : NOWHERE;
  const u32 eoff = ext ? (u32)(15 * W_NT + nt * 2048 + (shr >> 2) * W_KS + ((shr >> 1) & 1) * 1024 +
                               (((shr & 1) * 2 + wH) * 16 + wi) * 16 + kq1 * 8) : NOWHERE;
  const __amdgpu_buffer_rsrc_t wrs =
      __builtin_amdgcn_make_buffer_rsrc(myws + WB_OFF, 0, (int)WB_BYTES, 0x00020000);
  GAS u32* const wfl = (GAS u32*)(myws + WFL_OFF) + nt * 256;
  const __amdgpu_buffer_rsrc_t brs = __builtin_amdgcn_make_buffer_rsrc(
      myws + WB_OFF + (size_t)nt * W_NT, 0, (int)(WB_BYTES - nt * W_NT), 0x00020000);
  const u32 l16 = lane * 16;
  const u32 ecol = (u32)((15 - nt) * W_NT + nt * 2048);
  const bool own = src == rank;
  auto tell = [&](u32 v, u32 seq) {
    u64 q;
    asm volatile("ds_read_b64 %0, %1\n\ts_waitcnt lgkmcnt(0)" : "=v"(q) : "v"(ctl + 32 * 4 + (lane & 7) * 8) : "memory");
    raise(q + (u32)(lane >> 3) * 128u, v, seq);
  };

  if (wid == 8) {
    __builtin_amdgcn_s_setprio(3);
    const u32 seq = seq8;
    bool failed = false, cleaner = false;
    int xn = 0, cw = 0;
#define TR(t) do { } while (0)
#define KBAR(t) do { BARRIER(); SCHED(); } while (0)
    f32x4 wv[9];
#pragma unroll
    for (int j = 0; j < 9; ++j) wv[j] = (f32x4){0.f, 0.f, 0.f, 0.f};
    u32 lh = lane;
    asm volatile("" : "+v"(lh));
    const u32 eoffh = ext ? (u32)(15 * W_NT + nt * 2048 + (shr >> 2) * W_KS + ((shr >> 1) & 1) * 1024 + (shr & 1) * 512) +
                            (lh & 31) * 16 + (lh >> 5) * 8 : NOWHERE;
    if (H_SLEEP) __builtin_amdgcn_s_sleep(H_SLEEP);
#pragma nounroll
    for (int t = -2; t <= T0; ++t) {
      const int c = (t >> 2) + 3;
      if ((t & 3) == 0) {
        __builtin_amdgcn_s_waitcnt(0x0f70);
        asm volatile("" ::: "memory");
        if (lane == 0 && c <= NCH) __hip_atomic_store(t ? wfl + (c - 1) * 16 + shr : cnt, seq, RLX_AGENT);
        if (c < NCH) {
          w_load(wv, wfr, wfo, c * 256 + shr * 16);
          wv[8] = e_load(wfr, efo, c * 256 + shr * 16);
        }
      } else if ((t & 3) == 2 && t > 0 && c < NCH) {
        w_store(wv, wrs, woff, c);
        e_store(wv[8], wrs, eoffh, c);
      }
      if (t == 1 && lane < 2)
        __hip_atomic_store(lane ? ownfl + (b & 1) * 512 + 384 + (b >> 1) : wfl + 2 * 16 + shr, seq, RLX_SYSTEM);
      if (t == 25 + idx && sword(tick) != seq) {
        u32 old = 0;
        if (lane == 0) old = __hip_atomic_exchange(tick, seq, RLX_SYSTEM);
        cleaner = (u32)__builtin_amdgcn_readfirstlane((int)old) != seq;
      }
      SCHED();
      const int s = t == T0 ? NKS - 1 : t + 3, gw = t < -1 ? 0 : (s >> 2) + 1;
      if (xn < NKS || cw <= min(gw, NCH - 1)) {
        u32 t0 = 0, spins = 0;
        for (;;) {
          u32x16 f0, f1, fw;
          const int on = xn + cw;
          sflags3(f0, f1, fw, xfl, (u32)min(xn, X0_END - 1) * 64u, wfl, (u32)min(cw, NCH - 1) * 64u);
          xn = x_known(xn, f0, f1, seq);
          if (min16(fw) >= seq) cw = min(cw + 1, NCH);
          if (xn > s && cw >= gw) break;
          if (xn + cw != on) continue;
          if (spins == 0) t0 = (u32)__builtin_amdgcn_s_memrealtime();
          if ((spins & 63u) == 63u) {
            GAS u64* const xf = (GAS u64*)xfl;
            const u32 vw = __hip_atomic_load(wfl + min(cw, NCH - 1) * 16 + (lane & 15), RLX_AGENT);
            const u64 v0 = __hip_atomic_load(xf + lane, RLX_SYSTEM), v1 = __hip_atomic_load(xf + 64, RLX_SYSTEM);
            if (__all(vw >= seq)) cw = min(cw + 1, NCH);
            int n = lead_ones(both_ge(v0, seq)) >> 3;
            if (n == X0_END && __all((u32)v1 >= seq)) n = __all((u32)(v1 >> 32) >= seq) ? NKS : X1_END;
            xn = max(xn, n);
          }
          ++spins;
          if (give_up(t0, tmo, seq)) { failed = true; xn = NKS; cw = NCH; break; }
          if (POLL_GAP) __builtin_amdgcn_s_sleep(POLL_GAP);
        }
      }
      if (t == T0 - 3) {
        wv[0] = __builtin_bit_cast(f32x4, __builtin_amdgcn_raw_buffer_load_b128(mrs, l16, PRE, 17));
        wv[1] = __builtin_bit_cast(f32x4, __builtin_amdgcn_raw_buffer_load_b128(mrs, min(l16 + 1024u, (u32)NGW * 8u - 16u), PRE, 17));
      } else if (t == T0 - 1) {
        const u32x4 g0 = __builtin_bit_cast(u32x4, wv[0]), g1 = __builtin_bit_cast(u32x4, wv[1]);
        const u32 c = wave_max_u32(max(max(g0.y >= seq ? g0.x : 0u, g0.w >= seq ? g0.z : 0u), max(g1.y >= seq ? g1.x : 0u, g1.w >= seq ? g1.z : 0u)));
        const u32 thr = __builtin_bit_cast(u32, __builtin_bit_cast(float, c) * ALPHA);
        asm volatile("ds_write_b128 %0, %1\n\ts_waitcnt lgkmcnt(0)" ::"v"(ctl + 20 * 4), "v"((u32x4){seq, own || c == 0 || !CAND ? ~0u : thr, 0u, 0u}) : "memory");
      }
      ARRV(t == -2 ? -1 : t);
      KBAR(t == -2 ? -1 : t);
    }
    static_assert(T0 >= 55 && T0 <= 57, "everything is published by now");
    asm volatile("ds_write2_b32 %0, %1, %2 offset0:16 offset1:21" ::"v"(ctl), "v"(failed ? BAD : 0u), "v"(0u) : "memory");
    GAS u32* const best = (GAS u32*)(myws + AMX_OFF + BEST) + (b & 7) * 32;
    u32x4 rp;
    asm volatile("ds_read_b128 %0, %1\n\ts_waitcnt lgkmcnt(0)" : "=v"(rp) : "v"(ctl + 256 + (lane & 7) * 16) : "memory");
    const u32 top = wave_max_u32(rp.x);
    if (top) {
      const int w = __builtin_ctzll(__ballot(rp.x == top));
      const u32 ao = (u32)__builtin_amdgcn_readlane((int)rp.y, w), bo = (u32)__builtin_amdgcn_readlane((int)rp.z, w);
      const u32 first = __builtin_amdgcn_readlane((int)rp.w, w) ? NOWHERE : l16;
      if (HINT) __builtin_amdgcn_raw_buffer_store_b32(seq, mrs, lane == 0 ? 0u : NOWHERE, HNT + ((b & 7) * 2 + half) * 64 + (ao >> 9), HNT_AUX);
      f32x4 cc;
      asm volatile("ds_read_b128 %0, %1\n\ts_waitcnt lgkmcnt(0)" : "=v"(cc) : "v"(ctl + 512 + w * 1024 + l16) : "memory");
      bf16x8 ca[NH], cb[NH];
      auto fetch = [&](int h0, int h1) __attribute__((always_inline)) {
#pragma unroll
        for (int h = h0; h < h1; ++h) {
          const int t = (2 * T0 + 1 + h) >> 1, kh = (h + 1) & 1;
          cb[h] = __builtin_bit_cast(bf16x8, __builtin_amdgcn_raw_buffer_load_b128(brs, l16 + kh * 1024, bo + t * W_KS, 0));
          ca[h] = __builtin_bit_cast(bf16x8, __builtin_amdgcn_raw_buffer_load_b128(ars, (h ? l16 : first) + kh * 1024, ao + t * A_BYTES, 0));
        }
        SCHED();
      };
      fetch(0, NNEAR);
      KBAR(T0 + 1);
      KBAR(T0 + 2);
#pragma unroll
      for (int h = 0; h < NNEAR; ++h) cc = __builtin_amdgcn_mfma_f32_16x16x32_bf16(cb[h], ca[h], cc, 0, 0, 0);
      SCHED();
      fetch(NNEAR, NH);
      const u32x2 known = __builtin_amdgcn_raw_buffer_load_b64(mrs, 0, BEST + (b & 7) * 128, 17);
      SCHED();
      KBAR(T0 + 3);
#pragma unroll
      for (int h = NNEAR; h < NH; ++h) cc = __builtin_amdgcn_mfma_f32_16x16x32_bf16(cb[h], ca[h], cc, 0, 0, 0);
      const u32 c = wave_max_u32(__builtin_bit_cast(u32, fmaxf(fmaxf(fmaxf(fmaxf(0.f, fabsf(cc[0])), fabsf(cc[1])), fabsf(cc[2])), fabsf(cc[3]))));
      if (__any(known.y < seq || c > known.x)) tell(c, seq);
      TR(45);
      SCHED();
    } else {
      KBAR(T0 + 1);
      KBAR(T0 + 2);
      KBAR(T0 + 3);
    }
#pragma unroll
    for (int t = T0 + 4; t <= NKS - 4; ++t) KBAR(t);
    constexpr u32 LAST = ((NKS - 1) % 3) * STAGE;
    auto four = [&](__amdgpu_buffer_rsrc_t rs, u32 to, u32 from) {
      __builtin_amdgcn_raw_ptr_buffer_load_lds(rs, (LAS void*)(lds + to), 16, l16, from, 0, 0);
      __builtin_amdgcn_raw_ptr_buffer_load_lds(rs, (LAS void*)(lds + to), 16, l16, from, 1024, 0);
      __builtin_amdgcn_raw_ptr_buffer_load_lds(rs, (LAS void*)(lds + to), 16, l16, from, 2048, 0);
      __builtin_amdgcn_raw_ptr_buffer_load_lds(rs, (LAS void*)(lds + to), 16, l16, from, 3072, 0);
      SCHED();
    };
#pragma unroll
    for (int k = 0; k < 4; ++k) four(brs, LAST + A_BYTES + k * 4096, (NKS - 1) * W_KS + k * 4096);
    __builtin_amdgcn_raw_ptr_buffer_load_lds(brs, (LAS void*)(lds + LAST + E_POS), 16, ext ? l16 : NOWHERE, (NKS - 1) * W_KS + ecol, 0, 0);
    __builtin_amdgcn_raw_ptr_buffer_load_lds(brs, (LAS void*)(lds + LAST + E_POS), 16, ext ? l16 : NOWHERE, (NKS - 1) * W_KS + ecol, 1024, 0);
    SCHED();
#pragma unroll
    for (int k = 0; k < APRE; ++k) four(ars, LAST + k * 4096, (NKS - 1) * A_BYTES + k * 4096);
    TR(40);
    BARRIER();
    TR(41);
#pragma unroll
    for (int k = APRE; k < 8; ++k) four(ars, LAST + k * 4096, (NKS - 1) * A_BYTES + k * 4096);
    u32 off = lane * 64;
    u32x2 g, bv;
    u32 tv;
    asm volatile("buffer_load_dwordx2 %0, %3, %4, %5 offen sc0 sc1\n\tbuffer_load_dwordx2 %1, off, %4, %6 sc0 sc1\n\tglobal_load_dword %2, %7, off sc1"
                 : "=&v"(g), "=&v"(bv), "=&v"(tv) : "v"(off), "s"(mrs), "s"(GRP), "s"(__builtin_amdgcn_readfirstlane((int)(BEST + (b & 7) * 128))), "v"(tmo) : "memory");
    const bool now = !own && SPEC_REM == 0;
    WAIT_VM(27);
    ctl_write(ctl + 22 * 4, 1u);
    TR(42);
    WAIT_VM(19);
    ctl_write(ctl + 22 * 4, 2u);
    TR(43);
    WAIT_VM(11);
    ctl_write(ctl + 22 * 4, 3u);
    WAIT_VM(3);
    ctl_write(ctl + 22 * 4, 4u);
    TR(44);
    asm volatile("s_waitcnt vmcnt(0)" : "+v"(g), "+v"(bv), "+v"(tv)::"memory");
    u32 m = tv == seq ? BAD : bv.y >= seq ? bv.x : 0u;
    if (now) ctl_write(ctl + 23 * 4, m);
    ARRV_OUT();
    if (WGATE == 1) BARRIER();
    else if (HP_SLEEP) __builtin_amdgcn_s_sleep(HP_SLEEP);
    const u32 t0 = (u32)__builtin_amdgcn_s_memrealtime();
    const int thr = own ? SPEC_THR : SPEC_REM;
    u64 got = 0;
    bool go = now;
    for (u32 spins = 1;; ++spins) {
      const bool ok = g.y >= seq;
      const u32 c = ok ? g.x : 0u;
      off = ok ? BEST - GRP + (b & 7) * 128 : off;
      got |= __ballot(ok);
      if (got == ~0ull) {
        const u32 a = wave_max_63(max(m, c)) | FIN;
        if (lane == 63) ctl_write(ctl + 23 * 4, a);
        break;
      }
      const bool up = __any(c > m);
      m = max(m, c);
      if (__builtin_popcountll(got) >= thr && (up || !go)) {
        go = true;
        m = wave_max_u32(m);
        ctl_write(ctl + 23 * 4, m);
      }
      if ((spins & 15u) == 0 && give_up(t0, tmo, seq)) {
        ctl_write(ctl + 23 * 4, FIN | BAD);
        break;
      }
      g = __builtin_amdgcn_raw_buffer_load_b64(mrs, off, GRP, 17);
    }
    if (cleaner) drop();
    return;
  }

  f32x4 dv[2][9];
  if (wid >= 4) {
    if (DW_SLEEP) __builtin_amdgcn_s_sleep(DW_SLEEP);
#pragma unroll
    for (int d = 0; d < 2; ++d) {
      w_load(dv[d], wfr, wfo, d * 64 + (wid - 4) * 16);
      dv[d][8] = e_load(wfr, efo, d * 64 + (wid - 4) * 16);
    }
    SCHED();
  }
  f32x4 cv[9];
  const int xl = (b >> 1) * 6 + (wid - 4) * 2;
  u32 seqw = 0;
  if (wid == 2 || wid == 3) {
    const bool skip = wid == 2 && shr < 8;
    if (wid == 3) __builtin_amdgcn_s_sleep(P1_SLEEP);
    else if (W2_SLEEP) __builtin_amdgcn_s_sleep(W2_SLEEP);
    if (!skip) {
      w_load(cv, wfr, wfo, (wid - 2) * 256 + shr * 16);
      cv[8] = e_load(wfr, efo, (wid - 2) * 256 + shr * 16);
    }
    seqw = __hip_atomic_load(cnt, RLX_AGENT) + 1u;
    SCHED();
    if (!skip) {
      w_store(cv, wrs, woff, wid - 2);
      e_store(cv[8], wrs, eoff, wid - 2);
    }
    SCHED();
    if (wid == 2) {
      WAIT_VM(0);
      if (lane == 0) __hip_atomic_store(wfl + shr, seqw, RLX_AGENT);
      STAMP(12);
    }
  } else if (wid == 7) {
    w_load(cv, wfr, wfo, 2 * 256 + shr * 16);
    cv[8] = e_load(wfr, efo, 2 * 256 + shr * 16);
    SCHED();
  } else if (wid >= 4) {
    x_load(cv, px, b & 1, X1_END + (xl >> 4), xl & 15, lane);
    x_load(cv + 4, px, b & 1, X1_END + (xl >> 4), (xl & 15) + 1, lane);
    SCHED();
  }

  const __amdgpu_buffer_rsrc_t prs = __builtin_amdgcn_make_buffer_rsrc(myws + XP_OFF + (size_t)mt * (XP_SRC / 2), 0, (int)(XP_SRC / 2), 0x00020000);
  const u32 akb = (u32)((wid + AROT * nt) & 7) * 1024u;
  const u32 avo = akb + l16, bvo = wid * 2048 + l16;
  const u32 evo = ext ? ecol + (u32)(wid * 256 + lane * 4) : NOWHERE;
  auto piece = [&](int t, u32 slot, int j, bool pushed = false) {
    if (j == 2) {
      __builtin_amdgcn_raw_ptr_buffer_load_lds(brs, (LAS void*)(lds + slot + E_POS + wid * 256), 4, evo, t * W_KS, 0, 0);
    } else if (j < 2) {
      LAS void* lb = (LAS void*)(lds + slot + A_BYTES + wid * 2048);
      if (j == 0) __builtin_amdgcn_raw_ptr_buffer_load_lds(brs, lb, 16, bvo, t * W_KS, 0, 0);
      else __builtin_amdgcn_raw_ptr_buffer_load_lds(brs, lb, 16, bvo, t * W_KS, 1024, 0);
    } else {
      LAS void* la = (LAS void*)(lds + slot + akb + (j - 3) * 8192);
      __builtin_amdgcn_raw_ptr_buffer_load_lds(pushed ? prs : ars, la, 16, avo, t * A_BYTES + (j - 3) * 8192, 0, 0);
    }
    SCHED();
  };

  const int wr = wid >> 1, wc = wid & 1;
  const u32 aoff0 = wr * 8192 + wc * 4096 + l16, aoff1 = wr * 8192 + (1 - wc) * 4096 + l16, boff = A_BYTES + wc * 8192 + l16;
  f32x4 acc[4][4], ace[2];
#pragma unroll
  for (int i = 0; i < 4; ++i)
#pragma unroll
    for (int j = 0; j < 4; ++j) acc[i][j] = (f32x4){0.f, 0.f, 0.f, 0.f};
  ace[0] = ace[1] = (f32x4){0.f, 0.f, 0.f, 0.f};
  bf16x8 af[2][4], bfr[2][4], ef[2];
  auto frags = [&](u32 slot, int kh) {
#pragma unroll
    for (int i = 0; i < 4; ++i) {
      bfr[kh][i] = *(const bf16x8*)(lds + slot + boff + i * 2048 + kh * 1024);
      af[kh][i] = *(const bf16x8*)(lds + slot + (i < 2 ? aoff0 : aoff1) + (i & 1) * 2048 + kh * 1024);
    }
    ef[kh] = *(const bf16x8*)(lds + slot + E_POS + l16 + kh * 1024);
    SCHED();
  };
  auto mfma4 = [&](int kh, int mi) {
#pragma unroll
    for (int r = 0; r < 4; ++r)
      acc[mi][r] = __builtin_amdgcn_mfma_f32_16x16x32_bf16(bfr[kh][r], af[kh][mi], acc[mi][r], 0, 0, 0);
    if (mi >= 2) ace[mi - 2] = __builtin_amdgcn_mfma_f32_16x16x32_bf16(ef[kh], af[kh][mi - 2], ace[mi - 2], 0, 0, 0);
    SCHED();
  };

  BARRIER();
#pragma unroll
  for (int j = 0; j < 4; ++j) piece(0, 0, j + 3, j < PQ);
#pragma unroll
  for (int j = 0; j < 4; ++j) piece(1, STAGE, j + 3, j + 4 < PQ);
  if (wid >= 4) {
    char* const dst = lds + A_BYTES + wH * 8192 + ((wid >> 1) & 1) * 1024 + (((wid & 1) * 2 + kq1) * 16 + wi) * 16;
    char* const dse = lds + E_POS + ((wid >> 1) & 1) * 1024 + (((wid & 1) * 2 + wH) * 16 + wi) * 16 + kq1 * 8;
#pragma unroll
    for (int d = 0; d < 2; ++d) {
#pragma unroll
      for (int r = 0; r < 4; ++r) *(u32x4*)(dst + d * STAGE + r * 2048) = w_pack(dv[d], r);
      *(u32x2*)(dse + d * STAGE) = e_pack(dv[d][8]);
    }
    SCHED();
    if (wid == 7) {
      w_store(cv, wrs, woff, 2);
      e_store(cv[8], wrs, eoff, 2);
      SCHED();
      WAIT_VM_LGKM(9);
    } else {
      x_store(cv, xrs, b & 1, X1_END + (xl >> 4), xl & 15, lane);
      x_store(cv + 4, xrs, b & 1, X1_END + (xl >> 4), (xl & 15) + 1, lane);
      SCHED();
      WAIT_VM_LGKM(8);
    }
  } else if (wid == 3) {
    WAIT_VM(8);
    if (lane == 0) __hip_atomic_store(wfl + 16 + shr, seqw, RLX_AGENT);
    STAMP(13);
    WAIT_VM(5);
  } else {
    WAIT_VM_LGKM(4);
  }
  ARRV(-1);
  BARRIER();
  frags(0, 0);
#pragma unroll
  for (int j = 0; j < 4 - BAR_POS; ++j) piece(2, 2 * STAGE, j);

  u32 seq = 0, t0 = 0;
  auto look = [&]() __attribute__((always_inline)) {
    u32 at = ctl, l = l16;
    int wv = wid;
    asm volatile("" : "+s"(at), "+s"(wv), "+v"(l));
    u32x2 st;
    u64 now;
    asm volatile("ds_read_b64 %0, %2 offset:80\n\ts_memrealtime %1" : "=v"(st), "=s"(now) : "v"(at) : "memory");
    float bm[18];
#pragma unroll
    for (int i = 0; i < 18; ++i) {
      const f32x4& v = i < 16 ? acc[i >> 2][i & 3] : ace[i & 1];
      bm[i] = fmaxf(fmaxf(fmaxf(fmaxf(0.f, fabsf(v[0])), fabsf(v[1])), fabsf(v[2])), fabsf(v[3]));
    }
    if (!ext) bm[16] = bm[17] = 0.f;
    float mx = 0.f;
#pragma unroll
    for (int i = 0; i < 18; ++i) mx = fmaxf(mx, bm[i]);
    asm volatile("s_waitcnt lgkmcnt(0)" : "+v"(st), "+s"(now)::"memory");
    seq = (u32)__builtin_amdgcn_readfirstlane((int)st.x);
    t0 = (u32)now;
    u32x4 rp = {0u, 0u, 0u, 0u};
    if (own) {
      const u32x2 r = {wave_max_63(__builtin_bit_cast(u32, mx)), st.x};
      __builtin_amdgcn_raw_buffer_store_b64(r, mrs, l == 63 * 16 ? 0u : NOWHERE, PRE + idx * 64 + wv * 8, 17);
    } else if (__any(__builtin_bit_cast(u32, mx) > st.y)) {
      const u32 wm = wave_max_u32(__builtin_bit_cast(u32, mx)), to = at + wv * 1024 + l;
      int i0 = 0;
#pragma unroll
      for (int i = 0; i < 18; ++i)
        asm volatile("v_cmp_eq_u32 vcc, %2, %3\n\ts_cbranch_vccz 1f\n\tds_write_b128 %4, %5 offset:512\n\ts_mov_b32 %0, %1\n1:"
                     : "+s"(i0) : "n"(i), "s"(wm), "v"(bm[i]), "v"(to), "v"(i < 16 ? acc[i >> 2][i & 3] : ace[i & 1]) : "vcc", "memory");
      const int mi = i0 < 16 ? i0 >> 2 : i0 & 1;
      rp = (u32x4){wm, (u32)((wv >> 1) * 8192 + (((mi >> 1) ^ wv) & 1) * 4096 + (mi & 1) * 2048), i0 < 16 ? (u32)((wv & 1) * 8192 + (i0 & 3) * 2048) : ecol,
                   i0 < 4 * BAR_POS ? 1u : 0u};
    }
    asm volatile("ds_write_b128 %0, %1 offset:256" ::"v"(at + wv * 16), "v"(rp) : "memory");
    SCHED();
  };

  u32 cur = 0, nx1 = STAGE, nx2 = 2 * STAGE;
  auto step = [&](int t, auto more) {
    constexpr int MORE = decltype(more)::value;
    frags(cur, 1);
    if (MORE >= 2) piece(t + 2, nx2, 2);
#pragma unroll
    for (int g = 0; g < 4; ++g) {
      mfma4(0, g);
      if (MORE >= 2) piece(t + 2, nx2, 3 + g);
    }
    constexpr int BP = MORE == 1 ? BP61 : BAR_POS;
#pragma unroll
    for (int g = 0; g < BP; ++g) mfma4(1, g);
    if (MORE >= 1) {
      if (MORE >= 3 && __builtin_expect(t == T0, 0)) look();
      if (MORE >= 2) WAIT_VM_LGKM(7);
      else WAIT_VM_LGKM(0);
      ARRV(t);
      BARRIER();
      SCHED();
      frags(nx1, 0);
    }
#pragma unroll
    for (int g = BP; g < 4; ++g) {
      mfma4(1, g);
      if (MORE >= 3) piece(t + 3, cur, g - BP);
    }
    const u32 o = cur;
    cur = nx1; nx1 = nx2; nx2 = o;
  };
  static_assert(PQ <= 8, "the K loop reads the peers");
  for (int t = 0; t < NKS - 4; ++t) step(t, std::integral_constant<int, 3>());
  step(NKS - 4, std::integral_constant<int, 2>());
  step(NKS - 3, std::integral_constant<int, 1>());
  u32 late = 0, f3, ro;
  float mx = 0.f, me = 0.f;
  auto staged = [&](int n) {
    int v, have, spins;
    asm volatile("s_mov_b32 %2, 0\n"
                 "1:\n\tds_read_b32 %0, %3 offset:88\n\ts_waitcnt lgkmcnt(0)\n\tv_readfirstlane_b32 %1, %0\n\t"
                 "s_cmp_ge_i32 %1, %4\n\ts_cbranch_scc1 2f\n\ts_sleep %7\n\ts_add_u32 %2, %2, 0x1000000\n\ts_cbranch_scc0 1b\n\t"
                 "s_memrealtime vcc\n\ts_waitcnt lgkmcnt(0)\n\ts_sub_u32 vcc_lo, vcc_lo, %5\n\ts_cmp_lt_u32 vcc_lo, %6\n\ts_cbranch_scc1 1b\n"
                 "2:"
                 : "=&v"(v), "=&s"(have), "=&s"(spins) : "v"(ctl), "s"(n), "s"(t0), "n"((u32)DEADLINE_TICKS), "n"(RD_SLEEP) : "vcc", "scc", "memory");
    late = have < n ? BAD : late;
  };
  frags(cur, 1);
#pragma unroll
  for (int g = 0; g < 4; ++g) mfma4(0, g);
#pragma unroll
  for (int g = 0; g < BAR_POS; ++g) mfma4(1, g);
  asm volatile("ds_read_b32 %0, %1 offset:64" : "=v"(f3) : "v"(ctl) : "memory");
  staged(wr + 1);
  asm volatile("" : "+v"(f3));
  SCHED();
  frags(nx1, 0);
#pragma unroll
  for (int g = BAR_POS; g < 4; ++g) mfma4(1, g);
  frags(nx1, 1);
  f3 |= late;
  ro = lane == 63 ? (u32)((b & 7) * 2048 + idx * 64 + wid * 8) : NOWHERE;
  asm volatile("" : "+v"(f3), "+v"(ro));
  SCHED();
#pragma unroll
  for (int g = 0; g < 4; ++g) mfma4(0, g);
#pragma unroll
  for (int mi = 0; mi <= 4; ++mi) {
#pragma unroll
    for (int r = 0; r < 4; ++r) {
      if (mi < 4) {
        acc[mi][r] = __builtin_amdgcn_mfma_f32_16x16x32_bf16(bfr[1][r], af[1][mi], acc[mi][r], 0, 0, 0);
        if (mi < 2 && r == 3) ace[mi] = __builtin_amdgcn_mfma_f32_16x16x32_bf16(ef[1], af[1][mi], ace[mi], 0, 0, 0);
      }
      if (mi > 0) mx = fmaxf(fmaxf(fmaxf(fmaxf(mx, fabsf(acc[mi - 1][r][0])), fabsf(acc[mi - 1][r][1])), fabsf(acc[mi - 1][r][2])), fabsf(acc[mi - 1][r][3]));
      if (mi == 3 && r < 2) me = fmaxf(fmaxf(fmaxf(fmaxf(me, fabsf(ace[r][0])), fabsf(ace[r][1])), fabsf(ace[r][2])), fabsf(ace[r][3]));
      if (mi == 3 && r == 2) mx = fmaxf(mx, ext ? me : 0.f);
      SCHED();
    }
  }
  ARRV_OUT();

  const u32x2 fs = {wave_max_63(max(__builtin_bit_cast(u32, mx), f3)), seq};
  __builtin_amdgcn_raw_buffer_store_b64(fs, mrs, ro, LOC, 17);
  SCHED();
  STAMP(4);
  if (WGATE == 1) BARRIER();
  if (WGATE == 2 && lane == 63) asm volatile("ds_add_u32 %0, %1 offset:84" ::"v"(ctl), "v"(1u) : "memory");
  if (late) __hip_atomic_store(tmo, seq, RLX_AGENT);
  if (WDELAY) __builtin_amdgcn_s_sleep(WDELAY);
  const u32 wm = (u32)__builtin_amdgcn_readlane((int)fs.x, 63);
  const __amdgpu_buffer_rsrc_t ors = __builtin_amdgcn_make_buffer_rsrc(pout, 0, 4096 * NL, 0x00020000);
  const u32 orow = (u32)(mt * 256 + wr * 64 + (lane & 15)) * NL;
  const u32 oo0 = orow + wc * 32 * NL + nt * 128 + wc * 64 + (lane >> 4) * 16, oo1 = oo0 + (1 - 2 * wc) * 32 * NL;
  const u32 oe = ext ? orow + wc * 32 * NL + 15 * 128 + nt * 16 + (lane >> 4) * 4 : NOWHERE;
  u32 done = FIN;
#pragma nounroll
  for (u32 spins = 1;; ++spins) {
    u32 a;
    if (WGATE == 2) {
      u32x2 c2;
      asm volatile("ds_read2_b32 %0, %1 offset0:21 offset1:23\n\ts_waitcnt lgkmcnt(0)" : "=v"(c2) : "v"(ctl) : "memory");
      a = (u32)__builtin_amdgcn_readfirstlane((int)c2.y);
      if (__builtin_amdgcn_readfirstlane((int)c2.x) < 8 && a < FIN) a = 0;
    } else {
      a = (u32)__builtin_amdgcn_readfirstlane((int)ctl_read(ctl + 23 * 4));
    }
    if ((spins & 255u) == 0 && give_up(t0, tmo, seq)) a = FIN | BAD;
    const u32 gmax = max(a & ~FIN, wm);
    if (gmax != done && a != 0u) {
      if (a < wm) tell(wm, seq);
      done = gmax;
      const float amax = __builtin_bit_cast(float, gmax);
      const float inv = amax > 0.f ? 448.0f / amax : 0.f;
      auto scaled = [&](const f32x4& v) {
        f32x2 lo, hi;
        asm("v_pk_mul_f32 %0, %1, %2 op_sel_hi:[1,0]" : "=v"(lo) : "v"(__builtin_shufflevector(v, v, 0, 1)), "v"((f32x2){inv, inv}));
        asm("v_pk_mul_f32 %0, %1, %2 op_sel_hi:[1,0]" : "=v"(hi) : "v"(__builtin_shufflevector(v, v, 2, 3)), "v"((f32x2){inv, inv}));
        return __builtin_shufflevector(lo, hi, 0, 1, 2, 3);
      };
      auto q4 = [&](float a, float c, float d, float e) {
        const int q = __builtin_amdgcn_cvt_pk_fp8_f32(a, c, __builtin_bit_cast(int, a), false);
        return (u32)__builtin_amdgcn_cvt_pk_fp8_f32(d, e, q, true);
      };
      if (gmax > 0x7f800000u) {
        const u32 z = 0x7f7f7f7fu;
#pragma unroll
        for (int mi = 0; mi < 4; ++mi)
          __builtin_amdgcn_raw_buffer_store_b128((u32x4){z, z, z, z}, ors, (mi < 2 ? oo0 : oo1) + (mi & 1) * 16 * NL, 0, 16);
#pragma unroll
        for (int i = 0; i < 2; ++i) __builtin_amdgcn_raw_buffer_store_b32(z, ors, oe, i * 16 * NL, E_AUX);
      } else {
#pragma unroll
        for (int mi = 0; mi < 4; ++mi) {
          const f32x4 s0 = scaled(acc[mi][0]), s1 = scaled(acc[mi][1]), s2 = scaled(acc[mi][2]), s3 = scaled(acc[mi][3]);
          u32x4 d;
#pragma unroll
          for (int j = 0; j < 4; ++j) d[j] = q4(s0[j], s1[j], s2[j], s3[j]);
          if (WPACE && mi) WAIT_VM(WPACE - 1);
          __builtin_amdgcn_raw_buffer_store_b128(d, ors, (mi < 2 ? oo0 : oo1) + (mi & 1) * 16 * NL, 0, OUT_AUX);
          SCHED();
        }
#pragma unroll
        for (int i = 0; i < 2; ++i) {
          const f32x4 s = scaled(ace[i]);
          __builtin_amdgcn_raw_buffer_store_b32(q4(s[0], s[1], s[2], s[3]), ors, oe, i * 16 * NL, E_AUX);
        }
      }
    }
    if (a >= FIN) break;
    __builtin_amdgcn_s_sleep(GS_SLEEP);
  }
  if (done > 0x7f800000u) asm volatile("buffer_inv sc0 sc1" ::: "memory");
}

}

extern "C" void kernel_launch(int rank, int nranks, ncclComm_t comm, hipStream_t stream,
                              void* const* d_in, const int* in_sizes, int n_in,
                              void* d_out, int out_size,
                              const int* in_dtypes, int out_dtype,
                              void* const* d_out_all, void* const* d_ws_all, size_t ws_bytes) {
  (void)comm; (void)in_dtypes; (void)out_dtype; (void)d_out_all;
  if (nranks != NR || n_in < 2 || ws_bytes < WS_END || in_sizes[0] != 512 * KD || in_sizes[1] != KD * NL ||
      out_size != 4096 * NL || (unsigned)rank >= (unsigned)nranks) {
    fprintf(stderr, "unexpected problem geometry\n");
    return;
  }
  hipError_t e = hipFuncSetAttribute((const void*)ag_gemm_q8, hipFuncAttributeMaxDynamicSharedMemorySize, (int)LDS_BYTES);
  if (e != hipSuccess) { fprintf(stderr, "hipFuncSetAttribute: %s\n", hipGetErrorString(e)); return; }
  Params p;
  p.x = (const float*)d_in[0];
  p.w = (const float*)d_in[1];
  p.out = (unsigned char*)d_out;
  p.myws = (char*)d_ws_all[rank];
  for (int i = 0; i < NR; ++i) p.ws[i] = (char*)d_ws_all[i];
  p.rank = rank;
  p.perm = 0;
  for (int i = 0, lo = 0, hi = 4; i < NR; ++i) {
    const bool same = ((0x4d >> i) & 1) == ((0x4d >> rank) & 1);
    const int xc = PERM ? (same ? lo++ : hi++) : i;
    p.sws[xc] = p.ws[i];
    p.perm |= (u32)i << (4 * xc);
  }
  hipLaunchKernelGGL(ag_gemm_q8, dim3(NWG), dim3(NTHREADS), LDS_BYTES, stream, p);
  e = hipGetLastError();
  if (e != hipSuccess) fprintf(stderr, "launch: %s\n", hipGetErrorString(e));
}
```

```cpp
#include <hip/hip_runtime.h>
#include <rccl/rccl.h>
#include <cstdio>
#include <type_traits>

namespace {

typedef unsigned int u32;
typedef unsigned long long u64;
typedef __bf16 bf16x8 __attribute__((ext_vector_type(8)));
typedef __bf16 bf16x2 __attribute__((ext_vector_type(2)));
typedef float f32x4 __attribute__((ext_vector_type(4)));
typedef float f32x2 __attribute__((ext_vector_type(2)));
typedef u32 u32x2 __attribute__((ext_vector_type(2)));
typedef u32 u32x3 __attribute__((ext_vector_type(3)));
typedef u32 u32x4 __attribute__((ext_vector_type(4)));
typedef u32 u32x16 __attribute__((ext_vector_type(16)));
#define GAS __attribute__((address_space(1)))
#define LAS __attribute__((address_space(3)))

constexpr int NR = 8;
constexpr int KD = 4096;
constexpr int NL = 2048;
constexpr int NKS = 64;
constexpr int NCH = 16;
constexpr int NWG = 256;
constexpr int NGW = 240;
#define SPEC_THR 9
#define SPEC_REM 0
#define CAND 1
#define ALPHA 0.94f
#define TMO_S 0
#define BEST_S 1
#define WGATE 2
#define WDELAY 0
#define APRE 4
#define BP61 2
#define WPACE 0
#define OUT_AUX 16
#define WBL2 0
#define RD_SLEEP 1
#define HP_SLEEP 0
#define AG_SLEEP 6
#define GS_SLEEP 1
#define AROT 1
#define DROPV 0
#define HNT_AUX 16
#define NPW 2
#define PWIN 3
#define XMAP 0
#define PUE_SLEEP 0
#define OWNX -1
#define PSLOT 54
#define PWIN0 PWIN
#define PW_U 20
#define W_AUX 2
#define DW_AUX 0
#define X_NT 0
#define PQ 6
#define PUSH (PQ / 4)
#define D_SLEEP 24
#define X1_SLEEP 50
#define P1_SLEEP 80
#define W2_SLEEP 50
#define DW_SLEEP 80
#define H_SLEEP 0
#define POLL_GAP 2
#define C_SLEEP 12
#define E_AUX 0
#define PU_SLEEP 36
#define LS_SLEEP 1
#define PACE 0
#define PACE_U 24
#define PU0_SLEEP 0
#define L_SLEEP 100
#define XP_SLEEP 2
#define PERM 1
constexpr int BAR_POS = 2;
constexpr u32 W_KS = 16 * 16384, W_NT = 16384;
constexpr int NTHREADS = 576;
constexpr int X0_END = 8;
constexpr int X1_END = 16;
constexpr u32 A_BYTES = 32768, E_POS = 49152, STAGE = 51200;
#define LOOK_AT 57
#define HINT 1
#define HT_SLEEP 2
constexpr int T0 = LOOK_AT;
constexpr int NH = 2 * (NKS - T0) - 1;
constexpr int NNEAR = 7;
constexpr u32 LDS_CTL = 3 * STAGE;
constexpr u32 LDS_BYTES = LDS_CTL + 512 + 8192;
constexpr u32 NOWHERE = 0x80000000u;

constexpr size_t XB_OFF = 0;
constexpr size_t XB_BYTES = 4ull << 20;
constexpr size_t WB_OFF = 4ull << 20;
constexpr size_t WB_BYTES = 16ull << 20;
constexpr size_t CT_OFF = 20ull << 20;
constexpr size_t CNT_OFF = CT_OFF;
constexpr size_t TMO_OFF = CT_OFF + 4096;
constexpr size_t INV_OFF = CT_OFF + 5120;
constexpr size_t XFL_OFF = CT_OFF + 8192;
constexpr size_t WFL_OFF = CT_OFF + 40960;
constexpr size_t DBG_OFF = CT_OFF + 65536;
constexpr size_t AMX_OFF = CT_OFF + 196608;
constexpr u32 LOC = 0;
constexpr u32 GRP = 32768;
constexpr u32 PDN = 40960;
constexpr u32 PRE = 45056;
constexpr u32 HNT = 47104;
constexpr u32 BEST = 49152;
constexpr u32 FIN = 0x80000000u, BAD = 0x7fffffffu;
constexpr size_t WS_NEED = CT_OFF + 262144;
constexpr size_t XP_OFF = 28ull << 20;
constexpr size_t XP_SRC = 262144;
constexpr size_t WS_END = XP_OFF + NR * XP_SRC;

constexpr u64 DEADLINE_TICKS = 100000000ull;

struct Params {
  const float* x;
  const float* w;
  unsigned char* out;
  char* myws;
  char* ws[NR];
  char* sws[NR];
  u32 perm;
  int rank;
};

#define WAIT_VM(n) asm volatile("s_waitcnt vmcnt(%0)" ::"n"(n) : "memory")
#define WAIT_VM_LGKM(n) asm volatile("s_waitcnt vmcnt(" #n ") lgkmcnt(0)" ::: "memory")
#define WAIT_LGKM0() asm volatile("s_waitcnt lgkmcnt(0)" ::: "memory")
#define BARRIER() do { asm volatile("" ::: "memory"); __builtin_amdgcn_s_barrier(); asm volatile("" ::: "memory"); } while (0)
#define SCHED() __builtin_amdgcn_sched_barrier(0)
#define NAP(n) do { if ((n) > 254) __builtin_amdgcn_s_sleep(127); if ((n) > 127) __builtin_amdgcn_s_sleep(127); \
                    __builtin_amdgcn_s_sleep((n) > 254 ? (n) - 254 : (n) > 127 ? (n) - 127 : (n)); } while (0)
#define RLX_AGENT __ATOMIC_RELAXED, __HIP_MEMORY_SCOPE_AGENT
#define RLX_SYSTEM __ATOMIC_RELAXED, __HIP_MEMORY_SCOPE_SYSTEM

#define ARRV(t) do { } while (0)
#define ARRV_OUT() do { } while (0)
#define STAMP(i) do { } while (0)

__device__ __forceinline__ u32 pack_bf16(float a, float b) {
  f32x2 f = {a, b};
  bf16x2 h = __builtin_convertvector(f, bf16x2);
  return __builtin_bit_cast(u32, h);
}

__device__ __forceinline__ u32 wave_max_u32(u32 v) {
  v = max(v, (u32)__builtin_amdgcn_update_dpp(0, (int)v, 0x111, 0xf, 0xf, false));
  v = max(v, (u32)__builtin_amdgcn_update_dpp(0, (int)v, 0x112, 0xf, 0xf, false));
  v = max(v, (u32)__builtin_amdgcn_update_dpp(0, (int)v, 0x114, 0xf, 0xf, false));
  v = max(v, (u32)__builtin_amdgcn_update_dpp(0, (int)v, 0x118, 0xf, 0xf, false));
  const u32 a = (u32)__builtin_amdgcn_readlane((int)v, 15), b = (u32)__builtin_amdgcn_readlane((int)v, 31);
  const u32 c = (u32)__builtin_amdgcn_readlane((int)v, 47), d = (u32)__builtin_amdgcn_readlane((int)v, 63);
  return max(max(a, b), max(c, d));
}

__device__ __forceinline__ u32 wave_max_63(u32 v) {
  v = max(v, (u32)__builtin_amdgcn_update_dpp(0, (int)v, 0x111, 0xf, 0xf, false));
  v = max(v, (u32)__builtin_amdgcn_update_dpp(0, (int)v, 0x112, 0xf, 0xf, false));
  v = max(v, (u32)__builtin_amdgcn_update_dpp(0, (int)v, 0x114, 0xf, 0xf, false));
  v = max(v, (u32)__builtin_amdgcn_update_dpp(0, (int)v, 0x118, 0xf, 0xf, false));
  v = max(v, (u32)__builtin_amdgcn_update_dpp(0, (int)v, 0x142, 0xa, 0xf, false));
  v = max(v, (u32)__builtin_amdgcn_update_dpp(0, (int)v, 0x143, 0xc, 0xf, false));
  return v;
}

__device__ __forceinline__ u32 sword(const GAS u32* p) {
  u32 v;
  asm volatile("s_load_dword %0, %1, 0x0 glc\n\ts_waitcnt lgkmcnt(0)" : "=&s"(v) : "s"(p) : "memory");
  return v;
}
__device__ __forceinline__ void sflags3(u32x16& v, u32x16& w, u32x16& z, const GAS u32* p, u32 off, const GAS u32* q, u32 off2) {
  asm volatile("s_load_dwordx16 %0, %3, %4 glc\n\ts_load_dwordx16 %1, %3, %4 offset:0x40 glc\n\ts_load_dwordx16 %2, %5, %6 glc\n\ts_waitcnt lgkmcnt(0)"
               : "=&s"(v), "=&s"(w), "=&s"(z)
               : "s"(p), "s"(__builtin_amdgcn_readfirstlane((int)off)), "s"(q), "s"(__builtin_amdgcn_readfirstlane((int)off2)) : "memory");
}
__device__ __forceinline__ void sflags3p(u32x16& v, u32x16& w, u32x16& z, u32& m0, u32& m1, const GAS u32* p, u32 off) {
  u64 m;
  asm volatile("s_load_dwordx16 %0, %4, %5 glc\n\ts_load_dwordx16 %1, %4, %5 offset:0x40 glc\n\ts_load_dwordx16 %2, %4, %5 offset:0x80 glc\n\t"
               "s_load_dwordx2 %3, %4, 0x200 glc\n\ts_waitcnt lgkmcnt(0)"
               : "=&s"(v), "=&s"(w), "=&s"(z), "=&s"(m)
               : "s"(p), "s"(__builtin_amdgcn_readfirstlane((int)off)) : "memory");
  m0 = (u32)m;
  m1 = (u32)(m >> 32);
}
__device__ __forceinline__ int lead4(const u32x16& v, u32 seq, int skip = 0) {
  const bool a = skip > 0 || min(min(v[0], v[1]), min(v[2], v[3])) >= seq, b = skip > 1 || min(min(v[4], v[5]), min(v[6], v[7])) >= seq;
  const bool c = skip > 2 || min(min(v[8], v[9]), min(v[10], v[11])) >= seq, d = min(min(v[12], v[13]), min(v[14], v[15])) >= seq;
  return a ? (b ? (c ? (d ? 4 : 3) : 2) : 1) : 0;
}
__device__ __forceinline__ u32 min16(const u32x16& v) {
  u32 m, n;
  asm("s_min_u32 %0, %2, %3\n\ts_min_u32 %1, %4, %5\n\ts_min_u32 %0, %0, %6\n\ts_min_u32 %1, %1, %7\n\t"
      "s_min_u32 %0, %0, %8\n\ts_min_u32 %1, %1, %9\n\ts_min_u32 %0, %0, %10\n\ts_min_u32 %1, %1, %11\n\t"
      "s_min_u32 %0, %0, %12\n\ts_min_u32 %1, %1, %13\n\ts_min_u32 %0, %0, %14\n\ts_min_u32 %1, %1, %15\n\t"
      "s_min_u32 %0, %0, %16\n\ts_min_u32 %1, %1, %17\n\ts_min_u32 %0, %0, %1"
      : "=&s"(m), "=&s"(n)
      : "s"(v[0]), "s"(v[1]), "s"(v[2]), "s"(v[3]), "s"(v[4]), "s"(v[5]), "s"(v[6]), "s"(v[7]), "s"(v[8]), "s"(v[9]), "s"(v[10]),
        "s"(v[11]), "s"(v[12]), "s"(v[13]), "s"(v[14]), "s"(v[15])
      : "scc");
  return m;
}

__device__ __forceinline__ int x_known(int xn, const u32x16& v, const u32x16& w, u32 seq) {
  const int x0 = min(xn, X0_END - 1);
  if (xn < X0_END && min16(v) >= seq) xn = (x0 < X0_END - 1 && min16(w) >= seq) ? x0 + 2 : x0 + 1;
  if (xn >= X0_END && x0 == X0_END - 1 && w[0] >= seq) xn = w[1] >= seq ? NKS : X1_END;
  return xn;
}

__device__ __forceinline__ bool give_up(u32 t0, GAS u32* tmo, u32 seq) {
  const u32 dt = (u32)__builtin_amdgcn_s_memrealtime() - t0;
  if (__builtin_expect(dt < 100000u, 1)) return false;
  __builtin_amdgcn_s_sleep(64);
  const bool stop = dt > (u32)DEADLINE_TICKS || sword(tmo) == seq;
  if (stop) __hip_atomic_store(tmo, seq, RLX_AGENT);
  return stop;
}

template <int AUX = W_AUX>
__device__ __forceinline__ void w_load(f32x4* v, __amdgpu_buffer_rsrc_t rs, u32 off, int row) {
#pragma unroll
  for (int j = 0; j < 8; ++j)
    v[j] = __builtin_bit_cast(f32x4, __builtin_amdgcn_raw_buffer_load_b128(rs, off, (row + j) * (NL * 4), AUX));
}

__device__ __forceinline__ u32x4 w_pack(const f32x4* v, int r) {
  u32x4 d;
  d.x = pack_bf16(v[0][r], v[1][r]);
  d.y = pack_bf16(v[2][r], v[3][r]);
  d.z = pack_bf16(v[4][r], v[5][r]);
  d.w = pack_bf16(v[6][r], v[7][r]);
  return d;
}

__device__ __forceinline__ void w_store(const f32x4* v, __amdgpu_buffer_rsrc_t rs, u32 woff, int chunk) {
#pragma unroll
  for (int r = 0; r < 4; ++r) __builtin_amdgcn_raw_buffer_store_b128(w_pack(v, r), rs, woff + r * 2048, chunk * (4 * W_KS), 16  );
}

__device__ __forceinline__ f32x4 e_load(__amdgpu_buffer_rsrc_t rs, u32 off, int row) {
  f32x4 v;
#pragma unroll
  for (int j = 0; j < 4; ++j) v[j] = __builtin_bit_cast(float, __builtin_amdgcn_raw_buffer_load_b32(rs, off, (row + j) * (NL * 4), 0));
  return v;
}
__device__ __forceinline__ u32x2 e_pack(const f32x4& v) { return (u32x2){pack_bf16(v[0], v[1]), pack_bf16(v[2], v[3])}; }
__device__ __forceinline__ void e_store(const f32x4& v, __amdgpu_buffer_rsrc_t rs, u32 off, int chunk) {
  __builtin_amdgcn_raw_buffer_store_b64(e_pack(v), rs, off, chunk * (4 * W_KS), 16  );
}
__device__ __forceinline__ void ticket(u32& old, GAS u32* p, u32 v) {
  asm volatile("global_atomic_swap %0, %1, %2, off sc0 sc1" : "+v"(old) : "v"(p), "v"(v) : "memory");
}

__device__ __forceinline__ void raise(u64 p, u32 v, u32 seq) {
  asm volatile("global_atomic_umax_x2 %0, %1, off sc1" ::"v"(p), "v"((u32x2){v, seq}) : "memory");
}
__device__ __forceinline__ u32 valid(u64 v, u32 seq) { return (u32)(v >> 32) >= seq ? (u32)v : 0u; }

__device__ __forceinline__ void ctl_write(u32 addr, u32 v) {
  asm volatile("ds_write_b32 %0, %1" ::"v"(addr), "v"(v) : "memory");
}
__device__ __forceinline__ u32 ctl_read(u32 addr) {
  u32 v;
  asm volatile("ds_read_b32 %0, %1\n\ts_waitcnt lgkmcnt(0)" : "=v"(v) : "v"(addr) : "memory");
  return v;
}
__device__ __forceinline__ void x_load(f32x4* v, const float* __restrict__ x, int xh, int xks, int msub, int lane) {
  const float* xs = x + (size_t)(xh * 256 + msub * 16 + (lane & 15)) * KD + xks * 64 + (lane >> 4) * 8;
#pragma unroll
  for (int j = 0; j < 4; ++j) {
    const f32x4* q = (const f32x4*)(xs + (j >> 1) * 32 + (j & 1) * 4);
    v[j] = X_NT ? __builtin_nontemporal_load(q) : *q;
  }
}

__device__ __forceinline__ u32x4 x_pack(const f32x4* v, int kh) {
  u32x4 d;
  d.x = pack_bf16(v[kh * 2][0], v[kh * 2][1]);
  d.y = pack_bf16(v[kh * 2][2], v[kh * 2][3]);
  d.z = pack_bf16(v[kh * 2 + 1][0], v[kh * 2 + 1][1]);
  d.w = pack_bf16(v[kh * 2 + 1][2], v[kh * 2 + 1][3]);
  return d;
}

__device__ __forceinline__ void x_store(const f32x4* v, __amdgpu_buffer_rsrc_t rs, int xh, int xks, int msub, int lane) {
  const u32 xoff = (u32)(((xh * 64 + xks) * 16 + msub) * 2048 + lane * 16);
#pragma unroll
  for (int kh = 0; kh < 2; ++kh) __builtin_amdgcn_raw_buffer_store_b128(x_pack(v, kh), rs, xoff + kh * 1024, 0, 17  );
}

__device__ __forceinline__ u64 both_ge(u64 f, u32 seq) { return __ballot((u32)f >= seq && (u32)(f >> 32) >= seq); }
__device__ __forceinline__ int lead_ones(u64 m) { return m == ~0ull ? 64 : __builtin_ctzll(~m); }

__global__ void __launch_bounds__(NTHREADS) __attribute__((amdgpu_waves_per_eu(3, 3))) ag_gemm_q8(Params p) {
  extern __shared__ __attribute__((aligned(16))) char lds[];
  const int b = blockIdx.x;
  const int tid = threadIdx.x, lane = tid & 63;
  const int wid = __builtin_amdgcn_readfirstlane(tid >> 6);
  const int idx = b >> 3;
  const int half = idx & 1, nt = idx >> 1, shr = (b & 7) * 2 + half;
  const float* const px = p.x;
  const float* const pw = p.w;
  unsigned char* const pout = p.out;
  char* const myws = p.myws;
  char* const srcws = p.sws[b & 7];
  const int rank = p.rank;
  const u32 perm = p.perm;
  const u64 w0 = (u64)p.ws[0], w1 = (u64)p.ws[1], w2 = (u64)p.ws[2], w3 = (u64)p.ws[3];
  const u64 w4 = (u64)p.ws[4], w5 = (u64)p.ws[5], w6 = (u64)p.ws[6], w7 = (u64)p.ws[7];
  asm volatile("" ::"s"(px), "s"(pw), "s"(pout), "s"(myws), "s"(srcws), "s"(rank), "s"(perm), "s"(w0), "s"(w1), "s"(w2), "s"(w3),
               "s"(w4), "s"(w5), "s"(w6), "s"(w7));
  const int src = (perm >> (4 * (b & 7))) & 7, mt = src * 2 + half;
  auto peer_ws = [&]() {
    int lo = (int)w0, hi = (int)(w0 >> 32);
#define PUT(i) asm("v_writelane_b32 %0, %2, " #i "\n\tv_writelane_b32 %1, %3, " #i : "+v"(lo), "+v"(hi) : "s"((int)w##i), "s"((int)(w##i >> 32)))
    PUT(1); PUT(2); PUT(3); PUT(4); PUT(5); PUT(6); PUT(7);
#undef PUT
    return (char*)(((u64)(u32)hi << 32) | (u32)lo);
  };
  GAS u32* const cnt = (GAS u32*)(myws + CNT_OFF) + b;
  GAS u32* const ownfl = (GAS u32*)(myws + XFL_OFF) + rank * 1024;
  const __amdgpu_buffer_rsrc_t xrs = __builtin_amdgcn_make_buffer_rsrc(myws + XB_OFF, 0, (int)XB_BYTES, 0x00020000);

  const u32 ctl = (u32)(size_t)(LAS char*)lds + LDS_CTL;
  if (nt == 15) {
    if (wid == 4) {
      ctl_write(ctl, PQ);
      ctl_write(ctl + 4, 0u);
      WAIT_LGKM0();
    }
    BARRIER();
  }
  const int xe = (b & 4) ? idx * 4 + (b & 3) : idx * 2 + (b & 1);
  const int xq = wid || !XMAP ? wid * X0_END * 16 + (b >> 5) * 16 + (b & 15) : PQ * 4 + min((b & 7) >> 1, 2) * 32 + (xe >> 1);
  if (wid < 2 && (wid || (xq >= PQ * 4 && xq < X0_END * 16))) {
    f32x4 v[4];
    const int xh = wid || !XMAP ? (b >> 4) & 1 : xe & 1, xks = xq >> 4, msub = xq & 15;
    if (wid) NAP(X1_SLEEP);
    x_load(v, px, xh, xks, msub, lane);
    const u32 seq = __hip_atomic_load(cnt, RLX_AGENT);
    SCHED();
    GAS u32* pub = wid ? ownfl + xh * 512 + 128 + xks * 16 + msub : (GAS u32*)(peer_ws() + XFL_OFF) + (rank * 2 + xh) * 512 + xks * 16 + msub;
    asm volatile("" : "+v"(pub));
    SCHED();
    x_store(v, xrs, xh, xks, msub, lane);
    SCHED();
    WAIT_VM(0);
    if (lane < (wid ? 1 : 8)) __hip_atomic_store(pub, seq + 1u, RLX_SYSTEM);
  }
  u32 seq8 = 0;
  if (wid == 8) {
    f32x4 v[8];
    const int xh = idx >> 4, msub = idx & 15;
    const bool one = msub < PQ * 4, two = msub + 16 < PQ * 4;
    STAMP(0);
    if (one) x_load(v, px, xh, 0, msub, lane);
    if (two) x_load(v + 4, px, xh, 1, msub, lane);
    seq8 = __hip_atomic_load(cnt, RLX_AGENT) + 1u;
    {
      char* const pw8 = peer_ws() + AMX_OFF + BEST;
      if (lane < 8) asm volatile("ds_write_b64 %0, %1" ::"v"(ctl + 32 * 4 + lane * 8), "v"(pw8) : "memory");
    }
    SCHED();
    if (one) {
      const u32 off = (u32)(XP_OFF + rank * XP_SRC) + (u32)(xh * (XP_SRC / 2) + msub * 2048) + lane * 16;
      const u32 flo = lane == 0 ? (u32)XFL_OFF + (u32)((rank * 2 + xh) * 512 + msub) * 4u : NOWHERE;
      const __amdgpu_buffer_rsrc_t r0 = __builtin_amdgcn_make_buffer_rsrc(srcws, 0, (int)WS_END, 0x00020000);
      u32x4 d0 = x_pack(v, 0), d1 = x_pack(v, 1);
      asm volatile("" : "+v"(d0), "+v"(d1));
      SCHED();
      const u32 tp = (u32)__builtin_amdgcn_s_memrealtime(), at = (u32)(msub * 2 + xh) * PSLOT >> 4;
      if (PSLOT) while ((u32)__builtin_amdgcn_s_memrealtime() - tp < at) { }
      if (PU0_SLEEP) __builtin_amdgcn_s_sleep(PU0_SLEEP);
      if (PUE_SLEEP && (b & 6) == 0) __builtin_amdgcn_s_sleep(PUE_SLEEP);
      __builtin_amdgcn_raw_buffer_store_b128(d0, r0, off, 0, 17);
      __builtin_amdgcn_raw_buffer_store_b128(d1, r0, off + 1024, 0, 17);
      if (two) {
        const u32x4 d2 = x_pack(v + 4, 0), d3 = x_pack(v + 4, 1);
        SCHED();
        if (PSLOT) while ((u32)__builtin_amdgcn_s_memrealtime() - tp < at + 2 * PSLOT) { }
        else __builtin_amdgcn_s_sleep(PU_SLEEP);
        __builtin_amdgcn_raw_buffer_store_b128(d2, r0, off, A_BYTES, 17);
        __builtin_amdgcn_raw_buffer_store_b128(d3, r0, off + 1024, A_BYTES, 17);
        WAIT_VM(2);
        __builtin_amdgcn_raw_buffer_store_b32(seq8, r0, flo, 0, 17);
        WAIT_VM(1);
        __builtin_amdgcn_raw_buffer_store_b32(seq8, r0, flo, 64, 17);
      } else {
        WAIT_VM(0);
        __builtin_amdgcn_raw_buffer_store_b32(seq8, r0, flo, 0, 17);
      }
      STAMP(1);
    }
    seq8 = (u32)__builtin_amdgcn_readfirstlane((int)seq8);
  }

  GAS u32* const xfl = (GAS u32*)(myws + XFL_OFF) + mt * 512;
  GAS u32* const tmo = (GAS u32*)(myws + TMO_OFF);
  GAS u32* const tick = (GAS u32*)(myws + INV_OFF) + (__builtin_amdgcn_s_getreg(20 | (3 << 11)) & 15) * 32;
  const __amdgpu_buffer_rsrc_t mrs = __builtin_amdgcn_make_buffer_rsrc(myws + AMX_OFF, 0, 65536, 0x00020000);
  auto drop = [&]() {
    WAIT_VM(0);
    asm volatile("buffer_inv sc0 sc1" ::: "memory");
    WAIT_VM(0);
  };
  char* const xsrc = srcws + XB_OFF + (size_t)half * (XB_BYTES / 2);
  const __amdgpu_buffer_rsrc_t ars = __builtin_amdgcn_make_buffer_rsrc(xsrc, 0, (int)(XB_BYTES / 2), 0x00020000);

  if (nt == 15) {
    if (wid < 2 || wid > 5 || (wid < 4 && src == rank)) return;
    const u32 seq = (u32)__builtin_amdgcn_readfirstlane((int)__hip_atomic_load(cnt, RLX_AGENT)) + 1u;
    if (wid == 5) {
      f32x4 cv[24];
      const int xl = (b >> 1) * 6;
      NAP(L_SLEEP);
#pragma unroll
      for (int j = 0; j < 6; ++j) x_load(cv + 4 * j, px, b & 1, X1_END + ((xl + j) >> 4), (xl + j) & 15, lane);
      SCHED();
#pragma unroll
      for (int j = 0; j < 6; ++j) x_store(cv + 4 * j, xrs, b & 1, X1_END + ((xl + j) >> 4), (xl + j) & 15, lane);
      WAIT_VM(0);
      if (lane == 0) __hip_atomic_store(ownfl + (b & 1) * 512 + 384 + (b >> 1), seq, RLX_SYSTEM);
      return;
    }
    __builtin_amdgcn_s_setprio(3);
    const u32 t0 = (u32)__builtin_amdgcn_s_memrealtime();
    if (wid < 4) {
      const int pv = wid - 2;
      LAS void* dst = (LAS void*)(lds + pv * 256);
      const u32 vo = lane * 128;
      int np = PQ;
#pragma nounroll
      for (int u = PQ + pv; u < NKS * 4; u += NPW) {
        for (u32 spins = 1; u >= np; ++spins) {
          np = __builtin_amdgcn_readfirstlane((int)ctl_read(ctl));
          if (u < np) break;
          __builtin_amdgcn_s_sleep(LS_SLEEP);
          if ((spins & 255u) == 0 && give_up(t0, tmo, seq)) np = NKS * 4 + 1;
        }
        if (np > NKS * 4) break;
        if (PACE && u < PACE_U) __builtin_amdgcn_s_sleep(PACE);
        if (PWIN0 != PWIN && u < PW_U) WAIT_VM(PWIN0 - 1);
        else WAIT_VM(PWIN - 1);
        __builtin_amdgcn_raw_ptr_buffer_load_lds(ars, dst, 4, vo, u * 8192, 0, 0);
        if (HINT && pv == 0) ctl_write(ctl + 4, (u32)u);
      }
      if (HINT && pv == 0) ctl_write(ctl + 4, NKS * 4);
      WAIT_VM(0);
      if (np > NKS * 4) asm volatile("buffer_inv sc0 sc1" ::: "memory");
      BARRIER();
      return;
    }
    if (src != rank) {
      if (D_SLEEP) __builtin_amdgcn_s_sleep(D_SLEEP);
      for (int np = PQ; np < NKS * 4;) {
        u32x16 f0, f1, f2;
        u32 m0, m1;
        const int l0 = min(np >> 2, X0_END - 3);
        sflags3p(f0, f1, f2, m0, m1, xfl, (u32)l0 * 64u);
        int n = l0 * 4 + lead4(f0, seq, l0 == PQ / 4 ? PQ & 3 : 0);
        if (n == l0 * 4 + 4) n += lead4(f1, seq);
        if (n == l0 * 4 + 8) n += lead4(f2, seq);
        if (n == X0_END * 4 && m0 >= seq) n = m1 >= seq ? NKS * 4 : X1_END * 4;
        if (n > np) {
          np = n;
          ctl_write(ctl, (u32)np);
          continue;
        }
        if (give_up(t0, tmo, seq)) {
          ctl_write(ctl, NKS * 4 + 1);
          break;
        }
        if (POLL_GAP) __builtin_amdgcn_s_sleep(POLL_GAP);
      }
    } else {
      GAS u32* const pub = (GAS u32*)(p.ws[lane & 7] + XFL_OFF) + (rank * 2 + half) * 512 + 128;
      for (int ph = 0; ph < 2;) {
        if (both_ge(__hip_atomic_load((GAS u64*)(ownfl + half * 512 + 256 + ph * 128) + lane, RLX_SYSTEM), seq) == ~0ull) {
          if (lane < 8) __hip_atomic_store(pub + ph, seq, RLX_SYSTEM);
          ++ph;
          continue;
        }
        if (give_up(t0, tmo, seq)) break;
        __builtin_amdgcn_s_sleep(4);
      }
    }
    u32 old = seq;
    if (lane == 0) ticket(old, tick, seq);
    if (HINT && src != rank) {
      const GAS u32* const hint = (const GAS u32*)(myws + AMX_OFF + HNT) + ((b & 7) * 2 + half) * 16;
      const u32 far = (u32)(NKS - 4 + (lane >> 4)) * A_BYTES + (lane & 15) * 128;
      u32 seen = 0;
      for (u32 spins = 1;; ++spins) {
        const int u = __builtin_amdgcn_readfirstlane((int)ctl_read(ctl + 4));
        if (u >= NKS * 4 - 4 || ((spins & 63u) == 0 && give_up(t0, tmo, seq))) break;
        if (u < (T0 + 1) * 4) {
          __builtin_amdgcn_s_sleep(16);
          continue;
        }
        u32x16 f;
        asm volatile("s_load_dwordx16 %0, %1, 0x0 glc\n\ts_waitcnt lgkmcnt(0)" : "=&s"(f) : "s"(hint) : "memory");
        u32 want = 0;
#pragma unroll
        for (int i = 0; i < 16; ++i) want |= f[i] == seq ? 1u << i : 0u;
        want &= ~seen;
        seen |= want;
        for (; want; want &= want - 1)
          __builtin_amdgcn_raw_ptr_buffer_load_lds(ars, (LAS void*)(lds + 512), 4, far, __builtin_ctz(want) * 2048, 0, 0);
        if (HT_SLEEP) __builtin_amdgcn_s_sleep(HT_SLEEP);
      }
    }
    BARRIER();
    if (lane == 0) __hip_atomic_store(cnt, seq, RLX_AGENT);
    const u32 g = b & 7;
    if (half) {
      __builtin_amdgcn_raw_buffer_store_b32(seq, mrs, lane == 0 ? 0u : NOWHERE, PDN + g * 64, 17);
    } else {
      GAS u64* post = (GAS u64*)(p.ws[lane & 7] + AMX_OFF + GRP + (rank * 8 + g) * 64);
      asm volatile("" : "+v"(post));
      bool bad = false;
      while (!bad && __any(__builtin_amdgcn_raw_buffer_load_b32(mrs, 0, PDN + g * 64, 17) < seq)) {
        bad = give_up(t0, tmo, seq);
        __builtin_amdgcn_s_sleep(AG_SLEEP);
      }
      const u32 l1 = lane * 16, l2 = min(l1 + 1024u, (u32)NGW * 8u - 16u);
      u32 m = 0, s0 = 0, s1 = 0, s2 = 0, s3 = 0;
      for (u32 spins = 1; !bad; ++spins) {
        const u32 so = LOC + g * 2048;
        const u32x4 g0 = __builtin_amdgcn_raw_buffer_load_b128(mrs, l1, so, 17);
        const u32x4 g1 = __builtin_amdgcn_raw_buffer_load_b128(mrs, l2, so, 17);
        m = max(max(m, g0.y >= seq ? g0.x : 0u), g0.w >= seq ? g0.z : 0u);
        m = max(max(m, g1.y >= seq ? g1.x : 0u), g1.w >= seq ? g1.z : 0u);
        s0 = max(s0, g0.y); s1 = max(s1, g0.w); s2 = max(s2, g1.y); s3 = max(s3, g1.w);
        if (__all(min(min(s0, s1), min(s2, s3)) >= seq)) break;
        if ((spins & 31u) == 0) bad = give_up(t0, tmo, seq);
        if (!__any(max(max(s0, s1), max(s2, s3)) >= seq)) __builtin_amdgcn_s_sleep(AG_SLEEP);
      }
      m = bad ? BAD : wave_max_u32(m);
      if (lane < 8) __hip_atomic_store(post, ((u64)seq << 32) | m, RLX_SYSTEM);
      if (src == rank) raise((u64)post + (BEST - GRP + (lane >> 3) * 128 - (rank * 8 + g) * 64), m, seq);
      STAMP(11);
      if (WBL2) asm volatile("buffer_wbl2 sc1" ::: "memory");
    }
    WAIT_VM(0);
    asm volatile("" : "+v"(old));
    if ((u32)__builtin_amdgcn_readfirstlane((int)old) != seq) {
      while (!__all(__builtin_amdgcn_raw_buffer_load_b32(mrs, (DROPV == 2 ? lane : lane & 7) * 64, GRP + (DROPV == 2 ? 0 : rank * 512) + 4, 17) >= seq)) {
        if (give_up(t0, tmo, seq)) break;
        __builtin_amdgcn_s_sleep(C_SLEEP);
      }
      if (DROPV != 1) drop();
    }
    STAMP(15);
    return;
  }

  const int kq1 = lane >> 5, wH = (lane >> 4) & 1, wi = lane & 15;
  const __amdgpu_buffer_rsrc_t wfr = __builtin_amdgcn_make_buffer_rsrc((void*)pw, 0, KD * NL * 4, 0x00020000);
  const u32 wfo = (u32)(kq1 * 8 * NL + nt * 128 + wH * 64 + wi * 4) * 4u;
  const u32 woff = (u32)(nt * W_NT + (shr >> 2) * W_KS + wH * 8192 + ((shr >> 1) & 1) * 1024 +
                         (((shr & 1) * 2 + kq1) * 16 + wi) * 16);
  const bool ext = nt < 8;
  const u32 efo = ext ? (u32)((wH * 8 + kq1 * 4) * NL + 15 * 128 + nt * 16 + wi) * 4u : NOWHERE;
  const u32 eoff = ext ? (u32)(15 * W_NT + nt * 2048 + (shr >> 2) * W_KS + ((shr >> 1) & 1) * 1024 +
                               (((shr & 1) * 2 + wH) * 16 + wi) * 16 + kq1 * 8) : NOWHERE;
  const __amdgpu_buffer_rsrc_t wrs =
      __builtin_amdgcn_make_buffer_rsrc(myws + WB_OFF, 0, (int)WB_BYTES, 0x00020000);
  GAS u32* const wfl = (GAS u32*)(myws + WFL_OFF) + nt * 256;
  const __amdgpu_buffer_rsrc_t brs = __builtin_amdgcn_make_buffer_rsrc(
      myws + WB_OFF + (size_t)nt * W_NT, 0, (int)(WB_BYTES - nt * W_NT), 0x00020000);
  const u32 l16 = lane * 16;
  const u32 ecol = (u32)((15 - nt) * W_NT + nt * 2048);
  const bool own = src == rank;
  auto tell = [&](u32 v, u32 seq) {
    u64 q;
    asm volatile("ds_read_b64 %0, %1\n\ts_waitcnt lgkmcnt(0)" : "=v"(q) : "v"(ctl + 32 * 4 + (lane & 7) * 8) : "memory");
    raise(q + (u32)(lane >> 3) * 128u, v, seq);
  };

  if (wid == 8) {
    __builtin_amdgcn_s_setprio(3);
    const u32 seq = seq8;
    bool failed = false, cleaner = false;
    int xn = 0, cw = 0;
#define TR(t) do { } while (0)
#define KBAR(t) do { BARRIER(); SCHED(); } while (0)
    f32x4 wv[9];
#pragma unroll
    for (int j = 0; j < 9; ++j) wv[j] = (f32x4){0.f, 0.f, 0.f, 0.f};
    u32 lh = lane;
    asm volatile("" : "+v"(lh));
    const u32 eoffh = ext ? (u32)(15 * W_NT + nt * 2048 + (shr >> 2) * W_KS + ((shr >> 1) & 1) * 1024 + (shr & 1) * 512) +
                            (lh & 31) * 16 + (lh >> 5) * 8 : NOWHERE;
    if (H_SLEEP) __builtin_amdgcn_s_sleep(H_SLEEP);
#pragma nounroll
    for (int t = -2; t <= T0; ++t) {
      const int c = (t >> 2) + 3;
      if ((t & 3) == 0) {
        __builtin_amdgcn_s_waitcnt(0x0f70);
        asm volatile("" ::: "memory");
        if (lane == 0 && c <= NCH) __hip_atomic_store(t ? wfl + (c - 1) * 16 + shr : cnt, seq, RLX_AGENT);
        if (c < NCH) {
          w_load(wv, wfr, wfo, c * 256 + shr * 16);
          wv[8] = e_load(wfr, efo, c * 256 + shr * 16);
        }
      } else if ((t & 3) == 2 && t > 0 && c < NCH) {
        w_store(wv, wrs, woff, c);
        e_store(wv[8], wrs, eoffh, c);
      }
      if (t == 1 && lane < 2)
        __hip_atomic_store(lane ? ownfl + (b & 1) * 512 + 384 + (b >> 1) : wfl + 2 * 16 + shr, seq, RLX_SYSTEM);
      if (t == 25 + idx && sword(tick) != seq) {
        u32 old = 0;
        if (lane == 0) old = __hip_atomic_exchange(tick, seq, RLX_SYSTEM);
        cleaner = (u32)__builtin_amdgcn_readfirstlane((int)old) != seq;
      }
      SCHED();
      const int s = t == T0 ? NKS - 1 : t + 3, gw = t < -1 ? 0 : (s >> 2) + 1;
      if (xn < NKS || cw <= min(gw, NCH - 1)) {
        u32 t0 = 0, spins = 0;
        for (;;) {
          u32x16 f0, f1, fw;
          const int on = xn + cw;
          sflags3(f0, f1, fw, xfl, (u32)min(xn, X0_END - 1) * 64u, wfl, (u32)min(cw, NCH - 1) * 64u);
          xn = x_known(xn, f0, f1, seq);
          if (min16(fw) >= seq) cw = min(cw + 1, NCH);
          if (xn > s && cw >= gw) break;
          if (xn + cw != on) continue;
          if (spins == 0) t0 = (u32)__builtin_amdgcn_s_memrealtime();
          if ((spins & 63u) == 63u) {
            GAS u64* const xf = (GAS u64*)xfl;
            const u32 vw = __hip_atomic_load(wfl + min(cw, NCH - 1) * 16 + (lane & 15), RLX_AGENT);
            const u64 v0 = __hip_atomic_load(xf + lane, RLX_SYSTEM), v1 = __hip_atomic_load(xf + 64, RLX_SYSTEM);
            if (__all(vw >= seq)) cw = min(cw + 1, NCH);
            int n = lead_ones(both_ge(v0, seq)) >> 3;
            if (n == X0_END && __all((u32)v1 >= seq)) n = __all((u32)(v1 >> 32) >= seq) ? NKS : X1_END;
            xn = max(xn, n);
          }
          ++spins;
          if (give_up(t0, tmo, seq)) { failed = true; xn = NKS; cw = NCH; break; }
          if (POLL_GAP) __builtin_amdgcn_s_sleep(POLL_GAP);
        }
      }
      if (t == T0 - 3) {
        wv[0] = __builtin_bit_cast(f32x4, __builtin_amdgcn_raw_buffer_load_b128(mrs, l16, PRE, 17));
        wv[1] = __builtin_bit_cast(f32x4, __builtin_amdgcn_raw_buffer_load_b128(mrs, min(l16 + 1024u, (u32)NGW * 8u - 16u), PRE, 17));
      } else if (t == T0 - 1) {
        const u32x4 g0 = __builtin_bit_cast(u32x4, wv[0]), g1 = __builtin_bit_cast(u32x4, wv[1]);
        const u32 c = wave_max_u32(max(max(g0.y >= seq ? g0.x : 0u, g0.w >= seq ? g0.z : 0u), max(g1.y >= seq ? g1.x : 0u, g1.w >= seq ? g1.z : 0u)));
        const u32 thr = __builtin_bit_cast(u32, __builtin_bit_cast(float, c) * ALPHA);
        asm volatile("ds_write_b128 %0, %1\n\ts_waitcnt lgkmcnt(0)" ::"v"(ctl + 20 * 4), "v"((u32x4){seq, own || c == 0 || !CAND ? ~0u : thr, 0u, 0u}) : "memory");
      }
      ARRV(t == -2 ? -1 : t);
      KBAR(t == -2 ? -1 : t);
    }
    static_assert(T0 >= 55 && T0 <= 57, "everything is published by now");
    asm volatile("ds_write2_b32 %0, %1, %2 offset0:16 offset1:21" ::"v"(ctl), "v"(failed ? BAD : 0u), "v"(0u) : "memory");
    GAS u32* const best = (GAS u32*)(myws + AMX_OFF + BEST) + (b & 7) * 32;
    u32x4 rp;
    asm volatile("ds_read_b128 %0, %1\n\ts_waitcnt lgkmcnt(0)" : "=v"(rp) : "v"(ctl + 256 + (lane & 7) * 16) : "memory");
    const u32 top = wave_max_u32(rp.x);
    if (top) {
      const int w = __builtin_ctzll(__ballot(rp.x == top));
      const u32 ao = (u32)__builtin_amdgcn_readlane((int)rp.y, w), bo = (u32)__builtin_amdgcn_readlane((int)rp.z, w);
      const u32 first = __builtin_amdgcn_readlane((int)rp.w, w) ? NOWHERE : l16;
      if (HINT) __builtin_amdgcn_raw_buffer_store_b32(seq, mrs, lane == 0 ? 0u : NOWHERE, HNT + ((b & 7) * 2 + half) * 64 + (ao >> 9), HNT_AUX);
      f32x4 cc;
      asm volatile("ds_read_b128 %0, %1\n\ts_waitcnt lgkmcnt(0)" : "=v"(cc) : "v"(ctl + 512 + w * 1024 + l16) : "memory");
      bf16x8 ca[NH], cb[NH];
      auto fetch = [&](int h0, int h1) __attribute__((always_inline)) {
#pragma unroll
        for (int h = h0; h < h1; ++h) {
          const int t = (2 * T0 + 1 + h) >> 1, kh = (h + 1) & 1;
          cb[h] = __builtin_bit_cast(bf16x8, __builtin_amdgcn_raw_buffer_load_b128(brs, l16 + kh * 1024, bo + t * W_KS, 0));
          ca[h] = __builtin_bit_cast(bf16x8, __builtin_amdgcn_raw_buffer_load_b128(ars, (h ? l16 : first) + kh * 1024, ao + t * A_BYTES, 0));
        }
        SCHED();
      };
      fetch(0, NNEAR);
      KBAR(T0 + 1);
      KBAR(T0 + 2);
#pragma unroll
      for (int h = 0; h < NNEAR; ++h) cc = __builtin_amdgcn_mfma_f32_16x16x32_bf16(cb[h], ca[h], cc, 0, 0, 0);
      SCHED();
      fetch(NNEAR, NH);
      const u32x2 known = __builtin_amdgcn_raw_buffer_load_b64(mrs, 0, BEST + (b & 7) * 128, 17);
      SCHED();
      KBAR(T0 + 3);
#pragma unroll
      for (int h = NNEAR; h < NH; ++h) cc = __builtin_amdgcn_mfma_f32_16x16x32_bf16(cb[h], ca[h], cc, 0, 0, 0);
      const u32 c = wave_max_u32(__builtin_bit_cast(u32, fmaxf(fmaxf(fmaxf(fmaxf(0.f, fabsf(cc[0])), fabsf(cc[1])), fabsf(cc[2])), fabsf(cc[3]))));
      if (__any(known.y < seq || c > known.x)) tell(c, seq);
      TR(45);
      SCHED();
    } else {
      KBAR(T0 + 1);
      KBAR(T0 + 2);
      KBAR(T0 + 3);
    }
#pragma unroll
    for (int t = T0 + 4; t <= NKS - 4; ++t) KBAR(t);
    constexpr u32 LAST = ((NKS - 1) % 3) * STAGE;
    auto four = [&](__amdgpu_buffer_rsrc_t rs, u32 to, u32 from) {
      __builtin_amdgcn_raw_ptr_buffer_load_lds(rs, (LAS void*)(lds + to), 16, l16, from, 0, 0);
      __builtin_amdgcn_raw_ptr_buffer_load_lds(rs, (LAS void*)(lds + to), 16, l16, from, 1024, 0);
      __builtin_amdgcn_raw_ptr_buffer_load_lds(rs, (LAS void*)(lds + to), 16, l16, from, 2048, 0);
      __builtin_amdgcn_raw_ptr_buffer_load_lds(rs, (LAS void*)(lds + to), 16, l16, from, 3072, 0);
      SCHED();
    };
#pragma unroll
    for (int k = 0; k < 4; ++k) four(brs, LAST + A_BYTES + k * 4096, (NKS - 1) * W_KS + k * 4096);
    __builtin_amdgcn_raw_ptr_buffer_load_lds(brs, (LAS void*)(lds + LAST + E_POS), 16, ext ? l16 : NOWHERE, (NKS - 1) * W_KS + ecol, 0, 0);
    __builtin_amdgcn_raw_ptr_buffer_load_lds(brs, (LAS void*)(lds + LAST + E_POS), 16, ext ? l16 : NOWHERE, (NKS - 1) * W_KS + ecol, 1024, 0);
    SCHED();
#pragma unroll
    for (int k = 0; k < APRE; ++k) four(ars, LAST + k * 4096, (NKS - 1) * A_BYTES + k * 4096);
    TR(40);
    BARRIER();
    TR(41);
#pragma unroll
    for (int k = APRE; k < 8; ++k) four(ars, LAST + k * 4096, (NKS - 1) * A_BYTES + k * 4096);
    u32 off = lane * 64;
    u32x2 g, bv;
    u32 tv;
    asm volatile("buffer_load_dwordx2 %0, %3, %4, %5 offen sc0 sc1\n\tbuffer_load_dwordx2 %1, off, %4, %6 sc0 sc1\n\tglobal_load_dword %2, %7, off sc1"
                 : "=&v"(g), "=&v"(bv), "=&v"(tv) : "v"(off), "s"(mrs), "s"(GRP), "s"(__builtin_amdgcn_readfirstlane((int)(BEST + (b & 7) * 128))), "v"(tmo) : "memory");
    const bool now = !own && SPEC_REM == 0;
    WAIT_VM(27);
    ctl_write(ctl + 22 * 4, 1u);
    TR(42);
    WAIT_VM(19);
    ctl_write(ctl + 22 * 4, 2u);
    TR(43);
    WAIT_VM(11);
    ctl_write(ctl + 22 * 4, 3u);
    WAIT_VM(3);
    ctl_write(ctl + 22 * 4, 4u);
    TR(44);
    asm volatile("s_waitcnt vmcnt(0)" : "+v"(g), "+v"(bv), "+v"(tv)::"memory");
    u32 m = tv == seq ? BAD : bv.y >= seq ? bv.x : 0u;
    if (now) ctl_write(ctl + 23 * 4, m);
    ARRV_OUT();
    if (WGATE == 1) BARRIER();
    else if (HP_SLEEP) __builtin_amdgcn_s_sleep(HP_SLEEP);
    const u32 t0 = (u32)__builtin_amdgcn_s_memrealtime();
    const int thr = own ? SPEC_THR : SPEC_REM;
    u64 got = 0;
    bool go = now;
    for (u32 spins = 1;; ++spins) {
      const bool ok = g.y >= seq;
      const u32 c = ok ? g.x : 0u;
      off = ok ? BEST - GRP + (b & 7) * 128 : off;
      got |= __ballot(ok);
      if (got == ~0ull) {
        const u32 a = wave_max_63(max(m, c)) | FIN;
        if (lane == 63) ctl_write(ctl + 23 * 4, a);
        break;
      }
      const bool up = __any(c > m);
      m = max(m, c);
      if (__builtin_popcountll(got) >= thr && (up || !go)) {
        go = true;
        m = wave_max_u32(m);
        ctl_write(ctl + 23 * 4, m);
      }
      if ((spins & 15u) == 0 && give_up(t0, tmo, seq)) {
        ctl_write(ctl + 23 * 4, FIN | BAD);
        break;
      }
      g = __builtin_amdgcn_raw_buffer_load_b64(mrs, off, GRP, 17);
    }
    if (cleaner) drop();
    return;
  }

  f32x4 dv[2][9];
  if (wid >= 4) {
    if (DW_SLEEP) NAP(DW_SLEEP);
#pragma unroll
    for (int d = 0; d < 2; ++d) {
      w_load<DW_AUX>(dv[d], wfr, wfo, d * 64 + (wid - 4) * 16);
      dv[d][8] = e_load(wfr, efo, d * 64 + (wid - 4) * 16);
    }
    SCHED();
  }
  f32x4 cv[9];
  const int xl = (b >> 1) * 6 + (wid - 4) * 2;
  u32 seqw = 0;
  if (wid == 2 || wid == 3) {
    const bool skip = wid == 2 && shr < 8;
    if (wid == 3) NAP(P1_SLEEP);
    else if (W2_SLEEP) NAP(W2_SLEEP);
    if (!skip) {
      w_load(cv, wfr, wfo, (wid - 2) * 256 + shr * 16);
      cv[8] = e_load(wfr, efo, (wid - 2) * 256 + shr * 16);
    }
    seqw = __hip_atomic_load(cnt, RLX_AGENT) + 1u;
    SCHED();
    if (!skip) {
      w_store(cv, wrs, woff, wid - 2);
      e_store(cv[8], wrs, eoff, wid - 2);
    }
    SCHED();
    if (wid == 2) {
      WAIT_VM(0);
      if (lane == 0) __hip_atomic_store(wfl + shr, seqw, RLX_AGENT);
      STAMP(12);
    }
  } else if (wid == 7) {
    w_load(cv, wfr, wfo, 2 * 256 + shr * 16);
    cv[8] = e_load(wfr, efo, 2 * 256 + shr * 16);
    SCHED();
  } else if (wid >= 4) {
    x_load(cv, px, b & 1, X1_END + (xl >> 4), xl & 15, lane);
    x_load(cv + 4, px, b & 1, X1_END + (xl >> 4), (xl & 15) + 1, lane);
    SCHED();
  }

  const __amdgpu_buffer_rsrc_t prs = __builtin_amdgcn_make_buffer_rsrc(myws + XP_OFF + (size_t)mt * (XP_SRC / 2), 0, (int)(XP_SRC / 2), 0x00020000);
  const u32 akb = (u32)((wid + AROT * nt) & 7) * 1024u;
  const u32 avo = akb + l16, bvo = wid * 2048 + l16;
  const u32 evo = ext ? ecol + (u32)(wid * 256 + lane * 4) : NOWHERE;
  auto piece = [&](int t, u32 slot, int j, bool pushed = false) {
    if (j == 2) {
      __builtin_amdgcn_raw_ptr_buffer_load_lds(brs, (LAS void*)(lds + slot + E_POS + wid * 256), 4, evo, t * W_KS, 0, 0);
    } else if (j < 2) {
      LAS void* lb = (LAS void*)(lds + slot + A_BYTES + wid * 2048);
      if (j == 0) __builtin_amdgcn_raw_ptr_buffer_load_lds(brs, lb, 16, bvo, t * W_KS, 0, 0);
      else __builtin_amdgcn_raw_ptr_buffer_load_lds(brs, lb, 16, bvo, t * W_KS, 1024, 0);
    } else {
      LAS void* la = (LAS void*)(lds + slot + akb + (j - 3) * 8192);
      __builtin_amdgcn_raw_ptr_buffer_load_lds(pushed ? prs : ars, la, 16, avo, t * A_BYTES + (j - 3) * 8192, 0, 0);
    }
    SCHED();
  };

  const int wr = wid >> 1, wc = wid & 1;
  const u32 aoff0 = wr * 8192 + wc * 4096 + l16, aoff1 = wr * 8192 + (1 - wc) * 4096 + l16, boff = A_BYTES + wc * 8192 + l16;
  f32x4 acc[4][4], ace[2];
#pragma unroll
  for (int i = 0; i < 4; ++i)
#pragma unroll
    for (int j = 0; j < 4; ++j) acc[i][j] = (f32x4){0.f, 0.f, 0.f, 0.f};
  ace[0] = ace[1] = (f32x4){0.f, 0.f, 0.f, 0.f};
  bf16x8 af[2][4], bfr[2][4], ef[2];
  auto frags = [&](u32 slot, int kh) {
#pragma unroll
    for (int i = 0; i < 4; ++i) {
      bfr[kh][i] = *(const bf16x8*)(lds + slot + boff + i * 2048 + kh * 1024);
      af[kh][i] = *(const bf16x8*)(lds + slot + (i < 2 ? aoff0 : aoff1) + (i & 1) * 2048 + kh * 1024);
    }
    ef[kh] = *(const bf16x8*)(lds + slot + E_POS + l16 + kh * 1024);
    SCHED();
  };
  auto mfma4 = [&](int kh, int mi) {
#pragma unroll
    for (int r = 0; r < 4; ++r)
      acc[mi][r] = __builtin_amdgcn_mfma_f32_16x16x32_bf16(bfr[kh][r], af[kh][mi], acc[mi][r], 0, 0, 0);
    if (mi >= 2) ace[mi - 2] = __builtin_amdgcn_mfma_f32_16x16x32_bf16(ef[kh], af[kh][mi - 2], ace[mi - 2], 0, 0, 0);
    SCHED();
  };

  BARRIER();
#pragma unroll
  for (int j = 0; j < 4; ++j) piece(0, 0, j + 3, j < PQ);
#pragma unroll
  for (int j = 0; j < 4; ++j) piece(1, STAGE, j + 3, j + 4 < PQ);
  if (wid >= 4) {
    char* const dst = lds + A_BYTES + wH * 8192 + ((wid >> 1) & 1) * 1024 + (((wid & 1) * 2 + kq1) * 16 + wi) * 16;
    char* const dse = lds + E_POS + ((wid >> 1) & 1) * 1024 + (((wid & 1) * 2 + wH) * 16 + wi) * 16 + kq1 * 8;
#pragma unroll
    for (int d = 0; d < 2; ++d) {
#pragma unroll
      for (int r = 0; r < 4; ++r) *(u32x4*)(dst + d * STAGE + r * 2048) = w_pack(dv[d], r);
      *(u32x2*)(dse + d * STAGE) = e_pack(dv[d][8]);
    }
    SCHED();
    if (wid == 7) {
      w_store(cv, wrs, woff, 2);
      e_store(cv[8], wrs, eoff, 2);
      SCHED();
      WAIT_VM_LGKM(9);
    } else {
      x_store(cv, xrs, b & 1, X1_END + (xl >> 4), xl & 15, lane);
      x_store(cv + 4, xrs, b & 1, X1_END + (xl >> 4), (xl & 15) + 1, lane);
      SCHED();
      WAIT_VM_LGKM(8);
    }
  } else if (wid == 3) {
    WAIT_VM(8);
    if (lane == 0) __hip_atomic_store(wfl + 16 + shr, seqw, RLX_AGENT);
    STAMP(13);
    WAIT_VM(5);
  } else {
    WAIT_VM_LGKM(4);
  }
  ARRV(-1);
  BARRIER();
  frags(0, 0);
#pragma unroll
  for (int j = 0; j < 4 - BAR_POS; ++j) piece(2, 2 * STAGE, j);

  u32 seq = 0, t0 = 0;
  auto look = [&]() __attribute__((always_inline)) {
    u32 at = ctl, l = l16;
    int wv = wid;
    asm volatile("" : "+s"(at), "+s"(wv), "+v"(l));
    u32x2 st;
    u64 now;
    asm volatile("ds_read_b64 %0, %2 offset:80\n\ts_memrealtime %1" : "=v"(st), "=s"(now) : "v"(at) : "memory");
    float bm[18];
#pragma unroll
    for (int i = 0; i < 18; ++i) {
      const f32x4& v = i < 16 ? acc[i >> 2][i & 3] : ace[i & 1];
      bm[i] = fmaxf(fmaxf(fmaxf(fmaxf(0.f, fabsf(v[0])), fabsf(v[1])), fabsf(v[2])), fabsf(v[3]));
    }
    if (!ext) bm[16] = bm[17] = 0.f;
    float mx = 0.f;
#pragma unroll
    for (int i = 0; i < 18; ++i) mx = fmaxf(mx, bm[i]);
    asm volatile("s_waitcnt lgkmcnt(0)" : "+v"(st), "+s"(now)::"memory");
    seq = (u32)__builtin_amdgcn_readfirstlane((int)st.x);
    t0 = (u32)now;
    u32x4 rp = {0u, 0u, 0u, 0u};
    if (own) {
      const u32x2 r = {wave_max_63(__builtin_bit_cast(u32, mx)), st.x};
      __builtin_amdgcn_raw_buffer_store_b64(r, mrs, l == 63 * 16 ? 0u : NOWHERE, PRE + idx * 64 + wv * 8, 17);
    } else if (__any(__builtin_bit_cast(u32, mx) > st.y)) {
      const u32 wm = wave_max_u32(__builtin_bit_cast(u32, mx)), to = at + wv * 1024 + l;
      int i0 = 0;
#pragma unroll
      for (int i = 0; i < 18; ++i)
        asm volatile("v_cmp_eq_u32 vcc, %2, %3\n\ts_cbranch_vccz 1f\n\tds_write_b128 %4, %5 offset:512\n\ts_mov_b32 %0, %1\n1:"
                     : "+s"(i0) : "n"(i), "s"(wm), "v"(bm[i]), "v"(to), "v"(i < 16 ? acc[i >> 2][i & 3] : ace[i & 1]) : "vcc", "memory");
      const int mi = i0 < 16 ? i0 >> 2 : i0 & 1;
      rp = (u32x4){wm, (u32)((wv >> 1) * 8192 + (((mi >> 1) ^ wv) & 1) * 4096 + (mi & 1) * 2048), i0 < 16 ? (u32)((wv & 1) * 8192 + (i0 & 3) * 2048) : ecol,
                   i0 < 4 * BAR_POS ? 1u : 0u};
    }
    asm volatile("ds_write_b128 %0, %1 offset:256" ::"v"(at + wv * 16), "v"(rp) : "memory");
    SCHED();
  };

  u32 cur = 0, nx1 = STAGE, nx2 = 2 * STAGE;
  auto step = [&](int t, auto more) {
    constexpr int MORE = decltype(more)::value;
    frags(cur, 1);
    if (MORE >= 2) piece(t + 2, nx2, 2);
#pragma unroll
    for (int g = 0; g < 4; ++g) {
      mfma4(0, g);
      if (MORE >= 2) piece(t + 2, nx2, 3 + g);
    }
    constexpr int BP = MORE == 1 ? BP61 : BAR_POS;
#pragma unroll
    for (int g = 0; g < BP; ++g) mfma4(1, g);
    if (MORE >= 1) {
      if (MORE >= 3 && __builtin_expect(t == T0, 0)) look();
      if (MORE >= 2) WAIT_VM_LGKM(7);
      else WAIT_VM_LGKM(0);
      ARRV(t);
      BARRIER();
      SCHED();
      frags(nx1, 0);
    }
#pragma unroll
    for (int g = BP; g < 4; ++g) {
      mfma4(1, g);
      if (MORE >= 3) piece(t + 3, cur, g - BP);
    }
    const u32 o = cur;
    cur = nx1; nx1 = nx2; nx2 = o;
  };
  static_assert(PQ <= 8, "the K loop reads the peers");
  for (int t = 0; t < NKS - 4; ++t) step(t, std::integral_constant<int, 3>());
  step(NKS - 4, std::integral_constant<int, 2>());
  step(NKS - 3, std::integral_constant<int, 1>());
  u32 late = 0, f3, ro;
  float mx = 0.f, me = 0.f;
  auto staged = [&](int n) {
    int v, have, spins;
    asm volatile("s_mov_b32 %2, 0\n"
                 "1:\n\tds_read_b32 %0, %3 offset:88\n\ts_waitcnt lgkmcnt(0)\n\tv_readfirstlane_b32 %1, %0\n\t"
                 "s_cmp_ge_i32 %1, %4\n\ts_cbranch_scc1 2f\n\ts_sleep %7\n\ts_add_u32 %2, %2, 0x1000000\n\ts_cbranch_scc0 1b\n\t"
                 "s_memrealtime vcc\n\ts_waitcnt lgkmcnt(0)\n\ts_sub_u32 vcc_lo, vcc_lo, %5\n\ts_cmp_lt_u32 vcc_lo, %6\n\ts_cbranch_scc1 1b\n"
                 "2:"
                 : "=&v"(v), "=&s"(have), "=&s"(spins) : "v"(ctl), "s"(n), "s"(t0), "n"((u32)DEADLINE_TICKS), "n"(RD_SLEEP) : "vcc", "scc", "memory");
    late = have < n ? BAD : late;
  };
  frags(cur, 1);
#pragma unroll
  for (int g = 0; g < 4; ++g) mfma4(0, g);
#pragma unroll
  for (int g = 0; g < BAR_POS; ++g) mfma4(1, g);
  asm volatile("ds_read_b32 %0, %1 offset:64" : "=v"(f3) : "v"(ctl) : "memory");
  staged(wr + 1);
  asm volatile("" : "+v"(f3));
  SCHED();
  frags(nx1, 0);
#pragma unroll
  for (int g = BAR_POS; g < 4; ++g) mfma4(1, g);
  frags(nx1, 1);
  f3 |= late;
  ro = lane == 63 ? (u32)((b & 7) * 2048 + idx * 64 + wid * 8) : NOWHERE;
  asm volatile("" : "+v"(f3), "+v"(ro));
  SCHED();
#pragma unroll
  for (int g = 0; g < 4; ++g) mfma4(0, g);
#pragma unroll
  for (int mi = 0; mi <= 4; ++mi) {
#pragma unroll
    for (int r = 0; r < 4; ++r) {
      if (mi < 4) {
        acc[mi][r] = __builtin_amdgcn_mfma_f32_16x16x32_bf16(bfr[1][r], af[1][mi], acc[mi][r], 0, 0, 0);
        if (mi < 2 && r == 3) ace[mi] = __builtin_amdgcn_mfma_f32_16x16x32_bf16(ef[1], af[1][mi], ace[mi], 0, 0, 0);
      }
      if (mi > 0) mx = fmaxf(fmaxf(fmaxf(fmaxf(mx, fabsf(acc[mi - 1][r][0])), fabsf(acc[mi - 1][r][1])), fabsf(acc[mi - 1][r][2])), fabsf(acc[mi - 1][r][3]));
      if (mi == 3 && r < 2) me = fmaxf(fmaxf(fmaxf(fmaxf(me, fabsf(ace[r][0])), fabsf(ace[r][1])), fabsf(ace[r][2])), fabsf(ace[r][3]));
      if (mi == 3 && r == 2) mx = fmaxf(mx, ext ? me : 0.f);
      SCHED();
    }
  }
  ARRV_OUT();

  const u32x2 fs = {wave_max_63(max(__builtin_bit_cast(u32, mx), f3)), seq};
  __builtin_amdgcn_raw_buffer_store_b64(fs, mrs, ro, LOC, 17);
  SCHED();
  STAMP(4);
  if (WGATE == 1) BARRIER();
  if (WGATE == 2 && lane == 63) asm volatile("ds_add_u32 %0, %1 offset:84" ::"v"(ctl), "v"(1u) : "memory");
  if (late) __hip_atomic_store(tmo, seq, RLX_AGENT);
  if (WDELAY) __builtin_amdgcn_s_sleep(WDELAY);
  const u32 wm = (u32)__builtin_amdgcn_readlane((int)fs.x, 63);
  const __amdgpu_buffer_rsrc_t ors = __builtin_amdgcn_make_buffer_rsrc(pout, 0, 4096 * NL, 0x00020000);
  const u32 orow = (u32)(mt * 256 + wr * 64 + (lane & 15)) * NL;
  const u32 oo0 = orow + wc * 32 * NL + nt * 128 + wc * 64 + (lane >> 4) * 16, oo1 = oo0 + (1 - 2 * wc) * 32 * NL;
  const u32 oe = ext ? orow + wc * 32 * NL + 15 * 128 + nt * 16 + (lane >> 4) * 4 : NOWHERE;
  u32 done = FIN;
#pragma nounroll
  for (u32 spins = 1;; ++spins) {
    u32 a;
    if (WGATE == 2) {
      u32x2 c2;
      asm volatile("ds_read2_b32 %0, %1 offset0:21 offset1:23\n\ts_waitcnt lgkmcnt(0)" : "=v"(c2) : "v"(ctl) : "memory");
      a = (u32)__builtin_amdgcn_readfirstlane((int)c2.y);
      if (__builtin_amdgcn_readfirstlane((int)c2.x) < 8 && a < FIN) a = 0;
    } else {
      a = (u32)__builtin_amdgcn_readfirstlane((int)ctl_read(ctl + 23 * 4));
    }
    if ((spins & 255u) == 0 && give_up(t0, tmo, seq)) a = FIN | BAD;
    const u32 gmax = max(a & ~FIN, wm);
    if (gmax != done && a != 0u) {
      if (a < wm) tell(wm, seq);
      done = gmax;
      const float amax = __builtin_bit_cast(float, gmax);
      const float inv = amax > 0.f ? 448.0f / amax : 0.f;
      auto scaled = [&](const f32x4& v) {
        f32x2 lo, hi;
        asm("v_pk_mul_f32 %0, %1, %2 op_sel_hi:[1,0]" : "=v"(lo) : "v"(__builtin_shufflevector(v, v, 0, 1)), "v"((f32x2){inv, inv}));
        asm("v_pk_mul_f32 %0, %1, %2 op_sel_hi:[1,0]" : "=v"(hi) : "v"(__builtin_shufflevector(v, v, 2, 3)), "v"((f32x2){inv, inv}));
        return __builtin_shufflevector(lo, hi, 0, 1, 2, 3);
      };
      auto q4 = [&](float a, float c, float d, float e) {
        const int q = __builtin_amdgcn_cvt_pk_fp8_f32(a, c, __builtin_bit_cast(int, a), false);
        return (u32)__builtin_amdgcn_cvt_pk_fp8_f32(d, e, q, true);
      };
      if (gmax > 0x7f800000u) {
        const u32 z = 0x7f7f7f7fu;
#pragma unroll
        for (int mi = 0; mi < 4; ++mi)
          __builtin_amdgcn_raw_buffer_store_b128((u32x4){z, z, z, z}, ors, (mi < 2 ? oo0 : oo1) + (mi & 1) * 16 * NL, 0, 16);
#pragma unroll
        for (int i = 0; i < 2; ++i) __builtin_amdgcn_raw_buffer_store_b32(z, ors, oe, i * 16 * NL, E_AUX);
      } else {
#pragma unroll
        for (int mi = 0; mi < 4; ++mi) {
          const f32x4 s0 = scaled(acc[mi][0]), s1 = scaled(acc[mi][1]), s2 = scaled(acc[mi][2]), s3 = scaled(acc[mi][3]);
          u32x4 d;
#pragma unroll
          for (int j = 0; j < 4; ++j) d[j] = q4(s0[j], s1[j], s2[j], s3[j]);
          if (WPACE && mi) WAIT_VM(WPACE - 1);
          __builtin_amdgcn_raw_buffer_store_b128(d, ors, (mi < 2 ? oo0 : oo1) + (mi & 1) * 16 * NL, 0, OUT_AUX);
          SCHED();
        }
#pragma unroll
        for (int i = 0; i < 2; ++i) {
          const f32x4 s = scaled(ace[i]);
          __builtin_amdgcn_raw_buffer_store_b32(q4(s[0], s[1], s[2], s[3]), ors, oe, i * 16 * NL, E_AUX);
        }
      }
    }
    if (a >= FIN) break;
    __builtin_amdgcn_s_sleep(GS_SLEEP);
  }
  if (done > 0x7f800000u) asm volatile("buffer_inv sc0 sc1" ::: "memory");
}

}

extern "C" void kernel_launch(int rank, int nranks, ncclComm_t comm, hipStream_t stream,
                              void* const* d_in, const int* in_sizes, int n_in,
                              void* d_out, int out_size,
                              const int* in_dtypes, int out_dtype,
                              void* const* d_out_all, void* const* d_ws_all, size_t ws_bytes) {
  (void)comm; (void)in_dtypes; (void)out_dtype; (void)d_out_all;
  if (nranks != NR || n_in < 2 || ws_bytes < WS_END || in_sizes[0] != 512 * KD || in_sizes[1] != KD * NL ||
      out_size != 4096 * NL || (unsigned)rank >= (unsigned)nranks) {
    fprintf(stderr, "unexpected problem geometry\n");
    return;
  }
  hipError_t e = hipFuncSetAttribute((const void*)ag_gemm_q8, hipFuncAttributeMaxDynamicSharedMemorySize, (int)LDS_BYTES);
  if (e != hipSuccess) { fprintf(stderr, "hipFuncSetAttribute: %s\n", hipGetErrorString(e)); return; }
  Params p;
  p.x = (const float*)d_in[0];
  p.w = (const float*)d_in[1];
  p.out = (unsigned char*)d_out;
  p.myws = (char*)d_ws_all[rank];
  for (int i = 0; i < NR; ++i) p.ws[i] = (char*)d_ws_all[i];
  p.rank = rank;
  p.perm = 0;
  for (int i = 0, lo = 0, hi = 4; i < NR; ++i) {
    const bool same = ((0x4d >> i) & 1) == ((0x4d >> rank) & 1);
    if (lo == OWNX) ++lo;
    const int xc = PERM ? (i == rank && OWNX >= 0 ? OWNX : same ? lo++ : hi++) : i;
    p.sws[xc] = p.ws[i];
    p.perm |= (u32)i << (4 * xc);
  }
  hipLaunchKernelGGL(ag_gemm_q8, dim3(NWG), dim3(NTHREADS), LDS_BYTES, stream, p);
  e = hipGetLastError();
  if (e != hipSuccess) fprintf(stderr, "launch: %s\n", hipGetErrorString(e));
}
```
